```python
import numpy as np

kernel_name = "gpu_ag_gemm_m4096_k4096_n4096_w8a8_dqrelu"


ABSMAX_RTOL = 0.001

COMPUTE_DTYPE = "int8"
EPILOGUE = "dequant_relu"
IN_QUANT = "int8"
OUT_QUANT = None
IN_DTYPES = {"x": "int8", "w_mat": "int8",
             "scale_x": "float32", "scale_w": "float32"}
WORLD_SIZE = 8

def setup_inputs():

    x = np.random.randint(-127, 128, size=(4096, 4096)).astype(np.int8)
    w_mat = np.random.randint(-127, 128, size=(4096, 4096)).astype(np.int8)
    scale_x = (0.02 * (1.0 + np.random.rand(1))).astype(np.float32)
    scale_w = (0.02 * (1.0 + np.random.rand(1))).astype(np.float32)
    return {"x": x, "w_mat": w_mat, "scale_x": scale_x, "scale_w": scale_w}

def reference(x, w_mat, scale_x, scale_w):
    acc = x.astype(np.float64) @ w_mat.astype(np.float64)
    y = (acc.astype(np.float32) * (scale_x * scale_w)).astype(np.float32)
    return np.maximum(y, 0.0).astype(np.float32)
```

```cpp
#include <hip/hip_runtime.h>
#include <rccl/rccl.h>
#include <cstdio>

#define HIP_CHECK(expr) do { hipError_t _e = (expr); if (_e != hipSuccess) { \
    fprintf(stderr, "HIP error %s at %s:%d\n", hipGetErrorString(_e), __FILE__, __LINE__); \
    return; } } while (0)

typedef int i32x2 __attribute__((ext_vector_type(2)));
typedef int i32x4 __attribute__((ext_vector_type(4)));
typedef int i32x16 __attribute__((ext_vector_type(16)));
typedef unsigned u32x4 __attribute__((ext_vector_type(4)));
#define LDS_AS __attribute__((address_space(3)))
#define GLB_AS __attribute__((address_space(1)))

#define PCU 4
#define PWV 4
#define NPW (8 * PCU * PWV)
#define WIN 3
#define SELFPOS 1
#define PRIO 0
#define NOCOMP 0

constexpr int NRANK = 8;
constexpr int KDIM = 4096, MSH = 512, NSH = 512;
constexpr unsigned SHARD_BYTES = MSH * KDIM;
constexpr unsigned UNIT_BYTES = 8192;
constexpr int NUNIT = SHARD_BYTES / UNIT_BYTES;
constexpr int UPW = NUNIT / NPW;
constexpr size_t STAGE_PARITY = (size_t)NRANK * SHARD_BYTES;
constexpr size_t FLAG_OFF = 2 * STAGE_PARITY;
constexpr size_t CNT_OFF = FLAG_OFF + 65536;
constexpr size_t TMO_OFF = CNT_OFF + 65536;
constexpr size_t WS_NEED = TMO_OFF + 4096;
constexpr unsigned long long DEADLINE_TICKS = 20000000ull;
constexpr int LDS_W_BYTES = KDIM * 32;
constexpr int LDS_BYTES = LDS_W_BYTES + 64;
constexpr int NTHREADS = 256 + 64 * PWV;
#define STAMP(i)

struct Args {
    const signed char* x;
    const signed char* w;
    const float* sx;
    const float* sw;
    float* out;
    char* myws;
    char* ws[NRANK];
    int rank;
    int pad;
};

__global__ void __launch_bounds__(NTHREADS, (NTHREADS + 255) / 256) ag_gemm_kernel(Args a) {
    extern __shared__ __attribute__((aligned(16))) unsigned char smem[];
    const int lane = threadIdx.x & 63;
    const int wave = __builtin_amdgcn_readfirstlane(threadIdx.x >> 6);
    const int b = blockIdx.x;
    const int rank = a.rank;
    char* const myws = a.myws;
    STAMP(0);

    GLB_AS unsigned* cnt = (GLB_AS unsigned*)(myws + CNT_OFF) + b * 16 + wave;
    const unsigned call_v = __hip_atomic_load(cnt, __ATOMIC_RELAXED, __HIP_MEMORY_SCOPE_AGENT);

    LDS_AS unsigned* lds_ctr = (LDS_AS unsigned*)(smem + LDS_W_BYTES);

    if (wave >= 4) {
        const bool active = (b >> 3) < PCU;
        const int pw = ((b >> 3) * PWV + (wave - 4)) * 8 + (b & 7);
        u32x4 v[UPW][8];
        if (active) {
            auto xr = __builtin_amdgcn_make_buffer_rsrc((void*)a.x, 0, SHARD_BYTES, 0x00020000);
            const unsigned voff = (lane & 31) * KDIM + (lane >> 5) * 16;
#pragma unroll
            for (int s = 0; s < UPW; ++s) {
                const int u = pw + s * NPW, rb = u & 7, kc = u >> 3;
                const unsigned ub = rb * 64 * KDIM + kc * 128;
#pragma unroll
                for (int f = 0; f < 8; ++f)
                    v[s][f] = __builtin_amdgcn_raw_buffer_load_b128(xr, voff + (f & 3) * 32, ub + (f >> 2) * 32 * KDIM, 0);
            }
        }
        if (wave == 4 && lane == 0) *lds_ctr = 0;
        __builtin_amdgcn_s_waitcnt(0xC07F);
        __builtin_amdgcn_s_barrier();
        const unsigned call = __builtin_amdgcn_readfirstlane(call_v) + 1u;
        if (!active) {
            if (lane == 0) __hip_atomic_store(cnt, call, __ATOMIC_RELAXED, __HIP_MEMORY_SCOPE_AGENT);
            return;
        }
        const size_t slot = (size_t)(call & 1) * STAGE_PARITY + (size_t)rank * SHARD_BYTES + 4096;
        const unsigned voff = lane * 16;
        const int rot = pw % 7;
        char* peer[8];
#pragma unroll
        for (int j = 0; j < 7; ++j) {
            int jj = j + rot; jj = jj >= 7 ? jj - 7 : jj;
            peer[j < SELFPOS ? j : j + 1] = a.ws[(rank + 1 + jj) & 7];
        }
        peer[SELFPOS] = myws;
        __builtin_amdgcn_s_waitcnt(0x0F70);
        STAMP(1);
        auto flag_unit = [&](int t) {
            const int s1 = t / 8, j1 = t % 8;
            const int u1 = pw + s1 * NPW, rb1 = u1 & 7, kc1 = u1 >> 3;
            if (lane == 0)
                __hip_atomic_store((GLB_AS unsigned*)(peer[j1] + FLAG_OFF) + (rank * 8 + rb1) * 32 + kc1, call,
                                   __ATOMIC_RELAXED, __HIP_MEMORY_SCOPE_SYSTEM);
        };
#pragma unroll
        for (int t = 0; t < 8 * UPW; ++t) {
            const int s = t / 8, j = t % 8;
            const int u = pw + s * NPW, rb = u & 7, kc = u >> 3;
            const char* dst = peer[j] + slot + (size_t)(rb * 32 + kc) * UNIT_BYTES;
#pragma unroll
            for (int f = 0; f < 8; ++f) {
                asm volatile("global_store_dwordx4 %0, %2, %1 offset:%3 sc0 sc1\n\ts_nop 1"
                             :: "v"(voff), "s"(dst), "v"(v[s][f]), "i"(f * 1024 - 4096) : "memory");
                const int done = 8 * t + f - WIN;
                const bool fl = done >= 0 && (done & 7) == 7;
                if (j != SELFPOS || fl) asm volatile("s_waitcnt vmcnt(%0)" :: "i"(WIN) : "memory");
                if (fl) flag_unit(done >> 3);
            }
        }
        asm volatile("s_waitcnt vmcnt(0)" ::: "memory");
#pragma unroll
        for (int t = (8 * 8 * UPW - WIN) / 8; t < 8 * UPW; ++t) flag_unit(t);
        if (lane == 0) __hip_atomic_store(cnt, call, __ATOMIC_RELAXED, __HIP_MEMORY_SCOPE_AGENT);
        STAMP(2);
        return;
    }

    const int idx = b >> 3;
    int src, ct, rb;
    if (idx < 4) {
        const int t = (b & 7) * 4 + idx;
        src = rank; ct = t & 15; rb = (t >> 4) * 4 + wave;
    } else {
        const int g = (b & 7) * 7 + ((idx - 4) >> 2), rs = g >> 2;
        src = (rank + 1 + (rs >> 1)) & 7; ct = (g & 3) * 4 + (idx & 3); rb = (rs & 1) * 4 + wave;
    }
    __builtin_amdgcn_s_barrier();

    {
        const signed char* wsrc = a.w + (size_t)(wave * 1024 + (lane >> 1)) * NSH + ct * 32 + (lane & 1) * 16;
#pragma unroll 8
        for (int i = 0; i < (NOCOMP > 1 ? 0 : 32); ++i)
            __builtin_amdgcn_global_load_lds((const GLB_AS void*)(wsrc + (size_t)i * 32 * NSH),
                                             (LDS_AS void*)(smem + (wave * 32 + i) * 1024), 16, 0, 0);
    }
    const unsigned call = __builtin_amdgcn_readfirstlane(call_v) + 1u;
    if (lane == 0) __hip_atomic_store(cnt, call, __ATOMIC_RELAXED, __HIP_MEMORY_SCOPE_AGENT);
    const float scale = a.sx[0] * a.sw[0];
    asm volatile("s_waitcnt vmcnt(0)" ::: "memory");
    if (lane == 0) __hip_atomic_fetch_add(lds_ctr, 1u, __ATOMIC_RELEASE, __HIP_MEMORY_SCOPE_WORKGROUP);
    while (__hip_atomic_load(lds_ctr, __ATOMIC_ACQUIRE, __HIP_MEMORY_SCOPE_WORKGROUP) < 4u) __builtin_amdgcn_s_sleep(1);
    STAMP(1);

    const char* abase = myws + (size_t)(call & 1) * STAGE_PARITY + (size_t)src * SHARD_BYTES;
    auto ar = __builtin_amdgcn_make_buffer_rsrc((void*)abase, 0, NOCOMP ? 0 : SHARD_BYTES, 0x00020000);
    const unsigned avoff = lane * 16;
    const unsigned a_rb = rb * 32 * UNIT_BYTES;
    GLB_AS unsigned* flags = (GLB_AS unsigned*)(myws + FLAG_OFF) + (src * 8 + rb) * 32 + (lane & 31);
    GLB_AS unsigned* tmo = (GLB_AS unsigned*)(myws + TMO_OFF);
    const unsigned boff = (16 * (lane >> 5) + ((lane & 15) >> 1)) * 32 + 16 * ((lane >> 4) & 1) + 8 * (lane & 1);

    i32x16 acc[2] = {};
    auto load_unit = [&](u32x4 (&f)[8], int kc) {
#pragma unroll
        for (int i = 0; i < 8; ++i)
            f[i] = __builtin_amdgcn_raw_buffer_load_b128(ar, avoff + (i & 3) * 1024, a_rb + kc * UNIT_BYTES + (i >> 2) * 4096, 17);
    };
    auto mma_unit = [&](const u32x4 (&f)[8], int kc) {
        LDS_AS unsigned char* bp = (LDS_AS unsigned char*)smem + kc * 4096 + boff;
#pragma unroll
        for (int ks = 0; ks < 4; ++ks) {
            i32x2 lo = __builtin_amdgcn_ds_read_tr8_b64_v2i32((LDS_AS i32x2*)(bp + ks * 1024));
            i32x2 hi = __builtin_amdgcn_ds_read_tr8_b64_v2i32((LDS_AS i32x2*)(bp + ks * 1024 + 256));
            i32x4 bf = {lo[0], lo[1], hi[0], hi[1]};
            acc[0] = __builtin_amdgcn_mfma_i32_32x32x32_i8(__builtin_bit_cast(i32x4, f[ks]), bf, acc[0], 0, 0, 0);
            acc[1] = __builtin_amdgcn_mfma_i32_32x32x32_i8(__builtin_bit_cast(i32x4, f[4 + ks]), bf, acc[1], 0, 0, 0);
        }
    };

    unsigned pending = 0xFFFFFFFFu;
    bool failed = false;
    const unsigned long long t0 = __builtin_amdgcn_s_memrealtime();
    while (pending) {
        unsigned ready = pending;
        {
            const unsigned f = __hip_atomic_load(flags, __ATOMIC_RELAXED, __HIP_MEMORY_SCOPE_SYSTEM);
            ready &= (unsigned)__builtin_amdgcn_ballot_w64(f >= call);
            if (!ready) {
                if (__builtin_amdgcn_readfirstlane(__hip_atomic_load(tmo, __ATOMIC_RELAXED, __HIP_MEMORY_SCOPE_AGENT)) == call ||
                    __builtin_amdgcn_s_memrealtime() - t0 > DEADLINE_TICKS) {
                    __hip_atomic_store(tmo, call, __ATOMIC_RELAXED, __HIP_MEMORY_SCOPE_AGENT);
                    failed = true;
                    break;
                }
                __builtin_amdgcn_s_sleep(2);
                continue;
            }
            __builtin_amdgcn_fence(__ATOMIC_ACQUIRE, "wavefront");
            asm volatile("" ::: "memory");
        }
        const int k0 = __builtin_ctz(ready); ready &= ready - 1;
        const bool h1 = ready != 0;
        const int k1 = h1 ? __builtin_ctz(ready) : 0; ready &= ready - 1;
        const bool h2 = ready != 0;
        const int k2 = h2 ? __builtin_ctz(ready) : 0;
        u32x4 f0[8], f1[8], f2[8];
        load_unit(f0, k0);
        if (h1) load_unit(f1, k1);
        if (h2) load_unit(f2, k2);
        mma_unit(f0, k0);
        pending &= ~(1u << k0);
        if (h1) { mma_unit(f1, k1); pending &= ~(1u << k1); }
        if (h2) { mma_unit(f2, k2); pending &= ~(1u << k2); }
    }
    STAMP(4);
    if (!failed)
        failed = __builtin_amdgcn_readfirstlane(__hip_atomic_load(tmo, __ATOMIC_RELAXED, __HIP_MEMORY_SCOPE_AGENT)) == call;

    float* orow = a.out + (size_t)(src * MSH + rb * 64 + 4 * (lane >> 5)) * NSH + ct * 32 + (lane & 31);
#pragma unroll
    for (int h = 0; h < 2; ++h)
#pragma unroll
        for (int r = 0; r < 16; ++r) {
            float y = fmaxf((float)acc[h][r] * scale, 0.0f);
            if (failed) y = __builtin_nanf("");
            orow[(size_t)(h * 32 + (r & 3) + 8 * (r >> 2)) * NSH] = y;
        }
    STAMP(5);
}

extern "C" void kernel_launch(int rank, int nranks, ncclComm_t comm,
                              hipStream_t stream,
                              void* const* d_in, const int* in_sizes, int n_in,
                              void* d_out, int out_size,
                              const int* in_dtypes, int out_dtype,
                              void* const* d_out_all,
                              void* const* d_ws_all, size_t ws_bytes) {
    if (nranks != NRANK || (unsigned)rank >= (unsigned)NRANK || n_in != 4 || ws_bytes < WS_NEED ||
        in_sizes[0] != MSH * KDIM || in_sizes[1] != KDIM * NSH || out_size != NRANK * MSH * NSH) {
        fprintf(stderr, "unexpected problem shape\n");
        return;
    }
    HIP_CHECK(hipFuncSetAttribute((const void*)ag_gemm_kernel, hipFuncAttributeMaxDynamicSharedMemorySize, LDS_BYTES));
    Args a;
    a.x = (const signed char*)d_in[0];
    a.w = (const signed char*)d_in[1];
    a.sx = (const float*)d_in[2];
    a.sw = (const float*)d_in[3];
    a.out = (float*)d_out;
    for (int p = 0; p < NRANK; ++p) a.ws[p] = (char*)d_ws_all[p];
    a.myws = (char*)d_ws_all[rank];
    a.rank = rank;
    a.pad = 0;
    ag_gemm_kernel<<<dim3(256), dim3(NTHREADS), LDS_BYTES, stream>>>(a);
    HIP_CHECK(hipGetLastError());
}
```

```cpp
#include <hip/hip_runtime.h>
#include <rccl/rccl.h>
#include <cstdio>

#define HIP_CHECK(expr) do { hipError_t _e = (expr); if (_e != hipSuccess) { \
    fprintf(stderr, "HIP error %s at %s:%d\n", hipGetErrorString(_e), __FILE__, __LINE__); \
    return; } } while (0)

typedef int i32x2 __attribute__((ext_vector_type(2)));
typedef int i32x4 __attribute__((ext_vector_type(4)));
typedef int i32x16 __attribute__((ext_vector_type(16)));
typedef unsigned u32x2 __attribute__((ext_vector_type(2)));
typedef unsigned u32x4 __attribute__((ext_vector_type(4)));
typedef float f32x4 __attribute__((ext_vector_type(4)));
typedef unsigned long long u64;
#define LDS_AS __attribute__((address_space(3)))
#define GLB_AS __attribute__((address_space(1)))

#define N1 11
#define N3 1
#define WIN 9
#define PACE 900
#define PACEW PACE
#define PACEX 1500
#define SENDT0 11.5f
#define SENDDT 0.085f
#define FSTART 6.5f
#define MINGAP 150
#define NOCOMP 0
#define TCOST 1.5f
#define PCROSS 3.0f
#define OUTAUX ""
#define FSTRIDE 128
#define PSLEEP 1

constexpr int NRANK = 8;
constexpr int KDIM = 4096, MSH = 512, NSH = 512;
constexpr unsigned BLK_BYTES = 32 * KDIM;
constexpr int NBLK = 20;
constexpr unsigned OSLOT = 4608;
constexpr int NSLOT = 7, NTASK = 16;
constexpr size_t XS_PAR = (size_t)NRANK * NBLK * BLK_BYTES;
constexpr size_t OS_OFF = 2 * XS_PAR, OS_PAR = (size_t)16 * NSLOT * NTASK * OSLOT;
constexpr size_t FX_OFF = OS_OFF + 2 * OS_PAR;
constexpr size_t FO_OFF = FX_OFF + NRANK * NBLK * 32 * 128;
constexpr size_t CNT_OFF = FO_OFF + 16 * NSLOT * NTASK * 128;
constexpr size_t TMO_OFF = CNT_OFF + 16384;
constexpr size_t DUMMY_OFF = TMO_OFF + 1024;
constexpr size_t WS_NEED = DUMMY_OFF + 4096;
constexpr u64 DEADLINE_TICKS = 20000000ull;
constexpr int TILE_STRIDE = 4352;
constexpr int LDS_MAIN = 131072;
constexpr int LDS_UNP = LDS_MAIN;
constexpr int LDS_CTR = LDS_UNP + 2 * TILE_STRIDE;
constexpr int LDS_BYTES = LDS_CTR + 128;
constexpr int NTHREADS = 768;
static_assert(NTASK * TILE_STRIDE <= LDS_MAIN, "tiles");
#define STAMP(i)
#define TRACE(i, h)
#define TRACE_OUT()

struct Args {
    const signed char* x;
    const signed char* w;
    const float* sx;
    const float* sw;
    float* out;
    char* ws[NRANK];
    u64 tab[5][NSLOT][2];
    u64 tim[4][NSLOT][2];
    int rank;
    int pad;
};

#define STORE_WT(voff, base, val, off) \
    asm volatile("global_store_dwordx4 %0, %2, %1 offset:%3 sc0 sc1\n\ts_nop 1" \
                 :: "v"(voff), "s"(base), "v"(val), "i"(off) : "memory")
#define VMCNT(n) asm volatile("s_waitcnt vmcnt(%0)" :: "i"(n) : "memory")
#define LGKMCNT0() asm volatile("s_waitcnt lgkmcnt(0)" ::: "memory")
#define CBAR() asm volatile("" ::: "memory")

__device__ __forceinline__ void set_flag(char* p, unsigned call, int lane) {
    if (lane == 0) __hip_atomic_store((GLB_AS unsigned*)p, call, __ATOMIC_RELAXED, __HIP_MEMORY_SCOPE_SYSTEM);
}
__device__ __forceinline__ unsigned lds_get(LDS_AS unsigned* p) {
    return __builtin_amdgcn_readfirstlane(__hip_atomic_load(p, __ATOMIC_RELAXED, __HIP_MEMORY_SCOPE_WORKGROUP));
}
__device__ __forceinline__ void lds_wait(LDS_AS unsigned* p, unsigned v) {
    while (lds_get(p) < v) __builtin_amdgcn_s_sleep(1);
    CBAR();
}

#define TMOEVERY 8
#define DELAY0 0
__device__ __forceinline__ unsigned tmo_get(GLB_AS unsigned* tmo) {
    return __hip_atomic_load(tmo, __ATOMIC_RELAXED, __HIP_MEMORY_SCOPE_AGENT);
}
__device__ __forceinline__ bool wait_flags(GLB_AS unsigned* p, unsigned f, GLB_AS unsigned* tmo, unsigned call, int lane) {
    if (((unsigned)__builtin_amdgcn_ballot_w64(f >= call) & 15u) == 15u) return true;
    const u64 t0 = __builtin_amdgcn_s_memrealtime();
    for (unsigned it = 1;; ++it) {
        if (lane < 4) f = __hip_atomic_load(p, __ATOMIC_RELAXED, __HIP_MEMORY_SCOPE_SYSTEM);
        if (((unsigned)__builtin_amdgcn_ballot_w64(f >= call) & 15u) == 15u) return true;
        if (it % TMOEVERY == 0 &&
            (__builtin_amdgcn_readfirstlane(tmo_get(tmo)) == call || __builtin_amdgcn_s_memrealtime() - t0 > DEADLINE_TICKS)) {
            if (lane == 0) __hip_atomic_store(tmo, call, __ATOMIC_RELAXED, __HIP_MEMORY_SCOPE_AGENT);
            return false;
        }
        __builtin_amdgcn_s_sleep(PSLEEP);
    }
}

__global__ void __launch_bounds__(NTHREADS, 3) ag_gemm_kernel(Args a) {
    extern __shared__ __attribute__((aligned(16))) unsigned char smem_g[];
    LDS_AS unsigned char* smem = (LDS_AS unsigned char*)smem_g;
    const int lane = threadIdx.x & 63;
    const int wave = __builtin_amdgcn_readfirstlane(threadIdx.x >> 6);
    const int b = blockIdx.x;
    const int rank = a.rank;
    char* const myws = a.ws[0];
    STAMP(0);

    GLB_AS unsigned* cnt = (GLB_AS unsigned*)(myws + CNT_OFF) + b * 16 + wave;
    LDS_AS unsigned* ctr = (LDS_AS unsigned*)(smem + LDS_CTR);
    const unsigned voff = lane * 16;
    const unsigned xsrc_off = (lane >> 3) * KDIM + (((lane & 7) ^ (lane >> 3)) * 16);
    unsigned lo[4];
#pragma unroll
    for (int f = 0; f < 4; ++f) lo[f] = (lane & 31) * 128 + (((2 * f + (lane >> 5)) ^ (lane & 7)) * 16);

    if ((b >> 3) < 4) {
        if (wave >= 8) return;
        const int c = b;
        const int d = wave;
        if (d == 0) {
            const signed char* xs = a.x + c * 128 + xsrc_off;
#pragma unroll
            for (int i = 0; i < 60; ++i)
                __builtin_amdgcn_global_load_lds((const GLB_AS void*)(xs + (size_t)i * 8 * KDIM), (LDS_AS void*)(smem + i * 1024), 16, 0, 0);
            __builtin_amdgcn_s_barrier();
            VMCNT(56);
            asm volatile("ds_write_b32 %0, %1" :: "v"(ctr), "v"(1u) : "memory");
#pragma unroll
            for (int i = 60; i < 64; ++i)
                __builtin_amdgcn_global_load_lds((const GLB_AS void*)(xs + (size_t)i * 8 * KDIM), (LDS_AS void*)(smem + i * 1024), 16, 0, 0);
            VMCNT(0);
            asm volatile("ds_write_b32 %0, %1\n\ts_waitcnt lgkmcnt(0)" :: "v"(ctr), "v"(16u) : "memory");
            STAMP(2);
            return;
        }
        const unsigned call_v = __hip_atomic_load(cnt, __ATOMIC_RELAXED, __HIP_MEMORY_SCOPE_AGENT);
        if (d == 1) { ctr[0] = 0; LGKMCNT0(); }
        LDS_AS unsigned char* wl = smem + 65536 + (d & 3) * 16384;
        if (d >= 4) {
            const signed char* wsrc = a.w + (size_t)(c * 128 + (lane >> 3)) * NSH + (d & 3) * 128 + (lane & 7) * 16;
#pragma unroll
            for (int i = 0; i < 16; ++i)
                __builtin_amdgcn_global_load_lds((const GLB_AS void*)(wsrc + (size_t)i * 8 * NSH), (LDS_AS void*)(wl + i * 1024), 16, 0, 0);
        }
        char* peer = a.ws[d];
        asm volatile("" : "+s"(peer));
        __builtin_amdgcn_s_barrier();
        VMCNT(0);
        const unsigned call = __builtin_amdgcn_readfirstlane(call_v) + 1u;
        const unsigned par = call & 1u;
        char* prev_flag = myws + DUMMY_OFF + (c * 8 + wave) * 4;
        const char* xdst = peer + par * XS_PAR + (size_t)d * NBLK * BLK_BYTES + c * 4096;
        char* fdst = peer + FX_OFF + (size_t)(d * NBLK * 32 + c) * FSTRIDE;
        u32x4 v[4];
        unsigned tnext = 0;
        auto pace = [&](unsigned dt) {
            if (dt) {
                while ((int)((unsigned)__builtin_amdgcn_s_memrealtime() * 16u - tnext) < 0) __builtin_amdgcn_s_sleep(1);
                tnext += dt;
            }
        };
        auto send = [&](int blk, unsigned dt, auto&& next) {
            const char* dst = xdst + (size_t)blk * BLK_BYTES;
#pragma unroll
            for (int f = 0; f < 4; ++f) {
                pace(dt);
                STORE_WT(voff, dst, v[f], f * 1024);
                v[f] = next(f);
                if (WIN == 9) {
                    if (f == 0) { VMCNT(1); set_flag(prev_flag, call, lane); }
                } else {
                    if (f < WIN || f >= 2 * WIN) VMCNT(WIN); else VMCNT(WIN + 1);
                    if (f == WIN - 1) set_flag(prev_flag, call, lane);
                }
            }
            prev_flag = fdst + blk * 32 * FSTRIDE;
        };
        int blk, nblk, step;
        if (d >= 4) {
            const unsigned boff = (16 * (lane >> 5) + ((lane & 15) >> 1)) * 128 + 16 * ((lane >> 4) & 1) + 8 * (lane & 1);
            auto wfrag = [&](int t, int f) {
                i32x2 l2 = __builtin_amdgcn_ds_read_tr8_b64_v2i32((LDS_AS i32x2*)(wl + boff + f * 4096 + t * 32));
                i32x2 h2 = __builtin_amdgcn_ds_read_tr8_b64_v2i32((LDS_AS i32x2*)(wl + boff + f * 4096 + t * 32 + 1024));
                return u32x4{(unsigned)l2[0], (unsigned)l2[1], (unsigned)h2[0], (unsigned)h2[1]};
            };
#pragma unroll
            for (int f = 0; f < 4; ++f) v[f] = wfrag(0, f);
            STAMP(1);
            tnext = (unsigned)__builtin_amdgcn_s_memrealtime() * 16u;
#pragma unroll
            for (int t = 0; t < 3; ++t) send(16 + t, PACEW, [&](int f) { return wfrag(t + 1, f); });
            lds_wait(ctr, 16u);
            blk = 15; nblk = 16 - N1; step = -1;
            send(19, PACEW, [&](int f) { return *(LDS_AS u32x4*)(smem + 15 * 4096 + lo[f]); });
            STAMP(3);
        } else {
            lds_wait(ctr, 1u);
#pragma unroll
            for (int f = 0; f < 4; ++f) v[f] = *(LDS_AS u32x4*)(smem + lo[f]);
            STAMP(1);
            tnext = (unsigned)__builtin_amdgcn_s_memrealtime() * 16u;
            send(0, PACE, [&](int f) { return v[f]; });
            lds_wait(ctr, 16u);
#pragma unroll
            for (int f = 0; f < 4; ++f) v[f] = *(LDS_AS u32x4*)(smem + 4096 + lo[f]);
            blk = 1; nblk = N1 + N3 - 1; step = 1;
        }
        const unsigned dt = d >= 4 ? PACEX : PACE;
        for (int n = 0; n < nblk; ++n) {
            const int nb = (blk + step) & 15;
            send(blk, (PACE || PACEX) ? dt : 0, [&](int f) { return *(LDS_AS u32x4*)(smem + nb * 4096 + lo[f]); });
            blk = nb;
        }
        VMCNT(0);
        set_flag(prev_flag, call, lane);
        if (lane == 0) __hip_atomic_store(cnt, call, __ATOMIC_RELAXED, __HIP_MEMORY_SCOPE_AGENT);
        STAMP(2);
        return;
    }

    const u64 tstart = __builtin_amdgcn_s_memrealtime();
    const float scale = a.sx[0] * a.sw[0];
    const int j = (b & 7) * 28 + (b >> 3) - 4;
    const int grp = j >> 4, tile = j & 15;
    const int foreign = grp & 1, slot = grp >> 1;
    const int downer = 4 + (tile >> 2);
    GLB_AS unsigned* tmo = (GLB_AS unsigned*)(myws + TMO_OFF);
    if (!foreign && wave >= 10) return;
    const unsigned call_v = __hip_atomic_load(cnt, __ATOMIC_RELAXED, __HIP_MEMORY_SCOPE_AGENT);
    if (NOCOMP) {
        const unsigned call = __builtin_amdgcn_readfirstlane(call_v) + 1u;
        if (lane == 0) __hip_atomic_store(cnt, call, __ATOMIC_RELAXED, __HIP_MEMORY_SCOPE_AGENT);
        if (wave < 8 && tile == 0 && !foreign) {
            const int d = slot + 1;
            const int blk = d < 4 ? N1 + N3 - 1 : N1;
            GLB_AS unsigned* p = (GLB_AS unsigned*)(myws + FX_OFF + (size_t)((d * NBLK + blk) * 32 + 4 * wave + (lane & 3)) * FSTRIDE);
            wait_flags(p, 0, tmo, call, lane);
            STAMP(3);
        }
        return;
    }

    const int su = (slot * 16 + tile) * 8 + wave;
    const bool selfcopy = foreign && wave < 8 && su < 512;
    if (wave == 8 && lane < 32) ctr[lane] = 0;
    if (DELAY0 && !foreign) __builtin_amdgcn_s_sleep(DELAY0);
    if (wave < 8 && !foreign) {
        const signed char* wsrc = a.w + (size_t)(wave * 512 + (lane >> 1)) * NSH + tile * 32 + (lane & 1) * 16;
#pragma unroll
        for (int i = 0; i < 16; ++i)
            __builtin_amdgcn_global_load_lds((const GLB_AS void*)(wsrc + (size_t)i * 32 * NSH),
                                             (LDS_AS void*)(smem + (wave * 16 + i) * 1024), 16, 0, 0);
    }
    if (selfcopy) {
        const signed char* xs = a.x + (size_t)(su >> 5) * BLK_BYTES + (su & 31) * 128 + xsrc_off;
#pragma unroll
        for (int i = 0; i < 4; ++i)
            __builtin_amdgcn_global_load_lds((const GLB_AS void*)(xs + (size_t)i * 8 * KDIM),
                                             (LDS_AS void*)(smem + (wave * 16 + i) * 1024), 16, 0, 0);
    }
    LGKMCNT0();
    __builtin_amdgcn_s_barrier();
    CBAR();
    const unsigned call = __builtin_amdgcn_readfirstlane(call_v) + 1u;
    const unsigned par = call & 1u;
    if (lane == 0) __hip_atomic_store(cnt, call, __ATOMIC_RELAXED, __HIP_MEMORY_SCOPE_AGENT);
    const float fnan = __builtin_nanf("");
    const int tsel = foreign ? 1 + (tile & 3) : 0;
    u64 t_lo = a.tab[tsel][slot][0], t_hi = a.tab[tsel][slot][1];

    auto own_epilogue = [&](int i, int e) {
        const bool bad = lds_get(ctr + 17) != 0;
        LDS_AS unsigned char* tp = smem + i * TILE_STRIDE + 128;
        const int srank = rank ^ (e >> 4);
        float* op = a.out + (size_t)(srank * MSH + (e & 15) * 32 + (lane >> 3)) * NSH + tile * 32 + (lane & 7) * 4;
        i32x4 q[4];
#pragma unroll
        for (int k = 0; k < 4; ++k) q[k] = *(LDS_AS i32x4*)(tp + k * 1024 + voff);
#pragma unroll
        for (int k = 0; k < 4; ++k) {
            f32x4 o;
#pragma unroll
            for (int r = 0; r < 4; ++r) o[r] = bad ? fnan : fmaxf((float)q[k][r] * scale, 0.0f);
            asm volatile("global_store_dwordx4 %0, %1, off " OUTAUX "\n\ts_nop 1" :: "v"(op + (size_t)k * 8 * NSH), "v"(o) : "memory");
        }
    };

    if (wave >= 8) {
        unsigned tlast = 0;
        if (wave & 1) { t_lo = (t_lo >> 8) | (t_hi << 56); t_hi = (t_hi >> 8) | (0xFFull << 56); }
        const size_t oslot = OS_OFF + par * OS_PAR + (size_t)((tile * NSLOT + slot) * NTASK) * OSLOT;
        const size_t oflag = FO_OFF + (size_t)((tile * NSLOT + slot) * NTASK) * 128;
        for (int i = wave & 1; i < NTASK; i += 2) {
            const int e = (int)(t_lo & 0xFF), en = (int)((t_lo >> 8) & 0xFF);
            t_lo = (t_lo >> 16) | (t_hi << 48); t_hi = (t_hi >> 16) | (0xFFFFull << 48);
            if (e == 0xFF) break;
            const int d = e >> 4, blk = e & 15;
            if (!foreign) {
                if (en == 0xFF) break;
                lds_wait(ctr + i, 8u);
                TRACE(i, 0);
                own_epilogue(i, e);
                TRACE(i, 1);
                continue;
            }
            if (d == downer) continue;
            if (wave < 10) {
                lds_wait(ctr + i, 8u);
                const bool failed = lds_get(ctr + 17) != 0;
                LDS_AS unsigned char* tp = smem + i * TILE_STRIDE;
                i32x4 q[4];
#pragma unroll
                for (int k = 0; k < 4; ++k) q[k] = *(LDS_AS i32x4*)(tp + 128 + k * 1024 + voff);
                __builtin_amdgcn_s_waitcnt(0xC07F);
                CBAR();
                unsigned base = 0, hlo = 0, hhi = 0;
#pragma unroll
                for (int r = 0; r < 16; ++r) {
                    float y = fmaxf((float)q[r >> 2][r & 3] * scale, 0.0f);
                    if (failed) y = fnan;
                    const unsigned bits = __builtin_bit_cast(unsigned, y);
                    const u64 m = __builtin_amdgcn_ballot_w64(bits != 0);
                    const unsigned pos = __builtin_amdgcn_mbcnt_hi((unsigned)(m >> 32), __builtin_amdgcn_mbcnt_lo((unsigned)m, base));
                    if (bits != 0) *(LDS_AS unsigned*)(tp + 128 + pos * 4) = bits;
                    base += __builtin_popcountll(m);
                    hlo = lane == r ? (unsigned)m : hlo;
                    hhi = lane == r ? (unsigned)(m >> 32) : hhi;
                }
                if (lane < 16) *(LDS_AS u32x2*)(tp + lane * 8) = u32x2{hlo, hhi};
                CBAR();
                const unsigned nbytes = (128 + base * 4 + 63) & ~63u;
                const char* dst = a.ws[downer] + oslot + (size_t)i * OSLOT + 2048;
                u32x4 pv[5];
#pragma unroll
                for (int k = 0; k < 5; ++k) pv[k] = *(LDS_AS u32x4*)(tp + k * 1024 + voff);
                {
                    unsigned nb = (unsigned)((a.tim[tile & 3][slot][i >> 3] >> (8 * (i & 7))) & 0xFF) * 16u;
                    nb = nb > tlast ? nb : tlast;
                    unsigned now;
                    while ((now = (unsigned)(__builtin_amdgcn_s_memrealtime() - tstart)) < nb) __builtin_amdgcn_s_sleep(1);
                    tlast = now + MINGAP;
                }
                TRACE(i, 0);
#pragma unroll
                for (int k = 0; k < 5; ++k)
                    if (k * 1024 + voff < nbytes) STORE_WT(voff, dst, pv[k], k * 1024 - 2048);
                VMCNT(0);
                set_flag(a.ws[downer] + oflag + i * 128, call, lane);
                TRACE(i, 1);
            } else {
                GLB_AS unsigned* p = (GLB_AS unsigned*)(myws + oflag + i * 128);
                bool failed = !wait_flags(p, 0, tmo, call, lane);
                CBAR();
                TRACE(i, 0);
                LDS_AS unsigned char* tp = smem + LDS_UNP + (wave & 1) * TILE_STRIDE;
                const char* srcp = myws + oslot + (size_t)i * OSLOT + voff;
#pragma unroll
                for (int k = 0; k < 4; ++k)
                    __builtin_amdgcn_global_load_lds((const GLB_AS void*)(srcp + k * 1024), (LDS_AS void*)(tp + k * 1024), 16, 0, 17);
                if (lane < 8)
                    __builtin_amdgcn_global_load_lds((const GLB_AS void*)(srcp + 4096), (LDS_AS void*)(tp + 4096), 16, 0, 17);
                const unsigned tv = __hip_atomic_load(tmo, __ATOMIC_RELAXED, __HIP_MEMORY_SCOPE_AGENT);
                VMCNT(0);
                failed = failed || __builtin_amdgcn_readfirstlane(tv) == call;
                const u32x2 h = *(LDS_AS u32x2*)(tp + (lane & 15) * 8);
                unsigned base = 0;
                unsigned y[16];
#pragma unroll
                for (int r = 0; r < 16; ++r) {
                    const unsigned mlo = __builtin_amdgcn_readlane(h[0], r), mhi = __builtin_amdgcn_readlane(h[1], r);
                    const unsigned pos = __builtin_amdgcn_mbcnt_hi(mhi, __builtin_amdgcn_mbcnt_lo(mlo, base));
                    const unsigned val = *(LDS_AS unsigned*)(tp + 128 + (pos & 1023) * 4);
                    const bool nz = ((lane < 32 ? mlo : mhi) >> (lane & 31)) & 1;
                    y[r] = nz ? val : 0u;
                    if (failed) y[r] = __builtin_bit_cast(unsigned, fnan);
                    base += __builtin_popcount(mlo) + __builtin_popcount(mhi);
                }
                const int srank = rank ^ downer ^ d;
                float* op = a.out + (size_t)(srank * MSH + blk * 32 + (lane >> 3)) * NSH + tile * 32 + (lane & 7) * 4;
#pragma unroll
                for (int k = 0; k < 4; ++k) {
                    const u32x4 o = {y[4 * k], y[4 * k + 1], y[4 * k + 2], y[4 * k + 3]};
                    asm volatile("global_store_dwordx4 %0, %1, off " OUTAUX "\n\ts_nop 1" :: "v"(op + (size_t)k * 8 * NSH), "v"(o) : "memory");
                }
                TRACE(i, 1);
            }
        }
        STAMP(5);
        TRACE_OUT();
        return;
    }

    bool failed = false;
    const int dskip = foreign ? downer : 8;
    char* const fx = myws + FX_OFF + (size_t)(4 * wave + (lane & 3)) * FSTRIDE;
    auto flag_ptr = [&](int eb) { return (GLB_AS unsigned*)(fx + (size_t)eb * 32 * FSTRIDE); };
    auto ar = __builtin_amdgcn_make_buffer_rsrc((void*)(myws + par * XS_PAR), 0, (unsigned)XS_PAR, 0x00020000);
    VMCNT(0);
    i32x4 bf[16];
    if (selfcopy) {
        u32x4 v[4];
#pragma unroll
        for (int f = 0; f < 4; ++f) v[f] = *(LDS_AS u32x4*)(smem + wave * 16384 + lo[f]);
        const char* dst = myws + par * XS_PAR + (size_t)su * 4096;
#pragma unroll
        for (int f = 0; f < 4; ++f) STORE_WT(voff, dst, v[f], f * 1024);
        VMCNT(0);
        set_flag(myws + FX_OFF + (size_t)su * FSTRIDE, call, lane);
        STAMP(2);
    }
    if (!foreign) {
        const unsigned boff = (16 * (lane >> 5) + ((lane & 15) >> 1)) * 32 + 16 * ((lane >> 4) & 1) + 8 * (lane & 1);
        LDS_AS unsigned char* bp = smem + wave * 16384 + boff;
#pragma unroll
        for (int i = 0; i < 16; ++i) {
            i32x2 l2 = __builtin_amdgcn_ds_read_tr8_b64_v2i32((LDS_AS i32x2*)(bp + i * 1024));
            i32x2 h2 = __builtin_amdgcn_ds_read_tr8_b64_v2i32((LDS_AS i32x2*)(bp + i * 1024 + 256));
            bf[i] = i32x4{l2[0], l2[1], h2[0], h2[1]};
        }
        __builtin_amdgcn_s_waitcnt(0xC07F);
        CBAR();
    }
    if (wave * 16384 < NTASK * TILE_STRIDE) {
#pragma unroll
        for (int i = 0; i < 16; ++i) *(LDS_AS u32x4*)(smem + wave * 16384 + i * 1024 + voff) = u32x4{0, 0, 0, 0};
    }
    LGKMCNT0();
    if (lane == 0) __hip_atomic_fetch_add(ctr + 16, 1u, __ATOMIC_RELAXED, __HIP_MEMORY_SCOPE_WORKGROUP);
    if (foreign) {
        const int eb = downer * NBLK + 16 + (tile & 3);
        failed = !wait_flags(flag_ptr(eb), 0, tmo, call, lane);
        CBAR();
        STAMP(6);
        const unsigned bbase = (unsigned)eb * BLK_BYTES + wave * 16384;
#pragma unroll
        for (int k = 0; k < 16; ++k)
            bf[k] = __builtin_bit_cast(i32x4, __builtin_amdgcn_raw_buffer_load_b128(ar, voff + (k & 3) * 1024, bbase + (k >> 2) * 4096, 17));
    }
    lds_wait(ctr + 16, 8u);
    STAMP(1);
    unsigned fpre = 0;
    for (int i = 0; i < NTASK; ++i) {
        const int e = (int)(t_lo & 0xFF);
        t_lo = (t_lo >> 8) | (t_hi << 56); t_hi = (t_hi >> 8) | (0xFFull << 56);
        if (e == 0xFF) break;
        if ((e >> 4) == dskip) { fpre = 0; continue; }
        const int eb = (e >> 4) * NBLK + (e & 15);
        if (!failed) failed = !wait_flags(flag_ptr(eb), fpre, tmo, call, lane);
        CBAR();
        TRACE(i, 0);
        const int en = (int)(t_lo & 0xFF);
        if (lane < 4) fpre = __hip_atomic_load(flag_ptr(((en >> 4) & 7) * NBLK + (en & 15)), __ATOMIC_RELAXED, __HIP_MEMORY_SCOPE_SYSTEM);
        unsigned tv = 0;
        if (lane == 0) tv = tmo_get(tmo);
        u32x4 af[16];
        const unsigned abase = (unsigned)eb * BLK_BYTES + wave * 16384;
#pragma unroll
        for (int k = 0; k < 16; ++k)
            af[k] = __builtin_amdgcn_raw_buffer_load_b128(ar, voff + (k & 3) * 1024, abase + (k >> 2) * 4096, 17);
        __builtin_amdgcn_sched_barrier(0);
        i32x16 acc = {};
#pragma unroll
        for (int k = 0; k < 16; ++k)
            acc = __builtin_amdgcn_mfma_i32_32x32x32_i8(__builtin_bit_cast(i32x4, af[k]), bf[k], acc, 0, 0, 0);
        failed = failed || __builtin_amdgcn_readfirstlane(tv) == call;
        if (failed && lane == 0) __hip_atomic_store(ctr + 17, 1u, __ATOMIC_RELAXED, __HIP_MEMORY_SCOPE_WORKGROUP);
        LDS_AS unsigned char* tp = smem + i * TILE_STRIDE + 128;
#pragma unroll
        for (int r = 0; r < 16; ++r)
            __hip_atomic_fetch_add((LDS_AS int*)(tp + (((r & 3) + 8 * (r >> 2) + 4 * (lane >> 5)) * 32 + (lane & 31)) * 4), acc[r],
                                   __ATOMIC_RELAXED, __HIP_MEMORY_SCOPE_WORKGROUP);
        LGKMCNT0();
        unsigned old = 0;
        if (lane == 0) old = __hip_atomic_fetch_add(ctr + i, 1u, __ATOMIC_RELAXED, __HIP_MEMORY_SCOPE_WORKGROUP);
        TRACE(i, 1);
        if (!foreign && en == 0xFF && __builtin_amdgcn_readfirstlane(old) == 7u) {
            CBAR();
            own_epilogue(i, e);
        }
    }
    STAMP(5);
    TRACE_OUT();
}

static void build_table(int foreign, float tstart, u64 (*tab)[2], float (*fin)[NTASK] = nullptr) {
    struct T { float t; int e; } task[128];
    int n = 0;
    for (int d = 0; d < 8; ++d)
        for (int blk = 0; blk < 16; ++blk) {
            const int type = blk < N1 ? 1 : blk < N1 + N3 ? 3 : 2;
            float t;
            if (d == 0) { if (foreign && type != 1) continue; t = 3.0f; }
            else if (d < 4) { if (type == 2 || (foreign && type == 3)) continue; t = 3.0f + PACE * (4 * 0.01f / 16) * (blk + 1); }
            else { if (type == 1 || (foreign && type == 3)) continue; t = 12.0f + PACEX * (4 * 0.01f / 16) * (16 - blk); }
            task[n].t = t + 0.001f * d; task[n].e = d << 4 | blk; ++n;
        }
    for (int i = 1; i < n; ++i)
        for (int k = i; k > 0 && task[k].t < task[k - 1].t; --k) { T s = task[k]; task[k] = task[k - 1]; task[k - 1] = s; }
    float fr[NSLOT]; int cntv[NSLOT]; unsigned char lst[NSLOT][NTASK];
    for (int s = 0; s < NSLOT; ++s) { fr[s] = tstart; cntv[s] = 0; for (int i = 0; i < NTASK; ++i) lst[s][i] = 0xFF; }
    for (int i = 0; i < n; ++i) {
        int best = -1;
        for (int s = 0; s < NSLOT; ++s) if (cntv[s] < NTASK - 1 && (best < 0 || fr[s] < fr[best])) best = s;
        fr[best] = (fr[best] > task[i].t ? fr[best] : task[i].t) + TCOST;
        if (fin) fin[best][cntv[best]] = fr[best];
        lst[best][cntv[best]++] = (unsigned char)task[i].e;
    }
    for (int s = 0; s < NSLOT; ++s)
        for (int h = 0; h < 2; ++h) {
            u64 v = 0;
            for (int i = 7; i >= 0; --i) v = v << 8 | lst[s][h * 8 + i];
            tab[s][h] = v;
        }
}

static void build_all(Args& a) {
    build_table(0, 6.5f, a.tab[0]);
    float fin[4][NSLOT][NTASK];
    for (int t = 0; t < 4; ++t) {
        for (int s = 0; s < NSLOT; ++s) for (int i = 0; i < NTASK; ++i) fin[t][s][i] = 1e9f;
        build_table(1, FSTART + PACEW * (4 * 0.01f / 16) * t, a.tab[1 + t], fin[t]);
        for (int s = 0; s < NSLOT; ++s) a.tim[t][s][0] = a.tim[t][s][1] = 0;
    }
    float tnext = SENDT0;
    for (;;) {
        int bt = -1, bs = 0, bi = 0;
        for (int t = 0; t < 4; ++t) for (int s = 0; s < NSLOT; ++s) for (int i = 0; i < NTASK; ++i)
            if (fin[t][s][i] < 1e8f && (bt < 0 || fin[t][s][i] < fin[bt][bs][bi])) { bt = t; bs = s; bi = i; }
        if (bt < 0) break;
        if (tnext < fin[bt][bs][bi] - 1.0f) tnext = fin[bt][bs][bi] - 1.0f;
        int q = (int)(tnext / 0.16f); q = q > 255 ? 255 : q;
        a.tim[bt][bs][bi >> 3] |= (u64)q << (8 * (bi & 7));
        fin[bt][bs][bi] = 1e9f;
        tnext += SENDDT;
    }
}

extern "C" void kernel_launch(int rank, int nranks, ncclComm_t comm,
                              hipStream_t stream,
                              void* const* d_in, const int* in_sizes, int n_in,
                              void* d_out, int out_size,
                              const int* in_dtypes, int out_dtype,
                              void* const* d_out_all,
                              void* const* d_ws_all, size_t ws_bytes) {
    if (nranks != NRANK || (unsigned)rank >= (unsigned)NRANK || n_in != 4 || ws_bytes < WS_NEED ||
        in_sizes[0] != MSH * KDIM || in_sizes[1] != KDIM * NSH || out_size != NRANK * MSH * NSH) {
        fprintf(stderr, "unexpected problem shape\n");
        return;
    }
    HIP_CHECK(hipFuncSetAttribute((const void*)ag_gemm_kernel, hipFuncAttributeMaxDynamicSharedMemorySize, LDS_BYTES));
    Args a;
    a.x = (const signed char*)d_in[0];
    a.w = (const signed char*)d_in[1];
    a.sx = (const float*)d_in[2];
    a.sw = (const float*)d_in[3];
    a.out = (float*)d_out;
    for (int d = 0; d < NRANK; ++d) a.ws[d] = (char*)d_ws_all[rank ^ d];
    build_all(a);
    a.rank = rank;
    a.pad = 0;
    ag_gemm_kernel<<<dim3(256), dim3(NTHREADS), LDS_BYTES, stream>>>(a);
    HIP_CHECK(hipGetLastError());
}
```

```cpp
#include <hip/hip_runtime.h>
#include <rccl/rccl.h>
#include <cstdio>

#define HIP_CHECK(expr) do { hipError_t _e = (expr); if (_e != hipSuccess) { \
    fprintf(stderr, "HIP error %s at %s:%d\n", hipGetErrorString(_e), __FILE__, __LINE__); \
    return; } } while (0)

typedef int i32x2 __attribute__((ext_vector_type(2)));
typedef int i32x4 __attribute__((ext_vector_type(4)));
typedef int i32x16 __attribute__((ext_vector_type(16)));
typedef unsigned u32x2 __attribute__((ext_vector_type(2)));
typedef unsigned u32x4 __attribute__((ext_vector_type(4)));
typedef float f32x4 __attribute__((ext_vector_type(4)));
typedef unsigned long long u64;
#define LDS_AS __attribute__((address_space(3)))
#define GLB_AS __attribute__((address_space(1)))

#define N1 12
#define N3 0
#define WIN 9
#define PACE 900
#define PACEW PACE
#define PACEX PACE
#define SENDT0 20.8f
#define SENDDT 0.04f
#define FSTART 6.5f
#define MINGAP 150
#define NOCOMP 0
#define TCOST 1.5f
#define NRAW 2
#define UNPAUX OUTAUX
#define SGAP 1.8f
#define T0X 3.2f
#define OUTAUX "sc0 sc1"
#define FSTRIDE 128
#define PSLEEP 1

constexpr int NRANK = 8;
constexpr int KDIM = 4096, MSH = 512, NSH = 512;
constexpr unsigned BLK_BYTES = 32 * KDIM;
constexpr int NBLK = 20;
constexpr unsigned OSLOT = 4608;
constexpr int NSLOT = 7, NOWN = 6, NTASK = 16, NIDX = 256;
constexpr size_t XS_PAR = (size_t)NRANK * NBLK * BLK_BYTES;
constexpr size_t OS_OFF = 2 * XS_PAR, OS_PAR = (size_t)4 * NIDX * OSLOT;
constexpr size_t FX_OFF = OS_OFF + 2 * OS_PAR;
constexpr size_t FO_OFF = FX_OFF + NRANK * NBLK * 32 * 128;
constexpr size_t CNT_OFF = FO_OFF + 4 * NIDX * 128;
constexpr size_t TMO_OFF = CNT_OFF + 16384;
constexpr size_t DUMMY_OFF = TMO_OFF + 1024;
constexpr size_t WS_NEED = DUMMY_OFF + 4096;
constexpr u64 DEADLINE_TICKS = 20000000ull;
constexpr int TILE_STRIDE = 4352;
constexpr int LDS_MAIN = 131072;
constexpr int LDS_CTR = LDS_MAIN;
constexpr int LDS_BYTES = LDS_CTR + 128;
constexpr int NTHREADS = 768;
static_assert(NTASK * TILE_STRIDE <= LDS_MAIN, "tiles");
#define STAMP(i)
#define TRACE(i, h)
#define TRACE_OUT()

struct Args {
    const signed char* x;
    const signed char* w;
    const float* sx;
    const float* sw;
    float* out;
    char* ws[NRANK];
    float* outp[4];
    u64 tab[5][NSLOT][2];
    u64 tim[4][NSLOT][2];
    u64 ord[4][NSLOT][2];
    unsigned short ent[NIDX];
    int rank;
    int pad;
};

#define STORE_WT(voff, base, val, off) \
    asm volatile("global_store_dwordx4 %0, %2, %1 offset:%3 sc0 sc1\n\ts_nop 1" \
                 :: "v"(voff), "s"(base), "v"(val), "i"(off) : "memory")
#define VMCNT(n) asm volatile("s_waitcnt vmcnt(%0)" :: "i"(n) : "memory")
#define LGKMCNT0() asm volatile("s_waitcnt lgkmcnt(0)" ::: "memory")
#define CBAR() asm volatile("" ::: "memory")

__device__ __forceinline__ void set_flag(char* p, unsigned call, int lane) {
    if (lane == 0) __hip_atomic_store((GLB_AS unsigned*)p, call, __ATOMIC_RELAXED, __HIP_MEMORY_SCOPE_SYSTEM);
}
__device__ __forceinline__ unsigned lds_get(LDS_AS unsigned* p) {
    return __builtin_amdgcn_readfirstlane(__hip_atomic_load(p, __ATOMIC_RELAXED, __HIP_MEMORY_SCOPE_WORKGROUP));
}
__device__ __forceinline__ void lds_wait(LDS_AS unsigned* p, unsigned v) {
    while (lds_get(p) < v) __builtin_amdgcn_s_sleep(1);
    CBAR();
}

#define TMOEVERY 8
#define DELAYALL 0
#define DELAY0 0
__device__ __forceinline__ unsigned tmo_get(GLB_AS unsigned* tmo) {
    return __hip_atomic_load(tmo, __ATOMIC_RELAXED, __HIP_MEMORY_SCOPE_AGENT);
}
__device__ __forceinline__ bool wait_flags(GLB_AS unsigned* p, unsigned f, GLB_AS unsigned* tmo, unsigned call, int lane) {
    if (((unsigned)__builtin_amdgcn_ballot_w64(f >= call) & 15u) == 15u) return true;
    const u64 t0 = __builtin_amdgcn_s_memrealtime();
    for (unsigned it = 1;; ++it) {
        if (lane < 4) f = __hip_atomic_load(p, __ATOMIC_RELAXED, __HIP_MEMORY_SCOPE_SYSTEM);
        if (((unsigned)__builtin_amdgcn_ballot_w64(f >= call) & 15u) == 15u) return true;
        if (it % TMOEVERY == 0 &&
            (__builtin_amdgcn_readfirstlane(tmo_get(tmo)) == call || __builtin_amdgcn_s_memrealtime() - t0 > DEADLINE_TICKS)) {
            if (lane == 0) __hip_atomic_store(tmo, call, __ATOMIC_RELAXED, __HIP_MEMORY_SCOPE_AGENT);
            return false;
        }
        __builtin_amdgcn_s_sleep(PSLEEP);
    }
}

__global__ void __launch_bounds__(NTHREADS, 3) ag_gemm_kernel(Args a_) {
    const __attribute__((address_space(4))) Args* ap = (const __attribute__((address_space(4))) Args*)__builtin_amdgcn_kernarg_segment_ptr();
    const __attribute__((address_space(4))) Args& a = *ap;
    extern __shared__ __attribute__((aligned(16))) unsigned char smem_g[];
    LDS_AS unsigned char* smem = (LDS_AS unsigned char*)smem_g;
    const int lane = threadIdx.x & 63;
    const int wave = __builtin_amdgcn_readfirstlane(threadIdx.x >> 6);
    const int b = blockIdx.x;
    const int rank = a.rank;
    char* const myws = a.ws[0];
    STAMP(7);

    GLB_AS unsigned* cnt = (GLB_AS unsigned*)(myws + CNT_OFF) + b * 16 + wave;
    LDS_AS unsigned* ctr = (LDS_AS unsigned*)(smem + LDS_CTR);
    const unsigned voff = lane * 16;
    const unsigned xsrc_off = (lane >> 3) * KDIM + (((lane & 7) ^ (lane >> 3)) * 16);
    unsigned lo[4];
#pragma unroll
    for (int f = 0; f < 4; ++f) lo[f] = (lane & 31) * 128 + (((2 * f + (lane >> 5)) ^ (lane & 7)) * 16);

    if ((b >> 3) < 4) {
        if (wave >= 8) return;
        const int c = b;
        const int d = wave;
        if (d == 0) {
            const signed char* xs = a.x + c * 128 + xsrc_off;
#pragma unroll
            for (int i = 0; i < 60; ++i)
                __builtin_amdgcn_global_load_lds((const GLB_AS void*)(xs + (size_t)i * 8 * KDIM), (LDS_AS void*)(smem + i * 1024), 16, 0, 0);
            __builtin_amdgcn_s_barrier();
            VMCNT(56);
            asm volatile("ds_write_b32 %0, %1" :: "v"(ctr), "v"(1u) : "memory");
#pragma unroll
            for (int i = 60; i < 64; ++i)
                __builtin_amdgcn_global_load_lds((const GLB_AS void*)(xs + (size_t)i * 8 * KDIM), (LDS_AS void*)(smem + i * 1024), 16, 0, 0);
            VMCNT(0);
            asm volatile("ds_write_b32 %0, %1\n\ts_waitcnt lgkmcnt(0)" :: "v"(ctr), "v"(16u) : "memory");
            STAMP(2);
            return;
        }
        const unsigned call_v = __hip_atomic_load(cnt, __ATOMIC_RELAXED, __HIP_MEMORY_SCOPE_AGENT);
        if (d == 1) { ctr[0] = 0; LGKMCNT0(); }
        LDS_AS unsigned char* wl = smem + 65536 + (d & 3) * 16384;
        if (d >= 4) {
            const signed char* wsrc = a.w + (size_t)(c * 128 + (lane >> 3)) * NSH + (d & 3) * 128 + (lane & 7) * 16;
#pragma unroll
            for (int i = 0; i < 16; ++i)
                __builtin_amdgcn_global_load_lds((const GLB_AS void*)(wsrc + (size_t)i * 8 * NSH), (LDS_AS void*)(wl + i * 1024), 16, 0, 0);
        }
        char* peer = a.ws[d];
        asm volatile("" : "+s"(peer));
        __builtin_amdgcn_s_barrier();
        VMCNT(0);
        const unsigned call = __builtin_amdgcn_readfirstlane(call_v) + 1u;
        const unsigned par = call & 1u;
        char* prev_flag = myws + DUMMY_OFF + (c * 8 + wave) * 4;
        const char* xdst = peer + par * XS_PAR + (size_t)d * NBLK * BLK_BYTES + c * 4096;
        char* fdst = peer + FX_OFF + (size_t)(d * NBLK * 32 + c) * FSTRIDE;
        u32x4 v[4];
        unsigned tnext = 0;
        auto pace = [&](unsigned dt) {
            if (dt) {
                while ((int)((unsigned)__builtin_amdgcn_s_memrealtime() * 16u - tnext) < 0) __builtin_amdgcn_s_sleep(1);
                tnext += dt;
            }
        };
        auto send = [&](int blk, unsigned dt, auto&& next) {
            const char* dst = xdst + (size_t)blk * BLK_BYTES;
#pragma unroll
            for (int f = 0; f < 4; ++f) {
                pace(dt);
                STORE_WT(voff, dst, v[f], f * 1024);
                v[f] = next(f);
                if (WIN == 9) {
                    if (f == 0) { VMCNT(1); set_flag(prev_flag, call, lane); }
                } else {
                    if (f < WIN || f >= 2 * WIN) VMCNT(WIN); else VMCNT(WIN + 1);
                    if (f == WIN - 1) set_flag(prev_flag, call, lane);
                }
            }
            prev_flag = fdst + blk * 32 * FSTRIDE;
        };
        int blk, nblk, step;
        if (d >= 4) {
            const unsigned boff = (16 * (lane >> 5) + ((lane & 15) >> 1)) * 128 + 16 * ((lane >> 4) & 1) + 8 * (lane & 1);
            auto wfrag = [&](int t, int f) {
                i32x2 l2 = __builtin_amdgcn_ds_read_tr8_b64_v2i32((LDS_AS i32x2*)(wl + boff + f * 4096 + t * 32));
                i32x2 h2 = __builtin_amdgcn_ds_read_tr8_b64_v2i32((LDS_AS i32x2*)(wl + boff + f * 4096 + t * 32 + 1024));
                return u32x4{(unsigned)l2[0], (unsigned)l2[1], (unsigned)h2[0], (unsigned)h2[1]};
            };
#pragma unroll
            for (int f = 0; f < 4; ++f) v[f] = wfrag(0, f);
            STAMP(1);
            tnext = (unsigned)__builtin_amdgcn_s_memrealtime() * 16u;
#pragma unroll
            for (int t = 0; t < 3; ++t) send(16 + t, PACEW, [&](int f) { return wfrag(t + 1, f); });
            lds_wait(ctr, 16u);
            blk = 15; nblk = 16 - N1; step = -1;
            send(19, PACEW, [&](int f) { return *(LDS_AS u32x4*)(smem + 15 * 4096 + lo[f]); });
            STAMP(3);
        } else {
            lds_wait(ctr, 1u);
#pragma unroll
            for (int f = 0; f < 4; ++f) v[f] = *(LDS_AS u32x4*)(smem + lo[f]);
            STAMP(1);
            tnext = (unsigned)__builtin_amdgcn_s_memrealtime() * 16u;
            send(0, PACE, [&](int f) { return v[f]; });
            lds_wait(ctr, 16u);
#pragma unroll
            for (int f = 0; f < 4; ++f) v[f] = *(LDS_AS u32x4*)(smem + 4096 + lo[f]);
            blk = 1; nblk = N1 + N3 - 1; step = 1;
        }
        const unsigned dt = d >= 4 ? PACEX : PACE;
        for (int n = 0; n < nblk; ++n) {
            const int nb = (blk + step) & 15;
            send(blk, (PACE || PACEX) ? dt : 0, [&](int f) { return *(LDS_AS u32x4*)(smem + nb * 4096 + lo[f]); });
            blk = nb;
        }
        VMCNT(0);
        set_flag(prev_flag, call, lane);
        if (lane == 0) __hip_atomic_store(cnt, call, __ATOMIC_RELAXED, __HIP_MEMORY_SCOPE_AGENT);
        STAMP(2);
        return;
    }

    const u64 tstart = __builtin_amdgcn_s_memrealtime();
    if (DELAYALL) __builtin_amdgcn_s_sleep(DELAYALL);
    const float scale = a.sx[0] * a.sw[0];
    GLB_AS unsigned* tmo = (GLB_AS unsigned*)(myws + TMO_OFF);
    const unsigned call_v = __hip_atomic_load(cnt, __ATOMIC_RELAXED, __HIP_MEMORY_SCOPE_AGENT);
    const float fnan = __builtin_nanf("");

    if ((b >> 3) < 6) {
        const int u = (b & 7) * 2 + (b >> 3) - 4;
        const int link = u & 3, dn = 4 + link;
        const int k0 = (u >> 2) * 12 + wave;
        unsigned ev = 0xFFFF;
        if (lane < 8 && k0 + 48 * lane < NIDX) ev = a.ent[k0 + 48 * lane];
        const unsigned call = __builtin_amdgcn_readfirstlane(call_v) + 1u;
        const unsigned par = call & 1u;
        if (lane == 0) __hip_atomic_store(cnt, call, __ATOMIC_RELAXED, __HIP_MEMORY_SCOPE_AGENT);
        if (NOCOMP) return;
        LDS_AS unsigned char* tp = smem + wave * TILE_STRIDE;
        bool failed = false;
        for (int n = 0; n < 6; ++n) {
            const unsigned e = __builtin_amdgcn_readlane(ev, n);
            if (e == 0xFFFF) break;
            const int idx = link * NIDX + k0 + 48 * n;
            GLB_AS unsigned* p = (GLB_AS unsigned*)(myws + FO_OFF + (size_t)idx * 128);
            if (!failed) failed = !wait_flags(p, 0, tmo, call, lane);
            CBAR();
            TRACE(n, 0);
            const int code = (e >> 4) & 7;
            const int srank = rank ^ (code < 4 ? dn ^ code : code & 3);
            float* op = a.out + (size_t)(srank * MSH + (e & 15) * 32 + (lane >> 3)) * NSH + (link * 4 + (e >> 8)) * 32 + (lane & 7) * 4;
            if (e & 0x80) {
                if (failed || __builtin_amdgcn_readfirstlane(tmo_get(tmo)) == call) {
                    failed = true;
                    a.out[(size_t)rank * MSH * NSH + lane] = fnan;
                    op[0] = fnan;
                }
                TRACE(n, 1);
                continue;
            }
            const char* srcp = myws + OS_OFF + par * OS_PAR + (size_t)idx * OSLOT + voff;
#pragma unroll
            for (int k = 0; k < 4; ++k)
                __builtin_amdgcn_global_load_lds((const GLB_AS void*)(srcp + k * 1024), (LDS_AS void*)(tp + k * 1024), 16, 0, 17);
            if (lane < 8)
                __builtin_amdgcn_global_load_lds((const GLB_AS void*)(srcp + 4096), (LDS_AS void*)(tp + 4096), 16, 0, 17);
            const unsigned tv = tmo_get(tmo);
            VMCNT(0);
            failed = failed || __builtin_amdgcn_readfirstlane(tv) == call;
            const u32x2 h = *(LDS_AS u32x2*)(tp + (lane & 15) * 8);
            unsigned base = 0;
            unsigned y[16];
#pragma unroll
            for (int r = 0; r < 16; ++r) {
                const unsigned mlo = __builtin_amdgcn_readlane(h[0], r), mhi = __builtin_amdgcn_readlane(h[1], r);
                const unsigned pos = __builtin_amdgcn_mbcnt_hi(mhi, __builtin_amdgcn_mbcnt_lo(mlo, base));
                const unsigned val = *(LDS_AS unsigned*)(tp + 128 + (pos & 1023) * 4);
                const bool nz = ((lane < 32 ? mlo : mhi) >> (lane & 31)) & 1;
                y[r] = nz ? val : 0u;
                if (failed) y[r] = __builtin_bit_cast(unsigned, fnan);
                base += __builtin_popcount(mlo) + __builtin_popcount(mhi);
            }
#pragma unroll
            for (int k = 0; k < 4; ++k) {
                const u32x4 o = {y[4 * k], y[4 * k + 1], y[4 * k + 2], y[4 * k + 3]};
                asm volatile("global_store_dwordx4 %0, %1, off " UNPAUX "\n\ts_nop 1" :: "v"(op + (size_t)k * 8 * NSH), "v"(o) : "memory");
            }
            TRACE(n, 1);
        }
        STAMP(5);
        TRACE_OUT();
        return;
    }

    const int j = (b & 7) * 26 + (b >> 3) - 6;
    const int grp = j >> 4, tile = j & 15;
    const int foreign = !(grp & 1), slot = grp >> 1;
    const int downer = 4 + (tile >> 2);
    if (NOCOMP) {
        const unsigned call = __builtin_amdgcn_readfirstlane(call_v) + 1u;
        if (lane == 0) __hip_atomic_store(cnt, call, __ATOMIC_RELAXED, __HIP_MEMORY_SCOPE_AGENT);
        if (wave < 8 && tile == 0 && foreign) {
            const int d = slot + 1;
            const int blk = d < 4 ? N1 + N3 - 1 : N1;
            GLB_AS unsigned* p = (GLB_AS unsigned*)(myws + FX_OFF + (size_t)((d * NBLK + blk) * 32 + 4 * wave + (lane & 3)) * FSTRIDE);
            wait_flags(p, 0, tmo, call, lane);
            STAMP(3);
        }
        return;
    }

    const int su = (slot * 16 + tile) * 8 + wave;
    const bool selfcopy = foreign && wave < 8 && su < 512;
    if (wave == 8 && lane < 32) ctr[lane] = 0;
    if (DELAY0 && !foreign) __builtin_amdgcn_s_sleep(DELAY0);
    if (wave < 8 && !foreign) {
        const signed char* wsrc = a.w + (size_t)(wave * 512 + (lane >> 1)) * NSH + tile * 32 + (lane & 1) * 16;
#pragma unroll
        for (int i = 0; i < 16; ++i)
            __builtin_amdgcn_global_load_lds((const GLB_AS void*)(wsrc + (size_t)i * 32 * NSH),
                                             (LDS_AS void*)(smem + (wave * 16 + i) * 1024), 16, 0, 0);
    }
    if (selfcopy) {
        const signed char* xs = a.x + (size_t)(su >> 5) * BLK_BYTES + (su & 31) * 128 + xsrc_off;
#pragma unroll
        for (int i = 0; i < 4; ++i)
            __builtin_amdgcn_global_load_lds((const GLB_AS void*)(xs + (size_t)i * 8 * KDIM),
                                             (LDS_AS void*)(smem + (wave * 16 + i) * 1024), 16, 0, 0);
    }
    const int tsel = foreign ? 1 + (tile & 3) : 0;
    u64 t_lo = a.tab[tsel][slot][0], t_hi = a.tab[tsel][slot][1];
    u64 o_lo = a.ord[tile & 3][slot][0], o_hi = a.ord[tile & 3][slot][1];
    char* const peer = a.ws[downer];
    LGKMCNT0();
    __builtin_amdgcn_s_barrier();
    CBAR();
    const unsigned call = __builtin_amdgcn_readfirstlane(call_v) + 1u;
    const unsigned par = call & 1u;
    if (lane == 0) __hip_atomic_store(cnt, call, __ATOMIC_RELAXED, __HIP_MEMORY_SCOPE_AGENT);
    auto src_d = [&](int e) { const int code = (e >> 4) & 7; return foreign && code >= 4 ? downer ^ (code & 3) : code; };

    auto epilogue = [&](int i, int e) {
        const bool bad = lds_get(ctr + 17) != 0;
        LDS_AS unsigned char* tp = smem + i * TILE_STRIDE + 128;
        const int srank = rank ^ src_d(e);
        float* op = (foreign ? a.outp[tile >> 2] : a.out) + (size_t)(srank * MSH + (e & 15) * 32 + (lane >> 3)) * NSH + tile * 32 + (lane & 7) * 4;
        i32x4 q[4];
#pragma unroll
        for (int k = 0; k < 4; ++k) q[k] = *(LDS_AS i32x4*)(tp + k * 1024 + voff);
#pragma unroll
        for (int k = 0; k < 4; ++k) {
            f32x4 o;
#pragma unroll
            for (int r = 0; r < 4; ++r) o[r] = bad ? fnan : fmaxf((float)q[k][r] * scale, 0.0f);
            asm volatile("global_store_dwordx4 %0, %1, off sc0 sc1\n\ts_nop 1" :: "v"(op + (size_t)k * 8 * NSH), "v"(o) : "memory");
        }
    };

    if (wave >= 8) {
        const int i0 = wave - 8;
        u64 m_lo = a.tim[tile & 3][slot][0], m_hi = a.tim[tile & 3][slot][1];
        for (int i = i0; i < NTASK; i += 4) {
            const int sh = 8 * (i & 7);
            const int e = (int)(((i & 8 ? t_hi : t_lo) >> sh) & 0xFF);
            if (e == 0xFF) break;
            if (e & 0x80) continue;
            lds_wait(ctr + i, 8u);
            if (!foreign) {
                TRACE(i, 0);
                epilogue(i, e);
                TRACE(i, 1);
                continue;
            }
            const bool failed = lds_get(ctr + 17) != 0;
            LDS_AS unsigned char* tp = smem + i * TILE_STRIDE;
            i32x4 q[4];
#pragma unroll
            for (int k = 0; k < 4; ++k) q[k] = *(LDS_AS i32x4*)(tp + 128 + k * 1024 + voff);
            __builtin_amdgcn_s_waitcnt(0xC07F);
            CBAR();
            unsigned base = 0, hlo = 0, hhi = 0;
#pragma unroll
            for (int r = 0; r < 16; ++r) {
                float y = fmaxf((float)q[r >> 2][r & 3] * scale, 0.0f);
                if (failed) y = fnan;
                const unsigned bits = __builtin_bit_cast(unsigned, y);
                const u64 m = __builtin_amdgcn_ballot_w64(bits != 0);
                const unsigned pos = __builtin_amdgcn_mbcnt_hi((unsigned)(m >> 32), __builtin_amdgcn_mbcnt_lo((unsigned)m, base));
                if (bits != 0) *(LDS_AS unsigned*)(tp + 128 + pos * 4) = bits;
                base += __builtin_popcountll(m);
                hlo = lane == r ? (unsigned)m : hlo;
                hhi = lane == r ? (unsigned)(m >> 32) : hhi;
            }
            if (lane < 16) *(LDS_AS u32x2*)(tp + lane * 8) = u32x2{hlo, hhi};
            CBAR();
            const unsigned nbytes = (128 + base * 4 + 63) & ~63u;
            const int idx = (tile >> 2) * NIDX + (int)(((i & 8 ? o_hi : o_lo) >> sh) & 0xFF);
            const char* dst = peer + OS_OFF + par * OS_PAR + (size_t)idx * OSLOT + 2048;
            u32x4 pv[5];
#pragma unroll
            for (int k = 0; k < 5; ++k) pv[k] = *(LDS_AS u32x4*)(tp + k * 1024 + voff);
            const unsigned nb = (unsigned)(((i & 8 ? m_hi : m_lo) >> sh) & 0xFF) * 16u;
            while ((unsigned)(__builtin_amdgcn_s_memrealtime() - tstart) < nb) __builtin_amdgcn_s_sleep(1);
            TRACE(i, 0);
#pragma unroll
            for (int k = 0; k < 5; ++k)
                if (k * 1024 + voff < nbytes) STORE_WT(voff, dst, pv[k], k * 1024 - 2048);
            VMCNT(0);
            set_flag(peer + FO_OFF + (size_t)idx * 128, call, lane);
            TRACE(i, 1);
        }
        STAMP(5);
        TRACE_OUT();
        return;
    }

    bool failed = false;
    char* const fx = myws + FX_OFF + (size_t)(4 * wave + (lane & 3)) * FSTRIDE;
    auto flag_ptr = [&](int eb) { return (GLB_AS unsigned*)(fx + (size_t)eb * 32 * FSTRIDE); };
    auto ar = __builtin_amdgcn_make_buffer_rsrc((void*)(myws + par * XS_PAR), 0, (unsigned)XS_PAR, 0x00020000);
    VMCNT(0);
    i32x4 bf[16];
    if (selfcopy) {
        u32x4 v[4];
#pragma unroll
        for (int f = 0; f < 4; ++f) v[f] = *(LDS_AS u32x4*)(smem + wave * 16384 + lo[f]);
        const char* dst = myws + par * XS_PAR + (size_t)su * 4096;
#pragma unroll
        for (int f = 0; f < 4; ++f) STORE_WT(voff, dst, v[f], f * 1024);
        VMCNT(0);
        set_flag(myws + FX_OFF + (size_t)su * FSTRIDE, call, lane);
        STAMP(2);
    }
    if (!foreign) {
        const unsigned boff = (16 * (lane >> 5) + ((lane & 15) >> 1)) * 32 + 16 * ((lane >> 4) & 1) + 8 * (lane & 1);
        LDS_AS unsigned char* bp = smem + wave * 16384 + boff;
#pragma unroll
        for (int i = 0; i < 16; ++i) {
            i32x2 l2 = __builtin_amdgcn_ds_read_tr8_b64_v2i32((LDS_AS i32x2*)(bp + i * 1024));
            i32x2 h2 = __builtin_amdgcn_ds_read_tr8_b64_v2i32((LDS_AS i32x2*)(bp + i * 1024 + 256));
            bf[i] = i32x4{l2[0], l2[1], h2[0], h2[1]};
        }
        __builtin_amdgcn_s_waitcnt(0xC07F);
        CBAR();
    }
    if (wave * 16384 < NTASK * TILE_STRIDE) {
#pragma unroll
        for (int i = 0; i < 16; ++i) *(LDS_AS u32x4*)(smem + wave * 16384 + i * 1024 + voff) = u32x4{0, 0, 0, 0};
    }
    LGKMCNT0();
    if (lane == 0) __hip_atomic_fetch_add(ctr + 16, 1u, __ATOMIC_RELAXED, __HIP_MEMORY_SCOPE_WORKGROUP);
    if (foreign) {
        const int eb = downer * NBLK + 16 + (tile & 3);
        failed = !wait_flags(flag_ptr(eb), 0, tmo, call, lane);
        CBAR();
        STAMP(6);
        const unsigned bbase = (unsigned)eb * BLK_BYTES + wave * 16384;
#pragma unroll
        for (int k = 0; k < 16; ++k)
            bf[k] = __builtin_bit_cast(i32x4, __builtin_amdgcn_raw_buffer_load_b128(ar, voff + (k & 3) * 1024, bbase + (k >> 2) * 4096, 17));
    }
    lds_wait(ctr + 16, 8u);
    STAMP(1);
    unsigned fpre = 0;
    for (int i = 0; i < NTASK; ++i) {
        const int e = (int)(t_lo & 0xFF);
        t_lo = (t_lo >> 8) | (t_hi << 56); t_hi = (t_hi >> 8) | (0xFFull << 56);
        const int oi = (int)(o_lo & 0xFF);
        o_lo = (o_lo >> 8) | (o_hi << 56); o_hi >>= 8;
        if (e == 0xFF) break;
        const int eb = src_d(e) * NBLK + (e & 15);
        if (!failed) failed = !wait_flags(flag_ptr(eb), fpre, tmo, call, lane);
        CBAR();
        TRACE(i, 0);
        const int en = (int)(t_lo & 0xFF);
        if (lane < 4) fpre = __hip_atomic_load(flag_ptr(src_d(en) * NBLK + (en & 15)), __ATOMIC_RELAXED, __HIP_MEMORY_SCOPE_SYSTEM);
        unsigned tv = 0;
        if (lane == 0) tv = tmo_get(tmo);
        u32x4 af[16];
        const unsigned abase = (unsigned)eb * BLK_BYTES + wave * 16384;
#pragma unroll
        for (int k = 0; k < 16; ++k)
            af[k] = __builtin_amdgcn_raw_buffer_load_b128(ar, voff + (k & 3) * 1024, abase + (k >> 2) * 4096, 17);
        __builtin_amdgcn_sched_barrier(0);
        i32x16 acc = {};
#pragma unroll
        for (int k = 0; k < 16; ++k)
            acc = __builtin_amdgcn_mfma_i32_32x32x32_i8(__builtin_bit_cast(i32x4, af[k]), bf[k], acc, 0, 0, 0);
        failed = failed || __builtin_amdgcn_readfirstlane(tv) == call;
        if (failed && lane == 0) __hip_atomic_store(ctr + 17, 1u, __ATOMIC_RELAXED, __HIP_MEMORY_SCOPE_WORKGROUP);
        LDS_AS unsigned char* tp = smem + i * TILE_STRIDE + 128;
#pragma unroll
        for (int r = 0; r < 16; ++r)
            __hip_atomic_fetch_add((LDS_AS int*)(tp + (((r & 3) + 8 * (r >> 2) + 4 * (lane >> 5)) * 32 + (lane & 31)) * 4), acc[r],
                                   __ATOMIC_RELAXED, __HIP_MEMORY_SCOPE_WORKGROUP);
        LGKMCNT0();
        unsigned old = 0;
        if (lane == 0) old = __hip_atomic_fetch_add(ctr + i, 1u, __ATOMIC_RELAXED, __HIP_MEMORY_SCOPE_WORKGROUP);
        if ((e & 0x80) && __builtin_amdgcn_readfirstlane(old) == 7u) {
            CBAR();
            epilogue(i, e);
            if (foreign) {
                VMCNT(0);
                set_flag(peer + FO_OFF + (size_t)((tile >> 2) * NIDX + oi) * 128, call, lane);
            }
        }
        TRACE(i, 1);
    }
    STAMP(5);
    TRACE_OUT();
}

static void build_table(int foreign, float tstart, u64 (*tab)[2], float (*fin)[NTASK] = nullptr) {
    struct T { float t; int e; } task[128];
    const int nslot = foreign ? NSLOT : NOWN;
    const float tb = PACE * (4 * 0.01f / 16), tw = PACEW * (4 * 0.01f / 16), tx = PACEX * (4 * 0.01f / 16);
    int n = 0;
    for (int d = 0; d < 8; ++d)
        for (int blk = 0; blk < 16; ++blk) {
            const int type = blk < N1 ? 1 : blk < N1 + N3 ? 3 : 2;
            float t;
            int direct = 0;
            if (d == 0) { if (foreign && type != 1) continue; t = 3.0f; }
            else if (d < 4) {
                if (type == 2 || (foreign && type == 3)) continue;
                t = T0X + tb * (blk + 1);
                direct = foreign ? blk >= N1 - NRAW : blk == N1 + N3 - 1;
            } else {
                if (type == 1 || (foreign && (type == 3 || d == 4))) continue;
                t = T0X + 4 * tw + tx * (16 - blk);
            }
            task[n].t = t + 0.001f * d; task[n].e = direct << 7 | d << 4 | blk; ++n;
        }
    for (int i = 1; i < n; ++i)
        for (int k = i; k > 0 && task[k].t < task[k - 1].t; --k) { T s = task[k]; task[k] = task[k - 1]; task[k - 1] = s; }
    float fr[NSLOT]; int cntv[NSLOT]; unsigned char lst[NSLOT][NTASK];
    for (int s = 0; s < NSLOT; ++s) { fr[s] = tstart; cntv[s] = 0; for (int i = 0; i < NTASK; ++i) lst[s][i] = 0xFF; }
    for (int i = 0; i < n; ++i) {
        int best = -1;
        for (int s = 0; s < nslot; ++s) if (cntv[s] < NTASK - 1 && (best < 0 || fr[s] < fr[best])) best = s;
        fr[best] = (fr[best] > task[i].t ? fr[best] : task[i].t) + TCOST;
        if (fin) fin[best][cntv[best]] = fr[best];
        lst[best][cntv[best]++] = (unsigned char)task[i].e;
    }
    for (int s = 0; s < NSLOT; ++s)
        for (int h = 0; h < 2; ++h) {
            u64 v = 0;
            for (int i = 7; i >= 0; --i) v = v << 8 | lst[s][h * 8 + i];
            tab[s][h] = v;
        }
}

static void build_all(Args& a) {
    build_table(0, 6.5f, a.tab[0]);
    float fin[4][NSLOT][NTASK], wfree[4][NSLOT][4];
    int urgent[4][NSLOT], nxt[4][NSLOT][4];
    auto byte = [&](int t, int s, int i) { return (int)((a.tab[1 + t][s][i >> 3] >> (8 * (i & 7))) & 0xFF); };
    for (int t = 0; t < 4; ++t) {
        build_table(1, FSTART + PACEW * (4 * 0.01f / 16) * t, a.tab[1 + t], fin[t]);
        for (int s = 0; s < NSLOT; ++s) {
            a.tim[t][s][0] = a.tim[t][s][1] = a.ord[t][s][0] = a.ord[t][s][1] = 0;
            urgent[t][s] = 0;
            for (int i = 0; i < NTASK; ++i) if (byte(t, s, i) != 0xFF && (byte(t, s, i) & 0x80)) urgent[t][s] = 1;
            for (int w = 0; w < 4; ++w) { wfree[t][s][w] = 0; nxt[t][s][w] = w; }
        }
    }
    for (int i = 0; i < NIDX; ++i) a.ent[i] = 0xFFFF;
    float tnext = SENDT0;
    int idx = 0;
    for (;;) {
        int bt = -1, bs = 0, bw = 0; float bkey = 0, brdy = 0;
        for (int t = 0; t < 4; ++t) for (int s = 0; s < NSLOT; ++s) for (int w = 0; w < 4; ++w) {
            int& i = nxt[t][s][w];
            while (i < NTASK && byte(t, s, i) != 0xFF && (byte(t, s, i) & 0x80)) i += 4;
            if (i >= NTASK || byte(t, s, i) == 0xFF) continue;
            float rdy = fin[t][s][i] + 0.5f; if (rdy < wfree[t][s][w]) rdy = wfree[t][s][w];
            const float key = rdy <= tnext ? rdy - 100.f * (1 + urgent[t][s]) : rdy;
            if (bt < 0 || key < bkey) { bt = t; bs = s; bw = w; bkey = key; brdy = rdy; }
        }
        if (bt < 0) break;
        if (tnext < brdy - 1.0f) tnext = brdy - 1.0f;
        const int i = nxt[bt][bs][bw];
        int q = (int)(tnext / 0.16f); q = q > 255 ? 255 : q;
        a.tim[bt][bs][i >> 3] |= (u64)q << (8 * (i & 7));
        a.ord[bt][bs][i >> 3] |= (u64)idx << (8 * (i & 7));
        a.ent[idx++] = (unsigned short)(bt << 8 | byte(bt, bs, i));
        wfree[bt][bs][bw] = tnext + SGAP;
        nxt[bt][bs][bw] += 4;
        tnext += SENDDT;
    }
    for (int t = 0; t < 4; ++t) for (int s = 0; s < NSLOT; ++s) for (int i = 0; i < NTASK; ++i)
        if (byte(t, s, i) != 0xFF && (byte(t, s, i) & 0x80)) {
            a.ord[t][s][i >> 3] |= (u64)idx << (8 * (i & 7));
            a.ent[idx++] = (unsigned short)(t << 8 | byte(t, s, i));
        }
}

extern "C" void kernel_launch(int rank, int nranks, ncclComm_t comm,
                              hipStream_t stream,
                              void* const* d_in, const int* in_sizes, int n_in,
                              void* d_out, int out_size,
                              const int* in_dtypes, int out_dtype,
                              void* const* d_out_all,
                              void* const* d_ws_all, size_t ws_bytes) {
    if (nranks != NRANK || (unsigned)rank >= (unsigned)NRANK || n_in != 4 || ws_bytes < WS_NEED ||
        in_sizes[0] != MSH * KDIM || in_sizes[1] != KDIM * NSH || out_size != NRANK * MSH * NSH) {
        fprintf(stderr, "unexpected problem shape\n");
        return;
    }
    HIP_CHECK(hipFuncSetAttribute((const void*)ag_gemm_kernel, hipFuncAttributeMaxDynamicSharedMemorySize, LDS_BYTES));
    Args a;
    a.x = (const signed char*)d_in[0];
    a.w = (const signed char*)d_in[1];
    a.sx = (const float*)d_in[2];
    a.sw = (const float*)d_in[3];
    a.out = (float*)d_out;
    for (int d = 0; d < NRANK; ++d) a.ws[d] = (char*)d_ws_all[rank ^ d];
    for (int i = 0; i < 4; ++i) a.outp[i] = (float*)d_out_all[rank ^ (4 + i)];
    build_all(a);
    a.rank = rank;
    a.pad = 0;
    ag_gemm_kernel<<<dim3(256), dim3(NTHREADS), LDS_BYTES, stream>>>(a);
    HIP_CHECK(hipGetLastError());
}
```

```cpp
#include <hip/hip_runtime.h>
#include <rccl/rccl.h>
#include <cstdio>
#include <cstddef>

#define HIP_CHECK(expr) do { hipError_t _e = (expr); if (_e != hipSuccess) { \
    fprintf(stderr, "HIP error %s at %s:%d\n", hipGetErrorString(_e), __FILE__, __LINE__); \
    return; } } while (0)

typedef int i32x2 __attribute__((ext_vector_type(2)));
typedef int i32x4 __attribute__((ext_vector_type(4)));
typedef int i32x16 __attribute__((ext_vector_type(16)));
typedef unsigned u32x2 __attribute__((ext_vector_type(2)));
typedef unsigned u32x4 __attribute__((ext_vector_type(4)));
typedef float f32x4 __attribute__((ext_vector_type(4)));
typedef unsigned long long u64;
typedef unsigned __attribute__((aligned(1))) u32_u;
#define LDS_AS __attribute__((address_space(3)))
#define GLB_AS __attribute__((address_space(1)))
#define CONST_AS __attribute__((address_space(4)))

#define N1 11
#define N3 1
#define WIN 9
#define PACE 940
#define PACEW PACE
#define PACEX PACE
#define SENDT0 20.8f
#define SENDDT 0.04f
#define FSTART 8.6f
#define FSTEP 2.65f
#define MINGAP 150
#define NOCOMP 0
#define TCOST 1.3f
#define NRAW 1
#define UNPAUX OUTAUX
#define TSTAGE 26.0f
#define PACET PACE
#define LDDELAY 0
#define VARA 1
#define XOFF 0
#define PSLACK 150
#define CATCH 40
#define FLAGF 0
#define NSTAGE 192
#define LSLEEP 8
#define DYNQ 1
#define PACKV 1
#define DUMMYL 0
#define PTR 0
#define TFLAG2 1
#define LDPACE 1700
#define STARTBAR 1
#define CLAMP 100
#define SGAP 1.8f
#define T0X 4.0f
#define OUTAUX "sc0 sc1"
#define FSTRIDE 128
#define PSLEEP 1

constexpr int NRANK = 8;
constexpr int KDIM = 4096, MSH = 512, NSH = 512;
constexpr unsigned BLK_BYTES = 32 * KDIM;
constexpr int NBLK = 20;
constexpr unsigned OSLOT = 4608;
constexpr int NSLOT = 7, NOWN = 6, NTASK = 16, NIDX = 256;
constexpr size_t XS_PAR = (size_t)NRANK * NBLK * BLK_BYTES;
constexpr size_t OS_OFF = 2 * XS_PAR, OS_PAR = (size_t)4 * NIDX * OSLOT;
constexpr unsigned LSLOT = 2048;
constexpr size_t LS_OFF = OS_OFF + 2 * OS_PAR;
constexpr size_t FX_OFF = LS_OFF + 4 * NIDX * LSLOT;
constexpr size_t FO_OFF = FX_OFF + NRANK * NBLK * 32 * 128;
constexpr size_t FL_OFF = FO_OFF + 4 * NIDX * 128;
constexpr size_t CNT_OFF = FL_OFF + 4 * NIDX * 128;
constexpr size_t TMO_OFF = CNT_OFF + 16384;
constexpr size_t TK_OFF = TMO_OFF + 512;
constexpr size_t DUMMY_OFF = TMO_OFF + 1024;
constexpr size_t WS_NEED = DUMMY_OFF + 4096;
constexpr u64 DEADLINE_TICKS = 20000000ull;
constexpr int TILE_STRIDE = 4352;
constexpr int LDS_MAIN = 131072;
constexpr int LDS_CTR = LDS_MAIN;
constexpr int LDS_BYTES = LDS_CTR + 256;
constexpr int NTHREADS = 768;
static_assert(NTASK * TILE_STRIDE <= LDS_MAIN, "tiles");
#define STAMP(i)
#define TRACE(i, h)
#define TRACE_OUT()

struct Args {
    const signed char* x;
    const signed char* w;
    const float* sx;
    const float* sw;
    float* out;
    char* ws[NRANK];
    float* outp[4];
    u64 tab[5][NSLOT][2];
    u64 ord[4][NSLOT][2];
    int rank;
    int npacked, ntotal;
};

#define STORE_WT(voff, base, val, off) \
    asm volatile("global_store_dwordx4 %0, %2, %1 offset:%3 sc0 sc1\n\ts_nop 1" \
                 :: "v"(voff), "s"(base), "v"(val), "i"(off) : "memory")
#define VMCNT(n) asm volatile("s_waitcnt vmcnt(%0)" :: "i"(n) : "memory")
#define LGKMCNT0() asm volatile("s_waitcnt lgkmcnt(0)" ::: "memory")
#define CBAR() asm volatile("" ::: "memory")

__device__ __forceinline__ void set_flag(char* p, unsigned call, int lane) {
    if (lane == 0) __hip_atomic_store((GLB_AS unsigned*)p, call, __ATOMIC_RELAXED, __HIP_MEMORY_SCOPE_SYSTEM);
}
__device__ __forceinline__ unsigned lds_get(LDS_AS unsigned* p) {
    return __builtin_amdgcn_readfirstlane(__hip_atomic_load(p, __ATOMIC_RELAXED, __HIP_MEMORY_SCOPE_WORKGROUP));
}
__device__ __forceinline__ void lds_wait(LDS_AS unsigned* p, unsigned v) {
    while (lds_get(p) < v) __builtin_amdgcn_s_sleep(1);
    CBAR();
}

#define TMOEVERY 8
#define DELAYALL 70
#define DELAY0 0
__device__ __forceinline__ unsigned tmo_get(GLB_AS unsigned* tmo) {
    return __hip_atomic_load(tmo, __ATOMIC_RELAXED, __HIP_MEMORY_SCOPE_AGENT);
}
__device__ __forceinline__ bool wait_flags(GLB_AS unsigned* p, unsigned f, GLB_AS unsigned* tmo, unsigned call, int lane) {
    if (((unsigned)__builtin_amdgcn_ballot_w64(f >= call) & 15u) == 15u) return true;
    const u64 t0 = __builtin_amdgcn_s_memrealtime();
    for (unsigned it = 1;; ++it) {
        if (lane < 4) f = __hip_atomic_load(p, __ATOMIC_RELAXED, __HIP_MEMORY_SCOPE_SYSTEM);
        if (((unsigned)__builtin_amdgcn_ballot_w64(f >= call) & 15u) == 15u) return true;
        if (it % TMOEVERY == 0 &&
            (__builtin_amdgcn_readfirstlane(tmo_get(tmo)) == call || __builtin_amdgcn_s_memrealtime() - t0 > DEADLINE_TICKS)) {
            if (lane == 0) __hip_atomic_store(tmo, call, __ATOMIC_RELAXED, __HIP_MEMORY_SCOPE_AGENT);
            return false;
        }
        __builtin_amdgcn_s_sleep(PSLEEP);
    }
}

template <int V> struct IC { static constexpr int value = V; };
constexpr unsigned F_WIDE = 1u << 30, F_NAN = 1u << 31, F_SKIP = 1u << 29;
__device__ __forceinline__ unsigned wave_scan(unsigned x) {
    x += __builtin_amdgcn_update_dpp(0u, x, 0x111, 0xf, 0xf, false);
    x += __builtin_amdgcn_update_dpp(0u, x, 0x112, 0xf, 0xf, false);
    x += __builtin_amdgcn_update_dpp(0u, x, 0x114, 0xf, 0xf, false);
    x += __builtin_amdgcn_update_dpp(0u, x, 0x118, 0xf, 0xf, false);
    x += __builtin_amdgcn_update_dpp(0u, x, 0x142, 0xa, 0xf, false);
    x += __builtin_amdgcn_update_dpp(0u, x, 0x143, 0xc, 0xf, false);
    return x;
}
constexpr u64 ld_order() {
    u64 o = 0; int n = 0, a = 1, z = 15;
    for (int slot = 0; n < 15; ++slot) {
        const bool cross = z >= N1 + N3 && (a >= N1 + N3 || 4 + (15 - z) - (z < 15) <= a);
        if (cross) o |= (u64)z-- << (4 * n++); else o |= (u64)a++ << (4 * n++);
    }
    return o;
}
constexpr u64 ld_pos() {
    u64 p = 0;
    for (int n = 0; n < 15; ++n) p |= (u64)(n + 1) << (4 * ((ld_order() >> (4 * n)) & 15));
    return p;
}
#define LOAD_LDS_SYS(ldsaddr, voffs, base) \
    asm volatile("s_nop 0\n\tglobal_load_lds_dwordx4 %1, %2 sc0 sc1" :: "{m0}"(ldsaddr), "v"(voffs), "s"(base) : "memory")
#define LOAD_LDS(ldsaddr, voffs, base) \
    asm volatile("s_nop 0\n\tglobal_load_lds_dwordx4 %1, %2" :: "{m0}"(ldsaddr), "v"(voffs), "s"(base) : "memory")

__device__ __forceinline__ u64 flag64_get(const char* p) {
    return __hip_atomic_load((GLB_AS u64*)p, __ATOMIC_RELAXED, __HIP_MEMORY_SCOPE_SYSTEM);
}
__device__ __forceinline__ void flag64_set(char* p, unsigned call, unsigned info, int lane) {
    if (lane == 0) __hip_atomic_store((GLB_AS u64*)p, (u64)info << 32 | call, __ATOMIC_RELAXED, __HIP_MEMORY_SCOPE_SYSTEM);
}
__device__ __forceinline__ unsigned wait_flag64(const char* p, u64 f, GLB_AS unsigned* tmo, unsigned call, int lane) {
    if ((unsigned)__builtin_amdgcn_readfirstlane((unsigned)f) >= call) return __builtin_amdgcn_readfirstlane((unsigned)(f >> 32));
    const u64 t0 = __builtin_amdgcn_s_memrealtime();
    for (unsigned it = 1;; ++it) {
        f = flag64_get(p);
        if ((unsigned)__builtin_amdgcn_readfirstlane((unsigned)f) >= call) return __builtin_amdgcn_readfirstlane((unsigned)(f >> 32));
        if (it % TMOEVERY == 0 &&
            ((unsigned)__builtin_amdgcn_readfirstlane(tmo_get(tmo)) == call || __builtin_amdgcn_s_memrealtime() - t0 > DEADLINE_TICKS)) {
            if (lane == 0) __hip_atomic_store(tmo, call, __ATOMIC_RELAXED, __HIP_MEMORY_SCOPE_AGENT);
            return 0;
        }
        __builtin_amdgcn_s_sleep(PSLEEP);
    }
}

__global__ void __launch_bounds__(NTHREADS, 3) ag_gemm_kernel(Args a_) {
    const CONST_AS Args* ap = (const CONST_AS Args*)__builtin_amdgcn_kernarg_segment_ptr();
    const CONST_AS Args& a = *ap;
    extern __shared__ __attribute__((aligned(16))) unsigned char smem_g[];
    LDS_AS unsigned char* smem = (LDS_AS unsigned char*)smem_g;
    const int lane = threadIdx.x & 63;
    const int wave = __builtin_amdgcn_readfirstlane(threadIdx.x >> 6);
    const int b = blockIdx.x;
    LDS_AS unsigned* ctr = (LDS_AS unsigned*)(smem + LDS_CTR);
    if (wave == 0) ctr[lane] = 0;
    const int j = (b & 7) * 26 + (b >> 3) - 6;
    const int dpeer = b < 32 ? (wave < 8 ? wave : wave - 4) : 4 + ((j >> 2) & 3);
    const signed char *xin, *win;
    char *myws, *peer;
    asm volatile("s_load_dwordx2 %0, %4, 0x0\n\ts_load_dwordx2 %1, %4, 0x8\n\ts_load_dwordx2 %2, %4, 0x28\n\t"
                 "s_load_dwordx2 %3, %4, %5 offset:0x28\n\ts_waitcnt lgkmcnt(0)\n\ts_barrier"
                 : "=&s"(xin), "=&s"(win), "=&s"(myws), "=&s"(peer) : "s"(ap), "s"(dpeer * 8) : "memory");
    static_assert(offsetof(Args, x) == 0 && offsetof(Args, w) == 8 && offsetof(Args, ws) == 0x28, "kernarg offsets");
    STAMP(7);

    GLB_AS unsigned* cnt = (GLB_AS unsigned*)(myws + CNT_OFF) + b * 16 + wave;
    const CONST_AS unsigned* cnt_c = (const CONST_AS unsigned*)(myws + CNT_OFF + (b * 16 + wave) * 4);
    const unsigned voff = lane * 16;
    const unsigned xsrc_off = (lane >> 3) * KDIM + (((lane & 7) ^ (lane >> 3)) * 16);
    unsigned lo[4];
#pragma unroll
    for (int f = 0; f < 4; ++f) lo[f] = (lane & 31) * 128 + (((2 * f + (lane >> 5)) ^ (lane & 7)) * 16);

    if (__builtin_expect((b >> 3) < 4, 1)) {
        const int c = b;
        const int d = dpeer;
        const unsigned lds0 = (unsigned)(size_t)smem;
        auto now16 = [&]() { return (unsigned)__builtin_amdgcn_s_memrealtime() * 16u; };
        if (wave == 0) {
            const signed char* xs = xin + c * 128;
            unsigned tl = 0;
            for (int n = 0; n < 15; ++n) {
                const int blk = (int)((ld_order() >> (4 * n)) & 15);
                if (n >= 2) { while ((int)(now16() - tl) < 0) __builtin_amdgcn_s_sleep(2); }
                tl = now16() + LDPACE;
#pragma unroll
                for (int i = 0; i < 4; ++i) LOAD_LDS(lds0 + blk * 4096 + i * 1024, xsrc_off, xs + (size_t)(blk * 32 + i * 8) * KDIM);
                if (n == 0) continue;
                VMCNT(0);
                asm volatile("ds_write_b32 %0, %1" :: "v"(ctr), "v"(n + 1) : "memory");
            }
            LGKMCNT0();
            STAMP(2);
            __builtin_amdgcn_endpgm();
        }
        unsigned call;
        u32x4 v[4];
        const unsigned wlds = lds0 + 65536 + (d & 3) * 16384;
        const signed char* wsrc = win + (size_t)c * 128 * NSH + (d & 3) * 128;
        const unsigned wo = (lane >> 1) * NSH + (lane & 1) * 16;
        if (wave < 4) {
            const signed char* xs = xin + c * 128;
            const unsigned xo = (lane & 31) * KDIM + (lane >> 5) * 16;
            asm volatile("global_load_dwordx4 %0, %5, %6\n\tglobal_load_dwordx4 %1, %5, %6 offset:32\n\t"
                         "global_load_dwordx4 %2, %5, %6 offset:64\n\tglobal_load_dwordx4 %3, %5, %6 offset:96\n\t"
                         "s_load_dword %4, %7, 0x0\n\ts_waitcnt vmcnt(0) lgkmcnt(0)"
                         : "=&v"(v[0]), "=&v"(v[1]), "=&v"(v[2]), "=&v"(v[3]), "=&s"(call) : "v"(xo), "s"(xs), "s"(cnt_c) : "memory");
        } else {
            if (wave < 8) {
#pragma unroll
                for (int f = 0; f < 4; ++f) LOAD_LDS(wlds + f * 1024, wo, wsrc + f * 32 * NSH);
            }
            asm volatile("s_load_dword %0, %1, 0x0\n\ts_waitcnt vmcnt(0) lgkmcnt(0)" : "=&s"(call) : "s"(cnt_c) : "memory");
        }
        __builtin_amdgcn_s_waitcnt(0x0F70);
        call += 1u;
        const unsigned par = call & 1u;
        const int ntile = NOCOMP ? 0 : (NSTAGE - c + 31) >> 5;
        if (wave >= 8) {
            GLB_AS unsigned* tmo = (GLB_AS unsigned*)(myws + TMO_OFF);
            const char* lsrc = myws + LS_OFF + (size_t)((d & 3) * NIDX + c) * LSLOT;
            const char* lflg = myws + FL_OFF + (size_t)((d & 3) * NIDX + c) * 128;
            while (lds_get(ctr + d + 4) == 0) __builtin_amdgcn_s_sleep(16);
            CBAR();
            if (lane == 0) __hip_atomic_store(cnt, call, __ATOMIC_RELAXED, __HIP_MEMORY_SCOPE_AGENT);
            const u64 t0 = __builtin_amdgcn_s_memrealtime();
            unsigned pend = (1u << ntile) - 1u, fly = 0, qn = 0, finfo = 0;
            for (unsigned it = 1; pend | fly; ++it) {
                u32x2 f = {0, 0};
                asm volatile("s_mov_b64 exec, %3\n\tglobal_load_dwordx2 %0, %1, %2 sc0 sc1\n\ts_mov_b64 exec, -1\n\ts_waitcnt vmcnt(0)"
                             : "+v"(f) : "v"(lane * 4096), "s"(lflg), "s"((u64)pend) : "memory");
                if (fly) {
                    if ((fly >> (lane & 31)) & (lane < 8)) ctr[32 - 32 + d * 8 + qn + __builtin_amdgcn_mbcnt_lo(fly, 0)] = finfo | lane << 26;
                    qn += __builtin_popcount(fly);
                    LGKMCNT0();
                    if (lane == 0) __hip_atomic_store(ctr + d, qn, __ATOMIC_RELAXED, __HIP_MEMORY_SCOPE_WORKGROUP);
                }
                fly = (unsigned)__builtin_amdgcn_ballot_w64(f[0] >= call) & pend;
                finfo = f[1];
                if (!fly) {
                    if (it % TMOEVERY == 0 &&
                        ((unsigned)__builtin_amdgcn_readfirstlane(tmo_get(tmo)) == call || __builtin_amdgcn_s_memrealtime() - t0 > DEADLINE_TICKS)) {
                        if (lane == 0) __hip_atomic_store(tmo, call, __ATOMIC_RELAXED, __HIP_MEMORY_SCOPE_AGENT);
                        fly = pend; pend = 0; finfo = F_SKIP;
                        continue;
                    }
                    __builtin_amdgcn_s_sleep(LSLEEP);
                    continue;
                }
                pend &= ~fly;
                for (unsigned r = fly; r; r &= r - 1) {
                    const int n = __builtin_ctz(r);
                    TRACE(n, 0);
                    LOAD_LDS_SYS(wlds + n * LSLOT, voff, lsrc + (size_t)n * 32 * LSLOT);
                    LOAD_LDS_SYS(wlds + n * LSLOT + 1024, voff, lsrc + (size_t)n * 32 * LSLOT + 1024);
                }
            }
            LGKMCNT0();
            TRACE_OUT();
            __builtin_amdgcn_endpgm();
        }
        {
            char* prev_flag = myws + DUMMY_OFF + (c * 8 + wave) * 8;
            const char* xdst = peer + par * XS_PAR + (size_t)d * NBLK * BLK_BYTES + c * 4096;
            char* fdst = peer + FX_OFF + (size_t)(d * NBLK * 32 + c) * FSTRIDE;
            unsigned tnext = 0, tideal = 0;
            auto pace = [&](unsigned dt) {
                int late;
                while ((late = (int)(now16() - tnext)) < 0) __builtin_amdgcn_s_sleep(1);
                tnext += dt;
                if (PSLACK && late > PSLACK) tnext += late - PSLACK;
                if (CATCH) {
                    tideal += dt;
                    const int over = (int)(tnext - tideal);
                    tnext -= over < CATCH ? over : CATCH;
                }
            };
            auto send = [&](int blk, unsigned dt, auto nx, auto&& next) {
                const char* dst = xdst + (size_t)blk * BLK_BYTES;
#pragma unroll
                for (int f = 0; f < 4; ++f) {
                    pace(dt);
                    STORE_WT(voff, dst, v[f], f * 1024);
                    if (f == FLAGF) { VMCNT(FLAGF ? 2 + nx.value : 1); set_flag(prev_flag, call, lane); }
                    next(f);
                }
                prev_flag = fdst + blk * 32 * FSTRIDE;
            };
            auto xwait = [&](int blk) { lds_wait(ctr, (unsigned)(ld_pos() >> (4 * blk)) & 15u); };
            int blk, nblk, step;
            if (d >= 4) {
                const unsigned wl = wlds + (16 * (lane >> 5) + ((lane & 15) >> 1)) * 32 + 16 * ((lane >> 4) & 1) + 8 * (lane & 1);
                auto wfrag = [&](u32x4& o, int off) {
                    u32x2 l2, h2;
                    asm volatile("ds_read_b64_tr_b8 %0, %2\n\tds_read_b64_tr_b8 %1, %2 offset:256\n\ts_waitcnt lgkmcnt(0)"
                                 : "=&v"(l2), "=&v"(h2) : "v"(wl + off) : "memory");
                    o = u32x4{l2[0], l2[1], h2[0], h2[1]};
                };
                wfrag(v[0], 0);
                STAMP(1);
                tideal = tnext = now16();
                for (int t = 0; t < 3; ++t)
                    send(16 + t, PACEW, IC<2>{}, [&](int f) {
                        LOAD_LDS(wlds + (t + 1) * 4096 + f * 1024, wo, wsrc + f * 32 * NSH + (t + 1) * 32);
                        if (FLAGF) { if (f == 0) VMCNT(6); else VMCNT(7); }
                        else if (f == 3) VMCNT(6);
                        wfrag(v[(f + 1) & 3], t * 4096 + (f + 1) * 1024);
                    });
                send(19, PACEW, IC<1>{}, [&](int f) {
                    if (FLAGF) { if (f == 0) VMCNT(5); else if (f == 1) VMCNT(5); else if (f == 2) VMCNT(4); }
                    if (f < 3) wfrag(v[f + 1], 3 * 4096 + (f + 1) * 1024);
                });
                if (lane == 0) __hip_atomic_store(ctr + d + 4, 1u, __ATOMIC_RELAXED, __HIP_MEMORY_SCOPE_WORKGROUP);
                STAMP(3);
                blk = 15; nblk = 16 - N1; step = -1;
            } else {
                STAMP(1);
                tideal = tnext = now16();
                send(0, PACE, IC<0>{}, [&](int f) {});
                blk = 1; nblk = N1 + N3 - 1; step = 1;
            }
            xwait(blk);
#pragma unroll
            for (int f = 0; f < 4; ++f) v[f] = *(LDS_AS u32x4*)(smem + blk * 4096 + lo[f]);
            const unsigned dt = d >= 4 ? PACEX : PACE;
            for (int n = 0; n < nblk; ++n) {
                const int nb = n + 1 < nblk ? blk + step : blk;
                send(blk, dt, IC<0>{}, [&](int f) { if (f == 0) xwait(nb); v[f] = *(LDS_AS u32x4*)(smem + nb * 4096 + lo[f]); });
                blk = nb;
            }
            unsigned pinfo = 0;
            if (d >= 4) {
                STAMP(2);
                const char* rdst = peer + OS_OFF + par * OS_PAR + (size_t)((d & 3) * NIDX + c) * OSLOT;
                char* rflg = peer + FO_OFF + (size_t)((d & 3) * NIDX + c) * 128;
                for (int q = 0; q < ntile; ++q) {
                    if (lds_get(ctr + d) <= (unsigned)q) {
                        VMCNT(0);
                        flag64_set(prev_flag, call, pinfo, lane);
                        prev_flag = myws + DUMMY_OFF + (c * 8 + wave) * 8;
                        lds_wait(ctr + d, q + 1);
                    }
                    CBAR();
                    const unsigned info = lds_get(ctr + d * 8 + q);
                    if (info & F_SKIP) continue;
                    const unsigned n = (info >> 26) & 7, nbytes = info & 0xFFFF;
                    v[0] = *(LDS_AS u32x4*)(size_t)(wlds + n * LSLOT + voff);
                    v[1] = *(LDS_AS u32x4*)(size_t)(wlds + n * LSLOT + 1024 + voff);
                    const char* dst = rdst + (size_t)n * 32 * OSLOT;
                    tideal = tnext;
                    pace(PACET);
                    TRACE(n, 0);
                    asm volatile("v_cmpx_gt_u32 vcc, %3, %0\n\tglobal_store_dwordx4 %0, %2, %1 sc0 sc1\n\ts_mov_b64 exec, -1\n\ts_nop 0"
                                 :: "v"(voff), "s"(dst), "v"(v[0]), "s"(nbytes) : "memory", "vcc");
                    if (!TFLAG2) { VMCNT(1); flag64_set(prev_flag, call, pinfo, lane); }
                    const unsigned rest = max((int)nbytes - 1024, 16);
                    pace((rest + 64) * PACET / 1024);
                    asm volatile("v_cmpx_gt_u32 vcc, %3, %0\n\tglobal_store_dwordx4 %0, %2, %1 offset:1024 sc0 sc1\n\ts_mov_b64 exec, -1\n\ts_nop 0"
                                 :: "v"(voff), "s"(dst), "v"(v[1]), "s"(rest) : "memory", "vcc");
                    if (TFLAG2) { VMCNT(2); flag64_set(prev_flag, call, pinfo, lane); }
                    TRACE(n, 1);
                    prev_flag = rflg + (size_t)n * 32 * 128;
                    pinfo = info;
                }
            }
            VMCNT(0);
            flag64_set(prev_flag, call, pinfo, lane);
            if (lane == 0) __hip_atomic_store(cnt, call, __ATOMIC_RELAXED, __HIP_MEMORY_SCOPE_AGENT);
            STAMP(d >= 4 ? 4 : 2);
            TRACE_OUT();
        }
        __builtin_amdgcn_endpgm();
    }

    const u64 tstart = __builtin_amdgcn_s_memrealtime();
    if (DELAYALL) __builtin_amdgcn_s_sleep(DELAYALL);
    const float scale = __builtin_bit_cast(float, __builtin_amdgcn_readfirstlane(__builtin_bit_cast(int, *(const CONST_AS float*)a.sx * *(const CONST_AS float*)a.sw)));
    GLB_AS unsigned* tmo = (GLB_AS unsigned*)(myws + TMO_OFF);
    const float fnan = __builtin_nanf("");
    const unsigned call = *cnt_c + 1u;
    const unsigned par = call & 1u;
    const int rank = a.rank;

    if ((b >> 3) < 6) {
        const int u = (b & 7) * 2 + (b >> 3) - 4;
        const int link = u & 3, dn = 4 + link;
        const int k0 = (u >> 2) * 12 + wave;
        const int npacked = a.npacked, ntotal = a.ntotal;
        if (lane == 0) __hip_atomic_store(cnt, call, __ATOMIC_RELAXED, __HIP_MEMORY_SCOPE_AGENT);
        if (NOCOMP) __builtin_amdgcn_endpgm();
        LDS_AS unsigned char* tp = smem + wave * TILE_STRIDE;
        bool failed = false;
        u64 fpre = 0;
        for (int n = 0; k0 + 48 * n < ntotal; ++n) {
            const int idx = link * NIDX + k0 + 48 * n;
            const unsigned info = wait_flag64(myws + FO_OFF + (size_t)idx * 128, fpre, tmo, call, lane);
            CBAR();
            TRACE(n, 0);
            if (info == 0 || k0 + 48 * n >= npacked) {
                if (info == 0 || (unsigned)__builtin_amdgcn_readfirstlane(tmo_get(tmo)) == call) {
                    a.out[(size_t)rank * MSH * NSH + lane] = fnan;
                    break;
                }
                fpre = 0;
                TRACE(n, 1);
                continue;
            }
            const unsigned e = info >> 16;
            const int code = (e >> 4) & 7;
            const int srank = rank ^ (code < 4 ? dn ^ code : code & 3);
            float* op = a.out + (size_t)(srank * MSH + (e & 15) * 32 + (lane >> 3)) * NSH + (link * 4 + ((e >> 8) & 3)) * 32 + (lane & 7) * 4;
            const char* srcp = myws + OS_OFF + par * OS_PAR + (size_t)idx * OSLOT + voff;
            const unsigned nbytes = min(info & 0xFFFFu, 4224u);
#pragma unroll
            for (int k = 0; k < 5; ++k)
                if (k * 1024 + voff < nbytes)
                    __builtin_amdgcn_global_load_lds((const GLB_AS void*)(srcp + k * 1024), (LDS_AS void*)(tp + k * 1024), 16, 0, 17);
            unsigned tv = tmo_get(tmo);
            __builtin_amdgcn_s_waitcnt(0x0F70);
            tv = __builtin_amdgcn_readfirstlane(tv);
            asm volatile("" : "+s"(tv) :: "memory");
            fpre = flag64_get(myws + FO_OFF + (size_t)(idx + 48) * 128);
            failed = failed || tv == call;
            const float mul = failed || (info & F_NAN) ? fnan : __builtin_fabsf(scale);
            const unsigned m = *(LDS_AS unsigned short*)(tp + lane * 2);
            const unsigned c1 = __builtin_popcount(m), inc = wave_scan(c1);
            unsigned y[16];
            if (info & F_WIDE) {
                unsigned ix = inc - c1;
#pragma unroll
                for (int r = 0; r < 16; ++r) {
                    const int nb = __builtin_amdgcn_sbfe(m, r, 1);
                    const unsigned w = *(LDS_AS unsigned*)(tp + 128 + ix * 4);
                    ix -= nb;
                    y[r] = __builtin_bit_cast(unsigned, (float)(w & (unsigned)nb) * mul);
                }
            } else {
                const unsigned tot = __builtin_amdgcn_readlane(inc, 63);
                const unsigned tb = (unsigned)(size_t)tp + 128, c2 = -tb - 4 * tot;
                unsigned a1 = tb + 2 * tot + inc - c1, wh[16];
#pragma unroll
                for (int r = 0; r < 16; ++r) {
                    y[r] = *(LDS_AS unsigned short*)(size_t)(a1 * 2 + c2);
                    wh[r] = *(LDS_AS unsigned char*)(size_t)a1;
                    a1 -= __builtin_amdgcn_sbfe(m, r, 1);
                }
#pragma unroll
                for (int r = 0; r < 16; r += 4)
                    asm volatile("" : "+v"(y[r]), "+v"(y[r + 1]), "+v"(y[r + 2]), "+v"(y[r + 3]),
                                      "+v"(wh[r]), "+v"(wh[r + 1]), "+v"(wh[r + 2]), "+v"(wh[r + 3]) :: "memory");
#pragma unroll
                for (int r = 0; r < 16; ++r)
                    y[r] = __builtin_bit_cast(unsigned, (float)((wh[r] << 16 | y[r]) & (unsigned)__builtin_amdgcn_sbfe(m, r, 1)) * mul);
            }
            fpre = (u64)(unsigned)__builtin_amdgcn_readfirstlane((unsigned)(fpre >> 32)) << 32 | (unsigned)__builtin_amdgcn_readfirstlane((unsigned)fpre);
            asm volatile("" : "+s"(fpre));
#pragma unroll
            for (int k = 0; k < 4; ++k) {
                const u32x4 o = {y[4 * k], y[4 * k + 1], y[4 * k + 2], y[4 * k + 3]};
                asm volatile("global_store_dwordx4 %0, %1, off " UNPAUX "\n\ts_nop 1" :: "v"(op + (size_t)k * 8 * NSH), "v"(o) : "memory");
            }
            TRACE(n, 1);
        }
        STAMP(5);
        TRACE_OUT();
        __builtin_amdgcn_endpgm();
    }

    const int grp = j >> 4, tile = j & 15;
    const int foreign = !(grp & 1), slot = grp >> 1;
    const int downer = 4 + (tile >> 2);
    if (NOCOMP) {
        if (lane == 0) __hip_atomic_store(cnt, call, __ATOMIC_RELAXED, __HIP_MEMORY_SCOPE_AGENT);
        if (wave < 8 && tile == 0 && foreign) {
            const int d = slot + 1;
            const int blk = d < 4 ? N1 + N3 - 1 : N1;
            GLB_AS unsigned* p = (GLB_AS unsigned*)(myws + FX_OFF + (size_t)((d * NBLK + blk) * 32 + 4 * wave + (lane & 3)) * FSTRIDE);
            wait_flags(p, 0, tmo, call, lane);
            STAMP(3);
        }
        __builtin_amdgcn_endpgm();
    }

    const int su = (slot * 16 + tile) * 8 + wave;
    const bool selfcopy = foreign && wave < 8 && su < 512;
    if (DELAY0 && !foreign) __builtin_amdgcn_s_sleep(DELAY0);
    if (wave < 8 && !foreign) {
        const signed char* wsrc = win + (size_t)(wave * 512 + (lane >> 1)) * NSH + tile * 32 + (lane & 1) * 16;
#pragma unroll
        for (int i = 0; i < 16; ++i)
            __builtin_amdgcn_global_load_lds((const GLB_AS void*)(wsrc + (size_t)i * 32 * NSH),
                                             (LDS_AS void*)(smem + (wave * 16 + i) * 1024), 16, 0, 0);
    }
    if (selfcopy) {
        const signed char* xs = xin + (size_t)(su >> 5) * BLK_BYTES + (su & 31) * 128 + xsrc_off;
#pragma unroll
        for (int i = 0; i < 4; ++i)
            __builtin_amdgcn_global_load_lds((const GLB_AS void*)(xs + (size_t)i * 8 * KDIM),
                                             (LDS_AS void*)(smem + (wave * 16 + i) * 1024), 16, 0, 0);
    }
    const int tsel = foreign ? 1 + (tile & 3) : 0;
    u64 t_lo = a.tab[tsel][slot][0], t_hi = a.tab[tsel][slot][1];
    u64 o_lo = a.ord[tile & 3][slot][0], o_hi = a.ord[tile & 3][slot][1];
    if (lane == 0) __hip_atomic_store(cnt, call, __ATOMIC_RELAXED, __HIP_MEMORY_SCOPE_AGENT);
    auto src_d = [&](int e) { const int code = (e >> 4) & 7; return foreign && code >= 4 ? downer ^ (code & 3) : code; };

    auto epilogue = [&](int i, int e) {
        const float bad = lds_get(ctr + 17) != 0 ? fnan : 0.0f;
        LDS_AS unsigned char* tp = smem + i * TILE_STRIDE + 128;
        const int srank = rank ^ src_d(e);
        const float* op = (foreign ? a.outp[tile >> 2] : a.out) + (size_t)(srank * MSH + (e & 15) * 32) * NSH + tile * 32;
        unsigned vo = voff;
        asm volatile("" : "+v"(vo));
        i32x4 q[4];
#pragma unroll
        for (int k = 0; k < 4; ++k) q[k] = *(LDS_AS i32x4*)(tp + k * 1024 + vo);
        vo = (vo >> 7) * (NSH * 4) + (vo & 0x70);
#pragma unroll
        for (int k = 0; k < 4; ++k) {
            f32x4 o;
#pragma unroll
            for (int r = 0; r < 4; ++r) o[r] = fmaxf((float)q[k][r] * scale, 0.0f) + bad;
            STORE_WT(vo, op + (size_t)k * 8 * NSH, o, 0);
        }
    };

    if (wave >= 8) {
        const int i0 = wave - 8;
        const unsigned npacked = a.npacked;
        for (int i = i0; i < NTASK; i += 4) {
            const int sh = 8 * (i & 7);
            const int e = (int)(((i & 8 ? t_hi : t_lo) >> sh) & 0xFF);
            if (e == 0xFF) break;
            if (e & 0x80) continue;
            unsigned ticket = 0;
            if (foreign) {
                lds_wait(ctr + 32 + i, 8u);
                if (lane == 0) ticket = __hip_atomic_fetch_add((GLB_AS unsigned*)(myws + TK_OFF + (tile >> 2) * 128), 1u, __ATOMIC_RELAXED, __HIP_MEMORY_SCOPE_AGENT);
            }
            lds_wait(ctr + i, 8u);
            if (!foreign) {
                TRACE(i, 0);
                epilogue(i, e);
                TRACE(i, 1);
                continue;
            }
            if (PTR) TRACE(i, 0);
            const bool failed = lds_get(ctr + 17) != 0;
            LDS_AS unsigned char* tp = smem + i * TILE_STRIDE;
            i32x4 q[4];
#pragma unroll
            for (int k = 0; k < 4; ++k) q[k] = *(LDS_AS i32x4*)(tp + 128 + k * 1024 + voff);
            __builtin_amdgcn_s_waitcnt(0xC07F);
            CBAR();
            if (scale < 0.0f) {
#pragma unroll
                for (int k = 0; k < 4; ++k) q[k] = -q[k];
            }
            unsigned m = 0;
            int mx = 0;
#pragma unroll
            for (int r = 15; r >= 0; --r) {
                m = __builtin_amdgcn_alignbit(m, (unsigned)-q[r >> 2][r & 3], 31);
                mx = max(mx, q[r >> 2][r & 3]);
            }
            const bool wide = __builtin_amdgcn_ballot_w64(mx >= (1 << 24)) != 0;
            const unsigned c1 = __builtin_popcount(m), inc = wave_scan(c1);
            const unsigned es = wide ? 4 : 3;
            const unsigned tot = __builtin_amdgcn_readlane(inc, 63);
            const unsigned nbytes = (128 + tot * es + 63) & ~63u;
            *(LDS_AS unsigned short*)(tp + lane * 2) = (unsigned short)m;
            if (PTR == 2) TRACE(i, 1);
            if (wide) {
                unsigned ep = (unsigned)(size_t)tp + 128 + (inc - c1) * 4;
#pragma unroll
                for (int r = 0; r < 16; ++r)
                    asm volatile("v_cmpx_lt_i32 vcc, 0, %1\n\tds_write_b32 %0, %1\n\tv_add_u32 %0, 4, %0\n\ts_mov_b64 exec, -1"
                                 : "+v"(ep) : "v"(q[r >> 2][r & 3]) : "memory", "vcc");
            } else {
                const unsigned tb = (unsigned)(size_t)tp + 128, c2 = -tb - 4 * tot, ad = tb + 2 * tot + 1536 + DUMMYL * lane;
                unsigned a1 = tb + 2 * tot + inc - c1;
#pragma unroll
                for (int r = 0; r < 16; ++r) {
                    const unsigned nb = __builtin_amdgcn_sbfe(m, r, 1);
                    const unsigned ae = (a1 & nb) | (ad & ~nb);
                    a1 -= nb;
                    asm volatile("ds_write_b16 %0, %2\n\tds_write_b8_d16_hi %1, %2" :: "v"(ae * 2 + c2), "v"(ae), "v"(q[r >> 2][r & 3]) : "memory");
                }
            }
            const unsigned info = nbytes | (unsigned)((tile & 3) << 8 | e) << 16 | (wide ? F_WIDE : 0u) | (failed ? F_NAN : 0u);
            const unsigned oi = min((unsigned)__builtin_amdgcn_readfirstlane(ticket) - (call - 1u) * npacked, npacked - 1u);
            const int idx = (tile >> 2) * NIDX + oi;
            const bool staged = oi < NSTAGE && nbytes <= LSLOT;
            if (oi < NSTAGE && !staged) flag64_set(myws + FL_OFF + (size_t)idx * 128, call, F_SKIP, lane);
            const char* dst = (staged ? myws + LS_OFF + (size_t)idx * LSLOT : peer + OS_OFF + par * OS_PAR + (size_t)idx * OSLOT) + 2048;
            if (PTR != 2) TRACE(i, PTR);
#pragma unroll
            for (int k = 0; k < 5; ++k)
                if (k * 1024 < nbytes) {
                    const u32x4 pv = *(LDS_AS u32x4*)(tp + k * 1024 + voff);
                    if (k * 1024 + voff < nbytes) STORE_WT(voff, dst, pv, k * 1024 - 2048);
                }
            VMCNT(0);
            flag64_set((staged ? myws + FL_OFF : peer + FO_OFF) + (size_t)idx * 128, call, info, lane);
            if (!PTR) TRACE(i, 1);
        }
        STAMP(5);
        TRACE_OUT();
        __builtin_amdgcn_endpgm();
    }

    bool failed = false;
    const unsigned fxo = (lane & 3) * FSTRIDE;
    auto flag_ptr = [&](int eb) {
        char* p = myws + FX_OFF + (size_t)(eb * 32 + 4 * wave) * FSTRIDE;
        asm("" : "+s"(p));
        return (GLB_AS unsigned*)(p + fxo);
    };
    unsigned aoff = 128 + (4 * (lane >> 5) * 32 + (lane & 31)) * 4;
    asm("" : "+v"(aoff));
    auto ar = __builtin_amdgcn_make_buffer_rsrc((void*)(myws + par * XS_PAR), 0, (unsigned)XS_PAR, 0x00020000);
    __builtin_amdgcn_s_waitcnt(0x0F70);
    CBAR();
    i32x4 bf[16];
    if (selfcopy) {
        u32x4 v[4];
#pragma unroll
        for (int f = 0; f < 4; ++f) v[f] = *(LDS_AS u32x4*)(smem + wave * 16384 + lo[f]);
        const char* dst = myws + par * XS_PAR + (size_t)su * 4096;
#pragma unroll
        for (int f = 0; f < 4; ++f) STORE_WT(voff, dst, v[f], f * 1024);
        VMCNT(0);
        set_flag(myws + FX_OFF + (size_t)su * FSTRIDE, call, lane);
        STAMP(2);
    }
    if (!foreign) {
        const unsigned boff = (16 * (lane >> 5) + ((lane & 15) >> 1)) * 32 + 16 * ((lane >> 4) & 1) + 8 * (lane & 1);
        LDS_AS unsigned char* bp = smem + wave * 16384 + boff;
#pragma unroll
        for (int i = 0; i < 16; ++i) {
            i32x2 l2 = __builtin_amdgcn_ds_read_tr8_b64_v2i32((LDS_AS i32x2*)(bp + i * 1024));
            i32x2 h2 = __builtin_amdgcn_ds_read_tr8_b64_v2i32((LDS_AS i32x2*)(bp + i * 1024 + 256));
            bf[i] = i32x4{l2[0], l2[1], h2[0], h2[1]};
        }
        __builtin_amdgcn_s_waitcnt(0xC07F);
        CBAR();
    }
    if (wave * 16384 < NTASK * TILE_STRIDE) {
#pragma unroll
        for (int i = 0; i < 16; ++i) *(LDS_AS u32x4*)(smem + wave * 16384 + i * 1024 + voff) = u32x4{0, 0, 0, 0};
    }
    LGKMCNT0();
    if (lane == 0) __hip_atomic_fetch_add(ctr + 16, 1u, __ATOMIC_RELAXED, __HIP_MEMORY_SCOPE_WORKGROUP);
    auto blk_of = [&](int e) { return src_d(e) * NBLK + (e & 15); };
    u32x4 af[16];
    unsigned fpre;
    auto aload = [&](int k, unsigned abase) { af[k] = __builtin_amdgcn_raw_buffer_load_b128(ar, voff, abase + k * 1024, 17); };
    auto peek = [&](int e) {
        const unsigned o = lane & 4 ? (unsigned)(TMO_OFF - FX_OFF) : (blk_of(e) * 32 + 4 * wave) * FSTRIDE + fxo;
        fpre = __hip_atomic_load((GLB_AS unsigned*)(myws + FX_OFF + o), __ATOMIC_RELAXED, __HIP_MEMORY_SCOPE_SYSTEM);
    };
    {
        const int eb = blk_of((int)(t_lo & 0xFF));
        failed = !wait_flags(flag_ptr(eb), 0, tmo, call, lane);
        CBAR();
#pragma unroll
        for (int k = 0; k < 16; ++k) aload(k, (unsigned)eb * BLK_BYTES + wave * 16384);
    }
    if (foreign) {
        const int eb = downer * NBLK + 16 + (tile & 3);
        if (!failed) failed = !wait_flags(flag_ptr(eb), 0, tmo, call, lane);
        CBAR();
        STAMP(6);
        peek((int)(t_lo >> 8) & 0xFF);
        const unsigned bbase = (unsigned)eb * BLK_BYTES + wave * 16384;
#pragma unroll
        for (int k = 0; k < 16; ++k)
            bf[k] = __builtin_bit_cast(i32x4, __builtin_amdgcn_raw_buffer_load_b128(ar, voff, bbase + k * 1024, 17));
    } else {
        peek((int)(t_lo >> 8) & 0xFF);
    }
    if (lane == 0) __hip_atomic_fetch_add(ctr + 32, 1u, __ATOMIC_RELAXED, __HIP_MEMORY_SCOPE_WORKGROUP);
    TRACE(0, 0);
    lds_wait(ctr + 16, 8u);
    __builtin_amdgcn_s_waitcnt(0x0F70);
    STAMP(1);
    for (int i = 0; i < NTASK; ++i) {
        const int e = (int)(t_lo & 0xFF);
        t_lo = (t_lo >> 8) | (t_hi << 56); t_hi = (t_hi >> 8) | (0xFFull << 56);
        const int oi = (int)(o_lo & 0xFF);
        o_lo = (o_lo >> 8) | (o_hi << 56); o_hi >>= 8;
        const int en = (int)(t_lo & 0xFF);
        const unsigned nbase = (unsigned)blk_of(en) * BLK_BYTES + wave * 16384;
        failed = failed || (unsigned)__builtin_amdgcn_readlane(fpre, 4) == call;
        const bool ready = en != 0xFF && ((unsigned)__builtin_amdgcn_ballot_w64(fpre >= call) & 15u) == 15u;
        CBAR();
        i32x16 acc = {};
        if (ready) {
            TRACE(i + 1, 0);
            if (lane == 0) __hip_atomic_fetch_add(ctr + 33 + i, 1u, __ATOMIC_RELAXED, __HIP_MEMORY_SCOPE_WORKGROUP);
            peek((int)(t_lo >> 8) & 0xFF);
#pragma unroll
            for (int k = 0; k < 16; ++k) {
                __builtin_amdgcn_sched_barrier(0);
                acc = __builtin_amdgcn_mfma_i32_32x32x32_i8(__builtin_bit_cast(i32x4, af[k]), bf[k], acc, 0, 0, 0);
                __builtin_amdgcn_sched_barrier(0);
                aload(k, nbase);
            }
            __builtin_amdgcn_sched_barrier(0);
        } else {
#pragma unroll
            for (int k = 0; k < 16; ++k)
                acc = __builtin_amdgcn_mfma_i32_32x32x32_i8(__builtin_bit_cast(i32x4, af[k]), bf[k], acc, 0, 0, 0);
        }
        if (failed && lane == 0) __hip_atomic_store(ctr + 17, 1u, __ATOMIC_RELAXED, __HIP_MEMORY_SCOPE_WORKGROUP);
        LDS_AS unsigned char* tp = smem + i * TILE_STRIDE + aoff;
#pragma unroll
        for (int r = 0; r < 16; ++r)
            __hip_atomic_fetch_add((LDS_AS int*)(tp + ((r & 3) + 8 * (r >> 2)) * 128), acc[r],
                                   __ATOMIC_RELAXED, __HIP_MEMORY_SCOPE_WORKGROUP);
        LGKMCNT0();
        unsigned old = 0;
        if (lane == 0) old = __hip_atomic_fetch_add(ctr + i, 1u, __ATOMIC_RELAXED, __HIP_MEMORY_SCOPE_WORKGROUP);
        if ((e & 0x80) && __builtin_amdgcn_readfirstlane(old) == 7u) {
            CBAR();
            epilogue(i, e);
            if (foreign) {
                VMCNT(0);
                flag64_set(peer + FO_OFF + (size_t)((tile >> 2) * NIDX + oi) * 128, call, 1u, lane);
            }
        }
        TRACE(i, 1);
        if (!ready) {
            if (en == 0xFF) break;
            if (!failed) failed = !wait_flags(flag_ptr(blk_of(en)), 0, tmo, call, lane);
            CBAR();
            TRACE(i + 1, 0);
            if (lane == 0) __hip_atomic_fetch_add(ctr + 33 + i, 1u, __ATOMIC_RELAXED, __HIP_MEMORY_SCOPE_WORKGROUP);
            peek((int)(t_lo >> 8) & 0xFF);
            __builtin_amdgcn_sched_barrier(0);
#pragma unroll
            for (int k = 0; k < 16; ++k) aload(k, nbase);
            __builtin_amdgcn_sched_barrier(0);
        }
    }
    STAMP(5);
    TRACE_OUT();
}

static void build_table(int foreign, float tstart, u64 (*tab)[2], float (*fin)[NTASK] = nullptr) {
    struct T { float t; int e; } task[128];
    const int nslot = foreign ? NSLOT : NOWN;
    const float tb = PACE * (4 * 0.01f / 16), tw = PACEW * (4 * 0.01f / 16), tx = PACEX * (4 * 0.01f / 16);
    int n = 0;
    for (int d = 0; d < 8; ++d)
        for (int blk = 0; blk < 16; ++blk) {
            const int type = blk < N1 ? 1 : blk < N1 + N3 ? 3 : 2;
            float t;
            int direct = 0;
            if (d == 0) { if (foreign && type != 1) continue; t = 3.0f; }
            else if (d < 4) {
                if (type == 2 || (foreign && type == 3)) continue;
                t = T0X + tb * (blk + 1);
                direct = foreign ? blk >= N1 - NRAW : blk == N1 + N3 - 1;
            } else {
                if (type == 1 || (foreign && (type == 3 || d == 4))) continue;
                t = T0X + 4 * tw + tx * (16 - blk);
            }
            task[n].t = t + 0.001f * d; task[n].e = direct << 7 | d << 4 | blk; ++n;
        }
    for (int i = 1; i < n; ++i)
        for (int k = i; k > 0 && task[k].t < task[k - 1].t; --k) { T s = task[k]; task[k] = task[k - 1]; task[k - 1] = s; }
    float fr[NSLOT]; int cntv[NSLOT]; unsigned char lst[NSLOT][NTASK];
    for (int s = 0; s < NSLOT; ++s) { fr[s] = tstart; cntv[s] = 0; for (int i = 0; i < NTASK; ++i) lst[s][i] = 0xFF; }
    for (int i = 0; i < n; ++i) {
        int best = -1;
        for (int s = 0; s < nslot; ++s) if (cntv[s] < NTASK - 1 && (best < 0 || fr[s] < fr[best])) best = s;
        fr[best] = (fr[best] > task[i].t ? fr[best] : task[i].t) + TCOST;
        if (fin) fin[best][cntv[best]] = fr[best];
        lst[best][cntv[best]++] = (unsigned char)task[i].e;
    }
    for (int s = 0; s < NSLOT; ++s)
        for (int h = 0; h < 2; ++h) {
            u64 v = 0;
            for (int i = 7; i >= 0; --i) v = v << 8 | lst[s][h * 8 + i];
            tab[s][h] = v;
        }
}

static void build_all(Args& a) {
    build_table(0, 6.5f, a.tab[0]);
    auto byte = [&](int t, int s, int i) { return (int)((a.tab[1 + t][s][i >> 3] >> (8 * (i & 7))) & 0xFF); };
    int n = 0;
    for (int t = 0; t < 4; ++t) {
        build_table(1, FSTART + FSTEP * t, a.tab[1 + t]);
        for (int s = 0; s < NSLOT; ++s)
            for (int i = 0; i < NTASK; ++i) n += byte(t, s, i) < 0x80;
    }
    a.npacked = n;
    for (int t = 0; t < 4; ++t)
        for (int s = 0; s < NSLOT; ++s) a.ord[t][s][0] = a.ord[t][s][1] = 0;
    for (int blk = 0; blk < 16; ++blk)
        for (int t = 0; t < 4; ++t)
            for (int s = 0; s < NSLOT; ++s)
                for (int i = 0; i < NTASK; ++i)
                    if (byte(t, s, i) != 0xFF && (byte(t, s, i) & 0x8F) == (0x80 | blk)) a.ord[t][s][i >> 3] |= (u64)n++ << (8 * (i & 7));
    a.ntotal = n;
}

extern "C" void kernel_launch(int rank, int nranks, ncclComm_t comm,
                              hipStream_t stream,
                              void* const* d_in, const int* in_sizes, int n_in,
                              void* d_out, int out_size,
                              const int* in_dtypes, int out_dtype,
                              void* const* d_out_all,
                              void* const* d_ws_all, size_t ws_bytes) {
    if (nranks != NRANK || (unsigned)rank >= (unsigned)NRANK || n_in != 4 || ws_bytes < WS_NEED ||
        in_sizes[0] != MSH * KDIM || in_sizes[1] != KDIM * NSH || out_size != NRANK * MSH * NSH) {
        fprintf(stderr, "unexpected problem shape\n");
        return;
    }
    HIP_CHECK(hipFuncSetAttribute((const void*)ag_gemm_kernel, hipFuncAttributeMaxDynamicSharedMemorySize, LDS_BYTES));
    Args a;
    a.x = (const signed char*)d_in[0];
    a.w = (const signed char*)d_in[1];
    a.sx = (const float*)d_in[2];
    a.sw = (const float*)d_in[3];
    a.out = (float*)d_out;
    for (int d = 0; d < NRANK; ++d) a.ws[d] = (char*)d_ws_all[rank ^ d];
    for (int i = 0; i < 4; ++i) a.outp[i] = (float*)d_out_all[rank ^ (4 + i)];
    build_all(a);
    if (a.npacked < NSTAGE || a.ntotal > NIDX) return;
    a.rank = rank;
    ag_gemm_kernel<<<dim3(256), dim3(NTHREADS), LDS_BYTES, stream>>>(a);
    HIP_CHECK(hipGetLastError());
}
```

```cpp
#include <hip/hip_runtime.h>
#include <rccl/rccl.h>
#include <cstdio>
#include <cstddef>

#define HIP_CHECK(expr) do { hipError_t _e = (expr); if (_e != hipSuccess) { \
    fprintf(stderr, "HIP error %s at %s:%d\n", hipGetErrorString(_e), __FILE__, __LINE__); \
    return; } } while (0)

typedef int i32x2 __attribute__((ext_vector_type(2)));
typedef int i32x4 __attribute__((ext_vector_type(4)));
typedef int i32x16 __attribute__((ext_vector_type(16)));
typedef unsigned u32x2 __attribute__((ext_vector_type(2)));
typedef unsigned u32x4 __attribute__((ext_vector_type(4)));
typedef float f32x4 __attribute__((ext_vector_type(4)));
typedef unsigned long long u64;
typedef unsigned __attribute__((aligned(1))) u32_u;
#define LDS_AS __attribute__((address_space(3)))
#define GLB_AS __attribute__((address_space(1)))
#define CONST_AS __attribute__((address_space(4)))

#define N1 11
#define N3 1
#define WIN 9
#define PACE 900
#define PACEW PACE
#define PACEX PACE
#define SENDT0 20.8f
#define SENDDT 0.04f
#define FSTART 8.6f
#define FSTEP 2.65f
#define MINGAP 150
#define NOCOMP 0
#define TCOST 1.3f
#define NRAW 1
#define UNPAUX OUTAUX
#define TSTAGE 26.0f
#define PACET PACE
#define LDDELAY 20
#define VARA 1
#define XOFF 0
#define PSLACK 150
#define CATCH 0
#define FLAGF 0
#define NSTAGE 188
#define LSLEEP 8
#define DYNQ 1
#define TCUT 2600
#define WPF 0
#define WARM 0
#define CODE_LINES 184
#define WFIRST 1
#define PACKV 1
#define DUMMYL 0
#define PTR 0
#define TFLAG2 1
#define LDPACE 1700
#define STARTBAR 1
#define CLAMP 100
#define SGAP 1.8f
#define T0X 4.0f
#define OUTAUX "sc0 sc1"
#define FSTRIDE 128
#define PSLEEP 1

constexpr int NRANK = 8;
constexpr int KDIM = 4096, MSH = 512, NSH = 512;
constexpr unsigned BLK_BYTES = 32 * KDIM;
constexpr int NBLK = 20;
constexpr unsigned OSLOT = 4608;
constexpr int NSLOT = 7, NOWN = 6, NTASK = 16, NIDX = 256;
constexpr size_t XS_PAR = (size_t)NRANK * NBLK * BLK_BYTES;
constexpr size_t OS_OFF = 2 * XS_PAR, OS_PAR = (size_t)4 * NIDX * OSLOT;
constexpr unsigned LSLOT = 2048;
constexpr size_t LS_OFF = OS_OFF + 2 * OS_PAR;
constexpr size_t FX_OFF = LS_OFF + 4 * NIDX * LSLOT;
constexpr size_t FO_OFF = FX_OFF + NRANK * NBLK * 32 * 128;
constexpr size_t FL_OFF = FO_OFF + 4 * NIDX * 128;
constexpr size_t CNT_OFF = FL_OFF + 4 * NIDX * 128;
constexpr size_t TMO_OFF = CNT_OFF + 16384;
constexpr size_t TK_OFF = TMO_OFF + 512;
constexpr size_t DUMMY_OFF = TMO_OFF + 1024;
constexpr size_t WS_NEED = DUMMY_OFF + 4096;
constexpr u64 DEADLINE_TICKS = 20000000ull;
constexpr int TILE_STRIDE = 4352;
constexpr int LDS_MAIN = 131072;
constexpr int LDS_CTR = LDS_MAIN;
constexpr int LDS_BYTES = LDS_CTR + 256;
constexpr int NTHREADS = 768;
static_assert(NTASK * TILE_STRIDE <= LDS_MAIN, "tiles");
#define STAMP(i)
#define TRACE(i, h)
#define TRACE_OUT()

struct Args {
    const signed char* x;
    const signed char* w;
    const float* sx;
    const float* sw;
    float* out;
    char* ws[NRANK];
    float* outp[4];
    u64 tab[5][NSLOT][2];
    u64 ord[4][NSLOT][2];
    int rank;
    int npacked, ntotal;
};

#define STORE_WT(voff, base, val, off) \
    asm volatile("global_store_dwordx4 %0, %2, %1 offset:%3 sc0 sc1\n\ts_nop 1" \
                 :: "v"(voff), "s"(base), "v"(val), "i"(off) : "memory")
#define VMCNT(n) asm volatile("s_waitcnt vmcnt(%0)" :: "i"(n) : "memory")
#define LGKMCNT0() asm volatile("s_waitcnt lgkmcnt(0)" ::: "memory")
#define CBAR() asm volatile("" ::: "memory")

__device__ __forceinline__ void set_flag(char* p, unsigned call, int lane) {
    if (lane == 0) __hip_atomic_store((GLB_AS unsigned*)p, call, __ATOMIC_RELAXED, __HIP_MEMORY_SCOPE_SYSTEM);
}
__device__ __forceinline__ unsigned lds_get(LDS_AS unsigned* p) {
    return __builtin_amdgcn_readfirstlane(__hip_atomic_load(p, __ATOMIC_RELAXED, __HIP_MEMORY_SCOPE_WORKGROUP));
}
__device__ __forceinline__ void lds_wait(LDS_AS unsigned* p, unsigned v) {
    while (lds_get(p) < v) __builtin_amdgcn_s_sleep(1);
    CBAR();
}

#define TMOEVERY 8
#define DELAYALL 0
#define DELAY0 50
__device__ __forceinline__ unsigned tmo_get(GLB_AS unsigned* tmo) {
    return __hip_atomic_load(tmo, __ATOMIC_RELAXED, __HIP_MEMORY_SCOPE_AGENT);
}
__device__ __forceinline__ bool wait_flags(GLB_AS unsigned* p, unsigned f, GLB_AS unsigned* tmo, unsigned call, int lane) {
    if (((unsigned)__builtin_amdgcn_ballot_w64(f >= call) & 15u) == 15u) return true;
    const u64 t0 = __builtin_amdgcn_s_memrealtime();
    for (unsigned it = 1;; ++it) {
        if (lane < 4) f = __hip_atomic_load(p, __ATOMIC_RELAXED, __HIP_MEMORY_SCOPE_SYSTEM);
        if (((unsigned)__builtin_amdgcn_ballot_w64(f >= call) & 15u) == 15u) return true;
        if (it % TMOEVERY == 0 &&
            (__builtin_amdgcn_readfirstlane(tmo_get(tmo)) == call || __builtin_amdgcn_s_memrealtime() - t0 > DEADLINE_TICKS)) {
            if (lane == 0) __hip_atomic_store(tmo, call, __ATOMIC_RELAXED, __HIP_MEMORY_SCOPE_AGENT);
            return false;
        }
        __builtin_amdgcn_s_sleep(PSLEEP);
    }
}

template <int V> struct IC { static constexpr int value = V; };
constexpr unsigned F_WIDE = 1u << 30, F_NAN = 1u << 31, F_SKIP = 1u << 29;
__device__ __forceinline__ unsigned wave_scan(unsigned x) {
    x += __builtin_amdgcn_update_dpp(0u, x, 0x111, 0xf, 0xf, false);
    x += __builtin_amdgcn_update_dpp(0u, x, 0x112, 0xf, 0xf, false);
    x += __builtin_amdgcn_update_dpp(0u, x, 0x114, 0xf, 0xf, false);
    x += __builtin_amdgcn_update_dpp(0u, x, 0x118, 0xf, 0xf, false);
    x += __builtin_amdgcn_update_dpp(0u, x, 0x142, 0xa, 0xf, false);
    x += __builtin_amdgcn_update_dpp(0u, x, 0x143, 0xc, 0xf, false);
    return x;
}
constexpr u64 ld_order() {
    u64 o = 0; int n = 0, a = 1, z = 15;
    for (int slot = 0; n < 15; ++slot) {
        const bool cross = z >= N1 + N3 && (a >= N1 + N3 || 4 + (15 - z) - (z < 15) <= a);
        if (cross) o |= (u64)z-- << (4 * n++); else o |= (u64)a++ << (4 * n++);
    }
    return o;
}
constexpr u64 ld_pos() {
    u64 p = 0;
    for (int n = 0; n < 15; ++n) p |= (u64)(n + 1) << (4 * ((ld_order() >> (4 * n)) & 15));
    return p;
}
#define LOAD_LDS_SYS(ldsaddr, voffs, base) \
    asm volatile("s_nop 0\n\tglobal_load_lds_dwordx4 %1, %2 sc0 sc1" :: "{m0}"(ldsaddr), "v"(voffs), "s"(base) : "memory")
#define LOAD_LDS(ldsaddr, voffs, base) \
    asm volatile("s_nop 0\n\tglobal_load_lds_dwordx4 %1, %2" :: "{m0}"(ldsaddr), "v"(voffs), "s"(base) : "memory")

__device__ __forceinline__ u64 flag64_get(const char* p) {
    return __hip_atomic_load((GLB_AS u64*)p, __ATOMIC_RELAXED, __HIP_MEMORY_SCOPE_SYSTEM);
}
__device__ __forceinline__ void flag64_set(char* p, unsigned call, unsigned info, int lane) {
    if (lane == 0) __hip_atomic_store((GLB_AS u64*)p, (u64)info << 32 | call, __ATOMIC_RELAXED, __HIP_MEMORY_SCOPE_SYSTEM);
}
__device__ __forceinline__ unsigned wait_flag64(const char* p, u64 f, GLB_AS unsigned* tmo, unsigned call, int lane) {
    if ((unsigned)__builtin_amdgcn_readfirstlane((unsigned)f) >= call) return __builtin_amdgcn_readfirstlane((unsigned)(f >> 32));
    const u64 t0 = __builtin_amdgcn_s_memrealtime();
    for (unsigned it = 1;; ++it) {
        f = flag64_get(p);
        if ((unsigned)__builtin_amdgcn_readfirstlane((unsigned)f) >= call) return __builtin_amdgcn_readfirstlane((unsigned)(f >> 32));
        if (it % TMOEVERY == 0 &&
            ((unsigned)__builtin_amdgcn_readfirstlane(tmo_get(tmo)) == call || __builtin_amdgcn_s_memrealtime() - t0 > DEADLINE_TICKS)) {
            if (lane == 0) __hip_atomic_store(tmo, call, __ATOMIC_RELAXED, __HIP_MEMORY_SCOPE_AGENT);
            return 0;
        }
        __builtin_amdgcn_s_sleep(PSLEEP);
    }
}

__global__ void __launch_bounds__(NTHREADS, 3) ag_gemm_kernel(Args a_) {
    const CONST_AS Args* ap = (const CONST_AS Args*)__builtin_amdgcn_kernarg_segment_ptr();
    const CONST_AS Args& a = *ap;
    extern __shared__ __attribute__((aligned(16))) unsigned char smem_g[];
    LDS_AS unsigned char* smem = (LDS_AS unsigned char*)smem_g;
    const int lane = threadIdx.x & 63;
    const int wave = __builtin_amdgcn_readfirstlane(threadIdx.x >> 6);
    const int b = blockIdx.x;
    LDS_AS unsigned* ctr = (LDS_AS unsigned*)(smem + LDS_CTR);
    if (wave == 0) ctr[lane] = 0;
    const int j = (b & 7) * 26 + (b >> 3) - 6;
    const int dpeer = b < 32 ? (wave < 8 ? wave : wave - 4) : 4 + ((j >> 2) & 3);
    const signed char *xin, *win;
    char *myws, *peer;
    asm volatile("s_load_dwordx2 %0, %4, 0x0\n\ts_load_dwordx2 %1, %4, 0x8\n\ts_load_dwordx2 %2, %4, 0x28\n\t"
                 "s_load_dwordx2 %3, %4, %5 offset:0x28\n\ts_waitcnt lgkmcnt(0)\n\ts_barrier"
                 : "=&s"(xin), "=&s"(win), "=&s"(myws), "=&s"(peer) : "s"(ap), "s"(dpeer * 8) : "memory");
    static_assert(offsetof(Args, x) == 0 && offsetof(Args, w) == 8 && offsetof(Args, ws) == 0x28, "kernarg offsets");
    STAMP(7);

    GLB_AS unsigned* cnt = (GLB_AS unsigned*)(myws + CNT_OFF) + b * 16 + wave;
    const CONST_AS unsigned* cnt_c = (const CONST_AS unsigned*)(myws + CNT_OFF + (b * 16 + wave) * 4);
    const unsigned voff = lane * 16;
    const unsigned xsrc_off = (lane >> 3) * KDIM + (((lane & 7) ^ (lane >> 3)) * 16);
    unsigned lo[4];
#pragma unroll
    for (int f = 0; f < 4; ++f) lo[f] = (lane & 31) * 128 + (((2 * f + (lane >> 5)) ^ (lane & 7)) * 16);

    if (__builtin_expect((b >> 3) < 4, 1)) {
        const int c = b;
        const int d = dpeer;
        const unsigned lds0 = (unsigned)(size_t)smem;
        auto now16 = [&]() { return (unsigned)__builtin_amdgcn_s_memrealtime() * 16u; };
        if (wave == 0) {
            const signed char* xs = xin + c * 128;
            unsigned tl = 0;
            if (LDDELAY) __builtin_amdgcn_s_sleep(LDDELAY);
            for (int n = 0; n < 15; ++n) {
                const int blk = (int)((ld_order() >> (4 * n)) & 15);
                if (n >= 2) { while ((int)(now16() - tl) < 0) __builtin_amdgcn_s_sleep(2); }
                tl = now16() + LDPACE;
#pragma unroll
                for (int i = 0; i < 4; ++i) LOAD_LDS(lds0 + blk * 4096 + i * 1024, xsrc_off, xs + (size_t)(blk * 32 + i * 8) * KDIM);
                if (n == 0) continue;
                VMCNT(0);
                asm volatile("ds_write_b32 %0, %1" :: "v"(ctr), "v"(n + 1) : "memory");
            }
            LGKMCNT0();
            STAMP(2);
            __builtin_amdgcn_endpgm();
        }
        __builtin_amdgcn_s_waitcnt(0x0F70);
        unsigned call;
        u32x4 v[4];
        const unsigned wlds = lds0 + 65536 + (d & 3) * 16384;
        const signed char* wsrc = win + (size_t)c * 128 * NSH + (d & 3) * 32;
        const unsigned wo = (lane >> 1) * NSH + (lane & 1) * 16;
        if (wave < 4) {
            const signed char* xs = xin + c * 128;
            const unsigned xo = (lane & 31) * KDIM + (lane >> 5) * 16;
            asm volatile("global_load_dwordx4 %0, %5, %6\n\tglobal_load_dwordx4 %1, %5, %6 offset:32\n\t"
                         "global_load_dwordx4 %2, %5, %6 offset:64\n\tglobal_load_dwordx4 %3, %5, %6 offset:96\n\t"
                         "s_load_dword %4, %7, 0x0\n\ts_waitcnt vmcnt(0) lgkmcnt(0)"
                         : "=&v"(v[0]), "=&v"(v[1]), "=&v"(v[2]), "=&v"(v[3]), "=&s"(call) : "v"(xo), "s"(xs), "s"(cnt_c) : "memory");
        } else {
            if (wave < 8) {
#pragma unroll
                for (int f = 0; f < 4; ++f) LOAD_LDS(wlds + f * 1024, wo, wsrc + f * 32 * NSH);
            }
            asm volatile("s_load_dword %0, %1, 0x0\n\ts_waitcnt vmcnt(%2) lgkmcnt(0)" : "=&s"(call) : "s"(cnt_c), "i"(WFIRST ? 3 : 0) : "memory");
        }
        call += 1u;
        const unsigned par = call & 1u;
        const int ntile = NOCOMP ? 0 : (NSTAGE - c + 31) >> 5;
        if (wave >= 8) {
            GLB_AS unsigned* tmo = (GLB_AS unsigned*)(myws + TMO_OFF);
            const char* lsrc = myws + LS_OFF + (size_t)((d & 3) * NIDX + c) * LSLOT;
            const char* lflg = myws + FL_OFF + (size_t)((d & 3) * NIDX + c) * 128;
            while (lds_get(ctr + d + 4) == 0) __builtin_amdgcn_s_sleep(16);
            CBAR();
            if (lane == 0) __hip_atomic_store(cnt, call, __ATOMIC_RELAXED, __HIP_MEMORY_SCOPE_AGENT);
            const u64 t0 = __builtin_amdgcn_s_memrealtime();
            unsigned pend = (1u << ntile) - 1u, fly = 0, qn = 0, finfo = 0;
            for (unsigned it = 1; pend | fly; ++it) {
                u32x2 f = {0, 0};
                asm volatile("s_mov_b64 exec, %3\n\tglobal_load_dwordx2 %0, %1, %2 sc0 sc1\n\ts_mov_b64 exec, -1\n\ts_waitcnt vmcnt(0)"
                             : "+v"(f) : "v"(lane * 4096), "s"(lflg), "s"((u64)pend) : "memory");
                if (fly) {
                    if ((fly >> (lane & 31)) & (lane < 8)) ctr[32 - 32 + d * 8 + qn + __builtin_amdgcn_mbcnt_lo(fly, 0)] = finfo | lane << 26;
                    qn += __builtin_popcount(fly);
                    LGKMCNT0();
                    if (lane == 0) __hip_atomic_store(ctr + d, qn, __ATOMIC_RELAXED, __HIP_MEMORY_SCOPE_WORKGROUP);
                }
                fly = (unsigned)__builtin_amdgcn_ballot_w64(f[0] >= call) & pend;
                finfo = f[1];
                if (!fly) {
                    if (it % TMOEVERY == 0 &&
                        ((unsigned)__builtin_amdgcn_readfirstlane(tmo_get(tmo)) == call || __builtin_amdgcn_s_memrealtime() - t0 > DEADLINE_TICKS)) {
                        if (lane == 0) __hip_atomic_store(tmo, call, __ATOMIC_RELAXED, __HIP_MEMORY_SCOPE_AGENT);
                        fly = pend; pend = 0; finfo = F_SKIP;
                        continue;
                    }
                    __builtin_amdgcn_s_sleep(LSLEEP);
                    continue;
                }
                pend &= ~fly;
                for (unsigned r = fly; r; r &= r - 1) {
                    const int n = __builtin_ctz(r);
                    TRACE(n, 0);
                    LOAD_LDS_SYS(wlds + n * LSLOT, voff, lsrc + (size_t)n * 32 * LSLOT);
                    LOAD_LDS_SYS(wlds + n * LSLOT + 1024, voff, lsrc + (size_t)n * 32 * LSLOT + 1024);
                }
            }
            LGKMCNT0();
            TRACE_OUT();
            __builtin_amdgcn_endpgm();
        }
        {
            char* prev_flag = myws + DUMMY_OFF + (c * 8 + wave) * 8;
            const char* xdst = peer + par * XS_PAR + (size_t)d * NBLK * BLK_BYTES + c * 4096;
            char* fdst = peer + FX_OFF + (size_t)(d * NBLK * 32 + c) * FSTRIDE;
            unsigned tnext = 0, tideal = 0;
            auto pace = [&](unsigned dt) {
                int late;
                while ((late = (int)(now16() - tnext)) < 0) __builtin_amdgcn_s_sleep(1);
                tnext += dt;
                if (PSLACK && late > PSLACK) tnext += late - PSLACK;
                if (CATCH) {
                    tideal += dt;
                    const int over = (int)(tnext - tideal);
                    tnext -= over < CATCH ? over : CATCH;
                }
            };
            auto send = [&](int blk, unsigned dt, auto nx, auto&& next) {
                const char* dst = xdst + (size_t)blk * BLK_BYTES;
#pragma unroll
                for (int f = 0; f < 4; ++f) {
                    pace(dt);
                    STORE_WT(voff, dst, v[f], f * 1024);
                    if (f == FLAGF) { VMCNT(FLAGF ? 2 + nx.value : 1); set_flag(prev_flag, call, lane); }
                    next(f);
                }
                prev_flag = fdst + blk * 32 * FSTRIDE;
            };
            auto xwait = [&](int blk) { lds_wait(ctr, (unsigned)(ld_pos() >> (4 * blk)) & 15u); };
            int blk, nblk, step;
            if (d >= 4) {
                const unsigned wl = wlds + (16 * (lane >> 5) + ((lane & 15) >> 1)) * 32 + 16 * ((lane >> 4) & 1) + 8 * (lane & 1);
                auto wfrag = [&](u32x4& o, int off) {
                    u32x2 l2, h2;
                    asm volatile("ds_read_b64_tr_b8 %0, %2\n\tds_read_b64_tr_b8 %1, %2 offset:256\n\ts_waitcnt lgkmcnt(0)"
                                 : "=&v"(l2), "=&v"(h2) : "v"(wl + off) : "memory");
                    o = u32x4{l2[0], l2[1], h2[0], h2[1]};
                };
                wfrag(v[0], 0);
                STAMP(1);
                tideal = tnext = now16();
                for (int t = 0; t < 3; ++t)
                    send(16 + t, PACEW, IC<2>{}, [&](int f) {
                        LOAD_LDS(wlds + (t + 1) * 4096 + f * 1024, wo, wsrc + f * 32 * NSH + (t + 1) * 128);
                        if (FLAGF) { if (f == 0) VMCNT(6); else VMCNT(7); }
                        else if (f == 3) VMCNT(6);
                        wfrag(v[(f + 1) & 3], t * 4096 + (f + 1) * 1024);
                    });
                send(19, PACEW, IC<1>{}, [&](int f) {
                    if (FLAGF) { if (f == 0) VMCNT(5); else if (f == 1) VMCNT(5); else if (f == 2) VMCNT(4); }
                    if (f < 3) wfrag(v[f + 1], 3 * 4096 + (f + 1) * 1024);
                });
                if (lane == 0) __hip_atomic_store(ctr + d + 4, 1u, __ATOMIC_RELAXED, __HIP_MEMORY_SCOPE_WORKGROUP);
                STAMP(3);
                blk = 15; nblk = 16 - N1; step = -1;
            } else {
                STAMP(1);
                tideal = tnext = now16();
                send(0, PACE, IC<0>{}, [&](int f) {});
                blk = 1; nblk = N1 + N3 - 1; step = 1;
            }
            xwait(blk);
#pragma unroll
            for (int f = 0; f < 4; ++f) v[f] = *(LDS_AS u32x4*)(smem + blk * 4096 + lo[f]);
            const unsigned dt = d >= 4 ? PACEX : PACE;
            for (int n = 0; n < nblk; ++n) {
                const int nb = n + 1 < nblk ? blk + step : blk;
                send(blk, dt, IC<0>{}, [&](int f) { if (f == 0) xwait(nb); v[f] = *(LDS_AS u32x4*)(smem + nb * 4096 + lo[f]); });
                blk = nb;
            }
            unsigned pinfo = 0;
            if (d >= 4) {
                STAMP(2);
                const char* rdst = peer + OS_OFF + par * OS_PAR + (size_t)((d & 3) * NIDX + c) * OSLOT;
                char* rflg = peer + FO_OFF + (size_t)((d & 3) * NIDX + c) * 128;
                for (int q = 0; q < ntile; ++q) {
                    if (lds_get(ctr + d) <= (unsigned)q) {
                        VMCNT(0);
                        flag64_set(prev_flag, call, pinfo, lane);
                        prev_flag = myws + DUMMY_OFF + (c * 8 + wave) * 8;
                        lds_wait(ctr + d, q + 1);
                    }
                    CBAR();
                    const unsigned info = lds_get(ctr + d * 8 + q);
                    if (info & F_SKIP) continue;
                    const unsigned n = (info >> 26) & 7, nbytes = info & 0xFFFF;
                    v[0] = *(LDS_AS u32x4*)(size_t)(wlds + n * LSLOT + voff);
                    v[1] = *(LDS_AS u32x4*)(size_t)(wlds + n * LSLOT + 1024 + voff);
                    const char* dst = rdst + (size_t)n * 32 * OSLOT;
                    tideal = tnext;
                    pace(PACET);
                    TRACE(n, 0);
                    asm volatile("v_cmpx_gt_u32 vcc, %3, %0\n\tglobal_store_dwordx4 %0, %2, %1 sc0 sc1\n\ts_mov_b64 exec, -1\n\ts_nop 0"
                                 :: "v"(voff), "s"(dst), "v"(v[0]), "s"(nbytes) : "memory", "vcc");
                    if (!TFLAG2) { VMCNT(1); flag64_set(prev_flag, call, pinfo, lane); }
                    const unsigned rest = max((int)nbytes - 1024, 16);
                    pace((rest + 64) * PACET / 1024);
                    asm volatile("v_cmpx_gt_u32 vcc, %3, %0\n\tglobal_store_dwordx4 %0, %2, %1 offset:1024 sc0 sc1\n\ts_mov_b64 exec, -1\n\ts_nop 0"
                                 :: "v"(voff), "s"(dst), "v"(v[1]), "s"(rest) : "memory", "vcc");
                    if (TFLAG2) { VMCNT(2); flag64_set(prev_flag, call, pinfo, lane); }
                    TRACE(n, 1);
                    prev_flag = rflg + (size_t)n * 32 * 128;
                    pinfo = info;
                }
            }
            VMCNT(0);
            flag64_set(prev_flag, call, pinfo, lane);
            if (lane == 0) __hip_atomic_store(cnt, call, __ATOMIC_RELAXED, __HIP_MEMORY_SCOPE_AGENT);
            STAMP(d >= 4 ? 4 : 2);
            TRACE_OUT();
        }
        __builtin_amdgcn_endpgm();
    }

    const u64 tstart = __builtin_amdgcn_s_memrealtime();
    if (WPF && wave >= 8) {
        if (WPF > 1) __builtin_amdgcn_s_sleep(WPF);
        const unsigned l0 = min((((b >> 3) - 4) * 4 + wave - 8) * 192 + lane, 16384 - 129) * 128;
        unsigned j0, j1, j2;
        asm volatile("global_load_dword %0, %3, %4\n\tglobal_load_dword %1, %3, %5\n\tglobal_load_dword %2, %3, %6\n\ts_waitcnt vmcnt(0)"
                     : "=&v"(j0), "=&v"(j1), "=&v"(j2) : "v"(l0), "s"(win), "s"(win + 8192), "s"(win + 16384) : "memory");
    }
    if (DELAYALL) __builtin_amdgcn_s_sleep(DELAYALL);
    const float scale = __builtin_bit_cast(float, __builtin_amdgcn_readfirstlane(__builtin_bit_cast(int, *(const CONST_AS float*)a.sx * *(const CONST_AS float*)a.sw)));
    GLB_AS unsigned* tmo = (GLB_AS unsigned*)(myws + TMO_OFF);
    const float fnan = __builtin_nanf("");
    const unsigned call = *cnt_c + 1u;
    const unsigned par = call & 1u;
    const int rank = a.rank;

    if ((b >> 3) < 6) {
        const int u = (b & 7) * 2 + (b >> 3) - 4;
        const int link = u & 3, dn = 4 + link;
        const int k0 = (u >> 2) * 12 + wave;
        const int npacked = a.npacked, ntotal = a.ntotal;
        if (lane == 0) __hip_atomic_store(cnt, call, __ATOMIC_RELAXED, __HIP_MEMORY_SCOPE_AGENT);
        if (NOCOMP) __builtin_amdgcn_endpgm();
        LDS_AS unsigned char* tp = smem + wave * TILE_STRIDE;
        bool failed = false;
        u64 fpre = 0;
        for (int n = 0; k0 + 48 * n < ntotal; ++n) {
            const int idx = link * NIDX + k0 + 48 * n;
            const unsigned info = wait_flag64(myws + FO_OFF + (size_t)idx * 128, fpre, tmo, call, lane);
            CBAR();
            TRACE(n, 0);
            if (info == 0 || k0 + 48 * n >= npacked) {
                if (info == 0 || (unsigned)__builtin_amdgcn_readfirstlane(tmo_get(tmo)) == call) {
                    a.out[(size_t)rank * MSH * NSH + lane] = fnan;
                    break;
                }
                fpre = 0;
                TRACE(n, 1);
                continue;
            }
            const unsigned e = info >> 16;
            const int code = (e >> 4) & 7;
            const int srank = rank ^ (code < 4 ? dn ^ code : code & 3);
            float* op = a.out + (size_t)(srank * MSH + (e & 15) * 32 + (lane >> 3)) * NSH + (((e >> 8) & 3) * 4 + link) * 32 + (lane & 7) * 4;
            const char* srcp = myws + OS_OFF + par * OS_PAR + (size_t)idx * OSLOT + voff;
            const unsigned nbytes = min(info & 0xFFFFu, 4224u);
#pragma unroll
            for (int k = 0; k < 5; ++k)
                if (k * 1024 + voff < nbytes)
                    __builtin_amdgcn_global_load_lds((const GLB_AS void*)(srcp + k * 1024), (LDS_AS void*)(tp + k * 1024), 16, 0, 17);
            unsigned tv = tmo_get(tmo);
            __builtin_amdgcn_s_waitcnt(0x0F70);
            tv = __builtin_amdgcn_readfirstlane(tv);
            asm volatile("" : "+s"(tv) :: "memory");
            fpre = flag64_get(myws + FO_OFF + (size_t)(idx + 48) * 128);
            failed = failed || tv == call;
            const float mul = failed || (info & F_NAN) ? fnan : __builtin_fabsf(scale);
            const unsigned m = *(LDS_AS unsigned short*)(tp + lane * 2);
            const unsigned c1 = __builtin_popcount(m), inc = wave_scan(c1);
            unsigned y[16];
            if (info & F_WIDE) {
                unsigned ix = inc - c1;
#pragma unroll
                for (int r = 0; r < 16; ++r) {
                    const int nb = __builtin_amdgcn_sbfe(m, r, 1);
                    const unsigned w = *(LDS_AS unsigned*)(tp + 128 + ix * 4);
                    ix -= nb;
                    y[r] = __builtin_bit_cast(unsigned, (float)(w & (unsigned)nb) * mul);
                }
            } else {
                const unsigned tot = __builtin_amdgcn_readlane(inc, 63);
                const unsigned tb = (unsigned)(size_t)tp + 128, c2 = -tb - 4 * tot;
                unsigned a1 = tb + 2 * tot + inc - c1, wh[16];
#pragma unroll
                for (int r = 0; r < 16; ++r) {
                    y[r] = *(LDS_AS unsigned short*)(size_t)(a1 * 2 + c2);
                    wh[r] = *(LDS_AS unsigned char*)(size_t)a1;
                    a1 -= __builtin_amdgcn_sbfe(m, r, 1);
                }
#pragma unroll
                for (int r = 0; r < 16; r += 4)
                    asm volatile("" : "+v"(y[r]), "+v"(y[r + 1]), "+v"(y[r + 2]), "+v"(y[r + 3]),
                                      "+v"(wh[r]), "+v"(wh[r + 1]), "+v"(wh[r + 2]), "+v"(wh[r + 3]) :: "memory");
#pragma unroll
                for (int r = 0; r < 16; ++r)
                    y[r] = __builtin_bit_cast(unsigned, (float)((wh[r] << 16 | y[r]) & (unsigned)__builtin_amdgcn_sbfe(m, r, 1)) * mul);
            }
            fpre = (u64)(unsigned)__builtin_amdgcn_readfirstlane((unsigned)(fpre >> 32)) << 32 | (unsigned)__builtin_amdgcn_readfirstlane((unsigned)fpre);
            asm volatile("" : "+s"(fpre));
#pragma unroll
            for (int k = 0; k < 4; ++k) {
                const u32x4 o = {y[4 * k], y[4 * k + 1], y[4 * k + 2], y[4 * k + 3]};
                asm volatile("global_store_dwordx4 %0, %1, off " UNPAUX "\n\ts_nop 1" :: "v"(op + (size_t)k * 8 * NSH), "v"(o) : "memory");
            }
            TRACE(n, 1);
        }
        STAMP(5);
        TRACE_OUT();
        __builtin_amdgcn_endpgm();
    }

    const int grp = j >> 4, tile = j & 15;
    const int foreign = !(grp & 1), slot = grp >> 1;
    const int downer = 4 + (tile >> 2);
    if (NOCOMP) {
        if (lane == 0) __hip_atomic_store(cnt, call, __ATOMIC_RELAXED, __HIP_MEMORY_SCOPE_AGENT);
        if (wave < 8 && tile == 0 && foreign) {
            const int d = slot + 1;
            const int blk = d < 4 ? N1 + N3 - 1 : N1;
            GLB_AS unsigned* p = (GLB_AS unsigned*)(myws + FX_OFF + (size_t)((d * NBLK + blk) * 32 + 4 * wave + (lane & 3)) * FSTRIDE);
            wait_flags(p, 0, tmo, call, lane);
            STAMP(3);
        }
        __builtin_amdgcn_endpgm();
    }

    const int su = (slot * 16 + tile) * 8 + wave;
    const bool selfcopy = foreign && wave < 8 && su < 512;
    if (DELAY0 && !foreign) __builtin_amdgcn_s_sleep(DELAY0);
    if (wave < 8 && !foreign) {
        const signed char* wsrc = win + (size_t)(wave * 512 + (lane >> 1)) * NSH + tile * 32 + (lane & 1) * 16;
#pragma unroll
        for (int i = 0; i < 16; ++i)
            __builtin_amdgcn_global_load_lds((const GLB_AS void*)(wsrc + (size_t)i * 32 * NSH),
                                             (LDS_AS void*)(smem + (wave * 16 + i) * 1024), 16, 0, 0);
    }
    if (selfcopy) {
        const signed char* xs = xin + (size_t)(su >> 5) * BLK_BYTES + (su & 31) * 128 + xsrc_off;
#pragma unroll
        for (int i = 0; i < 4; ++i)
            __builtin_amdgcn_global_load_lds((const GLB_AS void*)(xs + (size_t)i * 8 * KDIM),
                                             (LDS_AS void*)(smem + (wave * 16 + i) * 1024), 16, 0, 0);
    }
    const int tsel = foreign ? 1 + (tile & 3) : 0;
    u64 t_lo = a.tab[tsel][slot][0], t_hi = a.tab[tsel][slot][1];
    u64 o_lo = a.ord[tile & 3][slot][0], o_hi = a.ord[tile & 3][slot][1];
    if (lane == 0) __hip_atomic_store(cnt, call, __ATOMIC_RELAXED, __HIP_MEMORY_SCOPE_AGENT);
    auto src_d = [&](int e) { const int code = (e >> 4) & 7; return foreign && code >= 4 ? downer ^ (code & 3) : code; };

    auto epilogue = [&](int i, int e) {
        const float bad = lds_get(ctr + 17) != 0 ? fnan : 0.0f;
        LDS_AS unsigned char* tp = smem + i * TILE_STRIDE + 128;
        const int srank = rank ^ src_d(e);
        const float* op = (foreign ? a.outp[tile >> 2] : a.out) + (size_t)(srank * MSH + (e & 15) * 32) * NSH + (foreign ? (tile & 3) * 4 + (tile >> 2) : tile) * 32;
        unsigned vo = voff;
        asm volatile("" : "+v"(vo));
        i32x4 q[4];
#pragma unroll
        for (int k = 0; k < 4; ++k) q[k] = *(LDS_AS i32x4*)(tp + k * 1024 + vo);
        vo = (vo >> 7) * (NSH * 4) + (vo & 0x70);
#pragma unroll
        for (int k = 0; k < 4; ++k) {
            f32x4 o;
#pragma unroll
            for (int r = 0; r < 4; ++r) o[r] = fmaxf((float)q[k][r] * scale, 0.0f) + bad;
            STORE_WT(vo, op + (size_t)k * 8 * NSH, o, 0);
        }
    };

    if (wave >= 8) {
        const int i0 = wave - 8;
        const unsigned npacked = a.npacked;
        for (int i = i0; i < NTASK; i += 4) {
            const int sh = 8 * (i & 7);
            const int e = (int)(((i & 8 ? t_hi : t_lo) >> sh) & 0xFF);
            if (e == 0xFF) break;
            if (e & 0x80) continue;
            unsigned ticket = 0;
            if (foreign) {
                lds_wait(ctr + 32 + i, 8u);
                if (lane == 0) ticket = __hip_atomic_fetch_add((GLB_AS unsigned*)(myws + TK_OFF + (tile >> 2) * 128), 1u, __ATOMIC_RELAXED, __HIP_MEMORY_SCOPE_AGENT);
            }
            lds_wait(ctr + i, 8u);
            if (!foreign) {
                TRACE(i, 0);
                epilogue(i, e);
                TRACE(i, 1);
                continue;
            }
            if (PTR) TRACE(i, 0);
            const bool failed = lds_get(ctr + 17) != 0;
            LDS_AS unsigned char* tp = smem + i * TILE_STRIDE;
            i32x4 q[4];
#pragma unroll
            for (int k = 0; k < 4; ++k) q[k] = *(LDS_AS i32x4*)(tp + 128 + k * 1024 + voff);
            __builtin_amdgcn_s_waitcnt(0xC07F);
            CBAR();
            if (scale < 0.0f) {
#pragma unroll
                for (int k = 0; k < 4; ++k) q[k] = -q[k];
            }
            unsigned m = 0;
            int mx = 0;
#pragma unroll
            for (int r = 15; r >= 0; --r) {
                m = __builtin_amdgcn_alignbit(m, (unsigned)-q[r >> 2][r & 3], 31);
                mx = max(mx, q[r >> 2][r & 3]);
            }
            const bool wide = __builtin_amdgcn_ballot_w64(mx >= (1 << 24)) != 0;
            const unsigned c1 = __builtin_popcount(m), inc = wave_scan(c1);
            const unsigned es = wide ? 4 : 3;
            const unsigned tot = __builtin_amdgcn_readlane(inc, 63);
            const unsigned nbytes = (128 + tot * es + 63) & ~63u;
            *(LDS_AS unsigned short*)(tp + lane * 2) = (unsigned short)m;
            if (PTR == 2) TRACE(i, 1);
            if (wide) {
                unsigned ep = (unsigned)(size_t)tp + 128 + (inc - c1) * 4;
#pragma unroll
                for (int r = 0; r < 16; ++r)
                    asm volatile("v_cmpx_lt_i32 vcc, 0, %1\n\tds_write_b32 %0, %1\n\tv_add_u32 %0, 4, %0\n\ts_mov_b64 exec, -1"
                                 : "+v"(ep) : "v"(q[r >> 2][r & 3]) : "memory", "vcc");
            } else {
                const unsigned tb = (unsigned)(size_t)tp + 128, c2 = -tb - 4 * tot, ad = tb + 2 * tot + 1536 + DUMMYL * lane;
                unsigned a1 = tb + 2 * tot + inc - c1;
#pragma unroll
                for (int r = 0; r < 16; ++r) {
                    const unsigned nb = __builtin_amdgcn_sbfe(m, r, 1);
                    const unsigned ae = (a1 & nb) | (ad & ~nb);
                    a1 -= nb;
                    asm volatile("ds_write_b16 %0, %2\n\tds_write_b8_d16_hi %1, %2" :: "v"(ae * 2 + c2), "v"(ae), "v"(q[r >> 2][r & 3]) : "memory");
                }
            }
            const unsigned info = nbytes | (unsigned)((tile & 3) << 8 | e) << 16 | (wide ? F_WIDE : 0u) | (failed ? F_NAN : 0u);
            const unsigned oi = min((unsigned)__builtin_amdgcn_readfirstlane(ticket) - (call - 1u) * npacked, npacked - 1u);
            const int idx = (tile >> 2) * NIDX + oi;
            const bool staged = oi < NSTAGE && nbytes <= LSLOT && (!TCUT || __builtin_amdgcn_s_memrealtime() - tstart < TCUT);
            if (oi < NSTAGE && !staged) flag64_set(myws + FL_OFF + (size_t)idx * 128, call, F_SKIP, lane);
            const char* dst = (staged ? myws + LS_OFF + (size_t)idx * LSLOT : peer + OS_OFF + par * OS_PAR + (size_t)idx * OSLOT) + 2048;
            if (PTR != 2) TRACE(i, PTR);
#pragma unroll
            for (int k = 0; k < 5; ++k)
                if (k * 1024 < nbytes) {
                    const u32x4 pv = *(LDS_AS u32x4*)(tp + k * 1024 + voff);
                    if (k * 1024 + voff < nbytes) STORE_WT(voff, dst, pv, k * 1024 - 2048);
                }
            VMCNT(0);
            flag64_set((staged ? myws + FL_OFF : peer + FO_OFF) + (size_t)idx * 128, call, info, lane);
            if (!PTR) TRACE(i, 1);
        }
        STAMP(5);
        TRACE_OUT();
        __builtin_amdgcn_endpgm();
    }

    bool failed = false;
    const unsigned fxo = (lane & 3) * FSTRIDE;
    auto flag_ptr = [&](int eb) {
        char* p = myws + FX_OFF + (size_t)(eb * 32 + 4 * wave) * FSTRIDE;
        asm("" : "+s"(p));
        return (GLB_AS unsigned*)(p + fxo);
    };
    unsigned aoff = 128 + (4 * (lane >> 5) * 32 + (lane & 31)) * 4;
    asm("" : "+v"(aoff));
    auto ar = __builtin_amdgcn_make_buffer_rsrc((void*)(myws + par * XS_PAR), 0, (unsigned)XS_PAR, 0x00020000);
    __builtin_amdgcn_s_waitcnt(0x0F70);
    CBAR();
    i32x4 bf[16];
    if (selfcopy) {
        u32x4 v[4];
#pragma unroll
        for (int f = 0; f < 4; ++f) v[f] = *(LDS_AS u32x4*)(smem + wave * 16384 + lo[f]);
        const char* dst = myws + par * XS_PAR + (size_t)su * 4096;
#pragma unroll
        for (int f = 0; f < 4; ++f) STORE_WT(voff, dst, v[f], f * 1024);
        VMCNT(0);
        set_flag(myws + FX_OFF + (size_t)su * FSTRIDE, call, lane);
        STAMP(2);
    }
    if (!foreign) {
        const unsigned boff = (16 * (lane >> 5) + ((lane & 15) >> 1)) * 32 + 16 * ((lane >> 4) & 1) + 8 * (lane & 1);
        LDS_AS unsigned char* bp = smem + wave * 16384 + boff;
#pragma unroll
        for (int i = 0; i < 16; ++i) {
            i32x2 l2 = __builtin_amdgcn_ds_read_tr8_b64_v2i32((LDS_AS i32x2*)(bp + i * 1024));
            i32x2 h2 = __builtin_amdgcn_ds_read_tr8_b64_v2i32((LDS_AS i32x2*)(bp + i * 1024 + 256));
            bf[i] = i32x4{l2[0], l2[1], h2[0], h2[1]};
        }
        __builtin_amdgcn_s_waitcnt(0xC07F);
        CBAR();
    }
    if (wave * 16384 < NTASK * TILE_STRIDE) {
#pragma unroll
        for (int i = 0; i < 16; ++i) *(LDS_AS u32x4*)(smem + wave * 16384 + i * 1024 + voff) = u32x4{0, 0, 0, 0};
    }
    LGKMCNT0();
    if (lane == 0) __hip_atomic_fetch_add(ctr + 16, 1u, __ATOMIC_RELAXED, __HIP_MEMORY_SCOPE_WORKGROUP);
    auto blk_of = [&](int e) { return src_d(e) * NBLK + (e & 15); };
    u32x4 af[16];
    unsigned fpre;
    auto aload = [&](int k, unsigned abase) { af[k] = __builtin_amdgcn_raw_buffer_load_b128(ar, voff, abase + k * 1024, 17); };
    auto peek = [&](int e) {
        const unsigned o = lane & 4 ? (unsigned)(TMO_OFF - FX_OFF) : (blk_of(e) * 32 + 4 * wave) * FSTRIDE + fxo;
        fpre = __hip_atomic_load((GLB_AS unsigned*)(myws + FX_OFF + o), __ATOMIC_RELAXED, __HIP_MEMORY_SCOPE_SYSTEM);
    };
    {
        const int eb = blk_of((int)(t_lo & 0xFF));
        failed = !wait_flags(flag_ptr(eb), 0, tmo, call, lane);
        CBAR();
#pragma unroll
        for (int k = 0; k < 16; ++k) aload(k, (unsigned)eb * BLK_BYTES + wave * 16384);
    }
    if (foreign) {
        const int eb = downer * NBLK + 16 + (tile & 3);
        if (!failed) failed = !wait_flags(flag_ptr(eb), 0, tmo, call, lane);
        CBAR();
        STAMP(6);
        peek((int)(t_lo >> 8) & 0xFF);
        const unsigned bbase = (unsigned)eb * BLK_BYTES + wave * 16384;
#pragma unroll
        for (int k = 0; k < 16; ++k)
            bf[k] = __builtin_bit_cast(i32x4, __builtin_amdgcn_raw_buffer_load_b128(ar, voff, bbase + k * 1024, 17));
    } else {
        peek((int)(t_lo >> 8) & 0xFF);
    }
    if (lane == 0) __hip_atomic_fetch_add(ctr + 32, 1u, __ATOMIC_RELAXED, __HIP_MEMORY_SCOPE_WORKGROUP);
    TRACE(0, 0);
    lds_wait(ctr + 16, 8u);
    __builtin_amdgcn_s_waitcnt(0x0F70);
    STAMP(1);
    for (int i = 0; i < NTASK; ++i) {
        const int e = (int)(t_lo & 0xFF);
        t_lo = (t_lo >> 8) | (t_hi << 56); t_hi = (t_hi >> 8) | (0xFFull << 56);
        const int oi = (int)(o_lo & 0xFF);
        o_lo = (o_lo >> 8) | (o_hi << 56); o_hi >>= 8;
        const int en = (int)(t_lo & 0xFF);
        const unsigned nbase = (unsigned)blk_of(en) * BLK_BYTES + wave * 16384;
        failed = failed || (unsigned)__builtin_amdgcn_readlane(fpre, 4) == call;
        const bool ready = en != 0xFF && ((unsigned)__builtin_amdgcn_ballot_w64(fpre >= call) & 15u) == 15u;
        CBAR();
        i32x16 acc = {};
        if (ready) {
            TRACE(i + 1, 0);
            if (lane == 0) __hip_atomic_fetch_add(ctr + 33 + i, 1u, __ATOMIC_RELAXED, __HIP_MEMORY_SCOPE_WORKGROUP);
            peek((int)(t_lo >> 8) & 0xFF);
#pragma unroll
            for (int k = 0; k < 16; ++k) {
                __builtin_amdgcn_sched_barrier(0);
                acc = __builtin_amdgcn_mfma_i32_32x32x32_i8(__builtin_bit_cast(i32x4, af[k]), bf[k], acc, 0, 0, 0);
                __builtin_amdgcn_sched_barrier(0);
                aload(k, nbase);
            }
            __builtin_amdgcn_sched_barrier(0);
        } else {
#pragma unroll
            for (int k = 0; k < 16; ++k)
                acc = __builtin_amdgcn_mfma_i32_32x32x32_i8(__builtin_bit_cast(i32x4, af[k]), bf[k], acc, 0, 0, 0);
        }
        if (failed && lane == 0) __hip_atomic_store(ctr + 17, 1u, __ATOMIC_RELAXED, __HIP_MEMORY_SCOPE_WORKGROUP);
        LDS_AS unsigned char* tp = smem + i * TILE_STRIDE + aoff;
#pragma unroll
        for (int r = 0; r < 16; ++r)
            __hip_atomic_fetch_add((LDS_AS int*)(tp + ((r & 3) + 8 * (r >> 2)) * 128), acc[r],
                                   __ATOMIC_RELAXED, __HIP_MEMORY_SCOPE_WORKGROUP);
        LGKMCNT0();
        unsigned old = 0;
        if (lane == 0) old = __hip_atomic_fetch_add(ctr + i, 1u, __ATOMIC_RELAXED, __HIP_MEMORY_SCOPE_WORKGROUP);
        if ((e & 0x80) && __builtin_amdgcn_readfirstlane(old) == 7u) {
            CBAR();
            epilogue(i, e);
            if (foreign) {
                VMCNT(0);
                flag64_set(peer + FO_OFF + (size_t)((tile >> 2) * NIDX + oi) * 128, call, 1u, lane);
            }
        }
        TRACE(i, 1);
        if (!ready) {
            if (en == 0xFF) break;
            if (!failed) failed = !wait_flags(flag_ptr(blk_of(en)), 0, tmo, call, lane);
            CBAR();
            TRACE(i + 1, 0);
            if (lane == 0) __hip_atomic_fetch_add(ctr + 33 + i, 1u, __ATOMIC_RELAXED, __HIP_MEMORY_SCOPE_WORKGROUP);
            peek((int)(t_lo >> 8) & 0xFF);
            __builtin_amdgcn_sched_barrier(0);
#pragma unroll
            for (int k = 0; k < 16; ++k) aload(k, nbase);
            __builtin_amdgcn_sched_barrier(0);
        }
    }
    STAMP(5);
    TRACE_OUT();
}

static void build_table(int foreign, float tstart, u64 (*tab)[2], float (*fin)[NTASK] = nullptr) {
    struct T { float t; int e; } task[128];
    const int nslot = foreign ? NSLOT : NOWN;
    const float tb = PACE * (4 * 0.01f / 16), tw = PACEW * (4 * 0.01f / 16), tx = PACEX * (4 * 0.01f / 16);
    int n = 0;
    for (int d = 0; d < 8; ++d)
        for (int blk = 0; blk < 16; ++blk) {
            const int type = blk < N1 ? 1 : blk < N1 + N3 ? 3 : 2;
            float t;
            int direct = 0;
            if (d == 0) { if (foreign && type != 1) continue; t = 3.0f; }
            else if (d < 4) {
                if (type == 2 || (foreign && type == 3)) continue;
                t = T0X + tb * (blk + 1);
                direct = foreign ? blk >= N1 - NRAW : blk == N1 + N3 - 1;
            } else {
                if (type == 1 || (foreign && (type == 3 || d == 4))) continue;
                t = T0X + 4 * tw + tx * (16 - blk);
            }
            task[n].t = t + 0.001f * d; task[n].e = direct << 7 | d << 4 | blk; ++n;
        }
    for (int i = 1; i < n; ++i)
        for (int k = i; k > 0 && task[k].t < task[k - 1].t; --k) { T s = task[k]; task[k] = task[k - 1]; task[k - 1] = s; }
    float fr[NSLOT]; int cntv[NSLOT]; unsigned char lst[NSLOT][NTASK];
    for (int s = 0; s < NSLOT; ++s) { fr[s] = tstart; cntv[s] = 0; for (int i = 0; i < NTASK; ++i) lst[s][i] = 0xFF; }
    for (int i = 0; i < n; ++i) {
        int best = -1;
        for (int s = 0; s < nslot; ++s) if (cntv[s] < NTASK - 1 && (best < 0 || fr[s] < fr[best])) best = s;
        fr[best] = (fr[best] > task[i].t ? fr[best] : task[i].t) + TCOST;
        if (fin) fin[best][cntv[best]] = fr[best];
        lst[best][cntv[best]++] = (unsigned char)task[i].e;
    }
    for (int s = 0; s < NSLOT; ++s)
        for (int h = 0; h < 2; ++h) {
            u64 v = 0;
            for (int i = 7; i >= 0; --i) v = v << 8 | lst[s][h * 8 + i];
            tab[s][h] = v;
        }
}

static void build_all(Args& a) {
    build_table(0, 6.5f, a.tab[0]);
    auto byte = [&](int t, int s, int i) { return (int)((a.tab[1 + t][s][i >> 3] >> (8 * (i & 7))) & 0xFF); };
    int n = 0;
    for (int t = 0; t < 4; ++t) {
        build_table(1, FSTART + FSTEP * t, a.tab[1 + t]);
        for (int s = 0; s < NSLOT; ++s)
            for (int i = 0; i < NTASK; ++i) n += byte(t, s, i) < 0x80;
    }
    a.npacked = n;
    for (int t = 0; t < 4; ++t)
        for (int s = 0; s < NSLOT; ++s) a.ord[t][s][0] = a.ord[t][s][1] = 0;
    for (int blk = 0; blk < 16; ++blk)
        for (int t = 0; t < 4; ++t)
            for (int s = 0; s < NSLOT; ++s)
                for (int i = 0; i < NTASK; ++i)
                    if (byte(t, s, i) != 0xFF && (byte(t, s, i) & 0x8F) == (0x80 | blk)) a.ord[t][s][i >> 3] |= (u64)n++ << (8 * (i & 7));
    a.ntotal = n;
}

extern "C" void kernel_launch(int rank, int nranks, ncclComm_t comm,
                              hipStream_t stream,
                              void* const* d_in, const int* in_sizes, int n_in,
                              void* d_out, int out_size,
                              const int* in_dtypes, int out_dtype,
                              void* const* d_out_all,
                              void* const* d_ws_all, size_t ws_bytes) {
    if (nranks != NRANK || (unsigned)rank >= (unsigned)NRANK || n_in != 4 || ws_bytes < WS_NEED ||
        in_sizes[0] != MSH * KDIM || in_sizes[1] != KDIM * NSH || out_size != NRANK * MSH * NSH) {
        fprintf(stderr, "unexpected problem shape\n");
        return;
    }
    HIP_CHECK(hipFuncSetAttribute((const void*)ag_gemm_kernel, hipFuncAttributeMaxDynamicSharedMemorySize, LDS_BYTES));
    Args a;
    a.x = (const signed char*)d_in[0];
    a.w = (const signed char*)d_in[1];
    a.sx = (const float*)d_in[2];
    a.sw = (const float*)d_in[3];
    a.out = (float*)d_out;
    for (int d = 0; d < NRANK; ++d) a.ws[d] = (char*)d_ws_all[rank ^ d];
    for (int i = 0; i < 4; ++i) a.outp[i] = (float*)d_out_all[rank ^ (4 + i)];
    build_all(a);
    if (a.npacked < NSTAGE || a.ntotal > NIDX) return;
    a.rank = rank;
    ag_gemm_kernel<<<dim3(256), dim3(NTHREADS), LDS_BYTES, stream>>>(a);
    HIP_CHECK(hipGetLastError());
}
```

```cpp
#include <hip/hip_runtime.h>
#include <rccl/rccl.h>
#include <cstdio>
#include <cstddef>

#define HIP_CHECK(expr) do { hipError_t _e = (expr); if (_e != hipSuccess) { \
    fprintf(stderr, "HIP error %s at %s:%d\n", hipGetErrorString(_e), __FILE__, __LINE__); \
    return; } } while (0)

typedef int i32x2 __attribute__((ext_vector_type(2)));
typedef int i32x4 __attribute__((ext_vector_type(4)));
typedef int i32x16 __attribute__((ext_vector_type(16)));
typedef unsigned u32x2 __attribute__((ext_vector_type(2)));
typedef unsigned u32x4 __attribute__((ext_vector_type(4)));
typedef float f32x4 __attribute__((ext_vector_type(4)));
typedef unsigned long long u64;
typedef unsigned __attribute__((aligned(1))) u32_u;
#define LDS_AS __attribute__((address_space(3)))
#define GLB_AS __attribute__((address_space(1)))
#define CONST_AS __attribute__((address_space(4)))

#define N1 11
#define N3 1
#define PACE 900
#define FSTART 8.6f
#define FSTEP 2.65f
#define NOCOMP 0
#define TCOST 1.3f
#define NRAW 1
#define PACET PACE
#define LDDELAY 20
#define PSLACK 50
#define NSTAGE 188
#define LSLEEP 8
#define DYNQ 1
#define TCUT 2600
#define PTR 0
#define LDPACE 1700
#define T0X 4.0f
#define FSTRIDE 128
#define PSLEEP 1

#define KSPLIT 144
#define LDORD 1
#define BURST1 0
#define SWZ 3
#define UPAGE(c, f, blk) ((unsigned)((c) ^ ((SWZ & 1 ? (f) : 0) ^ (SWZ & 2 ? (blk) & 3 : 0))) << 12)
#define WALL 4
#define ALIGN 4
#define AMAX 60
#define ASLEEP 60
#define STAG 1
#define TBIAS 0
#define P21 1
#define HAVE_ROLE 1
__host__ __device__ inline int role_of(int b, int& ix) {
    const int r = b >> 3;
    ix = r < 4 ? b : (b & 7) * (r < 6 ? 2 : 26) + r - (r < 6 ? 4 : 6);
    return r < 4 ? 0 : r < 6 ? 1 : 2;
}
constexpr int NRANK = 8;
constexpr int KDIM = 4096, MSH = 512, NSH = 512;
constexpr unsigned BLK_BYTES = 32 * KDIM;
constexpr int NBLK = 20;
constexpr unsigned OSLOT = 4608;
constexpr int NSLOT = 7, NOWN = 6, NTASK = 16, NIDX = 256;
constexpr size_t XS_PAR = (size_t)NRANK * NBLK * BLK_BYTES;
constexpr size_t OS_OFF = 2 * XS_PAR, OS_PAR = (size_t)4 * NIDX * OSLOT;
constexpr unsigned LSLOT = 2048;
constexpr size_t LS_OFF = OS_OFF + 2 * OS_PAR;
constexpr size_t FX_OFF = LS_OFF + 4 * NIDX * LSLOT;
constexpr size_t FO_OFF = FX_OFF + NRANK * NBLK * 32 * 128;
constexpr size_t FL_OFF = FO_OFF + 4 * NIDX * 128;
constexpr size_t CNT_OFF = FL_OFF + 4 * NIDX * 128;
constexpr size_t TMO_OFF = CNT_OFF + 16384;
constexpr size_t TK_OFF = TMO_OFF + 512;
constexpr size_t DUMMY_OFF = TMO_OFF + 1024;
constexpr size_t WS_NEED = DUMMY_OFF + 4096;
constexpr u64 DEADLINE_TICKS = 20000000ull;
constexpr int TILE_STRIDE = 4352;
constexpr int LDS_MAIN = 131072;
constexpr int LDS_CTR = LDS_MAIN;
constexpr int LDS_BYTES = LDS_CTR + 256;
constexpr int NTHREADS = 768;
static_assert(NTASK * TILE_STRIDE <= LDS_MAIN, "tiles");
#define STAMP(i)
#define TRACE(i, h)
#define TRACE_OUT()

struct Args {
    const signed char* x;
    const signed char* w;
    const float* sx;
    const float* sw;
    float* out;
    char* ws[NRANK];
    float* outp[4];
    u64 tab[5][NSLOT][2];
    u64 ord[4][NSLOT][2];
    int rank;
    int npacked, ntotal;
};

#define STORE_WT(voff, base, val, off) \
    asm volatile("global_store_dwordx4 %0, %2, %1 offset:%3 sc0 sc1\n\ts_nop 1" \
                 :: "v"(voff), "s"(base), "v"(val), "i"(off) : "memory")
#define VMCNT(n) asm volatile("s_waitcnt vmcnt(%0)" :: "i"(n) : "memory")
#define LGKMCNT0() asm volatile("s_waitcnt lgkmcnt(0)" ::: "memory")
#define CBAR() asm volatile("" ::: "memory")

__device__ __forceinline__ void set_flag(char* p, unsigned call, int lane) {
    if (lane == 0) __hip_atomic_store((GLB_AS unsigned*)p, call, __ATOMIC_RELAXED, __HIP_MEMORY_SCOPE_SYSTEM);
}
__device__ __forceinline__ unsigned lds_get(LDS_AS unsigned* p) {
    return __builtin_amdgcn_readfirstlane(__hip_atomic_load(p, __ATOMIC_RELAXED, __HIP_MEMORY_SCOPE_WORKGROUP));
}
__device__ __forceinline__ void lds_wait(LDS_AS unsigned* p, unsigned v) {
    while (lds_get(p) < v) __builtin_amdgcn_s_sleep(1);
    CBAR();
}

#define TMOEVERY 8
#define DELAY0 50
__device__ __forceinline__ unsigned tmo_get(GLB_AS unsigned* tmo) {
    return __hip_atomic_load(tmo, __ATOMIC_RELAXED, __HIP_MEMORY_SCOPE_AGENT);
}
__device__ __forceinline__ bool wait_flags(GLB_AS unsigned* p, unsigned f, GLB_AS unsigned* tmo, unsigned call, int lane) {
    if (((unsigned)__builtin_amdgcn_ballot_w64(f >= call) & 15u) == 15u) return true;
    const u64 t0 = __builtin_amdgcn_s_memrealtime();
    for (unsigned it = 1;; ++it) {
        if (lane < 4) f = __hip_atomic_load(p, __ATOMIC_RELAXED, __HIP_MEMORY_SCOPE_SYSTEM);
        if (((unsigned)__builtin_amdgcn_ballot_w64(f >= call) & 15u) == 15u) return true;
        if (it % TMOEVERY == 0 &&
            (__builtin_amdgcn_readfirstlane(tmo_get(tmo)) == call || __builtin_amdgcn_s_memrealtime() - t0 > DEADLINE_TICKS)) {
            if (lane == 0) __hip_atomic_store(tmo, call, __ATOMIC_RELAXED, __HIP_MEMORY_SCOPE_AGENT);
            return false;
        }
        __builtin_amdgcn_s_sleep(PSLEEP);
    }
}

template <int V> struct IC { static constexpr int value = V; };
constexpr unsigned F_WIDE = 1u << 30, F_NAN = 1u << 31, F_SKIP = 1u << 29;
#define BITMASK(nb, m, r) asm("v_bfe_i32 %0, %1, %2, 1" : "=v"(nb) : "v"(m), "i"(r))
#define STEP(x, nb, n) asm("v_mad_i32_i24 %0, %1, %2, %3" : "=v"(x) : "v"(nb), "i"(-(n)), "v"(x))
__device__ __forceinline__ unsigned wave_scan(unsigned x) {
    x += __builtin_amdgcn_update_dpp(0u, x, 0x111, 0xf, 0xf, false);
    x += __builtin_amdgcn_update_dpp(0u, x, 0x112, 0xf, 0xf, false);
    x += __builtin_amdgcn_update_dpp(0u, x, 0x114, 0xf, 0xf, false);
    x += __builtin_amdgcn_update_dpp(0u, x, 0x118, 0xf, 0xf, false);
    x += __builtin_amdgcn_update_dpp(0u, x, 0x142, 0xa, 0xf, false);
    x += __builtin_amdgcn_update_dpp(0u, x, 0x143, 0xc, 0xf, false);
    return x;
}
constexpr u64 ld_order() {
    u64 o = 0; int n = 0, a = 1, z = 15;
    for (int slot = 0; n < 15; ++slot) {
        const bool cross = z >= N1 && (a >= N1 || 4 + (15 - z) - (z < 15) <= a);
        if (cross) o |= (u64)z-- << (4 * n++); else o |= (u64)a++ << (4 * n++);
    }
    return o;
}
constexpr u64 ld_pos() {
    u64 p = 0;
    for (int n = 0; n < 15; ++n) p |= (u64)(n + 1) << (4 * ((ld_order() >> (4 * n)) & 15));
    return p;
}
#define LOAD_LDS_SYS(ldsaddr, voffs, base) \
    asm volatile("s_nop 0\n\tglobal_load_lds_dwordx4 %1, %2 sc0 sc1" :: "{m0}"(ldsaddr), "v"(voffs), "s"(base) : "memory")
#define LOAD_LDS(ldsaddr, voffs, base) \
    asm volatile("s_nop 0\n\tglobal_load_lds_dwordx4 %1, %2" :: "{m0}"(ldsaddr), "v"(voffs), "s"(base) : "memory")

__device__ __forceinline__ u64 flag64_get(const char* p) {
    return __hip_atomic_load((GLB_AS u64*)p, __ATOMIC_RELAXED, __HIP_MEMORY_SCOPE_SYSTEM);
}
__device__ __forceinline__ void flag64_set(char* p, unsigned call, unsigned info, int lane) {
    if (lane == 0) __hip_atomic_store((GLB_AS u64*)p, (u64)info << 32 | call, __ATOMIC_RELAXED, __HIP_MEMORY_SCOPE_SYSTEM);
}
__device__ __forceinline__ unsigned wait_flag64(const char* p, u64 f, GLB_AS unsigned* tmo, unsigned call, int lane) {
    if ((unsigned)__builtin_amdgcn_readfirstlane((unsigned)f) >= call) return __builtin_amdgcn_readfirstlane((unsigned)(f >> 32));
    const u64 t0 = __builtin_amdgcn_s_memrealtime();
    for (unsigned it = 1;; ++it) {
        f = flag64_get(p);
        if ((unsigned)__builtin_amdgcn_readfirstlane((unsigned)f) >= call) return __builtin_amdgcn_readfirstlane((unsigned)(f >> 32));
        if (it % TMOEVERY == 0 &&
            ((unsigned)__builtin_amdgcn_readfirstlane(tmo_get(tmo)) == call || __builtin_amdgcn_s_memrealtime() - t0 > DEADLINE_TICKS)) {
            if (lane == 0) __hip_atomic_store(tmo, call, __ATOMIC_RELAXED, __HIP_MEMORY_SCOPE_AGENT);
            return 0;
        }
        __builtin_amdgcn_s_sleep(PSLEEP);
    }
}

__global__ void __launch_bounds__(NTHREADS, 3) ag_gemm_kernel(Args a_) {
    const CONST_AS Args* ap = (const CONST_AS Args*)__builtin_amdgcn_kernarg_segment_ptr();
    const CONST_AS Args& a = *ap;
    extern __shared__ __attribute__((aligned(16))) unsigned char smem_g[];
    LDS_AS unsigned char* smem = (LDS_AS unsigned char*)smem_g;
    const int lane = threadIdx.x & 63;
    const int wave = __builtin_amdgcn_readfirstlane(threadIdx.x >> 6);
    const int b = blockIdx.x;
    LDS_AS unsigned* ctr = (LDS_AS unsigned*)(smem + LDS_CTR);
    if (wave == 0) ctr[lane] = 0;
    int j;
    const int role = role_of(b, j);
    const int dpeer = role == 0 ? (wave < 8 ? wave : wave - 4) : 4 + ((j >> 2) & 3);
    const signed char *xin, *win;
    char *myws, *peer;
    asm volatile("s_load_dwordx2 %0, %4, 0x0\n\ts_load_dwordx2 %1, %4, 0x8\n\ts_load_dwordx2 %2, %4, 0x28\n\t"
                 "s_load_dwordx2 %3, %4, %5 offset:0x28\n\ts_waitcnt lgkmcnt(0)\n\ts_barrier"
                 : "=&s"(xin), "=&s"(win), "=&s"(myws), "=&s"(peer) : "s"(ap), "s"(dpeer * 8) : "memory");
    static_assert(offsetof(Args, x) == 0 && offsetof(Args, w) == 8 && offsetof(Args, ws) == 0x28, "kernarg offsets");
    STAMP(7);

    GLB_AS unsigned* cnt = (GLB_AS unsigned*)(myws + CNT_OFF) + b * 16 + wave;
    const CONST_AS unsigned* cnt_c = (const CONST_AS unsigned*)(myws + CNT_OFF + (b * 16 + wave) * 4);
    const unsigned voff = lane * 16;
    const unsigned xsrc_off = (lane >> 3) * KDIM + (((lane & 7) ^ (lane >> 3)) * 16);
    unsigned lo[4];
#pragma unroll
    for (int f = 0; f < 4; ++f) lo[f] = (lane & 31) * 128 + (((2 * f + (lane >> 5)) ^ (lane & 7)) * 16);

    if (__builtin_expect(role == 0, 1)) {
        const int c = j;
        const int d = dpeer;
        const unsigned lds0 = (unsigned)(size_t)smem;
        auto now16 = [&]() { return (unsigned)__builtin_amdgcn_s_memrealtime() * 16u; };
        if (wave == 0) {
            const signed char* xs = xin + c * 128;
            unsigned tl = 0;
            if (LDDELAY) __builtin_amdgcn_s_sleep(LDDELAY);
            for (int n = 0; n < 15; ++n) {
                const int blk = (int)((ld_order() >> (4 * n)) & 15);
                if (n >= 2) { while ((int)(now16() - tl) < 0) __builtin_amdgcn_s_sleep(2); }
                tl = now16() + LDPACE;
#pragma unroll
                for (int i = 0; i < 4; ++i) LOAD_LDS(lds0 + blk * 4096 + i * 1024, xsrc_off, xs + (size_t)(blk * 32 + i * 8) * KDIM);
                if (n == 0) continue;
                VMCNT(0);
                asm volatile("ds_write_b32 %0, %1" :: "v"(ctr), "v"(n + 1) : "memory");
            }
            LGKMCNT0();
            STAMP(2);
            __builtin_amdgcn_endpgm();
        }
        __builtin_amdgcn_s_waitcnt(0x0F70);
        unsigned call;
        u32x4 v[4];
        const unsigned wlds = lds0 + 65536 + (d & 3) * 16384;
        const signed char* wsrc = win + (size_t)c * 128 * NSH + (d & 3) * 32;
        const unsigned wo = (lane >> 1) * NSH + (lane & 1) * 16;
        if (wave < 4) {
            const signed char* xs = xin + c * 128;
            const unsigned xo = (lane & 31) * KDIM + (lane >> 5) * 16;
            asm volatile("global_load_dwordx4 %0, %5, %6\n\tglobal_load_dwordx4 %1, %5, %6 offset:32\n\t"
                         "global_load_dwordx4 %2, %5, %6 offset:64\n\tglobal_load_dwordx4 %3, %5, %6 offset:96\n\t"
                         "s_load_dword %4, %7, 0x0\n\ts_waitcnt vmcnt(0) lgkmcnt(0)"
                         : "=&v"(v[0]), "=&v"(v[1]), "=&v"(v[2]), "=&v"(v[3]), "=&s"(call) : "v"(xo), "s"(xs), "s"(cnt_c) : "memory");
        } else {
            if (wave < 8) {
#pragma unroll
                for (int f = 0; f < 4; ++f) LOAD_LDS(wlds + f * 1024, wo, wsrc + f * 32 * NSH);
            }
            asm volatile("s_load_dword %0, %1, 0x0\n\ts_waitcnt vmcnt(%2) lgkmcnt(0)" : "=&s"(call) : "s"(cnt_c), "i"(3) : "memory");
        }
        call += 1u;
        const unsigned par = call & 1u;
        const int ntile = NOCOMP ? 0 : (NSTAGE - c + 31) >> 5;
        if (wave >= 8) {
            GLB_AS unsigned* tmo = (GLB_AS unsigned*)(myws + TMO_OFF);
            const char* lsrc = myws + LS_OFF + (size_t)((d & 3) * NIDX + c) * LSLOT;
            const char* lflg = myws + FL_OFF + (size_t)((d & 3) * NIDX + c) * 128;
            if (ALIGN) {
                const int dl = lane < 32 || d == 4 ? d : d - 4;
                char* st = myws + DUMMY_OFF + 2048 + dl * 256;
                unsigned t1;
                while (__builtin_amdgcn_ballot_w64((t1 = __hip_atomic_load(ctr + 24 + dl, __ATOMIC_RELAXED, __HIP_MEMORY_SCOPE_WORKGROUP)) == 0)) __builtin_amdgcn_s_sleep(4);
                if ((lane & 31) == 0) __hip_atomic_store((GLB_AS u64*)(st + c * 8), (u64)t1 << 32 | call, __ATOMIC_RELAXED, __HIP_MEMORY_SCOPE_AGENT);
                for (int it = 0; it < 8; ++it) {
                    __builtin_amdgcn_s_sleep(ASLEEP);
                    const u64 f = __hip_atomic_load((GLB_AS u64*)(st + (lane & 31) * 8), __ATOMIC_RELAXED, __HIP_MEMORY_SCOPE_AGENT);
                    if (__builtin_amdgcn_ballot_w64((unsigned)f == call) != ~0ull) continue;
                    const unsigned ref = __builtin_amdgcn_readfirstlane((unsigned)(f >> 32));
                    const unsigned sum = wave_scan((unsigned)(f >> 32) - ref);
                    const unsigned s0 = __builtin_amdgcn_readlane(sum, 31), s1 = __builtin_amdgcn_readlane(sum, 63) - s0;
                    const int ph = STAG * (((c + 5 * dl) & 31) - 16) * PACE / 32;
                    if ((lane & 31) == 0) ctr[16 + dl] = (ref + ((int)(lane ? s1 : s0) >> 5) - TBIAS + ph) | 1u;
                    break;
                }
            }
            while (lds_get(ctr + d + 4) == 0) __builtin_amdgcn_s_sleep(16);
            CBAR();
            if (lane == 0) __hip_atomic_store(cnt, call, __ATOMIC_RELAXED, __HIP_MEMORY_SCOPE_AGENT);
            const u64 t0 = __builtin_amdgcn_s_memrealtime();
            unsigned pend = (1u << ntile) - 1u, fly = 0, qn = 0, finfo = 0;
            for (unsigned it = 1; pend | fly; ++it) {
                u32x2 f = {0, 0};
                asm volatile("s_mov_b64 exec, %3\n\tglobal_load_dwordx2 %0, %1, %2 sc0 sc1\n\ts_mov_b64 exec, -1\n\ts_waitcnt vmcnt(0)"
                             : "+v"(f) : "v"(lane * 4096), "s"(lflg), "s"((u64)pend) : "memory");
                if (fly) {
                    if ((fly >> (lane & 31)) & (lane < 8)) ctr[32 - 32 + d * 8 + qn + __builtin_amdgcn_mbcnt_lo(fly, 0)] = finfo | lane << 26;
                    qn += __builtin_popcount(fly);
                    LGKMCNT0();
                    if (lane == 0) __hip_atomic_store(ctr + d, qn, __ATOMIC_RELAXED, __HIP_MEMORY_SCOPE_WORKGROUP);
                }
                fly = (unsigned)__builtin_amdgcn_ballot_w64(f[0] >= call) & pend;
                finfo = f[1];
                if (!fly) {
                    if (it % TMOEVERY == 0 &&
                        ((unsigned)__builtin_amdgcn_readfirstlane(tmo_get(tmo)) == call || __builtin_amdgcn_s_memrealtime() - t0 > DEADLINE_TICKS)) {
                        if (lane == 0) __hip_atomic_store(tmo, call, __ATOMIC_RELAXED, __HIP_MEMORY_SCOPE_AGENT);
                        fly = pend; pend = 0; finfo = F_SKIP;
                        continue;
                    }
                    __builtin_amdgcn_s_sleep(LSLEEP);
                    continue;
                }
                pend &= ~fly;
                for (unsigned r = fly; r; r &= r - 1) {
                    const int n = __builtin_ctz(r);
                    TRACE(n, 0);
                    LOAD_LDS_SYS(wlds + n * LSLOT, voff, lsrc + (size_t)n * 32 * LSLOT);
                    LOAD_LDS_SYS(wlds + n * LSLOT + 1024, voff, lsrc + (size_t)n * 32 * LSLOT + 1024);
                }
            }
            LGKMCNT0();
            TRACE_OUT();
            __builtin_amdgcn_endpgm();
        }
        {
            char* prev_flag = myws + DUMMY_OFF + (c * 8 + wave) * 8;
            const char* xdst = peer + par * XS_PAR + (size_t)d * NBLK * BLK_BYTES + (SWZ ? 0 : c * 4096);
            char* fdst = peer + FX_OFF + (size_t)(d * NBLK * 32 + c) * FSTRIDE;
            unsigned tnext = 0, tideal = 0, tfirst = 0;
            auto pace = [&](unsigned dt) {
                int late;
                while ((late = (int)(now16() - tnext)) < 0) __builtin_amdgcn_s_sleep(1);
                tnext += dt;
                if (PSLACK && late > PSLACK) tnext += late - PSLACK;
                if (ALIGN) {
                    tideal += dt;
                    tnext += max(-AMAX, min(AMAX, (int)(tideal - tnext) >> ALIGN));
                }
            };
            auto align = [&]() {
                if (ALIGN && tfirst) {
                    const unsigned t = lds_get(ctr + 16 + d);
                    if (t) { tideal += t - tfirst; tfirst = 0; }
                }
            };
            auto send = [&](int blk, unsigned dt, auto nx, auto&& next) {
                const char* dst = xdst + (size_t)blk * BLK_BYTES;
                constexpr int nl = nx.value > 1;
#pragma unroll
                for (int f = 0; f < 4; ++f) {
                    pace(dt);
                    STORE_WT(voff, dst + (SWZ ? UPAGE(c, f, blk) : 0), v[f], f * 1024);
                    if (f == 0) { VMCNT(1); set_flag(prev_flag, call, lane); }
                    if ((WALL >> f) & 1) VMCNT(1 + nl + (f == 1));
                    if (f == 3) align();
                    if (BURST1 && f == 0 && (blk == 0 || blk == 16)) tnext -= BURST1;
                    next(f);
                }
                prev_flag = fdst + blk * 32 * FSTRIDE;
            };
            auto xwait = [&](int blk) { lds_wait(ctr, (unsigned)(ld_pos() >> (4 * blk)) & 15u); };
            int blk, nblk, step;
            if (d >= 4) {
                const unsigned wl = wlds + (16 * (lane >> 5) + ((lane & 15) >> 1)) * 32 + 16 * ((lane >> 4) & 1) + 8 * (lane & 1);
                auto wfrag = [&](u32x4& o, int off) {
                    u32x2 l2, h2;
                    asm volatile("ds_read_b64_tr_b8 %0, %2\n\tds_read_b64_tr_b8 %1, %2 offset:256\n\ts_waitcnt lgkmcnt(0)"
                                 : "=&v"(l2), "=&v"(h2) : "v"(wl + off) : "memory");
                    o = u32x4{l2[0], l2[1], h2[0], h2[1]};
                };
                wfrag(v[0], 0);
                STAMP(1);
                tnext = now16();
                tfirst = tideal = tnext | 1u;
                if (ALIGN) ctr[24 + d] = tfirst;
#pragma clang loop unroll(disable)
                for (int t = 0; t < 3; ++t)
                    send(16 + t, PACE, IC<2>{}, [&](int f) {
                        LOAD_LDS(wlds + (t + 1) * 4096 + f * 1024, wo, wsrc + f * 32 * NSH + (t + 1) * 128);
                        if (f == 3) VMCNT(6);
                        wfrag(v[(f + 1) & 3], t * 4096 + (f + 1) * 1024);
                    });
                send(19, PACE, IC<1>{}, [&](int f) {
                    if (f < 3) wfrag(v[f + 1], 3 * 4096 + (f + 1) * 1024);
                });
                if (lane == 0) __hip_atomic_store(ctr + d + 4, 1u, __ATOMIC_RELAXED, __HIP_MEMORY_SCOPE_WORKGROUP);
                STAMP(3);
                blk = 15; nblk = 16 - N1; step = -1;
            } else {
                STAMP(1);
                tnext = now16();
                tfirst = tideal = tnext | 1u;
                if (ALIGN) ctr[24 + d] = tfirst;
                send(0, PACE, IC<0>{}, [&](int f) {});
                blk = 1; nblk = N1 + N3 - 1; step = 1;
            }
            xwait(blk);
#pragma unroll
            for (int f = 0; f < 4; ++f) v[f] = *(LDS_AS u32x4*)(smem + blk * 4096 + lo[f]);
            const unsigned dt = d >= 4 ? PACE : PACE;
            for (int n = 0; n < nblk; ++n) {
                const int nb = n + 1 < nblk ? blk + step : blk;
                send(blk, dt, IC<0>{}, [&](int f) { if (f == 0) xwait(nb); v[f] = *(LDS_AS u32x4*)(smem + nb * 4096 + lo[f]); });
                blk = nb;
            }
            unsigned pinfo = 0;
            if (d >= 4) {
                STAMP(2);
                const char* rdst = peer + OS_OFF + par * OS_PAR + (size_t)((d & 3) * NIDX + c) * OSLOT;
                char* rflg = peer + FO_OFF + (size_t)((d & 3) * NIDX + c) * 128;
                char* const dummy = myws + DUMMY_OFF + (c * 8 + wave) * 8;
                for (int q = 0; q < ntile; ++q) {
                    const bool none = lds_get(ctr + d) <= (unsigned)q;
                    if (none) {
                        VMCNT(0);
                        flag64_set(prev_flag, call, pinfo, lane);
                        prev_flag = dummy;
                        lds_wait(ctr + d, q + 1);
                    }
                    CBAR();
                    const unsigned info = lds_get(ctr + d * 8 + q);
                    if (info & F_SKIP) continue;
                    const unsigned n = (info >> 26) & 7, nbytes = info & 0xFFFF;
                    v[0] = *(LDS_AS u32x4*)(size_t)(wlds + n * LSLOT + voff);
                    v[1] = *(LDS_AS u32x4*)(size_t)(wlds + n * LSLOT + 1024 + voff);
                    const char* dst = rdst + (size_t)n * 32 * OSLOT;
                    tideal = tnext;
                    pace(PACET);
                    TRACE(n, 0);
                    asm volatile("v_cmpx_gt_u32 vcc, %3, %0\n\tglobal_store_dwordx4 %0, %2, %1 sc0 sc1\n\ts_mov_b64 exec, -1\n\ts_nop 0"
                                 :: "v"(voff), "s"(dst), "v"(v[0]), "s"(nbytes) : "memory", "vcc");
                    const unsigned rest = max((int)nbytes - 1024, 16);
                    pace((rest + 64) * PACET / 1024);
                    asm volatile("v_cmpx_gt_u32 vcc, %3, %0\n\tglobal_store_dwordx4 %0, %2, %1 offset:1024 sc0 sc1\n\ts_mov_b64 exec, -1\n\ts_nop 0"
                                 :: "v"(voff), "s"(dst), "v"(v[1]), "s"(rest) : "memory", "vcc");
                    VMCNT(2);
                    flag64_set(prev_flag, call, pinfo, lane);
                    TRACE(n, 1);
                    prev_flag = rflg + (size_t)n * 32 * 128;
                    pinfo = info;
                }
            }
            VMCNT(0);
            flag64_set(prev_flag, call, pinfo, lane);
            if (lane == 0) __hip_atomic_store(cnt, call, __ATOMIC_RELAXED, __HIP_MEMORY_SCOPE_AGENT);
            STAMP(d >= 4 ? 4 : 2);
            TRACE_OUT();
        }
        __builtin_amdgcn_endpgm();
    }

    const u64 tstart = __builtin_amdgcn_s_memrealtime();
    const float scale = __builtin_bit_cast(float, __builtin_amdgcn_readfirstlane(__builtin_bit_cast(int, *(const CONST_AS float*)a.sx * *(const CONST_AS float*)a.sw)));
    GLB_AS unsigned* tmo = (GLB_AS unsigned*)(myws + TMO_OFF);
    const float fnan = __builtin_nanf("");
    const unsigned call = *cnt_c + 1u;
    const unsigned par = call & 1u;
    const int rank = a.rank;

    const int grp = j >> 4, tile = j & 15;
    const int foreign = role == 2 ? !(grp & 1) : 2, slot = grp >> 1;
    const int downer = 4 + (tile >> 2);
    if (NOCOMP) {
        if (lane == 0) __hip_atomic_store(cnt, call, __ATOMIC_RELAXED, __HIP_MEMORY_SCOPE_AGENT);
        if (wave < 8 && tile == 0 && foreign == 1) {
            const int d = slot + 1;
            const int blk = d < 4 ? N1 + N3 - 1 : N1;
            GLB_AS unsigned* p = (GLB_AS unsigned*)(myws + FX_OFF + (size_t)((d * NBLK + blk) * 32 + 4 * wave + (lane & 3)) * FSTRIDE);
            wait_flags(p, 0, tmo, call, lane);
            STAMP(3);
        }
        __builtin_amdgcn_endpgm();
    }

    const int su = (slot * 16 + tile) * 8 + wave;
    const bool selfcopy = foreign == 1 && wave < 8 && su < 512;
    if (DELAY0 && !foreign) __builtin_amdgcn_s_sleep(DELAY0);
    if (wave < 8 && !foreign) {
        const signed char* wsrc = win + (size_t)(wave * 512 + (lane >> 1)) * NSH + tile * 32 + (lane & 1) * 16;
#pragma unroll
        for (int i = 0; i < 16; ++i)
            __builtin_amdgcn_global_load_lds((const GLB_AS void*)(wsrc + (size_t)i * 32 * NSH),
                                             (LDS_AS void*)(smem + (wave * 16 + i) * 1024), 16, 0, 0);
    }
    if (selfcopy) {
        const signed char* xs = xin + (size_t)(su >> 5) * BLK_BYTES + (su & 31) * 128 + xsrc_off;
#pragma unroll
        for (int i = 0; i < 4; ++i)
            __builtin_amdgcn_global_load_lds((const GLB_AS void*)(xs + (size_t)i * 8 * KDIM),
                                             (LDS_AS void*)(smem + (wave * 16 + i) * 1024), 16, 0, 0);
    }
    const int tsel = foreign ? 1 + (tile & 3) : 0;
    u64 t_lo = a.tab[tsel][slot][0], t_hi = a.tab[tsel][slot][1];
    u64 o_lo = a.ord[tile & 3][slot][0], o_hi = a.ord[tile & 3][slot][1];
    if (lane == 0) __hip_atomic_store(cnt, call, __ATOMIC_RELAXED, __HIP_MEMORY_SCOPE_AGENT);
    auto src_d = [&](int e) { const int code = (e >> 4) & 7; return foreign && code >= 4 ? downer ^ (code & 3) : code; };
    auto blk_of = [&](int e) { return src_d(e) * NBLK + (e & 15); };

    auto epilogue = [&](int i, int e) {
        const float bad = lds_get(ctr + 17) != 0 ? fnan : 0.0f;
        LDS_AS unsigned char* tp = smem + i * TILE_STRIDE + 128;
        const int srank = rank ^ src_d(e);
        const float* op = (foreign ? a.outp[tile >> 2] : a.out) + (size_t)(srank * MSH + (e & 15) * 32) * NSH + (foreign ? (tile & 3) * 4 + (tile >> 2) : tile) * 32;
        unsigned vo = voff;
        asm volatile("" : "+v"(vo));
        i32x4 q[4];
#pragma unroll
        for (int k = 0; k < 4; ++k) q[k] = *(LDS_AS i32x4*)(tp + k * 1024 + vo);
        vo = (vo >> 7) * (NSH * 4) + (vo & 0x70);
#pragma unroll
        for (int k = 0; k < 4; ++k) {
            f32x4 o;
#pragma unroll
            for (int r = 0; r < 4; ++r) o[r] = fmaxf((float)q[k][r] * scale, 0.0f) + bad;
            STORE_WT(vo, op + (size_t)k * 8 * NSH, o, 0);
        }
    };

    if (role == 1 || wave >= 8) {
    int link, k0, kend, kstep;
    LDS_AS unsigned char* tp;
    if (role == 2) {
    {
        const int i0 = wave - 8;
        const unsigned npacked = a.npacked;
        for (int i = i0; i < NTASK; i += 4) {
            const int sh = 8 * (i & 7);
            const int e = (int)(((i & 8 ? t_hi : t_lo) >> sh) & 0xFF);
            if (e == 0xFF) break;
            unsigned ticket = 0;
            if (foreign && !(e & 0x80)) {
                lds_wait(ctr + 32 + i, 8u);
                if (lane == 0) ticket = __hip_atomic_fetch_add((GLB_AS unsigned*)(myws + TK_OFF + (tile >> 2) * 128), 1u, __ATOMIC_RELAXED, __HIP_MEMORY_SCOPE_AGENT);
            }
            lds_wait(ctr + i, 8u);
            if (!foreign || (e & 0x80)) {
                TRACE(i, 0);
                epilogue(i, e);
                if (foreign) {
                    const int oi = (int)(((i & 8 ? o_hi : o_lo) >> sh) & 0xFF);
                    VMCNT(0);
                    flag64_set(peer + FO_OFF + (size_t)((tile >> 2) * NIDX + oi) * 128, call, 1u, lane);
                }
                TRACE(i, 1);
                continue;
            }
            if (PTR) TRACE(i, 0);
            const bool failed = lds_get(ctr + 17) != 0;
            LDS_AS unsigned char* tp = smem + i * TILE_STRIDE;
            i32x4 q[4];
#pragma unroll
            for (int k = 0; k < 4; ++k) q[k] = *(LDS_AS i32x4*)(tp + 128 + k * 1024 + voff);
            __builtin_amdgcn_s_waitcnt(0xC07F);
            CBAR();
            if (scale < 0.0f) {
#pragma unroll
                for (int k = 0; k < 4; ++k) q[k] = -q[k];
            }
            unsigned m = 0;
            int mx = 0;
#pragma unroll
            for (int r = 15; r >= 0; --r) {
                m = __builtin_amdgcn_alignbit(m, (unsigned)-q[r >> 2][r & 3], 31);
                mx = max(mx, q[r >> 2][r & 3]);
            }
            if (P21) *(LDS_AS u32x4*)(tp + 128 + voff) = u32x4{0, 0, 0, 0};
            const bool wide = __builtin_amdgcn_ballot_w64(mx >= (1 << (P21 ? 21 : 24))) != 0;
            const unsigned c1 = __builtin_popcount(m), inc = wave_scan(c1);
            const unsigned tot = __builtin_amdgcn_readlane(inc, 63);
            const unsigned hbytes = (5 * tot + 31) >> 5 << 2;
            const unsigned nbytes = (128 + (wide ? tot * 4 : P21 ? hbytes + 2 * tot : 3 * tot) + 63) & ~63u;
            *(LDS_AS unsigned short*)(tp + lane * 2) = (unsigned short)m;
            if (PTR == 2) TRACE(i, 1);
            if (wide) {
                unsigned ep = (unsigned)(size_t)tp + 128 + (inc - c1) * 4;
#pragma unroll
                for (int r = 0; r < 16; ++r)
                    asm volatile("v_cmpx_lt_i32 vcc, 0, %1\n\tds_write_b32 %0, %1\n\tv_add_u32 %0, 4, %0\n\ts_mov_b64 exec, -1"
                                 : "+v"(ep) : "v"(q[r >> 2][r & 3]) : "memory", "vcc");
            } else if (P21) {
                const unsigned tb = (unsigned)(size_t)tp + 128, ad = tb + 3072;
                unsigned a2 = tb + hbytes + 2 * (inc - c1), g = 5 * (inc - c1);
#pragma unroll
                for (int r0 = 0; r0 < 16; r0 += 6) {
                    unsigned acc = 0, sh5 = 0;
#pragma unroll
                    for (int r = r0; r < r0 + 6 && r < 16; ++r) {
                        int nb;
                        BITMASK(nb, m, r);
                        asm volatile("ds_write_b16 %0, %1" :: "v"((a2 & nb) | (ad & ~nb)), "v"(q[r >> 2][r & 3]) : "memory");
                        acc |= (((unsigned)q[r >> 2][r & 3] >> 16) & nb) << sh5;
                        STEP(a2, nb, 2);
                        STEP(sh5, nb, 5);
                    }
                    const u64 w = (u64)acc << (g & 31);
                    asm volatile("ds_or_b32 %0, %1\n\tds_or_b32 %0, %2 offset:4" :: "v"(tb + (g >> 5 << 2)), "v"((unsigned)w), "v"((unsigned)(w >> 32)) : "memory");
                    g += sh5;
                }
            } else {
                const unsigned tb = (unsigned)(size_t)tp + 128, c2 = -tb - 4 * tot, ad = tb + 2 * tot + 1536;
                unsigned a1 = tb + 2 * tot + inc - c1;
#pragma unroll
                for (int r = 0; r < 16; ++r) {
                    const unsigned nb = __builtin_amdgcn_sbfe(m, r, 1);
                    const unsigned ae = (a1 & nb) | (ad & ~nb);
                    a1 -= nb;
                    asm volatile("ds_write_b16 %0, %2\n\tds_write_b8_d16_hi %1, %2" :: "v"(ae * 2 + c2), "v"(ae), "v"(q[r >> 2][r & 3]) : "memory");
                }
            }
            const unsigned info = nbytes | (unsigned)((tile & 3) << 8 | e) << 16 | (wide ? F_WIDE : 0u) | (failed ? F_NAN : 0u);
            const unsigned oi = min((unsigned)__builtin_amdgcn_readfirstlane(ticket) - (call - 1u) * npacked, npacked - 1u);
            const int idx = (tile >> 2) * NIDX + oi;
            const bool staged = oi < NSTAGE && nbytes <= LSLOT && (!TCUT || __builtin_amdgcn_s_memrealtime() - tstart < TCUT);
            if (oi < NSTAGE && !staged) flag64_set(myws + FL_OFF + (size_t)idx * 128, call, F_SKIP, lane);
            const char* dst = (staged ? myws + LS_OFF + (size_t)idx * LSLOT : peer + OS_OFF + par * OS_PAR + (size_t)idx * OSLOT) + 2048;
            if (PTR != 2) TRACE(i, PTR);
#pragma unroll
            for (int k = 0; k < 5; ++k)
                if (k * 1024 < nbytes) {
                    const u32x4 pv = *(LDS_AS u32x4*)(tp + k * 1024 + voff);
                    if (k * 1024 + voff < nbytes) STORE_WT(voff, dst, pv, k * 1024 - 2048);
                }
            VMCNT(0);
            flag64_set((staged ? myws + FL_OFF : peer + FO_OFF) + (size_t)idx * 128, call, info, lane);
            if (!PTR) TRACE(i, 1);
        }
        STAMP(5);
        TRACE_OUT();
        if (!KSPLIT || !foreign || i0 == 3) __builtin_amdgcn_endpgm();
        const int id = i0 * (NSLOT * 16) + slot * 16 + tile;
        link = id & 3; k0 = KSPLIT + (id >> 2); kend = a.ntotal; kstep = NIDX;
        tp = smem + i0 * TILE_STRIDE;
    }
    } else {
        link = j & 3; k0 = (j >> 2) * 12 + wave; kstep = 48;
        kend = KSPLIT ? KSPLIT : a.ntotal;
        tp = smem + wave * TILE_STRIDE;
    }
    {
        const int dn = 4 + link;
        const int npacked = a.npacked;
        bool failed = false;
        u64 fpre = 0;
        for (int n = 0; k0 + kstep * n < kend; ++n) {
            const int idx = link * NIDX + k0 + kstep * n;
            const unsigned info = wait_flag64(myws + FO_OFF + (size_t)idx * 128, fpre, tmo, call, lane);
            CBAR();
            TRACE(n, 0);
            if (info == 0 || k0 + kstep * n >= npacked) {
                if (info == 0 || (unsigned)__builtin_amdgcn_readfirstlane(tmo_get(tmo)) == call) {
                    a.out[(size_t)rank * MSH * NSH + lane] = fnan;
                    break;
                }
                fpre = 0;
                TRACE(n, 1);
                continue;
            }
            const unsigned e = info >> 16;
            const int code = (e >> 4) & 7;
            const int srank = rank ^ (code < 4 ? dn ^ code : code & 3);
            float* op = a.out + (size_t)(srank * MSH + (e & 15) * 32 + (lane >> 3)) * NSH + (((e >> 8) & 3) * 4 + link) * 32 + (lane & 7) * 4;
            const char* srcp = myws + OS_OFF + par * OS_PAR + (size_t)idx * OSLOT + voff;
            const unsigned nbytes = min(info & 0xFFFFu, 4224u);
#pragma unroll
            for (int k = 0; k < 5; ++k)
                if (k * 1024 + voff < nbytes)
                    __builtin_amdgcn_global_load_lds((const GLB_AS void*)(srcp + k * 1024), (LDS_AS void*)(tp + k * 1024), 16, 0, 17);
            unsigned tv = tmo_get(tmo);
            __builtin_amdgcn_s_waitcnt(0x0F70);
            tv = __builtin_amdgcn_readfirstlane(tv);
            asm volatile("" : "+s"(tv) :: "memory");
            fpre = 0;
            if (k0 + kstep * (n + 1) < kend) fpre = flag64_get(myws + FO_OFF + (size_t)(idx + kstep) * 128);
            failed = failed || tv == call;
            const float mul = failed || (info & F_NAN) ? fnan : __builtin_fabsf(scale);
            const unsigned m = *(LDS_AS unsigned short*)(tp + lane * 2);
            const unsigned c1 = __builtin_popcount(m), inc = wave_scan(c1);
            unsigned y[16];
            if (info & F_WIDE) {
                unsigned ix = inc - c1;
#pragma unroll
                for (int r = 0; r < 16; ++r) {
                    const int nb = __builtin_amdgcn_sbfe(m, r, 1);
                    const unsigned w = *(LDS_AS unsigned*)(tp + 128 + ix * 4);
                    ix -= nb;
                    y[r] = __builtin_bit_cast(unsigned, (float)(w & (unsigned)nb) * mul);
                }
            } else if (P21) {
                const unsigned tot = __builtin_amdgcn_readlane(inc, 63);
                const unsigned tb = (unsigned)(size_t)tp + 128;
                unsigned a2 = tb + ((5 * tot + 31) >> 5 << 2) + 2 * (inc - c1), g[3], win[3];
                u32x2 hw[3];
                int nb[16];
#pragma unroll
                for (int k = 0; k < 3; ++k) {
                    g[k] = 5 * (inc - c1 + __builtin_popcount(m & ((1u << 6 * k) - 1u)));
                    const LDS_AS unsigned* hp = (const LDS_AS unsigned*)(size_t)(tb + (g[k] >> 5 << 2));
                    hw[k] = u32x2{hp[0], hp[1]};
                }
#pragma unroll
                for (int r = 0; r < 16; ++r) {
                    y[r] = *(LDS_AS unsigned short*)(size_t)a2;
                    BITMASK(nb[r], m, r);
                    STEP(a2, nb[r], 2);
                }
#pragma unroll
                for (int r = 0; r < 16; r += 8)
                    asm volatile("" : "+v"(y[r]), "+v"(y[r + 1]), "+v"(y[r + 2]), "+v"(y[r + 3]),
                                      "+v"(y[r + 4]), "+v"(y[r + 5]), "+v"(y[r + 6]), "+v"(y[r + 7]) :: "memory");
                asm volatile("" : "+v"(hw[0]), "+v"(hw[1]), "+v"(hw[2]));
#pragma unroll
                for (int k = 0; k < 3; ++k) win[k] = __builtin_amdgcn_alignbit(hw[k][1], hw[k][0], g[k]);
                unsigned sh5 = 0;
#pragma unroll
                for (int r = 0; r < 16; ++r) {
                    if (r % 6 == 0) sh5 = 0;
                    asm("v_lshrrev_b32_sdwa %0, %1, %2 dst_sel:WORD_1 dst_unused:UNUSED_PRESERVE src0_sel:DWORD src1_sel:DWORD" : "+v"(y[r]) : "v"(sh5), "v"(win[r / 6]));
                    y[r] = __builtin_bit_cast(unsigned, (float)(y[r] & (unsigned)nb[r] & 0x1FFFFFu) * mul);
                    STEP(sh5, nb[r], 5);
                }
            } else {
                const unsigned tot = __builtin_amdgcn_readlane(inc, 63);
                const unsigned tb = (unsigned)(size_t)tp + 128, c2 = -tb - 4 * tot;
                unsigned a1 = tb + 2 * tot + inc - c1, wh[16];
#pragma unroll
                for (int r = 0; r < 16; ++r) {
                    y[r] = *(LDS_AS unsigned short*)(size_t)(a1 * 2 + c2);
                    wh[r] = *(LDS_AS unsigned char*)(size_t)a1;
                    a1 -= __builtin_amdgcn_sbfe(m, r, 1);
                }
#pragma unroll
                for (int r = 0; r < 16; r += 4)
                    asm volatile("" : "+v"(y[r]), "+v"(y[r + 1]), "+v"(y[r + 2]), "+v"(y[r + 3]),
                                      "+v"(wh[r]), "+v"(wh[r + 1]), "+v"(wh[r + 2]), "+v"(wh[r + 3]) :: "memory");
#pragma unroll
                for (int r = 0; r < 16; ++r)
                    y[r] = __builtin_bit_cast(unsigned, (float)((wh[r] << 16 | y[r]) & (unsigned)__builtin_amdgcn_sbfe(m, r, 1)) * mul);
            }
            fpre = (u64)(unsigned)__builtin_amdgcn_readfirstlane((unsigned)(fpre >> 32)) << 32 | (unsigned)__builtin_amdgcn_readfirstlane((unsigned)fpre);
            asm volatile("" : "+s"(fpre));
#pragma unroll
            for (int k = 0; k < 4; ++k) {
                const u32x4 o = {y[4 * k], y[4 * k + 1], y[4 * k + 2], y[4 * k + 3]};
                asm volatile("global_store_dwordx4 %0, %1, off sc0 sc1\n\ts_nop 1" :: "v"(op + (size_t)k * 8 * NSH), "v"(o) : "memory");
            }
            TRACE(n, 1);
        }
        STAMP(role == 1 ? 5 : 4);
        TRACE_OUT();
        __builtin_amdgcn_endpgm();
    }
    }

    bool failed = false;
    const unsigned fxo = (lane & 3) * FSTRIDE;
    auto flag_ptr = [&](int eb) {
        char* p = myws + FX_OFF + (size_t)(eb * 32 + 4 * wave) * FSTRIDE;
        asm("" : "+s"(p));
        return (GLB_AS unsigned*)(p + fxo);
    };
    unsigned aoff = 128 + (4 * (lane >> 5) * 32 + (lane & 31)) * 4;
    asm("" : "+v"(aoff));
    auto ar = __builtin_amdgcn_make_buffer_rsrc((void*)(myws + par * XS_PAR), 0, (unsigned)XS_PAR, 0x00020000);
    __builtin_amdgcn_s_waitcnt(0x0F70);
    CBAR();
    i32x4 bf[16];
    if (selfcopy) {
        u32x4 v[4];
#pragma unroll
        for (int f = 0; f < 4; ++f) v[f] = *(LDS_AS u32x4*)(smem + wave * 16384 + lo[f]);
        const char* dst = myws + par * XS_PAR;
#pragma unroll
        for (int f = 0; f < 4; ++f) STORE_WT(voff, dst + UPAGE(su, f, su >> 5), v[f], f * 1024);
        VMCNT(0);
        set_flag(myws + FX_OFF + (size_t)su * FSTRIDE, call, lane);
        STAMP(2);
    }
    if (!foreign) {
        const unsigned boff = (16 * (lane >> 5) + ((lane & 15) >> 1)) * 32 + 16 * ((lane >> 4) & 1) + 8 * (lane & 1);
        LDS_AS unsigned char* bp = smem + wave * 16384 + boff;
#pragma unroll
        for (int i = 0; i < 16; ++i) {
            i32x2 l2 = __builtin_amdgcn_ds_read_tr8_b64_v2i32((LDS_AS i32x2*)(bp + (i & 3) * 4096 + (i >> 2) * 1024));
            i32x2 h2 = __builtin_amdgcn_ds_read_tr8_b64_v2i32((LDS_AS i32x2*)(bp + (i & 3) * 4096 + (i >> 2) * 1024 + 256));
            bf[i] = i32x4{l2[0], l2[1], h2[0], h2[1]};
        }
        __builtin_amdgcn_s_waitcnt(0xC07F);
        CBAR();
    }
    if (wave * 16384 < NTASK * TILE_STRIDE) {
#pragma unroll
        for (int i = 0; i < 16; ++i) *(LDS_AS u32x4*)(smem + wave * 16384 + i * 1024 + voff) = u32x4{0, 0, 0, 0};
    }
    LGKMCNT0();
    if (lane == 0) __hip_atomic_fetch_add(ctr + 16, 1u, __ATOMIC_RELAXED, __HIP_MEMORY_SCOPE_WORKGROUP);
    u32x4 af[16];
    unsigned fpre;
    auto aload = [&](int k, unsigned abase) { af[k] = __builtin_amdgcn_raw_buffer_load_b128(ar, voff, ((abase | UPAGE(0, 0, abase >> 17)) ^ UPAGE(k & 3, k >> 2, 0)) + (k >> 2) * 1024, 17); };
    auto peek = [&](int e) {
        const unsigned o = lane & 4 ? (unsigned)(TMO_OFF - FX_OFF) : (blk_of(e) * 32 + 4 * wave) * FSTRIDE + fxo;
        fpre = __hip_atomic_load((GLB_AS unsigned*)(myws + FX_OFF + o), __ATOMIC_RELAXED, __HIP_MEMORY_SCOPE_SYSTEM);
    };
    {
        const int eb = blk_of((int)(t_lo & 0xFF));
        failed = !wait_flags(flag_ptr(eb), 0, tmo, call, lane);
        CBAR();
#pragma unroll
        for (int k = 0; k < 16; ++k) aload(k, (unsigned)eb * BLK_BYTES + wave * 16384);
    }
    const int ebw = downer * NBLK + 16 + (tile & 3);
    if (foreign && !failed) failed = !wait_flags(flag_ptr(ebw), 0, tmo, call, lane);
    CBAR();
    STAMP(6);
    peek((int)(t_lo >> 8) & 0xFF);
    CBAR();
    if (foreign) {
        const unsigned bbase = (unsigned)ebw * BLK_BYTES + wave * 16384;
#pragma unroll
        for (int k = 0; k < 16; ++k)
            bf[k] = __builtin_bit_cast(i32x4, __builtin_amdgcn_raw_buffer_load_b128(ar, voff, ((bbase | UPAGE(0, 0, bbase >> 17)) ^ UPAGE(k & 3, k >> 2, 0)) + (k >> 2) * 1024, 17));
    }
    if (lane == 0) __hip_atomic_fetch_add(ctr + 32, 1u, __ATOMIC_RELAXED, __HIP_MEMORY_SCOPE_WORKGROUP);
    TRACE(0, 0);
    lds_wait(ctr + 16, 8u);
    __builtin_amdgcn_s_waitcnt(0x0F70);
    STAMP(1);
    for (int i = 0; i < NTASK; ++i) {
        t_lo = (t_lo >> 8) | (t_hi << 56); t_hi = (t_hi >> 8) | (0xFFull << 56);
        const int en = (int)(t_lo & 0xFF);
        const unsigned nbase = (unsigned)blk_of(en) * BLK_BYTES + wave * 16384;
        failed = failed || (unsigned)__builtin_amdgcn_readlane(fpre, 4) == call;
        const bool ready = en != 0xFF && ((unsigned)__builtin_amdgcn_ballot_w64(fpre >= call) & 15u) == 15u;
        CBAR();
        i32x16 acc = {};
        if (ready) {
            TRACE(i + 1, 0);
            if (lane == 0) __hip_atomic_fetch_add(ctr + 33 + i, 1u, __ATOMIC_RELAXED, __HIP_MEMORY_SCOPE_WORKGROUP);
            peek((int)(t_lo >> 8) & 0xFF);
#pragma unroll
            for (int k = 0; k < 16; ++k) {
                __builtin_amdgcn_sched_barrier(0);
                acc = __builtin_amdgcn_mfma_i32_32x32x32_i8(__builtin_bit_cast(i32x4, af[k]), bf[k], acc, 0, 0, 0);
                __builtin_amdgcn_sched_barrier(0);
                aload(k, nbase);
            }
            __builtin_amdgcn_sched_barrier(0);
        } else {
#pragma unroll
            for (int k = 0; k < 16; ++k)
                acc = __builtin_amdgcn_mfma_i32_32x32x32_i8(__builtin_bit_cast(i32x4, af[k]), bf[k], acc, 0, 0, 0);
        }
        if (failed && lane == 0) __hip_atomic_store(ctr + 17, 1u, __ATOMIC_RELAXED, __HIP_MEMORY_SCOPE_WORKGROUP);
        LDS_AS unsigned char* tp = smem + i * TILE_STRIDE + aoff;
#pragma unroll
        for (int r = 0; r < 16; ++r)
            __hip_atomic_fetch_add((LDS_AS int*)(tp + ((r & 3) + 8 * (r >> 2)) * 128), acc[r],
                                   __ATOMIC_RELAXED, __HIP_MEMORY_SCOPE_WORKGROUP);
        LGKMCNT0();
        if (lane == 0) __hip_atomic_fetch_add(ctr + i, 1u, __ATOMIC_RELAXED, __HIP_MEMORY_SCOPE_WORKGROUP);
        TRACE(i, 1);
        if (!ready) {
            if (en == 0xFF) break;
            if (!failed) failed = !wait_flags(flag_ptr(blk_of(en)), 0, tmo, call, lane);
            CBAR();
            TRACE(i + 1, 0);
            peek((int)(t_lo >> 8) & 0xFF);
            __builtin_amdgcn_sched_barrier(0);
#pragma unroll
            for (int k = 0; k < 16; ++k) aload(k, nbase);
            __builtin_amdgcn_sched_barrier(0);
            if (lane == 0) __hip_atomic_fetch_add(ctr + 33 + i, 1u, __ATOMIC_RELAXED, __HIP_MEMORY_SCOPE_WORKGROUP);
        }
    }
    STAMP(5);
    TRACE_OUT();
}

static void build_table(int foreign, float tstart, u64 (*tab)[2], float (*fin)[NTASK] = nullptr) {
    struct T { float t; int e; } task[128];
    const int nslot = foreign ? NSLOT : NOWN;
    const float tb = PACE * (4 * 0.01f / 16), tw = PACE * (4 * 0.01f / 16), tx = PACE * (4 * 0.01f / 16);
    int n = 0;
    for (int d = 0; d < 8; ++d)
        for (int blk = 0; blk < 16; ++blk) {
            const int type = blk < N1 ? 1 : blk < N1 + N3 ? 3 : 2;
            float t;
            int direct = 0;
            if (d == 0) { if (foreign && type != 1) continue; t = 3.0f; }
            else if (d < 4) {
                if (type == 2 || (foreign && type == 3)) continue;
                t = T0X + tb * (blk + 1);
                direct = foreign ? blk >= N1 - NRAW : blk == N1 + N3 - 1;
            } else {
                if (type == 1 || (foreign && (type == 3 || d == 4))) continue;
                t = T0X + 4 * tw + tx * (16 - blk);
            }
            task[n].t = t + 0.001f * d; task[n].e = direct << 7 | d << 4 | blk; ++n;
        }
    for (int i = 1; i < n; ++i)
        for (int k = i; k > 0 && task[k].t < task[k - 1].t; --k) { T s = task[k]; task[k] = task[k - 1]; task[k - 1] = s; }
    float fr[NSLOT]; int cntv[NSLOT]; unsigned char lst[NSLOT][NTASK];
    for (int s = 0; s < NSLOT; ++s) { fr[s] = tstart; cntv[s] = 0; for (int i = 0; i < NTASK; ++i) lst[s][i] = 0xFF; }
    for (int i = 0; i < n; ++i) {
        int best = -1;
        for (int s = 0; s < nslot; ++s) if (cntv[s] < NTASK - 1 && (best < 0 || fr[s] < fr[best])) best = s;
        fr[best] = (fr[best] > task[i].t ? fr[best] : task[i].t) + TCOST;
        if (fin) fin[best][cntv[best]] = fr[best];
        lst[best][cntv[best]++] = (unsigned char)task[i].e;
    }
    for (int s = 0; s < NSLOT; ++s)
        for (int h = 0; h < 2; ++h) {
            u64 v = 0;
            for (int i = 7; i >= 0; --i) v = v << 8 | lst[s][h * 8 + i];
            tab[s][h] = v;
        }
}

static void build_all(Args& a) {
    build_table(0, 6.5f, a.tab[0]);
    auto byte = [&](int t, int s, int i) { return (int)((a.tab[1 + t][s][i >> 3] >> (8 * (i & 7))) & 0xFF); };
    int n = 0;
    for (int t = 0; t < 4; ++t) {
        build_table(1, FSTART + FSTEP * t, a.tab[1 + t]);
        for (int s = 0; s < NSLOT; ++s)
            for (int i = 0; i < NTASK; ++i) n += byte(t, s, i) < 0x80;
    }
    a.npacked = n;
    for (int t = 0; t < 4; ++t)
        for (int s = 0; s < NSLOT; ++s) a.ord[t][s][0] = a.ord[t][s][1] = 0;
    for (int blk = 0; blk < 16; ++blk)
        for (int t = 0; t < 4; ++t)
            for (int s = 0; s < NSLOT; ++s)
                for (int i = 0; i < NTASK; ++i)
                    if (byte(t, s, i) != 0xFF && (byte(t, s, i) & 0x8F) == (0x80 | blk)) a.ord[t][s][i >> 3] |= (u64)n++ << (8 * (i & 7));
    a.ntotal = n;
}

extern "C" void kernel_launch(int rank, int nranks, ncclComm_t comm,
                              hipStream_t stream,
                              void* const* d_in, const int* in_sizes, int n_in,
                              void* d_out, int out_size,
                              const int* in_dtypes, int out_dtype,
                              void* const* d_out_all,
                              void* const* d_ws_all, size_t ws_bytes) {
    if (nranks != NRANK || (unsigned)rank >= (unsigned)NRANK || n_in != 4 || ws_bytes < WS_NEED ||
        in_sizes[0] != MSH * KDIM || in_sizes[1] != KDIM * NSH || out_size != NRANK * MSH * NSH) {
        fprintf(stderr, "unexpected problem shape\n");
        return;
    }
    HIP_CHECK(hipFuncSetAttribute((const void*)ag_gemm_kernel, hipFuncAttributeMaxDynamicSharedMemorySize, LDS_BYTES));
    Args a;
    a.x = (const signed char*)d_in[0];
    a.w = (const signed char*)d_in[1];
    a.sx = (const float*)d_in[2];
    a.sw = (const float*)d_in[3];
    a.out = (float*)d_out;
    for (int d = 0; d < NRANK; ++d) a.ws[d] = (char*)d_ws_all[rank ^ d];
    for (int i = 0; i < 4; ++i) a.outp[i] = (float*)d_out_all[rank ^ (4 + i)];
    build_all(a);
    if (a.npacked < NSTAGE || a.ntotal > NIDX || (KSPLIT && a.ntotal > KSPLIT + NSLOT * 12)) return;
    a.rank = rank;
    ag_gemm_kernel<<<dim3(256), dim3(NTHREADS), LDS_BYTES, stream>>>(a);
    HIP_CHECK(hipGetLastError());
}
```

```cpp
#include <hip/hip_runtime.h>
#include <rccl/rccl.h>
#include <cstdio>
#include <cstddef>

#define HIP_CHECK(expr) do { hipError_t _e = (expr); if (_e != hipSuccess) { \
    fprintf(stderr, "HIP error %s at %s:%d\n", hipGetErrorString(_e), __FILE__, __LINE__); \
    return; } } while (0)

typedef int i32x2 __attribute__((ext_vector_type(2)));
typedef int i32x4 __attribute__((ext_vector_type(4)));
typedef int i32x16 __attribute__((ext_vector_type(16)));
typedef unsigned u32x2 __attribute__((ext_vector_type(2)));
typedef unsigned u32x4 __attribute__((ext_vector_type(4)));
typedef float f32x4 __attribute__((ext_vector_type(4)));
typedef unsigned long long u64;
typedef unsigned __attribute__((aligned(1))) u32_u;
#define LDS_AS __attribute__((address_space(3)))
#define GLB_AS __attribute__((address_space(1)))
#define CONST_AS __attribute__((address_space(4)))

#define N1 11
#define N3 1
#define PACE 906
#define FSTART 8.6f
#define FSTEP 2.65f
#define NOCOMP 0
#define TCOST 1.3f
#define NRAW 1
#define PACET PACE
#define LDDELAY 4
#define PSLACK 160
#define NSTAGE 188
#define LSLEEP 8
#define DYNQ 1
#define TCUT 2600
#define PTR 0
#define LDPACE 1700
#define T0X 4.0f
#define FSTRIDE 128
#define PSLEEP 1

#define KSPLIT 144
#define LDORD 1
#define BURST1 0
#define SWZ 3
#define UPAGE(c, f, blk) ((unsigned)((c) ^ ((SWZ & 1 ? (f) : 0) ^ (SWZ & 2 ? (blk) & 3 : 0))) << 12)
#define WALL 4
#define ALIGN 4
#define AMAX 60
#define ASLEEP 60
#define STAG 1
#define TBIAS 0
#define P21 1
#define NTAG 0
#define TD2 50
#define TD3 105
#define TTAG 0.0f
#define DRYW 255
#define EMID 1
#define TDE 0
#define LINORD 0
#define KORD(n) (LINORD ? ((n) & 3) * 4 + (((n) >> 2) ^ ((n) & 3)) : (n))
#define SUEVEN 1
#define DELAY1 0
#define DELAYW 0
#define SLEEP0 2
#define SLEEP1 10
#define SLEEP2 8
#define OST0 6.5f
#define OST1 6.5f
constexpr int NTB = N1 + N3 - NTAG, EBLK = N1 + N3;
#define HAVE_ROLE 1
__host__ __device__ inline int role_of(int b, int& ix) {
    const int r = b >> 3;
    ix = r < 4 ? b : (b & 7) * (r < 6 ? 2 : 26) + r - (r < 6 ? 4 : 6);
    return r < 4 ? 0 : r < 6 ? 1 : 2;
}
constexpr int NRANK = 8;
constexpr int KDIM = 4096, MSH = 512, NSH = 512;
constexpr unsigned BLK_BYTES = 32 * KDIM;
constexpr int NBLK = 20;
constexpr unsigned OSLOT = 4608;
constexpr int NSLOT = 7, NOWN = 6, NTASK = 16, NIDX = 256;
constexpr size_t XS_PAR = (size_t)NRANK * NBLK * BLK_BYTES;
constexpr size_t OS_OFF = 2 * XS_PAR, OS_PAR = (size_t)4 * NIDX * OSLOT;
constexpr unsigned LSLOT = 2048;
constexpr size_t LS_OFF = OS_OFF + 2 * OS_PAR;
constexpr size_t FX_OFF = LS_OFF + 4 * NIDX * LSLOT;
constexpr size_t FO_OFF = FX_OFF + NRANK * NBLK * 32 * 128;
constexpr size_t FL_OFF = FO_OFF + 4 * NIDX * 128;
constexpr size_t CNT_OFF = FL_OFF + 4 * NIDX * 128;
constexpr size_t TMO_OFF = CNT_OFF + 16384;
constexpr size_t TK_OFF = TMO_OFF + 512;
constexpr size_t DUMMY_OFF = TMO_OFF + 1024;
constexpr size_t WS_NEED = DUMMY_OFF + 4096 + 1024;
constexpr unsigned DEADLINE_TICKS = 20000000u;
constexpr int TILE_STRIDE = 4352;
constexpr int LDS_MAIN = 131072;
constexpr int LDS_CTR = LDS_MAIN;
constexpr int LDS_BYTES = LDS_CTR + 256;
constexpr int NTHREADS = 768;
static_assert(NTASK * TILE_STRIDE <= LDS_MAIN, "tiles");
#define DIAG_N 0
#define STAMP(i)
#define TRACE(i, h)
#define TRACE_OUT()

struct Rec {
    const signed char* src;
    char* dst;
    const char* cnt;
    char* peer;
    const signed char* x;
    const signed char* w;
    char* ws;
    char* dst1;
};
struct Args {
    Rec rec[NTHREADS / 64];
    const float* sx;
    const float* sw;
    float* out;
    char* ws[NRANK];
    float* outp[4];
    u64 tab[5][NSLOT][2];
    u64 ord[4][NSLOT][2];
    int rank;
    int npacked, ntotal;
};

#define STORE_WT(voff, base, val, off) \
    asm volatile("global_store_dwordx4 %0, %2, %1 offset:%3 sc0 sc1\n\ts_nop 1" \
                 :: "v"(voff), "s"(base), "v"(val), "i"(off) : "memory")
#define VMCNT(n) asm volatile("s_waitcnt vmcnt(%0)" :: "i"(n) : "memory")
#define LGKMCNT0() asm volatile("s_waitcnt lgkmcnt(0)" ::: "memory")
#define CBAR() asm volatile("" ::: "memory")

__device__ __forceinline__ void set_flag(char* p, unsigned call, int lane) {
    if (lane == 0) __hip_atomic_store((GLB_AS unsigned*)p, call, __ATOMIC_RELAXED, __HIP_MEMORY_SCOPE_SYSTEM);
}
__device__ __forceinline__ unsigned lds_get(LDS_AS unsigned* p) {
    return __builtin_amdgcn_readfirstlane(__hip_atomic_load(p, __ATOMIC_RELAXED, __HIP_MEMORY_SCOPE_WORKGROUP));
}
__device__ __forceinline__ void lds_wait(LDS_AS unsigned* p, unsigned v) {
    while (lds_get(p) < v) __builtin_amdgcn_s_sleep(1);
    CBAR();
}

#define TMOEVERY 8
#define DELAY0 50
__device__ __forceinline__ unsigned tmo_get(GLB_AS unsigned* tmo) {
    return __hip_atomic_load(tmo, __ATOMIC_RELAXED, __HIP_MEMORY_SCOPE_AGENT);
}
__device__ __forceinline__ bool wait_flags(GLB_AS unsigned* p, unsigned f, GLB_AS unsigned* tmo, unsigned call, int lane) {
    if (((unsigned)__builtin_amdgcn_ballot_w64(f >= call) & 15u) == 15u) return true;
    const unsigned t0 = (unsigned)__builtin_amdgcn_s_memrealtime();
    for (unsigned it = 1;; ++it) {
        if (lane < 4) f = __hip_atomic_load(p, __ATOMIC_RELAXED, __HIP_MEMORY_SCOPE_SYSTEM);
        if (((unsigned)__builtin_amdgcn_ballot_w64(f >= call) & 15u) == 15u) return true;
        if (it % TMOEVERY == 0 &&
            (__builtin_amdgcn_readfirstlane(tmo_get(tmo)) == call || (unsigned)__builtin_amdgcn_s_memrealtime() - t0 > DEADLINE_TICKS)) {
            if (lane == 0) __hip_atomic_store(tmo, call, __ATOMIC_RELAXED, __HIP_MEMORY_SCOPE_AGENT);
            return false;
        }
        __builtin_amdgcn_s_sleep(PSLEEP);
    }
}

template <int V> struct IC { static constexpr int value = V; };
constexpr unsigned F_WIDE = 1u << 30, F_NAN = 1u << 31, F_SKIP = 1u << 29;
#define BITMASK(nb, m, r) asm("v_bfe_i32 %0, %1, %2, 1" : "=v"(nb) : "v"(m), "i"(r))
#define STEP(x, nb, n) asm("v_mad_i32_i24 %0, %1, %2, %3" : "=v"(x) : "v"(nb), "i"(-(n)), "v"(x))
__device__ __forceinline__ unsigned wave_scan(unsigned x) {
    x += __builtin_amdgcn_update_dpp(0u, x, 0x111, 0xf, 0xf, false);
    x += __builtin_amdgcn_update_dpp(0u, x, 0x112, 0xf, 0xf, false);
    x += __builtin_amdgcn_update_dpp(0u, x, 0x114, 0xf, 0xf, false);
    x += __builtin_amdgcn_update_dpp(0u, x, 0x118, 0xf, 0xf, false);
    x += __builtin_amdgcn_update_dpp(0u, x, 0x142, 0xa, 0xf, false);
    x += __builtin_amdgcn_update_dpp(0u, x, 0x143, 0xc, 0xf, false);
    return x;
}
constexpr u64 ld_order() {
    u64 o = 0; int n = 0, a = 1, z = 15;
    for (int slot = 0; n < 15; ++slot) {
        const bool cross = z >= N1 && (a >= N1 || 4 + (15 - z) - (z < 15) <= a);
        if (cross) o |= (u64)z-- << (4 * n++); else o |= (u64)a++ << (4 * n++);
    }
    return o;
}
constexpr u64 ld_pos() {
    u64 p = 0;
    for (int n = 0; n < 15; ++n) p |= (u64)(n + 1) << (4 * ((ld_order() >> (4 * n)) & 15));
    return p;
}
#define LOAD_LDS_SYS(ldsaddr, voffs, base) \
    asm volatile("s_nop 0\n\tglobal_load_lds_dwordx4 %1, %2 sc0 sc1" :: "{m0}"(ldsaddr), "v"(voffs), "s"(base) : "memory")
#define LOAD_LDS(ldsaddr, voffs, base) \
    asm volatile("s_nop 0\n\tglobal_load_lds_dwordx4 %1, %2" :: "{m0}"(ldsaddr), "v"(voffs), "s"(base) : "memory")

__device__ __forceinline__ u64 flag64_get(const char* p) {
    return __hip_atomic_load((GLB_AS u64*)p, __ATOMIC_RELAXED, __HIP_MEMORY_SCOPE_SYSTEM);
}
__device__ __forceinline__ void flag64_set(char* p, unsigned call, unsigned info, int lane) {
    if (lane == 0) __hip_atomic_store((GLB_AS u64*)p, (u64)info << 32 | call, __ATOMIC_RELAXED, __HIP_MEMORY_SCOPE_SYSTEM);
}
__device__ __forceinline__ unsigned wait_flag64(const char* p, u64 f, GLB_AS unsigned* tmo, unsigned call, int lane) {
    if ((unsigned)__builtin_amdgcn_readfirstlane((unsigned)f) >= call) return __builtin_amdgcn_readfirstlane((unsigned)(f >> 32));
    const unsigned t0 = (unsigned)__builtin_amdgcn_s_memrealtime();
    for (unsigned it = 1;; ++it) {
        f = flag64_get(p);
        if ((unsigned)__builtin_amdgcn_readfirstlane((unsigned)f) >= call) return __builtin_amdgcn_readfirstlane((unsigned)(f >> 32));
        if (it % TMOEVERY == 0 &&
            ((unsigned)__builtin_amdgcn_readfirstlane(tmo_get(tmo)) == call || (unsigned)__builtin_amdgcn_s_memrealtime() - t0 > DEADLINE_TICKS)) {
            if (lane == 0) __hip_atomic_store(tmo, call, __ATOMIC_RELAXED, __HIP_MEMORY_SCOPE_AGENT);
            return 0;
        }
        __builtin_amdgcn_s_sleep(PSLEEP);
    }
}

#define TG_STR_(x) #x
#define TG_STR(x) TG_STR_(x)
#define TG_DEADLINE 20000000
static_assert(TG_DEADLINE == DEADLINE_TICKS, "deadline");
#define TG_DIAG(...)
#define TG_TASK \
    "v_mov_b32 v139, 0\n\ts_cmp_eq_u32 %[dry], 0\n\ts_cbranch_scc1 1f\n\ts_mov_b64 exec, 0\n\t" \
    "s_cmp_eq_u32 %[dry], 2\n\ts_cbranch_scc1 10f\n\ts_cmp_eq_u32 %[dry], 3\n\ts_cbranch_scc1 20f\n\ts_cmp_eq_u32 %[dry], 4\n\ts_cbranch_scc1 11f\n\ts_cmp_eq_u32 %[dry], 5\n\ts_cbranch_scc1 21f\n\ts_cmp_eq_u32 %[dry], 6\n\ts_cbranch_scc1 12f\n\ts_cmp_eq_u32 %[dry], 7\n\ts_cbranch_scc1 13f\n\ts_cmp_eq_u32 %[dry], 8\n\ts_cbranch_scc1 30f\n" \
    "1:\n\tv_and_b32 v138, 0xffffcfff, %[vo]\n\tv_xor_b32 v143, 0x1000, %[vo]\n\tv_xor_b32 v144, 0x2000, %[vo]\n\tv_xor_b32 v145, 0x3000, %[vo]\n\t" \
    "buffer_load_dwordx2 v[130:131], v138, %[ar], %[eb] offen offset:0 sc0 sc1\n\tbuffer_load_dwordx2 v[132:133], v138, %[ar], %[eb] offen offset:1024 sc0 sc1\n\tbuffer_load_dwordx2 v[134:135], v138, %[ar], %[eb] offen offset:2048 sc0 sc1\n\tbuffer_load_dwordx2 v[136:137], v138, %[ar], %[eb] offen offset:3072 sc0 sc1\n\t" \
    "buffer_load_dwordx4 v[82:85], %[vo], %[ar], %[nb] offen offset:0 sc0 sc1\n\tbuffer_load_dwordx4 v[86:89], v143, %[ar], %[nb] offen offset:0 sc0 sc1\n\tbuffer_load_dwordx4 v[90:93], v144, %[ar], %[nb] offen offset:0 sc0 sc1\n\tbuffer_load_dwordx4 v[94:97], v145, %[ar], %[nb] offen offset:0 sc0 sc1\n\t" \
    "buffer_load_dwordx4 v[98:101], v143, %[ar], %[nb] offen offset:1024 sc0 sc1\n\tbuffer_load_dwordx4 v[102:105], %[vo], %[ar], %[nb] offen offset:1024 sc0 sc1\n\tbuffer_load_dwordx4 v[106:109], v145, %[ar], %[nb] offen offset:1024 sc0 sc1\n\tbuffer_load_dwordx4 v[110:113], v144, %[ar], %[nb] offen offset:1024 sc0 sc1\n\t" \
    "v_lshlrev_b32 v140, 24, %[call]\n" \
    "1:\n\ts_memrealtime vcc\n\ts_waitcnt lgkmcnt(0)\n\ts_sub_i32 vcc_lo, vcc_lo, %[t2]\n\ts_cmp_lt_i32 vcc_lo, " "0" "\n\ts_cbranch_scc1 1b\n\t" \
    "buffer_load_dwordx4 v[114:117], v144, %[ar], %[nb] offen offset:2048 sc0 sc1\n\tbuffer_load_dwordx4 v[118:121], v145, %[ar], %[nb] offen offset:2048 sc0 sc1\n\tbuffer_load_dwordx4 v[122:125], %[vo], %[ar], %[nb] offen offset:2048 sc0 sc1\n\tbuffer_load_dwordx4 v[126:129], v143, %[ar], %[nb] offen offset:2048 sc0 sc1\n\ts_cbranch_execz 30f\n" \
    "10:\n\tv_mov_b32 v141, 0x04020100\n\tv_mov_b32 v142, 0x05020100\n\ts_waitcnt vmcnt(8)\n" \
    "1:\n\tv_bitop3_b32 v138, v83, v85, v140 bitop3:0x7e\n\tv_bitop3_b32 v138, v138, v87, v140 bitop3:0xf6\n\tv_bitop3_b32 v138, v138, v89, v140 bitop3:0xf6\n\tv_bitop3_b32 v138, v138, v91, v140 bitop3:0xf6\n\tv_bitop3_b32 v138, v138, v93, v140 bitop3:0xf6\n\tv_bitop3_b32 v138, v138, v95, v140 bitop3:0xf6\n\tv_bitop3_b32 v138, v138, v97, v140 bitop3:0xf6\n\tv_cmp_lt_u32 vcc, 0xffffff, v138\n\ts_or_b32 vcc_lo, vcc_lo, vcc_hi\n\ts_or_b32 vcc_lo, vcc_lo, %[dry]\n\ts_cbranch_scc0 20f\n\ts_cmp_lg_u32 %[stop], 0\n\ts_cbranch_scc1 20f\n\t" \
    "buffer_load_dwordx4 v[82:85], %[vo], %[ar], %[nb] offen offset:0 sc0 sc1\n\tbuffer_load_dwordx4 v[86:89], v143, %[ar], %[nb] offen offset:0 sc0 sc1\n\tbuffer_load_dwordx4 v[90:93], v144, %[ar], %[nb] offen offset:0 sc0 sc1\n\tbuffer_load_dwordx4 v[94:97], v145, %[ar], %[nb] offen offset:0 sc0 sc1\n\t" \
    "v_mov_b32 v138, 0\n\tglobal_load_dword v139, v138, %[tmo] sc1\n\ts_memrealtime vcc\n\ts_waitcnt vmcnt(0) lgkmcnt(0)\n\tv_readfirstlane_b32 vcc_hi, v139\n\ts_sub_i32 vcc_lo, vcc_lo, %[t2]\n\ts_cmp_gt_u32 vcc_lo, " TG_STR(TG_DEADLINE) "\n\ts_cselect_b32 %[stop], 1, %[stop]\n\ts_cmp_eq_u32 vcc_hi, %[call]\n\ts_cselect_b32 %[stop], 1, %[stop]\n\ts_cbranch_execz 30f\n\t" TG_DIAG("s_add_u32 %[nr], %[nr], 1\n\t") "s_branch 1b\n" \
    "20:\n\tv_perm_b32 v83, v130, v83, v141\n\tv_perm_b32 v85, v130, v85, v142\n\tv_perm_b32 v87, v132, v87, v141\n\tv_perm_b32 v89, v132, v89, v142\n\tv_perm_b32 v91, v134, v91, v141\n\tv_perm_b32 v93, v134, v93, v142\n\tv_perm_b32 v95, v136, v95, v141\n\tv_perm_b32 v97, v136, v97, v142\n\t" \
    "v_mfma_i32_32x32x32_i8 %[acc], v[82:85], %[b0], 0\n\tv_mfma_i32_32x32x32_i8 %[acc], v[86:89], %[b1], %[acc]\n\tv_mfma_i32_32x32x32_i8 %[acc], v[90:93], %[b2], %[acc]\n\tv_mfma_i32_32x32x32_i8 %[acc], v[94:97], %[b3], %[acc]\n" \
    "1:\n\ts_memrealtime vcc\n\ts_waitcnt lgkmcnt(0)\n\ts_sub_i32 vcc_lo, vcc_lo, %[t2]\n\ts_cmp_lt_i32 vcc_lo, " TG_STR(TD3) "-" TG_STR(TD2) "\n\ts_cbranch_scc1 1b\n\t" \
    "buffer_load_dwordx4 v[82:85], v145, %[ar], %[nb] offen offset:3072 sc0 sc1\n\tbuffer_load_dwordx4 v[86:89], v144, %[ar], %[nb] offen offset:3072 sc0 sc1\n\tbuffer_load_dwordx4 v[90:93], v143, %[ar], %[nb] offen offset:3072 sc0 sc1\n\tbuffer_load_dwordx4 v[94:97], %[vo], %[ar], %[nb] offen offset:3072 sc0 sc1\n\t" \
    "v_mov_b32 v138, 0\n\tglobal_load_dword v139, v138, %[tmo] sc1\n\ts_cbranch_execz 30f\n" \
    "11:\n\tv_mov_b32 v141, 0x06020100\n\tv_mov_b32 v142, 0x07020100\n\ts_waitcnt vmcnt(9)\n" \
    "1:\n\tv_bitop3_b32 v138, v99, v101, v140 bitop3:0x7e\n\tv_bitop3_b32 v138, v138, v103, v140 bitop3:0xf6\n\tv_bitop3_b32 v138, v138, v105, v140 bitop3:0xf6\n\tv_bitop3_b32 v138, v138, v107, v140 bitop3:0xf6\n\tv_bitop3_b32 v138, v138, v109, v140 bitop3:0xf6\n\tv_bitop3_b32 v138, v138, v111, v140 bitop3:0xf6\n\tv_bitop3_b32 v138, v138, v113, v140 bitop3:0xf6\n\tv_cmp_lt_u32 vcc, 0xffffff, v138\n\ts_or_b32 vcc_lo, vcc_lo, vcc_hi\n\ts_or_b32 vcc_lo, vcc_lo, %[dry]\n\ts_cbranch_scc0 21f\n\ts_cmp_lg_u32 %[stop], 0\n\ts_cbranch_scc1 21f\n\t" \
    "buffer_load_dwordx4 v[98:101], v143, %[ar], %[nb] offen offset:1024 sc0 sc1\n\tbuffer_load_dwordx4 v[102:105], %[vo], %[ar], %[nb] offen offset:1024 sc0 sc1\n\tbuffer_load_dwordx4 v[106:109], v145, %[ar], %[nb] offen offset:1024 sc0 sc1\n\tbuffer_load_dwordx4 v[110:113], v144, %[ar], %[nb] offen offset:1024 sc0 sc1\n\t" \
    "v_mov_b32 v138, 0\n\tglobal_load_dword v139, v138, %[tmo] sc1\n\ts_memrealtime vcc\n\ts_waitcnt vmcnt(0) lgkmcnt(0)\n\tv_readfirstlane_b32 vcc_hi, v139\n\ts_sub_i32 vcc_lo, vcc_lo, %[t2]\n\ts_cmp_gt_u32 vcc_lo, " TG_STR(TG_DEADLINE) "\n\ts_cselect_b32 %[stop], 1, %[stop]\n\ts_cmp_eq_u32 vcc_hi, %[call]\n\ts_cselect_b32 %[stop], 1, %[stop]\n\ts_cbranch_execz 30f\n\t" TG_DIAG("s_add_u32 %[nr], %[nr], 256\n\t") "s_branch 1b\n" \
    "21:\n\tv_perm_b32 v99, v130, v99, v141\n\tv_perm_b32 v101, v130, v101, v142\n\tv_perm_b32 v103, v132, v103, v141\n\tv_perm_b32 v105, v132, v105, v142\n\tv_perm_b32 v107, v134, v107, v141\n\tv_perm_b32 v109, v134, v109, v142\n\tv_perm_b32 v111, v136, v111, v141\n\tv_perm_b32 v113, v136, v113, v142\n\t" \
    "v_mfma_i32_32x32x32_i8 %[acc], v[98:101], %[b4], %[acc]\n\tv_mfma_i32_32x32x32_i8 %[acc], v[102:105], %[b5], %[acc]\n\tv_mfma_i32_32x32x32_i8 %[acc], v[106:109], %[b6], %[acc]\n\tv_mfma_i32_32x32x32_i8 %[acc], v[110:113], %[b7], %[acc]\n\ts_cbranch_execz 22f\n" \
    "12:\n\tv_mov_b32 v141, 0x04020100\n\tv_mov_b32 v142, 0x05020100\n\ts_waitcnt vmcnt(5)\n" \
    "1:\n\tv_bitop3_b32 v138, v115, v117, v140 bitop3:0x7e\n\tv_bitop3_b32 v138, v138, v119, v140 bitop3:0xf6\n\tv_bitop3_b32 v138, v138, v121, v140 bitop3:0xf6\n\tv_bitop3_b32 v138, v138, v123, v140 bitop3:0xf6\n\tv_bitop3_b32 v138, v138, v125, v140 bitop3:0xf6\n\tv_bitop3_b32 v138, v138, v127, v140 bitop3:0xf6\n\tv_bitop3_b32 v138, v138, v129, v140 bitop3:0xf6\n\tv_cmp_lt_u32 vcc, 0xffffff, v138\n\ts_or_b32 vcc_lo, vcc_lo, vcc_hi\n\ts_or_b32 vcc_lo, vcc_lo, %[dry]\n\ts_cbranch_scc0 22f\n\ts_cmp_lg_u32 %[stop], 0\n\ts_cbranch_scc1 22f\n\t" \
    "buffer_load_dwordx4 v[114:117], v144, %[ar], %[nb] offen offset:2048 sc0 sc1\n\tbuffer_load_dwordx4 v[118:121], v145, %[ar], %[nb] offen offset:2048 sc0 sc1\n\tbuffer_load_dwordx4 v[122:125], %[vo], %[ar], %[nb] offen offset:2048 sc0 sc1\n\tbuffer_load_dwordx4 v[126:129], v143, %[ar], %[nb] offen offset:2048 sc0 sc1\n\t" \
    "v_mov_b32 v138, 0\n\tglobal_load_dword v139, v138, %[tmo] sc1\n\ts_memrealtime vcc\n\ts_waitcnt vmcnt(0) lgkmcnt(0)\n\tv_readfirstlane_b32 vcc_hi, v139\n\ts_sub_i32 vcc_lo, vcc_lo, %[t2]\n\ts_cmp_gt_u32 vcc_lo, " TG_STR(TG_DEADLINE) "\n\ts_cselect_b32 %[stop], 1, %[stop]\n\ts_cmp_eq_u32 vcc_hi, %[call]\n\ts_cselect_b32 %[stop], 1, %[stop]\n\ts_cbranch_execz 30f\n\t" TG_DIAG("s_add_u32 %[nr], %[nr], 65536\n\t") "s_branch 1b\n" \
    "22:\n\tv_perm_b32 v115, v131, v115, v141\n\tv_perm_b32 v117, v131, v117, v142\n\tv_perm_b32 v119, v133, v119, v141\n\tv_perm_b32 v121, v133, v121, v142\n\tv_perm_b32 v123, v135, v123, v141\n\tv_perm_b32 v125, v135, v125, v142\n\tv_perm_b32 v127, v137, v127, v141\n\tv_perm_b32 v129, v137, v129, v142\n\t" \
    "v_mfma_i32_32x32x32_i8 %[acc], v[114:117], %[b8], %[acc]\n\tv_mfma_i32_32x32x32_i8 %[acc], v[118:121], %[b9], %[acc]\n\tv_mfma_i32_32x32x32_i8 %[acc], v[122:125], %[b10], %[acc]\n\tv_mfma_i32_32x32x32_i8 %[acc], v[126:129], %[b11], %[acc]\n\ts_cbranch_execz 23f\n" \
    "13:\n\tv_mov_b32 v141, 0x06020100\n\tv_mov_b32 v142, 0x07020100\n\ts_waitcnt vmcnt(0)\n" \
    "1:\n\tv_bitop3_b32 v138, v83, v85, v140 bitop3:0x7e\n\tv_bitop3_b32 v138, v138, v87, v140 bitop3:0xf6\n\tv_bitop3_b32 v138, v138, v89, v140 bitop3:0xf6\n\tv_bitop3_b32 v138, v138, v91, v140 bitop3:0xf6\n\tv_bitop3_b32 v138, v138, v93, v140 bitop3:0xf6\n\tv_bitop3_b32 v138, v138, v95, v140 bitop3:0xf6\n\tv_bitop3_b32 v138, v138, v97, v140 bitop3:0xf6\n\tv_cmp_lt_u32 vcc, 0xffffff, v138\n\ts_or_b32 vcc_lo, vcc_lo, vcc_hi\n\ts_or_b32 vcc_lo, vcc_lo, %[dry]\n\ts_cbranch_scc0 23f\n\ts_cmp_lg_u32 %[stop], 0\n\ts_cbranch_scc1 23f\n\t" \
    "buffer_load_dwordx4 v[82:85], v145, %[ar], %[nb] offen offset:3072 sc0 sc1\n\tbuffer_load_dwordx4 v[86:89], v144, %[ar], %[nb] offen offset:3072 sc0 sc1\n\tbuffer_load_dwordx4 v[90:93], v143, %[ar], %[nb] offen offset:3072 sc0 sc1\n\tbuffer_load_dwordx4 v[94:97], %[vo], %[ar], %[nb] offen offset:3072 sc0 sc1\n\t" \
    "v_mov_b32 v138, 0\n\tglobal_load_dword v139, v138, %[tmo] sc1\n\ts_memrealtime vcc\n\ts_waitcnt vmcnt(0) lgkmcnt(0)\n\tv_readfirstlane_b32 vcc_hi, v139\n\ts_sub_i32 vcc_lo, vcc_lo, %[t2]\n\ts_cmp_gt_u32 vcc_lo, " TG_STR(TG_DEADLINE) "\n\ts_cselect_b32 %[stop], 1, %[stop]\n\ts_cmp_eq_u32 vcc_hi, %[call]\n\ts_cselect_b32 %[stop], 1, %[stop]\n\ts_cbranch_execz 30f\n\t" TG_DIAG("s_add_u32 %[nr], %[nr], 16777216\n\t") "s_branch 1b\n" \
    "23:\n\tv_perm_b32 v83, v131, v83, v141\n\tv_perm_b32 v85, v131, v85, v142\n\tv_perm_b32 v87, v133, v87, v141\n\tv_perm_b32 v89, v133, v89, v142\n\tv_perm_b32 v91, v135, v91, v141\n\tv_perm_b32 v93, v135, v93, v142\n\tv_perm_b32 v95, v137, v95, v141\n\tv_perm_b32 v97, v137, v97, v142\n\t" \
    "v_mfma_i32_32x32x32_i8 %[acc], v[82:85], %[b12], %[acc]\n\tv_mfma_i32_32x32x32_i8 %[acc], v[86:89], %[b13], %[acc]\n\tv_mfma_i32_32x32x32_i8 %[acc], v[90:93], %[b14], %[acc]\n\tv_mfma_i32_32x32x32_i8 %[acc], v[94:97], %[b15], %[acc]\n" \
    "30:\n\tv_readfirstlane_b32 vcc_hi, v139\n\ts_cmp_eq_u32 vcc_hi, %[call]\n\ts_cselect_b32 %[stop], 1, %[stop]\n\ts_cbranch_execnz 1f\n\ts_mov_b64 exec, -1\n\t" \
    "v_mov_b64 v[98:99], 0\n\tv_mov_b64 v[100:101], 0\n\ts_nop 15\n\tv_mfma_i32_32x32x32_i8 %[acc], v[98:101], %[b0], 0\n1:\n\ts_nop 15\n\ts_nop 1"
#define TG_AF() asm volatile("" : "={v[82:85]}"(af[0]), "={v[86:89]}"(af[1]), "={v[90:93]}"(af[2]), "={v[94:97]}"(af[3]), "={v[98:101]}"(af[4]), "={v[102:105]}"(af[5]), "={v[106:109]}"(af[6]), "={v[110:113]}"(af[7]), "={v[114:117]}"(af[8]), "={v[118:121]}"(af[9]), "={v[122:125]}"(af[10]), "={v[126:129]}"(af[11]), "={v[130:133]}"(af[12]), "={v[134:137]}"(af[13]), "={v[138:141]}"(af[14]), "={v[142:145]}"(af[15]))
#define TG_CLOBBER "v82", "v83", "v84", "v85", "v86", "v87", "v88", "v89", "v90", "v91", "v92", "v93", "v94", "v95", "v96", "v97", "v98", "v99", "v100", "v101", "v102", "v103", "v104", "v105", "v106", "v107", "v108", "v109", "v110", "v111", "v112", "v113", "v114", "v115", "v116", "v117", "v118", "v119", "v120", "v121", "v122", "v123", "v124", "v125", "v126", "v127", "v128", "v129", "v130", "v131", "v132", "v133", "v134", "v135", "v136", "v137", "v138", "v139", "v140", "v141", "v142", "v143", "v144", "v145"

__global__ void __launch_bounds__(NTHREADS, 3) ag_gemm_kernel(Args a_) {
    const CONST_AS Args* ap = (const CONST_AS Args*)__builtin_amdgcn_kernarg_segment_ptr();
    unsigned tid = threadIdx.x, w64, call, stmp;
    int b = blockIdx.x;
    const signed char *xin, *win;
    char *myws, *peer;
    u64 t_first, t_top;
#define PRO_(...)
    asm volatile(
        TG_DIAG("s_memrealtime %[t0]\n\t") "v_readfirstlane_b32 %[w], v0\n\ts_load_dwordx16 s[36:51], %[ap], %[w]\n\t"
        "s_cmp_gt_u32 %[b], 31\n\ts_cbranch_scc1 3f\n\ts_sub_u32 %[t], %[w], 64\n\ts_cmpk_lt_u32 %[t], 192\n\ts_cbranch_scc0 2f\n\t"
        "v_lshlrev_b32 v1, 12, v0\n\tv_lshrrev_b32 v10, 1, v0\n\ts_mov_b32 %[t], 0x1f010\n\tv_bitop3_b32 v1, v1, %[t], v10 bitop3:0xc8\n\t"
        "v_lshl_add_u32 v1, %[b], 7, v1\n\ts_waitcnt lgkmcnt(0)\n\tglobal_load_dwordx4 v[2:5], v1, s[36:37]\n\t" PRO_("s_memrealtime %[t2]\n\t")
        "s_lshl_b32 %[t], %[b], 6\n\ts_load_dword %[n], s[40:41], %[t]\n\ts_barrier\n\tglobal_load_dwordx4 v[6:9], v1, s[36:37] offset:32\n\t"
        "v_lshlrev_b32 v10, 4, v0\n\tv_and_b32 v10, 0x3f0, v10\n\tv_lshl_or_b32 v10, %[b], 12, v10\n\ts_movk_i32 m0, 2048-64\n\ts_sleep %[s0]\n\t"
        "global_load_lds_dwordx4 v1, s[36:37] offset:64\n\ts_movk_i32 m0, 3072-96\n\tv_xor_b32 v11, 0x1000, v10\n\tglobal_load_lds_dwordx4 v1, s[36:37] offset:96\n\t"
        "s_waitcnt lgkmcnt(0)\n\t" PRO_("s_memrealtime %[t3]\n\t") "s_bitcmp0_b32 %[n], 0\n\ts_cselect_b64 s[38:39], s[50:51], s[38:39]\n\t"
        "s_waitcnt vmcnt(3)\n\tglobal_store_dwordx4 v10, v[2:5], s[38:39] sc0 sc1\n\ts_memrealtime %[t1]\n\ts_sleep %[s1]\n\ts_waitcnt vmcnt(3)\n\t"
        "global_store_dwordx4 v11, v[6:9], s[38:39] offset:1024 sc0 sc1\n\t" PRO_("s_memrealtime %[t4]\n\t") "s_branch 4f\n\t.p2align 6\n"
        "2:\n\ts_cmpk_gt_u32 %[t], 447\n\ts_cbranch_scc1 3f\n\t"
        "v_lshlrev_b32 v1, 8, v0\n\tv_lshlrev_b32 v10, 4, v0\n\tv_and_b32 v1, 0x3e00, v1\n\tv_and_or_b32 v1, v10, 16, v1\n\t"
        "v_lshl_add_u32 v1, %[b], 16, v1\n\ts_lshl_b32 m0, %[w], 8\n\ts_waitcnt lgkmcnt(0)\n\tglobal_load_lds_dwordx4 v1, s[36:37]\n\t"
        "s_lshl_b32 %[t], %[b], 6\n\ts_barrier\n\t" PRO_("s_memrealtime %[t2]\n\t") "s_load_dword %[n], s[40:41], %[t]\n\t"
        "s_addk_i32 m0, 0x400\n\tv_add_u32 v13, 0x4000, v1\n\tglobal_load_lds_dwordx4 v13, s[36:37]\n\t"
        "v_and_b32 v12, 0x2e0, v10\n\tv_and_or_b32 v12, v0, 16, v12\n\tv_lshlrev_b32 v2, 3, v0\n\tv_and_or_b32 v12, v2, 8, v12\n\tv_lshl_add_u32 v12, %[w], 8, v12\n\t"
        "s_addk_i32 m0, 0x400\n\tv_add_u32 v14, 0x8000, v1\n\tglobal_load_lds_dwordx4 v14, s[36:37]\n\t"
        "s_addk_i32 m0, 0x400\n\tv_add_u32 v1, 0xc000, v1\n\tglobal_load_lds_dwordx4 v1, s[36:37]\n\t"
        "v_and_b32 v10, 0x3f0, v10\n\tv_lshl_or_b32 v10, %[b], 12, v10\n\tv_xor_b32 v11, 0x1000, v10\n\t"
        "s_waitcnt lgkmcnt(0)\n\t" PRO_("s_memrealtime %[t3]\n\t") "s_bitcmp0_b32 %[n], 0\n\ts_cselect_b64 s[38:39], s[50:51], s[38:39]\n\t"
        "s_waitcnt vmcnt(3)\n\tds_read_b64_tr_b8 v[2:3], v12\n\tds_read_b64_tr_b8 v[4:5], v12 offset:256\n\ts_waitcnt lgkmcnt(0)\n\t"
        "global_store_dwordx4 v10, v[2:5], s[38:39] sc0 sc1\n\ts_memrealtime %[t1]\n\ts_sleep %[s2]\n\ts_waitcnt vmcnt(3)\n\t"
        "ds_read_b64_tr_b8 v[6:7], v12 offset:1024\n\tds_read_b64_tr_b8 v[8:9], v12 offset:1280\n\ts_waitcnt lgkmcnt(0)\n\t"
        "global_store_dwordx4 v11, v[6:9], s[38:39] offset:1024 sc0 sc1\n\t" PRO_("s_memrealtime %[t4]\n\t") "s_branch 4f\n\t.p2align 6\n"
        "3:\n\tv_lshlrev_b32 v1, 2, v0\n\tv_and_b32 v1, 0xfc, v1\n\tv_add_u32 v1, %[ctr], v1\n\tv_mov_b32 v2, 0\n\tds_write_b32 v1, v2\n\t"
        "s_lshl_b32 %[t], %[b], 6\n\ts_waitcnt lgkmcnt(0)\n\ts_barrier\n\ts_load_dword %[n], s[40:41], %[t]\n\ts_memrealtime %[t1]\n"
        "4:\n\ts_waitcnt lgkmcnt(0)"
        : [w] "=&s"(w64), [n] "=&s"(call), [t] "=&s"(stmp), [t1] "=&s"(t_first), TG_DIAG([t0] "=&s"(t_top),) [b] "+s"(b), [ap] "+s"(ap),
          PRO_([t2] "=&s"(t_ka), [t3] "=&s"(t_cnt), [t4] "=&s"(t_ack),)
          "=&{s[42:43]}"(peer), "=&{s[44:45]}"(xin), "=&{s[46:47]}"(win), "=&{s[48:49]}"(myws)
        : [ctr] "i"(LDS_CTR), [s0] "i"(SLEEP0), [s1] "i"(SLEEP1), [s2] "i"(SLEEP2), "{v0}"(tid)
        : "memory", "scc", "v1", "v2", "v3", "v4", "v5", "v6", "v7", "v8", "v9", "v10", "v11", "v12", "v13", "v14", "s36", "s37", "s38", "s39", "s40", "s41", "s50", "s51");
    asm volatile("" : "+{v0}"(tid));
    call += 1u;
    const unsigned par = call & 1u;
    const unsigned tstart = (unsigned)t_first;
    static_assert(sizeof(Rec) == 64 && (SWZ & 1) && KDIM == 4096 && NSH == 512, "prologue");
    const CONST_AS Args& a = *ap;
    LDS_AS unsigned char* smem = (LDS_AS unsigned char*)0;
    int lane = tid & 63;
    asm volatile("" : "+v"(lane));
    __builtin_assume(lane >= 0 && lane < 64);
    const int wave = w64 >> 6;
    LDS_AS unsigned* ctr = (LDS_AS unsigned*)(smem + LDS_CTR);
    int j;
    const int role = role_of(b, j);
    if (role) peer = a.ws[4 + ((j >> 2) & 3)];
    STAMP(7);

    GLB_AS unsigned* cnt = (GLB_AS unsigned*)(myws + CNT_OFF) + b * 16 + wave;
    const unsigned voff = lane * 16;
    unsigned xsrc_off, lo[4];
    auto image = [&]() {
        unsigned l = lane;
        asm volatile("" : "+v"(l));
        xsrc_off = (l >> 3) * KDIM + (((l & 7) ^ (l >> 3)) * 16);
#pragma unroll
        for (int f = 0; f < 4; ++f) lo[f] = ((l & 31) * 128 + (((l >> 5) ^ (l & 7)) * 16)) ^ (f * 32);
        asm volatile("" : "+v"(xsrc_off), "+v"(lo[0]), "+v"(lo[1]), "+v"(lo[2]), "+v"(lo[3]));
    };

    if (__builtin_expect(role == 0, 1)) {
        const int c = j;
        image();
        const int d = wave < 8 ? wave : wave - 4;
        const unsigned lds0 = (unsigned)(size_t)smem;
        auto now16 = [&]() { return (unsigned)__builtin_amdgcn_s_memrealtime() * 16u; };
        if (wave == 0) {
            const signed char* xs = xin + c * 128;
            unsigned tl = 0;
            if (LDDELAY) __builtin_amdgcn_s_sleep(LDDELAY);
            for (int n = 0; n < 15; ++n) {
                const int blk = (int)((ld_order() >> (4 * n)) & 15);
                if (n >= 2) { while ((int)(now16() - tl) < 0) __builtin_amdgcn_s_sleep(2); }
                tl = now16() + LDPACE;
#pragma unroll
                for (int i = 0; i < 4; ++i) LOAD_LDS(lds0 + blk * 4096 + i * 1024, xsrc_off, xs + (size_t)(blk * 32 + i * 8) * KDIM);
                if (n == 0) continue;
                VMCNT(0);
                asm volatile("ds_write_b32 %0, %1" :: "v"(ctr), "v"(n + 1) : "memory");
            }
            LGKMCNT0();
            STAMP(2);
            __builtin_amdgcn_endpgm();
        }
        u32x4 v[4];
        const unsigned wlds = lds0 + 65536 + (d & 3) * 16384;
        const signed char* wsrc = win + (size_t)c * 128 * NSH + (d & 3) * 32;
        const unsigned wo = (lane >> 1) * NSH + (lane & 1) * 16;
        const int ntile = NOCOMP ? 0 : (NSTAGE - c + 31) >> 5;
        if (wave >= 8) {
            GLB_AS unsigned* tmo = (GLB_AS unsigned*)(myws + TMO_OFF);
            const char* lsrc = myws + LS_OFF + (size_t)((d & 3) * NIDX + c) * LSLOT;
            const char* lflg = myws + FL_OFF + (size_t)((d & 3) * NIDX + c) * 128;
            if (ALIGN) {
                const int dl = lane < 32 || d == 4 ? d : d - 4;
                char* st = myws + DUMMY_OFF + 2048 + dl * 256;
                unsigned t1;
                while (__builtin_amdgcn_ballot_w64((t1 = __hip_atomic_load(ctr + 24 + dl, __ATOMIC_RELAXED, __HIP_MEMORY_SCOPE_WORKGROUP)) == 0)) __builtin_amdgcn_s_sleep(4);
                if ((lane & 31) == 0) __hip_atomic_store((GLB_AS u64*)(st + c * 8), (u64)t1 << 32 | call, __ATOMIC_RELAXED, __HIP_MEMORY_SCOPE_AGENT);
                for (int it = 0; it < 8; ++it) {
                    __builtin_amdgcn_s_sleep(ASLEEP);
                    const u64 f = __hip_atomic_load((GLB_AS u64*)(st + (lane & 31) * 8), __ATOMIC_RELAXED, __HIP_MEMORY_SCOPE_AGENT);
                    if (__builtin_amdgcn_ballot_w64((unsigned)f == call) != ~0ull) continue;
                    const unsigned ref = __builtin_amdgcn_readfirstlane((unsigned)(f >> 32));
                    const unsigned sum = wave_scan((unsigned)(f >> 32) - ref);
                    const unsigned s0 = __builtin_amdgcn_readlane(sum, 31), s1 = __builtin_amdgcn_readlane(sum, 63) - s0;
                    const int ph = STAG * (((c + 5 * dl) & 31) - 16) * PACE / 32;
                    if ((lane & 31) == 0) ctr[16 + dl] = (ref + ((int)(lane ? s1 : s0) >> 5) - TBIAS + ph) | 1u;
                    break;
                }
            }
            while (lds_get(ctr + d + 4) == 0) __builtin_amdgcn_s_sleep(16);
            CBAR();
            if (lane == 0) __hip_atomic_store(cnt, call, __ATOMIC_RELAXED, __HIP_MEMORY_SCOPE_AGENT);
            const unsigned t0 = (unsigned)__builtin_amdgcn_s_memrealtime();
            unsigned pend = (1u << ntile) - 1u, fly = 0, qn = 0, finfo = 0;
            for (unsigned it = 1; pend | fly; ++it) {
                u32x2 f = {0, 0};
                asm volatile("s_mov_b64 exec, %3\n\tglobal_load_dwordx2 %0, %1, %2 sc0 sc1\n\ts_mov_b64 exec, -1\n\ts_waitcnt vmcnt(0)"
                             : "+v"(f) : "v"(lane * 4096), "s"(lflg), "s"((u64)pend) : "memory");
                if (fly) {
                    if ((fly >> (lane & 31)) & (lane < 8)) ctr[32 - 32 + d * 8 + qn + __builtin_amdgcn_mbcnt_lo(fly, 0)] = finfo | lane << 26;
                    qn += __builtin_popcount(fly);
                    LGKMCNT0();
                    if (lane == 0) __hip_atomic_store(ctr + d, qn, __ATOMIC_RELAXED, __HIP_MEMORY_SCOPE_WORKGROUP);
                }
                fly = (unsigned)__builtin_amdgcn_ballot_w64(f[0] >= call) & pend;
                finfo = f[1];
                if (!fly) {
                    if (it % TMOEVERY == 0 &&
                        ((unsigned)__builtin_amdgcn_readfirstlane(tmo_get(tmo)) == call || (unsigned)__builtin_amdgcn_s_memrealtime() - t0 > DEADLINE_TICKS)) {
                        if (lane == 0) __hip_atomic_store(tmo, call, __ATOMIC_RELAXED, __HIP_MEMORY_SCOPE_AGENT);
                        fly = pend; pend = 0; finfo = F_SKIP;
                        continue;
                    }
                    __builtin_amdgcn_s_sleep(LSLEEP);
                    continue;
                }
                pend &= ~fly;
                for (unsigned r = fly; r; r &= r - 1) {
                    const int n = __builtin_ctz(r);
                    TRACE(n, 0);
                    LOAD_LDS_SYS(wlds + n * LSLOT, voff, lsrc + (size_t)n * 32 * LSLOT);
                    LOAD_LDS_SYS(wlds + n * LSLOT + 1024, voff, lsrc + (size_t)n * 32 * LSLOT + 1024);
                }
            }
            LGKMCNT0();
            TRACE_OUT();
            __builtin_amdgcn_endpgm();
        }
        {
            char* prev_flag = nullptr;
            const char* xdst = peer + par * XS_PAR + (size_t)d * NBLK * BLK_BYTES + (SWZ ? 0 : c * 4096);
            char* fdst = peer + FX_OFF + (size_t)(d * NBLK * 32 + c) * FSTRIDE;
            unsigned tfirst = tstart * 16u + 1u, tnext = tfirst + 2 * PACE, tideal = tnext;
            if (ALIGN) ctr[24 + d] = tfirst;
            auto pace = [&](unsigned dt) {
                int late;
                while ((late = (int)(now16() - tnext)) < 0) __builtin_amdgcn_s_sleep(1);
                tnext += dt;
                if (PSLACK && late > PSLACK) tnext += late - PSLACK;
                if (ALIGN) {
                    tideal += dt;
                    tnext += max(-AMAX, min(AMAX, (int)(tideal - tnext) >> ALIGN));
                }
            };
            auto align = [&]() {
                if (ALIGN && tfirst) {
                    const unsigned t = lds_get(ctr + 16 + d);
                    if (t) { tideal += t - tfirst; tfirst = 0; }
                }
            };
            unsigned tmask = 0, tagv = call << 24;
            asm("" : "+v"(tagv));
            auto send = [&](int blk, unsigned dt, auto nx, auto&& next) {
                const char* dst = xdst + (size_t)blk * BLK_BYTES;
                constexpr int nl = nx.value > 1;
#pragma unroll
                for (int f = 0; f < 4; ++f) {
                    if (NTAG && nx.value < 0) {
                        v[f][1] = (v[f][1] & ~tmask) | (tagv & tmask);
                        v[f][3] = (v[f][3] & ~tmask) | (tagv & tmask);
                        asm volatile("" : "+v"(v[f]));
                    }
                    const bool sent = f < 2 && nx.value != 1 && blk == (nx.value < 0 ? 0 : 16);
                    if (!sent) {
                    pace(dt);
                    STORE_WT(voff, dst + (SWZ ? UPAGE(c, f, blk) : 0), v[f], f * 1024);
                    if (f == 0) { VMCNT(1); set_flag(prev_flag, call, lane); }
                    }
                    if ((WALL >> f) & 1) VMCNT(1 + nl + (f == 1));
                    if (f == 3) align();
                    if (BURST1 && f == 0 && (blk == 0 || blk == 16)) tnext -= BURST1;
                    next(f);
                }
                prev_flag = fdst + blk * 32 * FSTRIDE;
            };
            auto xwait = [&](int blk) { lds_wait(ctr, (unsigned)(ld_pos() >> (4 * blk)) & 15u); };
            int blk, nblk, step;
            VMCNT(DIAG_N + 2);
            if (d >= 4) {
                const unsigned wl = wlds + (16 * (lane >> 5) + ((lane & 15) >> 1)) * 32 + 16 * ((lane >> 4) & 1) + 8 * (lane & 1);
                auto wfrag = [&](u32x4& o, int off) {
                    u32x2 l2, h2;
                    asm volatile("ds_read_b64_tr_b8 %0, %2\n\tds_read_b64_tr_b8 %1, %2 offset:256\n\ts_waitcnt lgkmcnt(0)"
                                 : "=&v"(l2), "=&v"(h2) : "v"(wl + off) : "memory");
                    o = u32x4{l2[0], l2[1], h2[0], h2[1]};
                };
#pragma clang loop unroll(disable)
                for (int t = 0; t < 3; ++t)
                    send(16 + t, PACE, IC<2>{}, [&](int f) {
                        LOAD_LDS(wlds + (t + 1) * 4096 + f * 1024, wo, wsrc + f * 32 * NSH + (t + 1) * 128);
                        if (f == 3) VMCNT(6);
                        wfrag(v[(f + 1) & 3], t * 4096 + (f + 1) * 1024);
                    });
                send(19, PACE, IC<1>{}, [&](int f) {
                    if (f < 3) wfrag(v[f + 1], 3 * 4096 + (f + 1) * 1024);
                });
                if (lane == 0) __hip_atomic_store(ctr + d + 4, 1u, __ATOMIC_RELAXED, __HIP_MEMORY_SCOPE_WORKGROUP);
                STAMP(3);
                blk = 15; nblk = 16 - N1; step = -1;
                xwait(blk);
#pragma unroll
                for (int f = 0; f < 4; ++f) v[f] = *(LDS_AS u32x4*)(smem + blk * 4096 + lo[f]);
            } else {
                blk = 0; nblk = N1 + N3; step = 1;
#pragma unroll
                for (int f = 0; f < 4; ++f) v[f] = *(LDS_AS u32x4*)(smem + f * 1024 + voff);
            }
            const unsigned dt = PACE;
            static_assert(WALL & 4, "the flag of the displaced bytes follows this wait");
            char* edst = nullptr;
            u32x4 ev;
            for (int n = 0; n < nblk; ++n) {
                const int nb = n + 1 < nblk ? blk + step : blk;
                send(blk, dt, IC<-1>{}, [&](int f) {
                    if (f == 3) {
                        xwait(nb);
#pragma unroll
                        for (int g = 0; g < 4; ++g) v[g] = *(LDS_AS u32x4*)(smem + nb * 4096 + lo[g]);
                    }
                    if (EMID && NTAG && f == 0 && edst) { pace(d < 4 ? dt : 0); STORE_WT(voff, edst, ev, 0); }
                    if (EMID && NTAG && f == 2 && edst) { set_flag(d < 4 ? fdst + EBLK * 32 * FSTRIDE : edst, call, lane); edst = nullptr; }
                });
                blk = nb;
                if (NTAG && (d < 4 ? blk == NTB : d == 7 && n == 0)) {
                    if (d < 4) { xwait(NTB + 1); tmask = 0xFF000000u; }
#pragma unroll
                    for (int h = 0; h < 4; ++h) {
                        u32x4 p = v[h & 1 ? 2 : 0], q = v[h & 1 ? 3 : 1];
                        if (h >= 2) {
                            p = *(LDS_AS u32x4*)(smem + (NTB + 1) * 4096 + lo[h & 1 ? 2 : 0]);
                            q = *(LDS_AS u32x4*)(smem + (NTB + 1) * 4096 + lo[h & 1 ? 3 : 1]);
                        }
                        ev[h] = __builtin_amdgcn_perm(__builtin_amdgcn_perm(q[3], q[1], 0x0703u), __builtin_amdgcn_perm(p[3], p[1], 0x0703u), 0x05040100u);
                    }
                    edst = d < 4 ? (char*)xdst + (size_t)EBLK * BLK_BYTES + c * 1024 : myws + DUMMY_OFF + 4096;
                    if (!EMID) {
                        pace(d < 4 ? dt : 0);
                        STORE_WT(voff, edst, ev, 0);
                    }
                }
            }
            unsigned pinfo = 0;
            if (d >= 4) {
                STAMP(2);
                const char* rdst = peer + OS_OFF + par * OS_PAR + (size_t)((d & 3) * NIDX + c) * OSLOT;
                char* rflg = peer + FO_OFF + (size_t)((d & 3) * NIDX + c) * 128;
                char* const dummy = myws + DUMMY_OFF + (c * 8 + wave) * 8;
                for (int q = 0; q < ntile; ++q) {
                    const bool none = lds_get(ctr + d) <= (unsigned)q;
                    if (none) {
                        VMCNT(0);
                        flag64_set(prev_flag, call, pinfo, lane);
                        prev_flag = dummy;
                        lds_wait(ctr + d, q + 1);
                    }
                    CBAR();
                    const unsigned info = lds_get(ctr + d * 8 + q);
                    if (info & F_SKIP) continue;
                    const unsigned n = (info >> 26) & 7, nbytes = info & 0xFFFF;
                    v[0] = *(LDS_AS u32x4*)(size_t)(wlds + n * LSLOT + voff);
                    v[1] = *(LDS_AS u32x4*)(size_t)(wlds + n * LSLOT + 1024 + voff);
                    const char* dst = rdst + (size_t)n * 32 * OSLOT;
                    tideal = tnext;
                    pace(PACET);
                    TRACE(n, 0);
                    asm volatile("v_cmpx_gt_u32 vcc, %3, %0\n\tglobal_store_dwordx4 %0, %2, %1 sc0 sc1\n\ts_mov_b64 exec, -1\n\ts_nop 0"
                                 :: "v"(voff), "s"(dst), "v"(v[0]), "s"(nbytes) : "memory", "vcc");
                    const unsigned rest = max((int)nbytes - 1024, 16);
                    pace((rest + 64) * PACET / 1024);
                    asm volatile("v_cmpx_gt_u32 vcc, %3, %0\n\tglobal_store_dwordx4 %0, %2, %1 offset:1024 sc0 sc1\n\ts_mov_b64 exec, -1\n\ts_nop 0"
                                 :: "v"(voff), "s"(dst), "v"(v[1]), "s"(rest) : "memory", "vcc");
                    VMCNT(2);
                    flag64_set(prev_flag, call, pinfo, lane);
                    TRACE(n, 1);
                    prev_flag = rflg + (size_t)n * 32 * 128;
                    pinfo = info;
                }
            }
            VMCNT(0);
            flag64_set(prev_flag, call, pinfo, lane);
            if (lane == 0) __hip_atomic_store(cnt, call, __ATOMIC_RELAXED, __HIP_MEMORY_SCOPE_AGENT);
            STAMP(d >= 4 ? 4 : 2);
            TRACE_OUT();
        }
        __builtin_amdgcn_endpgm();
    }

    const float scale = __builtin_bit_cast(float, __builtin_amdgcn_readfirstlane(__builtin_bit_cast(int, *(const CONST_AS float*)a.sx * *(const CONST_AS float*)a.sw)));
    GLB_AS unsigned* tmo = (GLB_AS unsigned*)(myws + TMO_OFF);
    const float fnan = __builtin_nanf("");
    const int rank = a.rank;

    const int grp = j >> 4, tile = j & 15;
    const int foreign = role == 2 ? !(grp & 1) : 2, slot = grp >> 1;
    const int downer = 4 + (tile >> 2);
    if (NOCOMP) {
        if (lane == 0) __hip_atomic_store(cnt, call, __ATOMIC_RELAXED, __HIP_MEMORY_SCOPE_AGENT);
        if (wave < 8 && tile == 0 && foreign == 1) {
            const int d = slot + 1;
            const int blk = d < 4 ? N1 + N3 - 1 : N1;
            GLB_AS unsigned* p = (GLB_AS unsigned*)(myws + FX_OFF + (size_t)((d * NBLK + blk) * 32 + 4 * wave + (lane & 3)) * FSTRIDE);
            wait_flags(p, 0, tmo, call, lane);
            STAMP(3);
        }
        __builtin_amdgcn_endpgm();
    }

    const int su = SUEVEN ? wave * 16 * (foreign ? NSLOT : NOWN) + slot * 16 + tile : (slot * 16 + tile) * 8 + wave;
    image();
    const bool selfcopy = foreign < 2 && wave < 8 && su < 512;
    if (DELAY0 && !foreign) __builtin_amdgcn_s_sleep(DELAY0);
    if (DELAY1 && foreign) __builtin_amdgcn_s_sleep(DELAY1);
    if (DELAYW) for (int i = 0; i < wave && wave < 8; ++i) __builtin_amdgcn_s_sleep(DELAYW);
    if (selfcopy) {
        const signed char* src = foreign ? xin + (size_t)(su >> 5) * BLK_BYTES + (su & 31) * 128
                                         : win + (size_t)(su >> 4) * 128 * NSH + (su & 15) * 32;
        const unsigned so = foreign ? xsrc_off : (lane >> 1) * NSH + (lane & 1) * 16;
        static_assert(8 * KDIM == 2 * 32 * NSH, "row steps");
#pragma unroll
        for (int i = 0; i < 4; ++i) LOAD_LDS((wave * 16 + i) * 1024, so, src + (i * 32 * NSH << foreign));
    }
    const int tsel = foreign ? 1 + (tile & 3) : 0;
    u64 t_lo = a.tab[tsel][slot][0], t_hi = a.tab[tsel][slot][1];
    u64 o_lo = a.ord[tile & 3][slot][0], o_hi = a.ord[tile & 3][slot][1];
    if (lane == 0) __hip_atomic_store(cnt, call, __ATOMIC_RELAXED, __HIP_MEMORY_SCOPE_AGENT);
    auto src_d = [&](int e) { const int code = (e >> 4) & 7; return foreign && code >= 4 ? downer ^ (code & 3) : code; };
    auto blk_of = [&](int e) { return src_d(e) * NBLK + (e & 15); };
    auto tagged = [&](int e) { return NTAG && ((e >> 4) & 7) - 1u < 3u && (e & 15) >= NTB; };

    auto epilogue = [&](int i, int e) {
        const float bad = lds_get(ctr + 17) != 0 ? fnan : 0.0f;
        LDS_AS unsigned char* tp = smem + i * TILE_STRIDE + 128;
        const int srank = rank ^ src_d(e);
        const float* op = (foreign ? a.outp[tile >> 2] : a.out) + (size_t)(srank * MSH + (e & 15) * 32) * NSH + (foreign ? (tile & 3) * 4 + (tile >> 2) : tile) * 32;
        unsigned vo = voff;
        asm volatile("" : "+v"(vo));
        i32x4 q[4];
#pragma unroll
        for (int k = 0; k < 4; ++k) q[k] = *(LDS_AS i32x4*)(tp + k * 1024 + vo);
        vo = (vo >> 7) * (NSH * 4) + (vo & 0x70);
#pragma unroll
        for (int k = 0; k < 4; ++k) {
            f32x4 o;
#pragma unroll
            for (int r = 0; r < 4; ++r) o[r] = fmaxf((float)q[k][r] * scale, 0.0f) + bad;
            STORE_WT(vo, op + (size_t)k * 8 * NSH, o, 0);
        }
    };

    if (role == 1 || wave >= 8) {
    int link, k0, kend, kstep;
    LDS_AS unsigned char* tp;
    if (role == 2) {
    {
        const int i0 = wave - 8;
        const unsigned npacked = a.npacked;
        for (int i = i0; i < NTASK; i += 4) {
            const int sh = 8 * (i & 7);
            const int e = (int)(((i & 8 ? t_hi : t_lo) >> sh) & 0xFF);
            if (e == 0xFF) break;
            unsigned ticket = 0;
            if (foreign && !(e & 0x80)) {
                lds_wait(ctr + 32 + i, 8u);
                if (lane == 0) ticket = __hip_atomic_fetch_add((GLB_AS unsigned*)(myws + TK_OFF + (tile >> 2) * 128), 1u, __ATOMIC_RELAXED, __HIP_MEMORY_SCOPE_AGENT);
            }
            lds_wait(ctr + i, 8u);
            if (!foreign || (e & 0x80)) {
                TRACE(i, 0);
                epilogue(i, e);
                if (foreign) {
                    const int oi = (int)(((i & 8 ? o_hi : o_lo) >> sh) & 0xFF);
                    VMCNT(0);
                    flag64_set(peer + FO_OFF + (size_t)((tile >> 2) * NIDX + oi) * 128, call, 1u, lane);
                }
                TRACE(i, 1);
                continue;
            }
            if (PTR) TRACE(i, 0);
            const bool failed = lds_get(ctr + 17) != 0;
            LDS_AS unsigned char* tp = smem + i * TILE_STRIDE;
            i32x4 q[4];
#pragma unroll
            for (int k = 0; k < 4; ++k) q[k] = *(LDS_AS i32x4*)(tp + 128 + k * 1024 + voff);
            __builtin_amdgcn_s_waitcnt(0xC07F);
            CBAR();
            if (scale < 0.0f) {
#pragma unroll
                for (int k = 0; k < 4; ++k) q[k] = -q[k];
            }
            unsigned m = 0;
            int mx = 0;
#pragma unroll
            for (int r = 15; r >= 0; --r) {
                m = __builtin_amdgcn_alignbit(m, (unsigned)-q[r >> 2][r & 3], 31);
                mx = max(mx, q[r >> 2][r & 3]);
            }
            if (P21) *(LDS_AS u32x4*)(tp + 128 + voff) = u32x4{0, 0, 0, 0};
            const bool wide = __builtin_amdgcn_ballot_w64(mx >= (1 << (P21 ? 21 : 24))) != 0;
            const unsigned c1 = __builtin_popcount(m), inc = wave_scan(c1);
            const unsigned tot = __builtin_amdgcn_readlane(inc, 63);
            const unsigned hbytes = (5 * tot + 31) >> 5 << 2;
            const unsigned nbytes = (128 + (wide ? tot * 4 : P21 ? hbytes + 2 * tot : 3 * tot) + 63) & ~63u;
            *(LDS_AS unsigned short*)(tp + lane * 2) = (unsigned short)m;
            if (PTR == 2) TRACE(i, 1);
            if (wide) {
                unsigned ep = (unsigned)(size_t)tp + 128 + (inc - c1) * 4;
#pragma unroll
                for (int r = 0; r < 16; ++r)
                    asm volatile("v_cmpx_lt_i32 vcc, 0, %1\n\tds_write_b32 %0, %1\n\tv_add_u32 %0, 4, %0\n\ts_mov_b64 exec, -1"
                                 : "+v"(ep) : "v"(q[r >> 2][r & 3]) : "memory", "vcc");
            } else if (P21) {
                const unsigned tb = (unsigned)(size_t)tp + 128, ad = tb + 3072;
                unsigned a2 = tb + hbytes + 2 * (inc - c1), g = 5 * (inc - c1);
#pragma unroll
                for (int r0 = 0; r0 < 16; r0 += 6) {
                    unsigned acc = 0, sh5 = 0;
#pragma unroll
                    for (int r = r0; r < r0 + 6 && r < 16; ++r) {
                        int nb;
                        BITMASK(nb, m, r);
                        asm volatile("ds_write_b16 %0, %1" :: "v"((a2 & nb) | (ad & ~nb)), "v"(q[r >> 2][r & 3]) : "memory");
                        acc |= (((unsigned)q[r >> 2][r & 3] >> 16) & nb) << sh5;
                        STEP(a2, nb, 2);
                        STEP(sh5, nb, 5);
                    }
                    const u64 w = (u64)acc << (g & 31);
                    asm volatile("ds_or_b32 %0, %1\n\tds_or_b32 %0, %2 offset:4" :: "v"(tb + (g >> 5 << 2)), "v"((unsigned)w), "v"((unsigned)(w >> 32)) : "memory");
                    g += sh5;
                }
            } else {
                const unsigned tb = (unsigned)(size_t)tp + 128, c2 = -tb - 4 * tot, ad = tb + 2 * tot + 1536;
                unsigned a1 = tb + 2 * tot + inc - c1;
#pragma unroll
                for (int r = 0; r < 16; ++r) {
                    const unsigned nb = __builtin_amdgcn_sbfe(m, r, 1);
                    const unsigned ae = (a1 & nb) | (ad & ~nb);
                    a1 -= nb;
                    asm volatile("ds_write_b16 %0, %2\n\tds_write_b8_d16_hi %1, %2" :: "v"(ae * 2 + c2), "v"(ae), "v"(q[r >> 2][r & 3]) : "memory");
                }
            }
            const unsigned info = nbytes | (unsigned)((tile & 3) << 8 | e) << 16 | (wide ? F_WIDE : 0u) | (failed ? F_NAN : 0u);
            const unsigned oi = min((unsigned)__builtin_amdgcn_readfirstlane(ticket) - (call - 1u) * npacked, npacked - 1u);
            const int idx = (tile >> 2) * NIDX + oi;
            const bool staged = oi < NSTAGE && nbytes <= LSLOT && (!TCUT || (unsigned)__builtin_amdgcn_s_memrealtime() - tstart < TCUT);
            if (oi < NSTAGE && !staged) flag64_set(myws + FL_OFF + (size_t)idx * 128, call, F_SKIP, lane);
            const char* dst = (staged ? myws + LS_OFF + (size_t)idx * LSLOT : peer + OS_OFF + par * OS_PAR + (size_t)idx * OSLOT) + 2048;
            if (PTR != 2) TRACE(i, PTR);
#pragma unroll
            for (int k = 0; k < 5; ++k)
                if (k * 1024 < nbytes) {
                    const u32x4 pv = *(LDS_AS u32x4*)(tp + k * 1024 + voff);
                    if (k * 1024 + voff < nbytes) STORE_WT(voff, dst, pv, k * 1024 - 2048);
                }
            VMCNT(0);
            flag64_set((staged ? myws + FL_OFF : peer + FO_OFF) + (size_t)idx * 128, call, info, lane);
            if (!PTR) TRACE(i, 1);
        }
        STAMP(5);
        TRACE_OUT();
        if (!KSPLIT || !foreign || i0 == 3) __builtin_amdgcn_endpgm();
        const int id = i0 * (NSLOT * 16) + slot * 16 + tile;
        link = id & 3; k0 = KSPLIT + (id >> 2); kend = a.ntotal; kstep = NIDX;
        tp = smem + i0 * TILE_STRIDE;
    }
    } else {
        link = j & 3; k0 = (j >> 2) * 12 + wave; kstep = 48;
        kend = KSPLIT ? KSPLIT : a.ntotal;
        tp = smem + wave * TILE_STRIDE;
    }
    {
        const int dn = 4 + link;
        const int npacked = a.npacked;
        bool failed = false;
        u64 fpre = 0;
        for (int n = 0; k0 + kstep * n < kend; ++n) {
            const int idx = link * NIDX + k0 + kstep * n;
            const unsigned info = wait_flag64(myws + FO_OFF + (size_t)idx * 128, fpre, tmo, call, lane);
            CBAR();
            TRACE(n, 0);
            if (info == 0 || k0 + kstep * n >= npacked) {
                if (info == 0 || (unsigned)__builtin_amdgcn_readfirstlane(tmo_get(tmo)) == call) {
                    a.out[(size_t)rank * MSH * NSH + lane] = fnan;
                    break;
                }
                fpre = 0;
                TRACE(n, 1);
                continue;
            }
            const unsigned e = info >> 16;
            const int code = (e >> 4) & 7;
            const int srank = rank ^ (code < 4 ? dn ^ code : code & 3);
            float* op = a.out + (size_t)(srank * MSH + (e & 15) * 32 + (lane >> 3)) * NSH + (((e >> 8) & 3) * 4 + link) * 32 + (lane & 7) * 4;
            const char* srcp = myws + OS_OFF + par * OS_PAR + (size_t)idx * OSLOT;
            const unsigned nbytes = min(info & 0xFFFFu, 4224u);
#pragma unroll
            for (int k = 0; k < 5; ++k)
                if (k * 1024 + voff < nbytes) LOAD_LDS_SYS((unsigned)(size_t)tp + k * 1024, voff, srcp + k * 1024);
            unsigned tv = tmo_get(tmo);
            VMCNT(0);
            tv = __builtin_amdgcn_readfirstlane(tv);
            asm volatile("" : "+s"(tv) :: "memory");
            fpre = 0;
            if (k0 + kstep * (n + 1) < kend) fpre = flag64_get(myws + FO_OFF + (size_t)(idx + kstep) * 128);
            failed = failed || tv == call;
            const float mul = failed || (info & F_NAN) ? fnan : __builtin_fabsf(scale);
            const unsigned m = *(LDS_AS unsigned short*)(tp + lane * 2);
            const unsigned c1 = __builtin_popcount(m), inc = wave_scan(c1);
            unsigned y[16];
            if (info & F_WIDE) {
                unsigned ix = inc - c1;
#pragma unroll
                for (int r = 0; r < 16; ++r) {
                    const int nb = __builtin_amdgcn_sbfe(m, r, 1);
                    const unsigned w = *(LDS_AS unsigned*)(tp + 128 + ix * 4);
                    ix -= nb;
                    y[r] = __builtin_bit_cast(unsigned, (float)(w & (unsigned)nb) * mul);
                }
            } else if (P21) {
                const unsigned tot = __builtin_amdgcn_readlane(inc, 63);
                const unsigned tb = (unsigned)(size_t)tp + 128;
                unsigned a2 = tb + ((5 * tot + 31) >> 5 << 2) + 2 * (inc - c1), g[3], win[3];
                u32x2 hw[3];
                int nb[16];
#pragma unroll
                for (int k = 0; k < 3; ++k) {
                    g[k] = 5 * (inc - c1 + __builtin_popcount(m & ((1u << 6 * k) - 1u)));
                    const LDS_AS unsigned* hp = (const LDS_AS unsigned*)(size_t)(tb + (g[k] >> 5 << 2));
                    hw[k] = u32x2{hp[0], hp[1]};
                }
#pragma unroll
                for (int r = 0; r < 16; ++r) {
                    y[r] = *(LDS_AS unsigned short*)(size_t)a2;
                    BITMASK(nb[r], m, r);
                    STEP(a2, nb[r], 2);
                }
#pragma unroll
                for (int r = 0; r < 16; r += 8)
                    asm volatile("" : "+v"(y[r]), "+v"(y[r + 1]), "+v"(y[r + 2]), "+v"(y[r + 3]),
                                      "+v"(y[r + 4]), "+v"(y[r + 5]), "+v"(y[r + 6]), "+v"(y[r + 7]) :: "memory");
                asm volatile("" : "+v"(hw[0]), "+v"(hw[1]), "+v"(hw[2]));
#pragma unroll
                for (int k = 0; k < 3; ++k) win[k] = __builtin_amdgcn_alignbit(hw[k][1], hw[k][0], g[k]);
                unsigned sh5 = 0;
#pragma unroll
                for (int r = 0; r < 16; ++r) {
                    if (r % 6 == 0) sh5 = 0;
                    asm("v_lshrrev_b32_sdwa %0, %1, %2 dst_sel:WORD_1 dst_unused:UNUSED_PRESERVE src0_sel:DWORD src1_sel:DWORD" : "+v"(y[r]) : "v"(sh5), "v"(win[r / 6]));
                    y[r] = __builtin_bit_cast(unsigned, (float)(y[r] & (unsigned)nb[r] & 0x1FFFFFu) * mul);
                    STEP(sh5, nb[r], 5);
                }
            } else {
                const unsigned tot = __builtin_amdgcn_readlane(inc, 63);
                const unsigned tb = (unsigned)(size_t)tp + 128, c2 = -tb - 4 * tot;
                unsigned a1 = tb + 2 * tot + inc - c1, wh[16];
#pragma unroll
                for (int r = 0; r < 16; ++r) {
                    y[r] = *(LDS_AS unsigned short*)(size_t)(a1 * 2 + c2);
                    wh[r] = *(LDS_AS unsigned char*)(size_t)a1;
                    a1 -= __builtin_amdgcn_sbfe(m, r, 1);
                }
#pragma unroll
                for (int r = 0; r < 16; r += 4)
                    asm volatile("" : "+v"(y[r]), "+v"(y[r + 1]), "+v"(y[r + 2]), "+v"(y[r + 3]),
                                      "+v"(wh[r]), "+v"(wh[r + 1]), "+v"(wh[r + 2]), "+v"(wh[r + 3]) :: "memory");
#pragma unroll
                for (int r = 0; r < 16; ++r)
                    y[r] = __builtin_bit_cast(unsigned, (float)((wh[r] << 16 | y[r]) & (unsigned)__builtin_amdgcn_sbfe(m, r, 1)) * mul);
            }
            fpre = (u64)(unsigned)__builtin_amdgcn_readfirstlane((unsigned)(fpre >> 32)) << 32 | (unsigned)__builtin_amdgcn_readfirstlane((unsigned)fpre);
            asm volatile("" : "+s"(fpre));
#pragma unroll
            for (int k = 0; k < 4; ++k) {
                const u32x4 o = {y[4 * k], y[4 * k + 1], y[4 * k + 2], y[4 * k + 3]};
                asm volatile("global_store_dwordx4 %0, %1, off sc0 sc1\n\ts_nop 1" :: "v"(op + (size_t)k * 8 * NSH), "v"(o) : "memory");
            }
            TRACE(n, 1);
        }
        STAMP(role == 1 ? 5 : 4);
        TRACE_OUT();
        __builtin_amdgcn_endpgm();
    }
    }

    bool failed = false;
    const unsigned fxo = (lane & 3) * FSTRIDE;
    auto flag_ptr = [&](int eb) {
        char* p = myws + FX_OFF + (size_t)(eb * 32 + 4 * wave) * FSTRIDE;
        asm("" : "+s"(p));
        return (GLB_AS unsigned*)(p + fxo);
    };
    unsigned aoff = 128 + (4 * (lane >> 5) * 32 + (lane & 31)) * 4;
    asm("" : "+v"(aoff));
    auto ar = __builtin_amdgcn_make_buffer_rsrc((void*)(myws + par * XS_PAR), 0, (unsigned)XS_PAR, 0x00020000);
    VMCNT(0);
    i32x4 bf[16];
    if (selfcopy) {
        u32x4 v[4];
        if (foreign) {
#pragma unroll
            for (int f = 0; f < 4; ++f) v[f] = *(LDS_AS u32x4*)(smem + wave * 16384 + lo[f]);
        } else {
            LDS_AS unsigned char* bp = smem + wave * 16384 + (16 * (lane >> 5) + ((lane & 15) >> 1)) * 32 + 16 * ((lane >> 4) & 1) + 8 * (lane & 1);
#pragma unroll
            for (int f = 0; f < 4; ++f) {
                const i32x2 l2 = __builtin_amdgcn_ds_read_tr8_b64_v2i32((LDS_AS i32x2*)(bp + f * 1024));
                const i32x2 h2 = __builtin_amdgcn_ds_read_tr8_b64_v2i32((LDS_AS i32x2*)(bp + f * 1024 + 256));
                v[f] = __builtin_bit_cast(u32x4, i32x4{l2[0], l2[1], h2[0], h2[1]});
            }
        }
        const int uu = foreign ? su : (((su >> 2) & 3) * NBLK + 16 + (su & 3)) * 32 + (su >> 4);
        const char* dst = myws + par * XS_PAR;
#pragma unroll
        for (int f = 0; f < 4; ++f) STORE_WT(voff, dst + UPAGE(uu, f, uu >> 5), v[f], f * 1024);
        VMCNT(0);
        set_flag(myws + FX_OFF + (size_t)uu * FSTRIDE, call, lane);
        if (foreign) STAMP(2);
    }
    if (wave * 16384 < NTASK * TILE_STRIDE) {
#pragma unroll
        for (int i = 0; i < 16; ++i) *(LDS_AS u32x4*)(smem + wave * 16384 + i * 1024 + voff) = u32x4{0, 0, 0, 0};
    }
    LGKMCNT0();
    if (lane == 0) __hip_atomic_fetch_add(ctr + 16, 1u, __ATOMIC_RELAXED, __HIP_MEMORY_SCOPE_WORKGROUP);
    u32x4 af[16];
    unsigned fpre;
    auto aload = [&](int k, unsigned abase) { af[k] = __builtin_amdgcn_raw_buffer_load_b128(ar, voff, ((abase | UPAGE(0, 0, abase >> 17)) ^ UPAGE(k & 3, k >> 2, 0)) + (k >> 2) * 1024, 17); };
    auto peek = [&](int e) {
        const unsigned o = lane & 4 ? (unsigned)(TMO_OFF - FX_OFF) : (blk_of(e) * 32 + 4 * wave) * FSTRIDE + fxo;
        fpre = __hip_atomic_load((GLB_AS unsigned*)(myws + FX_OFF + o), __ATOMIC_RELAXED, __HIP_MEMORY_SCOPE_SYSTEM);
    };
    TG_AF();
    int dry = NTAG != 0 && DRYW;
    if (!foreign) STAMP(2);
#pragma clang loop unroll(disable)
    for (int i = -1; i < NTASK; ++i) {
        asm volatile("" : "+s"(i));
        if (i >= 0) { t_lo = (t_lo >> 8) | (t_hi << 56); t_hi = (t_hi >> 8) | (0xFFull << 56); }
        int en = (int)(t_lo & 0xFF);
        unsigned nbase = (unsigned)blk_of(en) * BLK_BYTES + wave * 16384;
        bool ready = false;
        if (i >= 0) {
        failed = failed || (unsigned)__builtin_amdgcn_readlane(fpre, 4) == call;
        ready = en != 0xFF && !tagged(en) && ((unsigned)__builtin_amdgcn_ballot_w64(fpre >= call) & 15u) == 15u;
        CBAR();
        i32x16 acc = {};
        if (ready) {
            TRACE(i + 1, 0);
            if (lane == 0) __hip_atomic_fetch_add(ctr + 33 + i, 1u, __ATOMIC_RELAXED, __HIP_MEMORY_SCOPE_WORKGROUP);
            peek((int)(t_lo >> 8) & 0xFF);
#pragma unroll
            for (int n = 0; n < 16; ++n) {
                const int k = KORD(n);
                __builtin_amdgcn_sched_barrier(0);
                acc = __builtin_amdgcn_mfma_i32_32x32x32_i8(__builtin_bit_cast(i32x4, af[k]), bf[k], acc, 0, 0, 0);
                __builtin_amdgcn_sched_barrier(0);
                aload(k, nbase);
            }
            __builtin_amdgcn_sched_barrier(0);
        } else {
#pragma unroll
            for (int n = 0; n < 16; ++n)
                acc = __builtin_amdgcn_mfma_i32_32x32x32_i8(__builtin_bit_cast(i32x4, af[KORD(n)]), bf[KORD(n)], acc, 0, 0, 0);
        }
        if (failed && lane == 0) __hip_atomic_store(ctr + 17, 1u, __ATOMIC_RELAXED, __HIP_MEMORY_SCOPE_WORKGROUP);
        LDS_AS unsigned char* tp = smem + i * TILE_STRIDE + aoff;
#pragma unroll
        for (int r = 0; r < 16; ++r)
            __hip_atomic_fetch_add((LDS_AS int*)(tp + ((r & 3) + 8 * (r >> 2)) * 128), acc[r],
                                   __ATOMIC_RELAXED, __HIP_MEMORY_SCOPE_WORKGROUP);
        LGKMCNT0();
        if (lane == 0) __hip_atomic_fetch_add(ctr + i, 1u, __ATOMIC_RELAXED, __HIP_MEMORY_SCOPE_WORKGROUP);
        TRACE(i, 1);
        }
        if (!ready) {
#pragma clang loop unroll(disable)
            for (; en != 0xFF; dry = 0) {
                asm volatile("" : "+s"(dry));
                const int tg = tagged(en);
                if (!dry && !failed) failed = !wait_flags(flag_ptr(__builtin_amdgcn_readfirstlane((int)(nbase >> 17) - tg + (EMID && tg && (en & 15) == NTB ? EBLK - NTB + 1 : 0))), 0, tmo, call, lane);
                CBAR();
                if (!tg && !dry) break;
                i += 1 - dry;
                if (!dry) TRACE(i, 0);
                if (lane == 0) __hip_atomic_fetch_add(ctr + 32 + i + dry, 1u - dry, __ATOMIC_RELAXED, __HIP_MEMORY_SCOPE_WORKGROUP);
                const unsigned t2 = (unsigned)__builtin_amdgcn_s_memrealtime() + (dry || (EMID && (en & 15) == NTB) ? TDE - (TD3 - TD2) : TD2);
                const unsigned ebase = (nbase >> 17 << 17) + wave * 4096 + (EBLK * BLK_BYTES - NTB * 8) - ((en & 15) << 17) + (en & 15) * 8;
                unsigned stop = failed ? 1u : 0u;
                stop = __builtin_amdgcn_readfirstlane(stop);
                i32x16 acc;
                asm volatile(TG_TASK
                             : [acc] "=&v"(acc), [stop] "+s"(stop) TG_DIAG(, [nr] "+s"(nretry))
                             : [nb] "s"(__builtin_amdgcn_readfirstlane(nbase)), [eb] "s"(__builtin_amdgcn_readfirstlane(ebase)),
                               [vo] "v"(voff | UPAGE(0, 0, nbase >> 17)), [ar] "s"(ar), [tmo] "s"(tmo), [call] "s"(call), [t2] "s"(t2), [dry] "s"(dry * (wave + 1)),
                               [b0] "v"(bf[0]), [b1] "v"(bf[1]), [b2] "v"(bf[2]), [b3] "v"(bf[3]), [b4] "v"(bf[4]), [b5] "v"(bf[5]), [b6] "v"(bf[6]), [b7] "v"(bf[7]),
                               [b8] "v"(bf[8]), [b9] "v"(bf[9]), [b10] "v"(bf[10]), [b11] "v"(bf[11]), [b12] "v"(bf[12]), [b13] "v"(bf[13]), [b14] "v"(bf[14]), [b15] "v"(bf[15])
                             : "memory", "vcc", "scc", TG_CLOBBER);
                TG_AF();
                if (stop) {
                    failed = true;
                    if (lane == 0) {
                        __hip_atomic_store(tmo, call, __ATOMIC_RELAXED, __HIP_MEMORY_SCOPE_AGENT);
                        __hip_atomic_store(ctr + 17, 1u, __ATOMIC_RELAXED, __HIP_MEMORY_SCOPE_WORKGROUP);
                    }
                }
                LDS_AS unsigned char* tq = smem + (i + dry) * TILE_STRIDE + aoff;
#pragma unroll
                for (int r = 0; r < 16; ++r)
                    __hip_atomic_fetch_add((LDS_AS int*)(tq + ((r & 3) + 8 * (r >> 2)) * 128), acc[r],
                                           __ATOMIC_RELAXED, __HIP_MEMORY_SCOPE_WORKGROUP);
                LGKMCNT0();
                if (lane == 0) __hip_atomic_fetch_add(ctr + i + dry, 1u - dry, __ATOMIC_RELAXED, __HIP_MEMORY_SCOPE_WORKGROUP);
                if (dry && !foreign) STAMP(6);
                if (dry) continue;
                TRACE(i, 1);
                t_lo = (t_lo >> 8) | (t_hi << 56); t_hi = (t_hi >> 8) | (0xFFull << 56);
                en = (int)(t_lo & 0xFF);
                nbase = (unsigned)blk_of(en) * BLK_BYTES + wave * 16384;
            }
            if (en == 0xFF) break;
            TRACE(i + 1, 0);
            peek((int)(t_lo >> 8) & 0xFF);
            __builtin_amdgcn_sched_barrier(0);
#pragma unroll
            for (int k = 0; k < 16; ++k) { aload(KORD(k), nbase); __builtin_amdgcn_sched_barrier(0); }
            __builtin_amdgcn_sched_barrier(0);
            if (i < 0) {
                {
                    const int ebw = (foreign ? downer : tile >> 2) * NBLK + 16 + (tile & 3);
                    if (!failed) failed = !wait_flags(flag_ptr(ebw), 0, tmo, call, lane);
                    CBAR();
                    if (foreign) STAMP(6);
                    peek((int)(t_lo >> 8) & 0xFF);
                    CBAR();
                    const unsigned bbase = (unsigned)ebw * BLK_BYTES + wave * 16384;
#pragma unroll
                    for (int k = 0; k < 16; ++k)
                        bf[k] = __builtin_bit_cast(i32x4, __builtin_amdgcn_raw_buffer_load_b128(ar, voff, ((bbase | UPAGE(0, 0, bbase >> 17)) ^ UPAGE(k & 3, k >> 2, 0)) + (k >> 2) * 1024, 17));
                }
                TRACE(0, 0);
            }
            if (lane == 0) __hip_atomic_fetch_add(ctr + 33 + i, 1u, __ATOMIC_RELAXED, __HIP_MEMORY_SCOPE_WORKGROUP);
            if (i < 0) {
                lds_wait(ctr + 16, 8u);
                __builtin_amdgcn_s_waitcnt(0x0F70);
                STAMP(1);
            }
        }
    }
    STAMP(5);
    TRACE_OUT();
    __builtin_amdgcn_s_waitcnt(0x0F70);
}

static void build_table(int foreign, float tstart, u64 (*tab)[2], float (*fin)[NTASK] = nullptr) {
    struct T { float t; int e; } task[128];
    const int nslot = foreign ? NSLOT : NOWN;
    const float tb = PACE * (4 * 0.01f / 16), tw = PACE * (4 * 0.01f / 16), tx = PACE * (4 * 0.01f / 16);
    int n = 0;
    for (int d = 0; d < 8; ++d)
        for (int blk = 0; blk < 16; ++blk) {
            const int type = blk < N1 ? 1 : blk < N1 + N3 ? 3 : 2;
            float t;
            int direct = 0;
            if (d == 0) { if (foreign && type != 1) continue; t = 3.0f; }
            else if (d < 4) {
                if (type == 2 || (foreign && type == 3)) continue;
                t = T0X + tb * (blk + 1) + (NTAG && blk >= NTB ? TTAG : 0.0f);
                direct = foreign ? blk >= N1 - NRAW : blk == N1 + N3 - 1;
            } else {
                if (type == 1 || (foreign && (type == 3 || d == 4))) continue;
                t = T0X + 4 * tw + tx * (16 - blk);
            }
            task[n].t = t + 0.001f * d; task[n].e = direct << 7 | d << 4 | blk; ++n;
        }
    for (int i = 1; i < n; ++i)
        for (int k = i; k > 0 && task[k].t < task[k - 1].t; --k) { T s = task[k]; task[k] = task[k - 1]; task[k - 1] = s; }
    float fr[NSLOT]; int cntv[NSLOT]; unsigned char lst[NSLOT][NTASK];
    for (int s = 0; s < NSLOT; ++s) { fr[s] = foreign || s < 4 ? tstart : OST1; cntv[s] = 0; for (int i = 0; i < NTASK; ++i) lst[s][i] = 0xFF; }
    for (int i = 0; i < n; ++i) {
        int best = -1;
        for (int s = 0; s < nslot; ++s) if (cntv[s] < NTASK - 1 && (best < 0 || fr[s] < fr[best])) best = s;
        fr[best] = (fr[best] > task[i].t ? fr[best] : task[i].t) + TCOST;
        if (fin) fin[best][cntv[best]] = fr[best];
        lst[best][cntv[best]++] = (unsigned char)task[i].e;
    }
    for (int s = 0; s < NSLOT; ++s)
        for (int h = 0; h < 2; ++h) {
            u64 v = 0;
            for (int i = 7; i >= 0; --i) v = v << 8 | lst[s][h * 8 + i];
            tab[s][h] = v;
        }
}

static void build_all(Args& a) {
    build_table(0, OST0, a.tab[0]);
    auto byte = [&](int t, int s, int i) { return (int)((a.tab[1 + t][s][i >> 3] >> (8 * (i & 7))) & 0xFF); };
    int n = 0;
    for (int t = 0; t < 4; ++t) {
        build_table(1, FSTART + FSTEP * t, a.tab[1 + t]);
        for (int s = 0; s < NSLOT; ++s)
            for (int i = 0; i < NTASK; ++i) n += byte(t, s, i) < 0x80;
    }
    a.npacked = n;
    for (int t = 0; t < 4; ++t)
        for (int s = 0; s < NSLOT; ++s) a.ord[t][s][0] = a.ord[t][s][1] = 0;
    for (int blk = 0; blk < 16; ++blk)
        for (int t = 0; t < 4; ++t)
            for (int s = 0; s < NSLOT; ++s)
                for (int i = 0; i < NTASK; ++i)
                    if (byte(t, s, i) != 0xFF && (byte(t, s, i) & 0x8F) == (0x80 | blk)) a.ord[t][s][i >> 3] |= (u64)n++ << (8 * (i & 7));
    a.ntotal = n;
}

extern "C" void kernel_launch(int rank, int nranks, ncclComm_t comm,
                              hipStream_t stream,
                              void* const* d_in, const int* in_sizes, int n_in,
                              void* d_out, int out_size,
                              const int* in_dtypes, int out_dtype,
                              void* const* d_out_all,
                              void* const* d_ws_all, size_t ws_bytes) {
    if (nranks != NRANK || (unsigned)rank >= (unsigned)NRANK || n_in != 4 || ws_bytes < WS_NEED ||
        in_sizes[0] != MSH * KDIM || in_sizes[1] != KDIM * NSH || out_size != NRANK * MSH * NSH) {
        fprintf(stderr, "unexpected problem shape\n");
        return;
    }
    HIP_CHECK(hipFuncSetAttribute((const void*)ag_gemm_kernel, hipFuncAttributeMaxDynamicSharedMemorySize, LDS_BYTES));
    Args a;
    a.sx = (const float*)d_in[2];
    a.sw = (const float*)d_in[3];
    a.out = (float*)d_out;
    for (int d = 0; d < NRANK; ++d) a.ws[d] = (char*)d_ws_all[rank ^ d];
    for (int i = 0; i < 4; ++i) a.outp[i] = (float*)d_out_all[rank ^ (4 + i)];
    for (int i = 0; i < NTHREADS / 64; ++i) {
        Rec& r = a.rec[i];
        const int d = i & 7;
        r.x = (const signed char*)d_in[0];
        r.w = (const signed char*)d_in[1];
        r.ws = a.ws[0];
        r.peer = a.ws[d];
        r.src = d < 4 ? r.x : r.w + (d & 3) * 32;
        r.dst = r.peer + (size_t)(d * NBLK + (d < 4 ? 0 : 16)) * BLK_BYTES;
        r.cnt = r.ws + CNT_OFF + i * 4;
        r.dst1 = r.dst + XS_PAR;
    }
    build_all(a);
    if (a.npacked < NSTAGE || a.ntotal > NIDX || (KSPLIT && a.ntotal > KSPLIT + NSLOT * 12)) return;
    a.rank = rank;
    ag_gemm_kernel<<<dim3(256), dim3(NTHREADS), LDS_BYTES, stream>>>(a);
    HIP_CHECK(hipGetLastError());
}
```

```cpp
#include <hip/hip_runtime.h>
#include <rccl/rccl.h>
#include <cstdio>
#include <cstddef>

#define HIP_CHECK(expr) do { hipError_t _e = (expr); if (_e != hipSuccess) { \
    fprintf(stderr, "HIP error %s at %s:%d\n", hipGetErrorString(_e), __FILE__, __LINE__); \
    return; } } while (0)

typedef int i32x2 __attribute__((ext_vector_type(2)));
typedef int i32x4 __attribute__((ext_vector_type(4)));
typedef int i32x16 __attribute__((ext_vector_type(16)));
typedef unsigned u32x2 __attribute__((ext_vector_type(2)));
typedef unsigned u32x4 __attribute__((ext_vector_type(4)));
typedef float f32x4 __attribute__((ext_vector_type(4)));
typedef unsigned long long u64;
typedef unsigned __attribute__((aligned(1))) u32_u;
#define LDS_AS __attribute__((address_space(3)))
#define GLB_AS __attribute__((address_space(1)))
#define CONST_AS __attribute__((address_space(4)))

#define N1 11
#define N3 1
#define PACE 906
#define FSTART 7.7f
#define FSTEP 2.2f
#define NOCOMP 0
#define TCOST 1.3f
#define NRAW 1
#define PACET PACE
#define LDDELAY 4
#define PSLACK 160
#define NSTAGE 188
#define LSLEEP 8
#define DYNQ 1
#define TCUT 2370
#define PTR 0
#define LDPACE 1700
#define T0X 4.0f
#define FSTRIDE 128
#define PSLEEP 1

#define KSPLIT 144
#define LDORD 1
#define BURST1 0
#define SWZ 3
#define UPAGE(c, f, blk) ((unsigned)((c) ^ ((SWZ & 1 ? (f) : 0) ^ (SWZ & 2 ? (blk) & 3 : 0))) << 12)
#define WALL 4
#define ALIGN 4
#define AMAX 60
#define ASLEEP 60
#define STAG 1
#define TBIAS 0
#define P21 1
#define NTAG 0
#define SLOTTED 2
#define RUSHB 99
#define RUSHC 99
#define RUSHM 900
#define RUSHS 0
#define RUSHW 1
#define TRAW (HS ? TDIR + 36 * TDSTEP / 16 - 20 : 0)
#define TRSTEP 6
#define TDSTEP (1600 * PACET / 32768)
#define TDIR ((36 + HS) * PACE / 16 + TSOFF + (NSTAGE * 1600 - (HS ? (48 + 4 * HS) * QNOM : 0)) / 1024 * PACET / 512)
#define DUEU 3
#define LASTP 200.0f
#define TRES (22.2f - 0.6f * HS)
#define PACKT 1.3f
#define WGAP 0.9f
#define USLEEP 1
#define TSOFF 80
#define QNOM 352
#define HB 4
#define HS 0
#define HPOS 15
#define FEARLY 0
#define NHELP 3
#define FASTX 16
#define FASTO 256
#define SUEVEN 1
#define DELAY1 0
#define DELAYW 0
#define SLEEP0 2
#define SLEEP1 10
#define SLEEP2 8
#define OST0 6.5f
#define OST1 6.5f
#define HAVE_ROLE 1
__host__ __device__ inline int role_of(int b, int& ix) {
    const int r = b >> 3;
    ix = r < 4 ? b : (b & 7) * (r < 6 ? 2 : 26) + r - (r < 6 ? 4 : 6);
    return r < 4 ? 0 : r < 6 ? 1 : 2;
}
constexpr int NRANK = 8;
constexpr int KDIM = 4096, MSH = 512, NSH = 512;
constexpr unsigned BLK_BYTES = 32 * KDIM;
constexpr int NBLK = 20;
constexpr unsigned OSLOT = 4608;
constexpr int NSLOT = 7, NOWN = 6, NTASK = 16, NIDX = 256;
constexpr size_t XS_PAR = (size_t)NRANK * NBLK * BLK_BYTES;
constexpr size_t OS_OFF = 2 * XS_PAR, OS_PAR = (size_t)4 * NIDX * OSLOT;
constexpr unsigned LSLOT = 2048;
constexpr size_t LS_OFF = OS_OFF + 2 * OS_PAR;
constexpr size_t FX_OFF = LS_OFF + 4 * NIDX * LSLOT;
constexpr size_t FO_OFF = FX_OFF + NRANK * NBLK * 32 * 128;
constexpr size_t FL_OFF = FO_OFF + 4 * NIDX * 128;
constexpr size_t CNT_OFF = FL_OFF + 4 * NIDX * 128;
constexpr size_t TMO_OFF = CNT_OFF + 16384;
constexpr size_t DUMMY_OFF = TMO_OFF + 1024;
constexpr size_t WS_NEED = DUMMY_OFF + 4096;
constexpr unsigned DEADLINE_TICKS = 20000000u;
constexpr int TILE_STRIDE = 4352;
constexpr int LDS_MAIN = 131072;
constexpr int LDS_CTR = LDS_MAIN;
constexpr int LDS_BYTES = LDS_CTR + 256;
constexpr int NTHREADS = 768;
static_assert((NTASK + 8) * TILE_STRIDE <= LDS_MAIN, "tiles");
#define DIAG_N 0
#define STAMP(i)
#define TRACE(i, h)
#define TRACE2(i, h)
#define TRACE_OUT()

struct Rec {
    const signed char* src;
    char* dst;
    const char* cnt;
    char* peer;
    const signed char* x;
    const signed char* w;
    char* ws;
    char* dst1;
};
struct Args {
    Rec rec[NTHREADS / 64];
    const float* sx;
    const float* sw;
    float* out;
    char* ws[NRANK];
    float* outp[4];
    u64 tab[5][NSLOT][2];
    u64 ord[4][NSLOT][2];
    int rank;
    int npacked, ntotal;
};

#define STORE_WT(voff, base, val, off) \
    asm volatile("global_store_dwordx4 %0, %2, %1 offset:%3 sc0 sc1\n\ts_nop 1" \
                 :: "v"(voff), "s"(base), "v"(val), "i"(off) : "memory")
#define VMCNT(n) asm volatile("s_waitcnt vmcnt(%0)" :: "i"(n) : "memory")
#define LGKMCNT0() asm volatile("s_waitcnt lgkmcnt(0)" ::: "memory")
#define CBAR() asm volatile("" ::: "memory")

__device__ __forceinline__ void set_flag(char* p, unsigned call, int lane) {
    if (lane == 0) __hip_atomic_store((GLB_AS unsigned*)p, call, __ATOMIC_RELAXED, __HIP_MEMORY_SCOPE_SYSTEM);
}
__device__ __forceinline__ unsigned lds_get(LDS_AS unsigned* p) {
    return __builtin_amdgcn_readfirstlane(__hip_atomic_load(p, __ATOMIC_RELAXED, __HIP_MEMORY_SCOPE_WORKGROUP));
}
__device__ __forceinline__ void lds_wait(LDS_AS unsigned* p, unsigned v) {
    while (lds_get(p) < v) __builtin_amdgcn_s_sleep(1);
    CBAR();
}

#define TMOEVERY 8
#define DELAY0 50
__device__ __forceinline__ unsigned tmo_get(GLB_AS unsigned* tmo) {
    return __hip_atomic_load(tmo, __ATOMIC_RELAXED, __HIP_MEMORY_SCOPE_AGENT);
}
__device__ __forceinline__ bool wait_flags(GLB_AS unsigned* p, unsigned f, GLB_AS unsigned* tmo, unsigned call, int lane) {
    if (((unsigned)__builtin_amdgcn_ballot_w64(f >= call) & 15u) == 15u) return true;
    const unsigned t0 = (unsigned)__builtin_amdgcn_s_memrealtime();
    for (unsigned it = 1;; ++it) {
        if (lane < 4) f = __hip_atomic_load(p, __ATOMIC_RELAXED, __HIP_MEMORY_SCOPE_SYSTEM);
        if (((unsigned)__builtin_amdgcn_ballot_w64(f >= call) & 15u) == 15u) return true;
        if (it % TMOEVERY == 0 &&
            (__builtin_amdgcn_readfirstlane(tmo_get(tmo)) == call || (unsigned)__builtin_amdgcn_s_memrealtime() - t0 > DEADLINE_TICKS)) {
            if (lane == 0) __hip_atomic_store(tmo, call, __ATOMIC_RELAXED, __HIP_MEMORY_SCOPE_AGENT);
            return false;
        }
        __builtin_amdgcn_s_sleep(PSLEEP);
    }
}

#define PDEPTH 7
#define PGAP 0
#define POLL(LOAD, CHECK) \
    "v_mov_b64 v[166:167], 0\n\ts_mov_b32 %[it], 0\n\ts_mov_b32 exec_hi, 0\n" \
    "1:\n\ts_add_u32 %[it], %[it], 1\n\ts_and_b32 vcc_lo, %[it], 31\n\ts_cselect_b32 vcc_lo, 0, 16\n\ts_or_b32 exec_lo, %[pend], vcc_lo\n\t" \
    LOAD " %[vo], %[base] sc0 sc1\n\ts_mov_b32 exec_lo, 31\n\ts_memrealtime vcc\n\ts_sleep %[gap]\n\t" \
    "s_cmp_lg_u32 %[fast], 0\n\ts_cbranch_scc1 2f\n\ts_waitcnt vmcnt(0)\n" \
    "2:\n\ts_waitcnt vmcnt(%[dep]) lgkmcnt(0)\n\ts_cmp_eq_u32 %[it], 1\n\ts_cselect_b32 %[t0], vcc_lo, %[t0]\n\ts_sub_u32 vcc_lo, vcc_lo, %[t0]\n\t" \
    "s_cmp_gt_u32 vcc_lo, %[dl]\n\ts_cbranch_scc1 3f\n\tv_cmp_le_u32 vcc, %[call], v166\n\ts_bitcmp1_b32 vcc_lo, 4\n\ts_cbranch_scc1 3f\n\t" \
    CHECK "s_andn2_b32 %[pend], %[pend], vcc_lo\n\ts_cbranch_scc1 1b\n" \
    "3:\n\ts_mov_b64 exec, -1"
__device__ __forceinline__ bool poll_flags(unsigned vo, const char* base, GLB_AS unsigned* tmo, unsigned call, unsigned fast, int lane) {
    unsigned pend = 15, it, t0;
    asm volatile(POLL("global_load_dword v166,", "")
                 : [pend] "+s"(pend), [it] "=&s"(it), [t0] "=&s"(t0)
                 : [vo] "v"(vo), [base] "s"(base), [call] "s"(call), [fast] "s"(fast), [dep] "i"(PDEPTH), [gap] "i"(PGAP), [dl] "i"(DEADLINE_TICKS)
                 : "memory", "vcc", "scc", "v166", "v167");
    if (pend && lane == 0) __hip_atomic_store(tmo, call, __ATOMIC_RELAXED, __HIP_MEMORY_SCOPE_AGENT);
    return !pend;
}
__device__ __forceinline__ unsigned poll_flag64(unsigned vo, const char* base, GLB_AS unsigned* tmo, unsigned call, unsigned fast, int lane) {
    unsigned pend = 1, it, t0, info;
    asm volatile(POLL("global_load_dwordx2 v[166:167],",
                      "v_readfirstlane_b32 %[info], v167\n\ts_xor_b32 vcc_hi, %[info], %[call]\n\ts_and_b32 vcc_hi, vcc_hi, 63\n\ts_cbranch_scc1 1b\n\t")
                 : [pend] "+s"(pend), [it] "=&s"(it), [t0] "=&s"(t0), [info] "=&s"(info)
                 : [vo] "v"(vo), [base] "s"(base), [call] "s"(call), [fast] "s"(fast), [dep] "i"(PDEPTH), [gap] "i"(PGAP), [dl] "i"(DEADLINE_TICKS)
                 : "memory", "vcc", "scc", "v166", "v167");
    if (pend && lane == 0) __hip_atomic_store(tmo, call, __ATOMIC_RELAXED, __HIP_MEMORY_SCOPE_AGENT);
    return pend ? 0 : info;
}

template <int V> struct IC { static constexpr int value = V; };
constexpr unsigned F_WIDE = 1u << 30, F_NAN = 1u << 31, F_SKIP = 1u << 29;
#define BITMASK(nb, m, r) asm("v_bfe_i32 %0, %1, %2, 1" : "=v"(nb) : "v"(m), "i"(r))
#define STEP(x, nb, n) asm("v_mad_i32_i24 %0, %1, %2, %3" : "=v"(x) : "v"(nb), "i"(-(n)), "v"(x))
__device__ __forceinline__ unsigned wave_scan(unsigned x) {
    x += __builtin_amdgcn_update_dpp(0u, x, 0x111, 0xf, 0xf, false);
    x += __builtin_amdgcn_update_dpp(0u, x, 0x112, 0xf, 0xf, false);
    x += __builtin_amdgcn_update_dpp(0u, x, 0x114, 0xf, 0xf, false);
    x += __builtin_amdgcn_update_dpp(0u, x, 0x118, 0xf, 0xf, false);
    x += __builtin_amdgcn_update_dpp(0u, x, 0x142, 0xa, 0xf, false);
    x += __builtin_amdgcn_update_dpp(0u, x, 0x143, 0xc, 0xf, false);
    return x;
}
constexpr u64 ld_order() {
    u64 o = 0; int n = 0, a = 1, z = 15;
    for (int slot = 0; n < 15; ++slot) {
        const bool cross = z >= N1 && (a >= N1 || 4 + (15 - z) - (z < 15) <= a);
        if (cross) o |= (u64)z-- << (4 * n++); else o |= (u64)a++ << (4 * n++);
    }
    return o;
}
constexpr u64 ld_pos() {
    u64 p = 0;
    for (int n = 0; n < 15; ++n) p |= (u64)(n + 1) << (4 * ((ld_order() >> (4 * n)) & 15));
    return p;
}
#define LOAD_LDS_SYS(ldsaddr, voffs, base) \
    asm volatile("s_nop 0\n\tglobal_load_lds_dwordx4 %1, %2 sc0 sc1" :: "{m0}"(ldsaddr), "v"(voffs), "s"(base) : "memory")
#define LOAD_LDS(ldsaddr, voffs, base) \
    asm volatile("s_nop 0\n\tglobal_load_lds_dwordx4 %1, %2" :: "{m0}"(ldsaddr), "v"(voffs), "s"(base) : "memory")

__device__ __forceinline__ u64 flag64_get(const char* p) {
    return __hip_atomic_load((GLB_AS u64*)p, __ATOMIC_RELAXED, __HIP_MEMORY_SCOPE_SYSTEM);
}
__device__ __forceinline__ void flag64_set(char* p, unsigned call, unsigned info, int lane) {
    if (lane == 0) __hip_atomic_store((GLB_AS u64*)p, (u64)info << 32 | call, __ATOMIC_RELAXED, __HIP_MEMORY_SCOPE_SYSTEM);
}
#define IF_DIAG(...)
__global__ void __launch_bounds__(NTHREADS, 3) __attribute__((amdgpu_num_vgpr(83))) ag_gemm_kernel(Args a_) {
    const CONST_AS Args* ap = (const CONST_AS Args*)__builtin_amdgcn_kernarg_segment_ptr();
    unsigned tid = threadIdx.x, w64, call, stmp;
    int b = blockIdx.x;
    const signed char *xin, *win;
    char *myws, *peer;
    u64 t_first, t_top;
#define PRO_(...)
    asm volatile(
        IF_DIAG("s_memrealtime %[t0]\n\t") "v_readfirstlane_b32 %[w], v0\n\ts_load_dwordx16 s[36:51], %[ap], %[w]\n\t"
        "s_cmp_gt_u32 %[b], 31\n\ts_cbranch_scc1 3f\n\ts_sub_u32 %[t], %[w], 64\n\ts_cmpk_lt_u32 %[t], 192\n\ts_cbranch_scc0 2f\n\t"
        "v_lshlrev_b32 v1, 12, v0\n\tv_lshrrev_b32 v10, 1, v0\n\ts_mov_b32 %[t], 0x1f010\n\tv_bitop3_b32 v1, v1, %[t], v10 bitop3:0xc8\n\t"
        "v_lshl_add_u32 v1, %[b], 7, v1\n\ts_waitcnt lgkmcnt(0)\n\tglobal_load_dwordx4 v[2:5], v1, s[36:37]\n\t" PRO_("s_memrealtime %[t2]\n\t")
        "s_lshl_b32 %[t], %[b], 6\n\ts_load_dword %[n], s[40:41], %[t]\n\ts_barrier\n\tglobal_load_dwordx4 v[6:9], v1, s[36:37] offset:32\n\t"
        "v_lshlrev_b32 v10, 4, v0\n\tv_and_b32 v10, 0x3f0, v10\n\tv_lshl_or_b32 v10, %[b], 12, v10\n\ts_movk_i32 m0, 2048-64\n\ts_sleep %[s0]\n\t"
        "global_load_lds_dwordx4 v1, s[36:37] offset:64\n\ts_movk_i32 m0, 3072-96\n\tv_xor_b32 v11, 0x1000, v10\n\tglobal_load_lds_dwordx4 v1, s[36:37] offset:96\n\t"
        "s_waitcnt lgkmcnt(0)\n\t" PRO_("s_memrealtime %[t3]\n\t") "s_bitcmp0_b32 %[n], 0\n\ts_cselect_b64 s[38:39], s[50:51], s[38:39]\n\t"
        "s_waitcnt vmcnt(3)\n\tglobal_store_dwordx4 v10, v[2:5], s[38:39] sc0 sc1\n\ts_memrealtime %[t1]\n\ts_sleep %[s1]\n\ts_waitcnt vmcnt(3)\n\t"
        "global_store_dwordx4 v11, v[6:9], s[38:39] offset:1024 sc0 sc1\n\t" PRO_("s_memrealtime %[t4]\n\t") "s_branch 4f\n\t.p2align 6\n"
        "2:\n\ts_cmpk_gt_u32 %[t], 447\n\ts_cbranch_scc1 3f\n\t"
        "v_lshlrev_b32 v1, 8, v0\n\tv_lshlrev_b32 v10, 4, v0\n\tv_and_b32 v1, 0x3e00, v1\n\tv_and_or_b32 v1, v10, 16, v1\n\t"
        "v_lshl_add_u32 v1, %[b], 16, v1\n\ts_lshl_b32 m0, %[w], 8\n\ts_waitcnt lgkmcnt(0)\n\tglobal_load_lds_dwordx4 v1, s[36:37]\n\t"
        "s_lshl_b32 %[t], %[b], 6\n\ts_barrier\n\t" PRO_("s_memrealtime %[t2]\n\t") "s_load_dword %[n], s[40:41], %[t]\n\t"
        "s_addk_i32 m0, 0x400\n\tv_add_u32 v13, 0x4000, v1\n\tglobal_load_lds_dwordx4 v13, s[36:37]\n\t"
        "v_and_b32 v12, 0x2e0, v10\n\tv_and_or_b32 v12, v0, 16, v12\n\tv_lshlrev_b32 v2, 3, v0\n\tv_and_or_b32 v12, v2, 8, v12\n\tv_lshl_add_u32 v12, %[w], 8, v12\n\t"
        "s_addk_i32 m0, 0x400\n\tv_add_u32 v14, 0x8000, v1\n\tglobal_load_lds_dwordx4 v14, s[36:37]\n\t"
        "s_addk_i32 m0, 0x400\n\tv_add_u32 v1, 0xc000, v1\n\tglobal_load_lds_dwordx4 v1, s[36:37]\n\t"
        "v_and_b32 v10, 0x3f0, v10\n\tv_lshl_or_b32 v10, %[b], 12, v10\n\tv_xor_b32 v11, 0x1000, v10\n\t"
        "s_waitcnt lgkmcnt(0)\n\t" PRO_("s_memrealtime %[t3]\n\t") "s_bitcmp0_b32 %[n], 0\n\ts_cselect_b64 s[38:39], s[50:51], s[38:39]\n\t"
        "s_waitcnt vmcnt(3)\n\tds_read_b64_tr_b8 v[2:3], v12\n\tds_read_b64_tr_b8 v[4:5], v12 offset:256\n\ts_waitcnt lgkmcnt(0)\n\t"
        "global_store_dwordx4 v10, v[2:5], s[38:39] sc0 sc1\n\ts_memrealtime %[t1]\n\ts_sleep %[s2]\n\ts_waitcnt vmcnt(3)\n\t"
        "ds_read_b64_tr_b8 v[6:7], v12 offset:1024\n\tds_read_b64_tr_b8 v[8:9], v12 offset:1280\n\ts_waitcnt lgkmcnt(0)\n\t"
        "global_store_dwordx4 v11, v[6:9], s[38:39] offset:1024 sc0 sc1\n\t" PRO_("s_memrealtime %[t4]\n\t") "s_branch 4f\n\t.p2align 6\n"
        "3:\n\tv_lshlrev_b32 v1, 2, v0\n\tv_and_b32 v1, 0xfc, v1\n\tv_add_u32 v1, %[ctr], v1\n\tv_mov_b32 v2, 0\n\tds_write_b32 v1, v2\n\t"
        "s_lshl_b32 %[t], %[b], 6\n\ts_waitcnt lgkmcnt(0)\n\ts_barrier\n\ts_load_dword %[n], s[40:41], %[t]\n\ts_memrealtime %[t1]\n"
        "4:\n\ts_waitcnt lgkmcnt(0)"
        : [w] "=&s"(w64), [n] "=&s"(call), [t] "=&s"(stmp), [t1] "=&s"(t_first), IF_DIAG([t0] "=&s"(t_top),) [b] "+s"(b), [ap] "+s"(ap),
          PRO_([t2] "=&s"(t_ka), [t3] "=&s"(t_cnt), [t4] "=&s"(t_ack),)
          "=&{s[42:43]}"(peer), "=&{s[44:45]}"(xin), "=&{s[46:47]}"(win), "=&{s[48:49]}"(myws)
        : [ctr] "i"(LDS_CTR), [s0] "i"(SLEEP0), [s1] "i"(SLEEP1), [s2] "i"(SLEEP2), "{v0}"(tid)
        : "memory", "scc", "v1", "v2", "v3", "v4", "v5", "v6", "v7", "v8", "v9", "v10", "v11", "v12", "v13", "v14", "s36", "s37", "s38", "s39", "s40", "s41", "s50", "s51");
    asm volatile("" : "+{v0}"(tid));
    call += 1u;
    const unsigned par = call & 1u;
    const unsigned tstart = (unsigned)t_first;
    static_assert(sizeof(Rec) == 64 && (SWZ & 1) && KDIM == 4096 && NSH == 512, "prologue");
    const CONST_AS Args& a = *ap;
    LDS_AS unsigned char* smem = (LDS_AS unsigned char*)0;
    int lane = tid & 63;
    asm volatile("" : "+v"(lane));
    __builtin_assume(lane >= 0 && lane < 64);
    const int wave = w64 >> 6;
    LDS_AS unsigned* ctr = (LDS_AS unsigned*)(smem + LDS_CTR);
    int j;
    const int role = role_of(b, j);
    STAMP(7);

    GLB_AS unsigned* cnt = (GLB_AS unsigned*)(myws + CNT_OFF) + b * 16 + wave;
    const unsigned voff = lane * 16;
    unsigned xsrc_off, lo[4];
    auto image = [&]() {
        unsigned l = lane;
        asm volatile("" : "+v"(l));
        xsrc_off = (l >> 3) * KDIM + (((l & 7) ^ (l >> 3)) * 16);
#pragma unroll
        for (int f = 0; f < 4; ++f) lo[f] = ((l & 31) * 128 + (((l >> 5) ^ (l & 7)) * 16)) ^ (f * 32);
        asm volatile("" : "+v"(xsrc_off), "+v"(lo[0]), "+v"(lo[1]), "+v"(lo[2]), "+v"(lo[3]));
    };

    if (__builtin_expect(role == 0, 1)) {
        const int c = j;
        image();
        const int d = wave < 8 ? wave : wave - 4;
        const unsigned lds0 = (unsigned)(size_t)smem;
        auto now16 = [&]() { return (unsigned)__builtin_amdgcn_s_memrealtime() * 16u; };
        if (wave == 0) {
            const signed char* xs = xin + c * 128;
            unsigned tl = 0;
            if (LDDELAY) __builtin_amdgcn_s_sleep(LDDELAY);
            for (int n = 0; n < 15; ++n) {
                const int blk = (int)((ld_order() >> (4 * n)) & 15);
                if (n >= 2) { while ((int)(now16() - tl) < 0) __builtin_amdgcn_s_sleep(2); }
                tl = now16() + LDPACE;
#pragma unroll
                for (int i = 0; i < 4; ++i) LOAD_LDS(lds0 + blk * 4096 + i * 1024, xsrc_off, xs + (size_t)(blk * 32 + i * 8) * KDIM);
                if (n == 0) continue;
                VMCNT(0);
                asm volatile("ds_write_b32 %0, %1" :: "v"(ctr), "v"(n + 1) : "memory");
            }
            LGKMCNT0();
            STAMP(2);
            __builtin_amdgcn_endpgm();
        }
        u32x4 v[4];
        const unsigned wlds = lds0 + 65536 + (d & 3) * 16384;
        const signed char* wsrc = win + (size_t)c * 128 * NSH + (d & 3) * 32;
        const unsigned wo = (lane >> 1) * NSH + (lane & 1) * 16;
        const int ntile = (NSTAGE - c + 31) >> 5;
        if (wave >= 8) {
            GLB_AS unsigned* tmo = (GLB_AS unsigned*)(myws + TMO_OFF);
            const char* lsrc = myws + LS_OFF + (size_t)((d & 3) * NIDX + c) * LSLOT;
            const char* lflg = myws + FL_OFF + (size_t)((d & 3) * NIDX + c) * 128;
            if (ALIGN) {
                const int dl = lane < 32 || d == 4 ? d : d - 4;
                char* st = myws + DUMMY_OFF + 2048 + dl * 256;
                unsigned t1;
                while (__builtin_amdgcn_ballot_w64((t1 = __hip_atomic_load(ctr + 24 + dl, __ATOMIC_RELAXED, __HIP_MEMORY_SCOPE_WORKGROUP)) == 0)) __builtin_amdgcn_s_sleep(4);
                if ((lane & 31) == 0) __hip_atomic_store((GLB_AS u64*)(st + c * 8), (u64)t1 << 32 | call, __ATOMIC_RELAXED, __HIP_MEMORY_SCOPE_AGENT);
                for (int it = 0; it < 8; ++it) {
                    __builtin_amdgcn_s_sleep(ASLEEP);
                    const u64 f = __hip_atomic_load((GLB_AS u64*)(st + (lane & 31) * 8), __ATOMIC_RELAXED, __HIP_MEMORY_SCOPE_AGENT);
                    if (__builtin_amdgcn_ballot_w64((unsigned)f == call) != ~0ull) continue;
                    const unsigned ref = __builtin_amdgcn_readfirstlane((unsigned)(f >> 32));
                    const unsigned sum = wave_scan((unsigned)(f >> 32) - ref);
                    const unsigned s0 = __builtin_amdgcn_readlane(sum, 31), s1 = __builtin_amdgcn_readlane(sum, 63) - s0;
                    const int ph = STAG * (((c + 5 * dl) & 31) - 16) * PACE / 32;
                    if ((lane & 31) == 0) ctr[16 + dl] = (ref + ((int)(lane ? s1 : s0) >> 5) - TBIAS + ph) | 1u;
                    break;
                }
            }
            while (lds_get(ctr + d + 4) == 0) __builtin_amdgcn_s_sleep(16);
            CBAR();
            if (lane == 0) __hip_atomic_store(cnt, call, __ATOMIC_RELAXED, __HIP_MEMORY_SCOPE_AGENT);
            const unsigned t0 = (unsigned)__builtin_amdgcn_s_memrealtime();
            unsigned pend = (1u << ntile) - 1u, fly = 0, qn = 0, finfo = 0;
            for (unsigned it = 1; pend | fly; ++it) {
                u32x2 f = {0, 0};
                asm volatile("s_mov_b64 exec, %3\n\tglobal_load_dwordx2 %0, %1, %2 sc0 sc1\n\ts_mov_b64 exec, -1\n\ts_waitcnt vmcnt(0)"
                             : "+v"(f) : "v"(lane * 4096), "s"(lflg), "s"((u64)pend) : "memory");
                if (fly) {
                    if ((fly >> (lane & 31)) & (lane < 8)) ctr[32 - 32 + d * 8 + qn + __builtin_amdgcn_mbcnt_lo(fly, 0)] = finfo | lane << 26;
                    qn += __builtin_popcount(fly);
                    LGKMCNT0();
                    if (lane == 0) __hip_atomic_store(ctr + d, qn, __ATOMIC_RELAXED, __HIP_MEMORY_SCOPE_WORKGROUP);
                }
                fly = (unsigned)__builtin_amdgcn_ballot_w64(f[0] >= call) & pend;
                finfo = f[1];
                if (!fly) {
                    if (it % TMOEVERY == 0 &&
                        ((unsigned)__builtin_amdgcn_readfirstlane(tmo_get(tmo)) == call || (unsigned)__builtin_amdgcn_s_memrealtime() - t0 > DEADLINE_TICKS)) {
                        if (lane == 0) __hip_atomic_store(tmo, call, __ATOMIC_RELAXED, __HIP_MEMORY_SCOPE_AGENT);
                        fly = pend; pend = 0; finfo = F_SKIP;
                        continue;
                    }
                    __builtin_amdgcn_s_sleep(LSLEEP);
                    continue;
                }
                pend &= ~fly;
                for (unsigned r = fly; r; r &= r - 1) {
                    const int n = __builtin_ctz(r);
                    TRACE(n, 0);
                    LOAD_LDS_SYS(wlds + n * LSLOT, voff, lsrc + (size_t)n * 32 * LSLOT);
                    LOAD_LDS_SYS(wlds + n * LSLOT + 1024, voff, lsrc + (size_t)n * 32 * LSLOT + 1024);
                }
            }
            LGKMCNT0();
            TRACE_OUT();
            __builtin_amdgcn_endpgm();
        }
        {
            char* prev_flag = nullptr;
            const char* xdst = peer + par * XS_PAR + (size_t)d * NBLK * BLK_BYTES + (SWZ ? 0 : c * 4096);
            char* fdst = peer + FX_OFF + (size_t)(d * NBLK * 32 + c) * FSTRIDE;
            unsigned tfirst = tstart * 16u + 1u, tnext = tfirst + 2 * PACE, tideal = tnext;
            if (ALIGN) ctr[24 + d] = tfirst;
            int amin = -AMAX, ash = ALIGN;
            auto pace = [&](unsigned dt) {
                int late;
                while ((late = (int)(now16() - tnext)) < 0) __builtin_amdgcn_s_sleep(1);
                tnext += dt;
                if (PSLACK && late > PSLACK) tnext += late - PSLACK;
                if (ALIGN) {
                    tideal += dt;
                    tnext += max(amin, min(AMAX, (int)(tideal - tnext) >> ash));
                }
            };
            auto align = [&]() {
                if (ALIGN && tfirst) {
                    const unsigned t = lds_get(ctr + 16 + d);
                    if (t) { tideal += t - tfirst; tfirst = 0; }
                }
            };
            auto send = [&](int blk, unsigned dt, auto nx, auto&& next) {
                const char* dst = xdst + (size_t)blk * BLK_BYTES;
                constexpr int nl = nx.value > 1;
#pragma unroll
                for (int f = 0; f < 4; ++f) {
                    const bool sent = f < 2 && nx.value != 1 && blk == (nx.value < 0 ? 0 : 16);
                    if (!sent) {
                    pace(dt);
                    STORE_WT(voff, dst + (SWZ ? UPAGE(c, f, blk) : 0), v[f], f * 1024);
                    if (f == 0) { VMCNT(1); set_flag(prev_flag, call, lane); }
                    }
                    if (((WALL >> f) & 1) && (RUSHW || nx.value >= 0 || ash == ALIGN)) VMCNT(1 + nl + (f == 1));
                    if (f == 3) align();
                    if (BURST1 && f == 0 && (blk == 0 || blk == 16)) tnext -= BURST1;
                    next(f);
                }
                prev_flag = fdst + blk * 32 * FSTRIDE;
            };
            auto xwait = [&](int blk) { lds_wait(ctr, (unsigned)(ld_pos() >> (4 * blk)) & 15u); };
            int blk, nblk, step;
            static_assert(WALL & 4, "covers the first loads of the second W tile");
            VMCNT(DIAG_N + 2);
            if (d >= 4) {
                const unsigned wl = wlds + (16 * (lane >> 5) + ((lane & 15) >> 1)) * 32 + 16 * ((lane >> 4) & 1) + 8 * (lane & 1);
                auto wfrag = [&](u32x4& o, int off) {
                    u32x2 l2, h2;
                    asm volatile("ds_read_b64_tr_b8 %0, %2\n\tds_read_b64_tr_b8 %1, %2 offset:256\n\ts_waitcnt lgkmcnt(0)"
                                 : "=&v"(l2), "=&v"(h2) : "v"(wl + off) : "memory");
                    o = u32x4{l2[0], l2[1], h2[0], h2[1]};
                };
#pragma clang loop unroll(disable)
                for (int t = 0; t < 3; ++t)
                    send(16 + t, PACE, IC<2>{}, [&](int f) {
                        LOAD_LDS(wlds + (t + 1) * 4096 + f * 1024, wo, wsrc + f * 32 * NSH + (t + 1) * 128);
                        if (f == 3) VMCNT(6);
                        wfrag(v[(f + 1) & 3], t * 4096 + (f + 1) * 1024);
                    });
                send(19, PACE, IC<1>{}, [&](int f) {
                    if (f < 3) wfrag(v[f + 1], 3 * 4096 + (f + 1) * 1024);
                });
                if (lane == 0) __hip_atomic_store(ctr + d + 4, 1u, __ATOMIC_RELAXED, __HIP_MEMORY_SCOPE_WORKGROUP);
                STAMP(3);
                blk = 15; nblk = 16 - N1; step = -1;
                xwait(blk);
#pragma unroll
                for (int f = 0; f < 4; ++f) v[f] = *(LDS_AS u32x4*)(smem + blk * 4096 + lo[f]);
            } else {
                blk = 0; nblk = N1 + N3 - (HS > 0); step = 1;
#pragma unroll
                for (int f = 0; f < 4; ++f) v[f] = *(LDS_AS u32x4*)(smem + f * 1024 + voff);
            }
            const unsigned dt = PACE;
            for (int n = 0; n < nblk; ++n) {
                int nb = n + 1 < nblk ? blk + step : blk;
                if (HS && nb == HB) ++nb;
                if (blk == (d < 4 ? RUSHB : RUSHC)) { amin = -RUSHM; ash = RUSHS; }
                if (HS && blk == (d < 4 ? HB + 1 : HPOS)) {
                    xwait(HB);
                    const unsigned po = UPAGE(c, lane >> 4, HB) + (lane >> 3) * 512 + (lane & 7) * 16;
                    const int g0 = d < 4 ? 0 : 4 - HS, g1 = d < 4 ? 4 - HS : 4;
#pragma clang loop unroll(disable)
                    for (int g = g0; g < g1; ++g) {
                        const u32x4 pv = *(LDS_AS u32x4*)(smem + HB * 4096 + g * 1024 + (lane & 7) * 128 + (((lane >> 3) ^ (lane & 7)) * 16));
                        pace(dt);
                        STORE_WT(po, xdst + (size_t)HB * BLK_BYTES + g * 128, pv, 0);
                        if (g == g0) { VMCNT(1); set_flag(prev_flag, call, lane); }
                    }
                    prev_flag = fdst + HB * 32 * FSTRIDE;
                }
                send(blk, dt, IC<-1>{}, [&](int f) {
                    if (f == 3) {
                        xwait(nb);
#pragma unroll
                        for (int g = 0; g < 4; ++g) v[g] = *(LDS_AS u32x4*)(smem + nb * 4096 + lo[g]);
                    }
                });
                blk = nb;
            }
            unsigned pinfo = 0;
            if (d >= 4) {
                STAMP(2);
                const char* rdst = peer + OS_OFF + par * OS_PAR + (size_t)((d & 3) * NIDX + c) * OSLOT;
                char* rflg = peer + FO_OFF + (size_t)((d & 3) * NIDX + c) * 128;
                char* const dummy = myws + DUMMY_OFF + (c * 8 + wave) * 8;
                for (int q = 0; q < ntile; ++q) {
                    const bool none = lds_get(ctr + d) <= (unsigned)q;
                    if (none) {
                        VMCNT(0);
                        flag64_set(prev_flag, call, pinfo, lane);
                        prev_flag = dummy;
                        lds_wait(ctr + d, q + 1);
                    }
                    CBAR();
                    const unsigned info = lds_get(ctr + d * 8 + q);
                    if (info & F_SKIP) continue;
                    const unsigned n = (info >> 26) & 7, nbytes = info & 0xFFC0;
                    v[0] = *(LDS_AS u32x4*)(size_t)(wlds + n * LSLOT + voff);
                    v[1] = *(LDS_AS u32x4*)(size_t)(wlds + n * LSLOT + 1024 + voff);
                    const char* dst = rdst + (size_t)n * 32 * OSLOT;
                    tideal = tnext;
                    pace(min(nbytes, 1024u) * PACET / 1024);
                    TRACE(n, 0);
                    asm volatile("v_cmpx_gt_u32 vcc, %3, %0\n\tglobal_store_dwordx4 %0, %2, %1 sc0 sc1\n\ts_mov_b64 exec, -1\n\ts_nop 0"
                                 :: "v"(voff), "s"(dst), "v"(v[0]), "s"(nbytes) : "memory", "vcc");
                    if (FEARLY) { VMCNT(1); flag64_set(prev_flag, call, pinfo, lane); }
                    const unsigned rest = max((int)nbytes - 1024, 16);
                    pace((rest + 64) * PACET / 1024);
                    asm volatile("v_cmpx_gt_u32 vcc, %3, %0\n\tglobal_store_dwordx4 %0, %2, %1 offset:1024 sc0 sc1\n\ts_mov_b64 exec, -1\n\ts_nop 0"
                                 :: "v"(voff), "s"(dst), "v"(v[1]), "s"(rest) : "memory", "vcc");
                    if (!FEARLY) { VMCNT(2); flag64_set(prev_flag, call, pinfo, lane); }
                    TRACE(n, 1);
                    prev_flag = rflg + (size_t)n * 32 * 128;
                    pinfo = info;
                }
            }
            VMCNT(0);
            flag64_set(prev_flag, call, pinfo, lane);
            if (lane == 0) __hip_atomic_store(cnt, call, __ATOMIC_RELAXED, __HIP_MEMORY_SCOPE_AGENT);
            STAMP(d >= 4 ? 4 : 2);
            TRACE_OUT();
        }
        __builtin_amdgcn_endpgm();
    }

    float scale;
    GLB_AS unsigned* tmo = (GLB_AS unsigned*)(myws + TMO_OFF);
    const float fnan = __builtin_nanf("");
    int rank;

    const int grp = j >> 4, tile = j & 15;
    const int foreign = role == 2 ? !(grp & 1) : 2, slot = grp >> 1;
    const int downer = 4 + (tile >> 2);
    const int su = SUEVEN ? wave * 16 * (foreign ? NSLOT : NOWN) + slot * 16 + tile : (slot * 16 + tile) * 8 + wave;
    image();
    const bool selfcopy = foreign < 2 && wave < 8 && su < 512;
    if (DELAY0 && !foreign) __builtin_amdgcn_s_sleep(DELAY0);
    if (DELAY1 && foreign == 1) __builtin_amdgcn_s_sleep(DELAY1);
    if (DELAYW) for (int i = 0; i < wave && wave < 8; ++i) __builtin_amdgcn_s_sleep(DELAYW);
    if (selfcopy) {
        const signed char* src = foreign ? xin + (size_t)(su >> 5) * BLK_BYTES + (su & 31) * 128
                                         : win + (size_t)(su >> 4) * 128 * NSH + (su & 15) * 32;
        const unsigned so = foreign ? xsrc_off : (lane >> 1) * NSH + (lane & 1) * 16;
        static_assert(8 * KDIM == 2 * 32 * NSH, "row steps");
#pragma unroll
        for (int i = 0; i < 4; ++i) LOAD_LDS((wave * 16 + i) * 1024, so, src + (i * 32 * NSH << foreign));
    }
    const int tsel = foreign ? 1 + (tile & 3) : 0;
    u64 t_lo = a.tab[tsel][slot][0], t_hi = a.tab[tsel][slot][1];
    if (lane == 0) __hip_atomic_store(cnt, call, __ATOMIC_RELAXED, __HIP_MEMORY_SCOPE_AGENT);
    auto src_d = [&](int e) { const int code = (e >> 4) & 7; return foreign && code >= 4 ? downer ^ (code & 3) : code; };
    auto blk_of = [&](int e) { return src_d(e) * NBLK + (e & 15); };

    auto rows_of = [&](unsigned e, bool frn) { return !HS || (e & 15) != HB || (!frn && !(e & 0x70)) ? 15u : e & 0x40 ? 15u & ~(15u >> HS) : 15u >> HS; };
    auto epilogue = [&](int i, int e) {
        const float bad = lds_get(ctr + 17) != 0 ? fnan : 0.0f;
        LDS_AS unsigned char* tp = smem + i * TILE_STRIDE + 128;
        const int srank = rank ^ src_d(e);
        const unsigned rows = rows_of(e, foreign);
        const float* op = (foreign ? a.outp[tile >> 2] : a.out) + (size_t)(srank * MSH + (e & 15) * 32) * NSH + (foreign ? (tile & 3) * 4 + (tile >> 2) : tile) * 32;
        unsigned vo = voff;
        asm volatile("" : "+v"(vo));
        i32x4 q[4];
#pragma unroll
        for (int k = 0; k < 4; ++k) q[k] = *(LDS_AS i32x4*)(tp + k * 1024 + vo);
        vo = (vo >> 7) * (NSH * 4) + (vo & 0x70);
#pragma unroll
        for (int k = 0; k < 4; ++k) {
            f32x4 o;
#pragma unroll
            for (int r = 0; r < 4; ++r) o[r] = fmaxf((float)q[k][r] * scale, 0.0f) + bad;
            if ((rows >> k) & 1) STORE_WT(vo, op + (size_t)k * 8 * NSH, o, 0);
        }
    };

    if (role == 2 && wave < 8) {
    bool failed = false;
    const unsigned fxo = (lane & 3) * FSTRIDE;
    auto flag_ptr = [&](int eb) {
        char* p = myws + FX_OFF + (size_t)(eb * 32 + 4 * wave) * FSTRIDE;
        asm("" : "+s"(p));
        return (GLB_AS unsigned*)(p + fxo);
    };
    unsigned aoff = 128 + (4 * (lane >> 5) * 32 + (lane & 31)) * 4;
    asm("" : "+v"(aoff));
    auto ar = __builtin_amdgcn_make_buffer_rsrc((void*)(myws + par * XS_PAR), 0, (unsigned)XS_PAR, 0x00020000);
    VMCNT(0);
    i32x4 bf[16];
    if (selfcopy) {
        u32x4 v[4];
        if (foreign) {
#pragma unroll
            for (int f = 0; f < 4; ++f) v[f] = *(LDS_AS u32x4*)(smem + wave * 16384 + lo[f]);
        } else {
            LDS_AS unsigned char* bp = smem + wave * 16384 + (16 * (lane >> 5) + ((lane & 15) >> 1)) * 32 + 16 * ((lane >> 4) & 1) + 8 * (lane & 1);
#pragma unroll
            for (int f = 0; f < 4; ++f) {
                const i32x2 l2 = __builtin_amdgcn_ds_read_tr8_b64_v2i32((LDS_AS i32x2*)(bp + f * 1024));
                const i32x2 h2 = __builtin_amdgcn_ds_read_tr8_b64_v2i32((LDS_AS i32x2*)(bp + f * 1024 + 256));
                v[f] = __builtin_bit_cast(u32x4, i32x4{l2[0], l2[1], h2[0], h2[1]});
            }
        }
        const int uu = foreign ? su : (((su >> 2) & 3) * NBLK + 16 + (su & 3)) * 32 + (su >> 4);
        const char* dst = myws + par * XS_PAR;
#pragma unroll
        for (int f = 0; f < 4; ++f) STORE_WT(voff, dst + UPAGE(uu, f, uu >> 5), v[f], f * 1024);
        VMCNT(0);
        set_flag(myws + FX_OFF + (size_t)uu * FSTRIDE, call, lane);
        if (foreign) STAMP(2);
    }
    if (wave * 16384 < NTASK * TILE_STRIDE) {
#pragma unroll
        for (int i = 0; i < 16; ++i) *(LDS_AS u32x4*)(smem + wave * 16384 + i * 1024 + voff) = u32x4{0, 0, 0, 0};
    }
    LGKMCNT0();
    if (lane == 0) __hip_atomic_fetch_add(ctr + 16, 1u, __ATOMIC_RELAXED, __HIP_MEMORY_SCOPE_WORKGROUP);
    u32x4 af[16];
    unsigned fpre;
    auto aload = [&](int k, unsigned abase) { af[k] = __builtin_amdgcn_raw_buffer_load_b128(ar, voff, ((abase | UPAGE(0, 0, abase >> 17)) ^ UPAGE(k & 3, k >> 2, 0)) + (k >> 2) * 1024, 17); };
    auto flag_off = [&](int e) { return lane & 4 ? (unsigned)(TMO_OFF - FX_OFF) : (blk_of(e) * 32 + 4 * wave) * FSTRIDE + fxo; };
    auto peek = [&](int e) { fpre = __hip_atomic_load((GLB_AS unsigned*)(myws + FX_OFF + flag_off(e)), __ATOMIC_RELAXED, __HIP_MEMORY_SCOPE_SYSTEM); };
    if (!foreign) STAMP(2);
#pragma clang loop unroll(disable)
    for (int i = -1; i < NTASK; ++i) {
        asm volatile("" : "+s"(i));
        if (i >= 0) { t_lo = (t_lo >> 8) | (t_hi << 56); t_hi = (t_hi >> 8) | (0xFFull << 56); }
        int en = (int)(t_lo & 0xFF);
        unsigned nbase = (unsigned)blk_of(en) * BLK_BYTES + wave * 16384;
        bool ready = false;
        if (i >= 0) {
        failed = failed || (unsigned)__builtin_amdgcn_readlane(fpre, 4) == call;
        ready = en != 0xFF && ((unsigned)__builtin_amdgcn_ballot_w64(fpre >= call) & 15u) == 15u;
        CBAR();
        i32x16 acc = {};
        if (ready) {
            TRACE(i + 1, 0);
            peek((int)(t_lo >> 8) & 0xFF);
#pragma unroll
            for (int k = 0; k < 16; ++k) {
                __builtin_amdgcn_sched_barrier(0);
                acc = __builtin_amdgcn_mfma_i32_32x32x32_i8(__builtin_bit_cast(i32x4, af[k]), bf[k], acc, 0, 0, 0);
                __builtin_amdgcn_sched_barrier(0);
                aload(k, nbase);
            }
            __builtin_amdgcn_sched_barrier(0);
        } else {
#pragma unroll
            for (int k = 0; k < 16; ++k)
                acc = __builtin_amdgcn_mfma_i32_32x32x32_i8(__builtin_bit_cast(i32x4, af[k]), bf[k], acc, 0, 0, 0);
        }
        if (failed && lane == 0) __hip_atomic_store(ctr + 17, 1u, __ATOMIC_RELAXED, __HIP_MEMORY_SCOPE_WORKGROUP);
        LDS_AS unsigned char* tp = smem + i * TILE_STRIDE + aoff;
#pragma unroll
        for (int r = 0; r < 16; ++r)
            __hip_atomic_fetch_add((LDS_AS int*)(tp + ((r & 3) + 8 * (r >> 2)) * 128), acc[r],
                                   __ATOMIC_RELAXED, __HIP_MEMORY_SCOPE_WORKGROUP);
        LGKMCNT0();
        if (lane == 0) __hip_atomic_fetch_add(ctr + i, 1u, __ATOMIC_RELAXED, __HIP_MEMORY_SCOPE_WORKGROUP);
        TRACE(i, 1);
        }
        if (!ready) {
            if (en == 0xFF) break;
            if (!failed) failed = !poll_flags(flag_off(en), myws + FX_OFF, tmo, call, FASTX < 0 ? 1u : (14u >> ((en >> 4) & 7)) & (((en & 15) + 16 - FASTX) >> 4), lane);
            CBAR();
            TRACE(i + 1, 0);
            peek((int)(t_lo >> 8) & 0xFF);
            __builtin_amdgcn_sched_barrier(0);
#pragma unroll
            for (int k = 0; k < 16; ++k) { aload(k, nbase); __builtin_amdgcn_sched_barrier(0); }
            __builtin_amdgcn_sched_barrier(0);
            if (i < 0) {
                {
                    const int ebw = (foreign ? downer : tile >> 2) * NBLK + 16 + (tile & 3);
                    if (!failed) failed = !wait_flags(flag_ptr(ebw), 0, tmo, call, lane);
                    CBAR();
                    if (foreign) STAMP(6);
                    peek((int)(t_lo >> 8) & 0xFF);
                    CBAR();
                    const unsigned bbase = (unsigned)ebw * BLK_BYTES + wave * 16384;
#pragma unroll
                    for (int k = 0; k < 16; ++k)
                        bf[k] = __builtin_bit_cast(i32x4, __builtin_amdgcn_raw_buffer_load_b128(ar, voff, ((bbase | UPAGE(0, 0, bbase >> 17)) ^ UPAGE(k & 3, k >> 2, 0)) + (k >> 2) * 1024, 17));
                }
                TRACE(0, 0);
            }
            if (i < 0) {
                lds_wait(ctr + 16, 8u);
                __builtin_amdgcn_s_waitcnt(0x0F70);
                STAMP(1);
            }
        }
    }
    STAMP(5);
    TRACE_OUT();
    __builtin_amdgcn_s_waitcnt(0x0F70);
    if (!foreign || slot < 3 * NRAW) __builtin_amdgcn_endpgm();
    }
    asm volatile("" : "+s"(ap));
    peer = ap->ws[4 + ((j >> 2) & 3)];
    rank = ap->rank;
    scale = __builtin_bit_cast(float, __builtin_amdgcn_readfirstlane(__builtin_bit_cast(int, *(const CONST_AS float*)ap->sx * *(const CONST_AS float*)ap->sw)));
    if (role == 2 && wave >= 8) {
        const u64 o_lo = ap->ord[tile & 3][slot][0], o_hi = ap->ord[tile & 3][slot][1];
        for (int i = wave - 8; i < NTASK; i += 4) {
            const int sh = 8 * (i & 7);
            const int e = (int)(((i & 8 ? t_hi : t_lo) >> sh) & 0xFF);
            if (e == 0xFF) break;
            const unsigned oi = (unsigned)(((i & 8 ? o_hi : o_lo) >> sh) & 0xFF), idx = (tile >> 2) * NIDX + oi;
            lds_wait(ctr + i, 8u);
            if (!foreign || (e & 0x80)) {
                TRACE(i, 0);
                if (TRAW && foreign)
                    while ((unsigned)__builtin_amdgcn_s_memrealtime() - tstart < TRAW + (oi - ap->npacked) * TRSTEP) __builtin_amdgcn_s_sleep(1);
                epilogue(i, e);
                if (foreign) {
                    VMCNT(0);
                    flag64_set(peer + FO_OFF + (size_t)idx * 128, call, 64u | (call & 63u), lane);
                }
                TRACE(i, 1);
                continue;
            }
            TRACE(i, 0);
            const bool failed = lds_get(ctr + 17) != 0;
            LDS_AS unsigned char* tp = smem + i * TILE_STRIDE;
            i32x4 q[4];
#pragma unroll
            for (int k = 0; k < 4; ++k) q[k] = *(LDS_AS i32x4*)(tp + 128 + k * 1024 + voff);
            __builtin_amdgcn_s_waitcnt(0xC07F);
            CBAR();
            if (scale < 0.0f) {
#pragma unroll
                for (int k = 0; k < 4; ++k) q[k] = -q[k];
            }
            if (HS && (e & 15) == HB) {
#pragma unroll
                for (int k = 0; k < 4; ++k)
                    if ((k >= 4 - HS) != ((e & 0x40) != 0)) q[k] = i32x4{0, 0, 0, 0};
            }
            unsigned m = 0;
            int mx = 0;
#pragma unroll
            for (int r = 15; r >= 0; --r) {
                m = __builtin_amdgcn_alignbit(m, (unsigned)-q[r >> 2][r & 3], 31);
                mx = max(mx, q[r >> 2][r & 3]);
            }
            if (P21) *(LDS_AS u32x4*)(tp + 128 + voff) = u32x4{0, 0, 0, 0};
            const bool wide = __builtin_amdgcn_ballot_w64(mx >= (1 << (P21 ? 21 : 24))) != 0;
            const unsigned c1 = __builtin_popcount(m), inc = wave_scan(c1);
            const unsigned tot = __builtin_amdgcn_readlane(inc, 63);
            const unsigned hbytes = (5 * tot + 31) >> 5 << 2;
            const unsigned nbytes = (128 + (wide ? tot * 4 : P21 ? hbytes + 2 * tot : 3 * tot) + 63) & ~63u;
            *(LDS_AS unsigned short*)(tp + lane * 2) = (unsigned short)m;
            if (wide) {
                unsigned ep = (unsigned)(size_t)tp + 128 + (inc - c1) * 4;
#pragma unroll
                for (int r = 0; r < 16; ++r)
                    asm volatile("v_cmpx_lt_i32 vcc, 0, %1\n\tds_write_b32 %0, %1\n\tv_add_u32 %0, 4, %0\n\ts_mov_b64 exec, -1"
                                 : "+v"(ep) : "v"(q[r >> 2][r & 3]) : "memory", "vcc");
            } else if (P21) {
                const unsigned tb = (unsigned)(size_t)tp + 128, ad = tb + 3072;
                unsigned a2 = tb + hbytes + 2 * (inc - c1), g = 5 * (inc - c1);
#pragma unroll
                for (int r0 = 0; r0 < 16; r0 += 6) {
                    unsigned acc = 0, sh5 = 0;
#pragma unroll
                    for (int r = r0; r < r0 + 6 && r < 16; ++r) {
                        int nb;
                        BITMASK(nb, m, r);
                        asm volatile("ds_write_b16 %0, %1" :: "v"((a2 & nb) | (ad & ~nb)), "v"(q[r >> 2][r & 3]) : "memory");
                        acc |= (((unsigned)q[r >> 2][r & 3] >> 16) & nb) << sh5;
                        STEP(a2, nb, 2);
                        STEP(sh5, nb, 5);
                    }
                    const u64 w = (u64)acc << (g & 31);
                    asm volatile("ds_or_b32 %0, %1\n\tds_or_b32 %0, %2 offset:4" :: "v"(tb + (g >> 5 << 2)), "v"((unsigned)w), "v"((unsigned)(w >> 32)) : "memory");
                    g += sh5;
                }
            } else {
                const unsigned tb = (unsigned)(size_t)tp + 128, c2 = -tb - 4 * tot, ad = tb + 2 * tot + 1536;
                unsigned a1 = tb + 2 * tot + inc - c1;
#pragma unroll
                for (int r = 0; r < 16; ++r) {
                    const unsigned nb = __builtin_amdgcn_sbfe(m, r, 1);
                    const unsigned ae = (a1 & nb) | (ad & ~nb);
                    a1 -= nb;
                    asm volatile("ds_write_b16 %0, %2\n\tds_write_b8_d16_hi %1, %2" :: "v"(ae * 2 + c2), "v"(ae), "v"(q[r >> 2][r & 3]) : "memory");
                }
            }
            const unsigned info = nbytes | (call & 63u) | (unsigned)((tile & 3) << 8 | e) << 16 | (wide ? F_WIDE : 0u) | (failed ? F_NAN : 0u);
            const bool staged = oi < NSTAGE && nbytes <= LSLOT && (unsigned)__builtin_amdgcn_s_memrealtime() - tstart < TCUT;
            if (oi < NSTAGE && !staged) flag64_set(myws + FL_OFF + (size_t)idx * 128, call, F_SKIP, lane);
            const char* dst = (staged ? myws + LS_OFF + (size_t)idx * LSLOT : peer + OS_OFF + par * OS_PAR + (size_t)idx * OSLOT) + 2048;
            TRACE(i, 1);
            if (!staged && nbytes - 1024u <= 1024u)
                while ((unsigned)__builtin_amdgcn_s_memrealtime() - tstart < TDIR + ((int)oi - NSTAGE) * TDSTEP / 16) __builtin_amdgcn_s_sleep(1);
            TRACE2(i, 0);
#pragma unroll
            for (int k = 0; k < 5; ++k)
                if (k * 1024 < nbytes) {
                    const u32x4 pv = *(LDS_AS u32x4*)(tp + k * 1024 + voff);
                    if (k * 1024 + voff < nbytes) STORE_WT(voff, dst, pv, k * 1024 - 2048);
                }
            VMCNT(0);
            flag64_set((staged ? myws + FL_OFF : peer + FO_OFF) + (size_t)idx * 128, call, info, lane);
            TRACE2(i, 1);
        }
        STAMP(5);
        TRACE_OUT();
        __builtin_amdgcn_endpgm();
    }
    {
        const int link = j & 3, kstep = role == 1 ? 48 : NIDX, kend = role == 1 ? KSPLIT : a.ntotal;
        const int k0 = role == 1 ? (j >> 2) * 12 + wave : KSPLIT + wave * 16 + (NSLOT - 1 - slot) * 4 + (tile >> 2);
        LDS_AS unsigned char* tp = smem + (role == 1 ? wave : wave + NTASK) * TILE_STRIDE;
        const int dn = 4 + link;
        const int npacked = a.npacked;
        bool failed = false;
        u64 fpre = 0;
        if (USLEEP) while ((unsigned)__builtin_amdgcn_s_memrealtime() - tstart < (36 + HS) * PACE / 16) __builtin_amdgcn_s_sleep(8);
        for (int n = 0; k0 + kstep * n < kend; ++n) {
            const int idx = link * NIDX + k0 + kstep * n;
            const unsigned info = (unsigned)fpre >= call ? (unsigned)(fpre >> 32)
                : poll_flag64(lane & 4 ? (unsigned)(TMO_OFF - FO_OFF) : idx * 128, myws + FO_OFF, tmo, call, (unsigned)(k0 + kstep * n + 256 - FASTO) >> 8, lane);
            CBAR();
            TRACE(n, 0);
            if (role == 1) STAMP(6);
            if (info == 0 || k0 + kstep * n >= npacked) {
                if (info == 0 || (unsigned)__builtin_amdgcn_readfirstlane(tmo_get(tmo)) == call) {
                    a.out[(size_t)rank * MSH * NSH + lane] = fnan;
                    break;
                }
                fpre = 0;
                TRACE(n, 1);
                continue;
            }
            const unsigned e = info >> 16;
            const int code = (e >> 4) & 7;
            const unsigned rows = rows_of(e, true);
            const int srank = rank ^ (code < 4 ? dn ^ code : code & 3);
            float* op = a.out + (size_t)(srank * MSH + (e & 15) * 32 + (lane >> 3)) * NSH + (((e >> 8) & 3) * 4 + link) * 32 + (lane & 7) * 4;
            const char* srcp = myws + OS_OFF + par * OS_PAR + (size_t)idx * OSLOT;
            const unsigned nbytes = min(info & 0xFFC0u, 4224u);
#pragma unroll
            for (int k = 0; k < 5; ++k)
                if (k * 1024 + voff < nbytes) LOAD_LDS_SYS((unsigned)(size_t)tp + k * 1024, voff, srcp + k * 1024);
            unsigned tv = tmo_get(tmo);
            VMCNT(0);
            tv = __builtin_amdgcn_readfirstlane(tv);
            asm volatile("" : "+s"(tv) :: "memory");
            fpre = 0;
            if (k0 + kstep * (n + 1) < kend) fpre = flag64_get(myws + FO_OFF + (size_t)(idx + kstep) * 128);
            failed = failed || tv == call;
            const float mul = failed || (info & F_NAN) ? fnan : __builtin_fabsf(scale);
            const unsigned m = *(LDS_AS unsigned short*)(tp + lane * 2);
            const unsigned c1 = __builtin_popcount(m), inc = wave_scan(c1);
            unsigned y[16];
            if (info & F_WIDE) {
                unsigned ix = inc - c1;
#pragma unroll
                for (int r = 0; r < 16; ++r) {
                    const int nb = __builtin_amdgcn_sbfe(m, r, 1);
                    const unsigned w = *(LDS_AS unsigned*)(tp + 128 + ix * 4);
                    ix -= nb;
                    y[r] = __builtin_bit_cast(unsigned, (float)(w & (unsigned)nb) * mul);
                }
            } else if (P21) {
                const unsigned tot = __builtin_amdgcn_readlane(inc, 63);
                const unsigned tb = (unsigned)(size_t)tp + 128;
                unsigned a2 = tb + ((5 * tot + 31) >> 5 << 2) + 2 * (inc - c1), g[3], win[3];
                u32x2 hw[3];
                int nb[16];
#pragma unroll
                for (int k = 0; k < 3; ++k) {
                    g[k] = 5 * (inc - c1 + __builtin_popcount(m & ((1u << 6 * k) - 1u)));
                    const LDS_AS unsigned* hp = (const LDS_AS unsigned*)(size_t)(tb + (g[k] >> 5 << 2));
                    hw[k] = u32x2{hp[0], hp[1]};
                }
#pragma unroll
                for (int r = 0; r < 16; ++r) {
                    y[r] = *(LDS_AS unsigned short*)(size_t)a2;
                    BITMASK(nb[r], m, r);
                    STEP(a2, nb[r], 2);
                }
#pragma unroll
                for (int r = 0; r < 16; r += 8)
                    asm volatile("" : "+v"(y[r]), "+v"(y[r + 1]), "+v"(y[r + 2]), "+v"(y[r + 3]),
                                      "+v"(y[r + 4]), "+v"(y[r + 5]), "+v"(y[r + 6]), "+v"(y[r + 7]) :: "memory");
                asm volatile("" : "+v"(hw[0]), "+v"(hw[1]), "+v"(hw[2]));
#pragma unroll
                for (int k = 0; k < 3; ++k) win[k] = __builtin_amdgcn_alignbit(hw[k][1], hw[k][0], g[k]);
                unsigned sh5 = 0;
#pragma unroll
                for (int r = 0; r < 16; ++r) {
                    if (r % 6 == 0) sh5 = 0;
                    asm("v_lshrrev_b32_sdwa %0, %1, %2 dst_sel:WORD_1 dst_unused:UNUSED_PRESERVE src0_sel:DWORD src1_sel:DWORD" : "+v"(y[r]) : "v"(sh5), "v"(win[r / 6]));
                    y[r] = __builtin_bit_cast(unsigned, (float)(y[r] & (unsigned)nb[r] & 0x1FFFFFu) * mul);
                    STEP(sh5, nb[r], 5);
                }
            } else {
                const unsigned tot = __builtin_amdgcn_readlane(inc, 63);
                const unsigned tb = (unsigned)(size_t)tp + 128, c2 = -tb - 4 * tot;
                unsigned a1 = tb + 2 * tot + inc - c1, wh[16];
#pragma unroll
                for (int r = 0; r < 16; ++r) {
                    y[r] = *(LDS_AS unsigned short*)(size_t)(a1 * 2 + c2);
                    wh[r] = *(LDS_AS unsigned char*)(size_t)a1;
                    a1 -= __builtin_amdgcn_sbfe(m, r, 1);
                }
#pragma unroll
                for (int r = 0; r < 16; r += 4)
                    asm volatile("" : "+v"(y[r]), "+v"(y[r + 1]), "+v"(y[r + 2]), "+v"(y[r + 3]),
                                      "+v"(wh[r]), "+v"(wh[r + 1]), "+v"(wh[r + 2]), "+v"(wh[r + 3]) :: "memory");
#pragma unroll
                for (int r = 0; r < 16; ++r)
                    y[r] = __builtin_bit_cast(unsigned, (float)((wh[r] << 16 | y[r]) & (unsigned)__builtin_amdgcn_sbfe(m, r, 1)) * mul);
            }
            fpre = (u64)(unsigned)__builtin_amdgcn_readfirstlane((unsigned)(fpre >> 32)) << 32 | (unsigned)__builtin_amdgcn_readfirstlane((unsigned)fpre);
            asm volatile("" : "+s"(fpre));
#pragma unroll
            for (int k = 0; k < 4; ++k) {
                const u32x4 o = {y[4 * k], y[4 * k + 1], y[4 * k + 2], y[4 * k + 3]};
                if ((rows >> k) & 1) asm volatile("global_store_dwordx4 %0, %1, off sc0 sc1\n\ts_nop 1" :: "v"(op + (size_t)k * 8 * NSH), "v"(o) : "memory");
            }
            TRACE(n, 1);
        }
        STAMP(role == 1 ? 5 : 4);
        TRACE_OUT();
    }
    __builtin_amdgcn_s_waitcnt(0x0F70);
}

static void build_table(int foreign, float tstart, u64 (*tab)[2], float (*fin)[NTASK] = nullptr) {
    struct T { float t; int e; } task[128];
    const int nslot = foreign ? NSLOT : NOWN;
    const float tb = PACE * (4 * 0.01f / 16), tw = PACE * (4 * 0.01f / 16), tx = PACE * (4 * 0.01f / 16);
    int n = 0;
    for (int d = 0; d < 8; ++d)
        for (int blk = 0; blk < 16; ++blk) {
            const int type = blk < N1 ? 1 : blk < N1 + N3 ? 3 : 2;
            float t;
            int direct = 0;
            if (d == 0) { if (foreign && type != 1) continue; t = 3.0f; }
            else if (d < 4) {
                if (type == 2 || (foreign && type == 3)) continue;
                t = T0X + tb * (blk + 1 - (blk >= HB ? HS * 0.25f : 0.0f));
                direct = foreign ? blk >= N1 - NRAW : blk == N1 + N3 - 1;
            } else {
                if ((type == 1 && !(HS && blk == HB)) || (foreign && (type == 3 || d == 4))) continue;
                t = T0X + 4 * tw + tx * (type == 1 ? 15 - HPOS : 16 - blk) + (blk <= HPOS ? tx * HS * 0.25f : 0.0f);
            }
            task[n].t = t + 0.001f * d; task[n].e = direct << 7 | d << 4 | blk; ++n;
        }
    for (int i = 1; i < n; ++i)
        for (int k = i; k > 0 && task[k].t < task[k - 1].t; --k) { T s = task[k]; task[k] = task[k - 1]; task[k - 1] = s; }
    float fr[NSLOT]; int cntv[NSLOT]; unsigned char lst[NSLOT][NTASK];
    for (int s = 0; s < NSLOT; ++s) { fr[s] = foreign || s < 4 ? tstart : OST1; cntv[s] = 0; for (int i = 0; i < NTASK; ++i) lst[s][i] = 0xFF; }
    for (int i = 0; i < n; ++i) {
        int best = -1;
        for (int s = 0; s < nslot; ++s) {
            if (foreign && task[i].t > TRES && (s < 3 * NRAW) != (task[i].e >> 7)) continue;
            if (cntv[s] < NTASK - 1 && (best < 0 || fr[s] < fr[best])) best = s;
        }
        fr[best] = (fr[best] > task[i].t ? fr[best] : task[i].t) + TCOST;
        if (fin) fin[best][cntv[best]] = fr[best];
        lst[best][cntv[best]++] = (unsigned char)task[i].e;
    }
    for (int s = 0; s < NSLOT; ++s)
        for (int h = 0; h < 2; ++h) {
            u64 v = 0;
            for (int i = 7; i >= 0; --i) v = v << 8 | lst[s][h * 8 + i];
            tab[s][h] = v;
        }
}

static void build_all(Args& a) {
    build_table(0, OST0, a.tab[0]);
    auto byte = [&](int t, int s, int i) { return i >= NTASK ? 0xFF : (int)((a.tab[1 + t][s][i >> 3] >> (8 * (i & 7))) & 0xFF); };
    int n = 0;
    float fin[4][NSLOT][NTASK];
    int next[4][NSLOT];
    for (int t = 0; t < 4; ++t) {
        build_table(1, FSTART + FSTEP * t, a.tab[1 + t], fin[t]);
        for (int s = 0; s < NSLOT; ++s) {
            for (int i = 0; i < NTASK; ++i) n += byte(t, s, i) < 0x80;
            a.ord[t][s][0] = a.ord[t][s][1] = 0;
            next[t][s] = 0;
        }
    }
    a.npacked = n;
    for (int k = 0, kf = 0, kp = 64; k < n; ++k) {
        int bt = -1, bs = 0;
        for (int t = 0; t < 4; ++t)
            for (int s = 0; s < NSLOT; ++s)
                if (byte(t, s, next[t][s]) < 0x80 && (bt < 0 || fin[t][s][next[t][s]] < fin[bt][bs][next[bt][bs]])) { bt = t; bs = s; }
        const int i = next[bt][bs]++;
        if (HS && kf == 64) kf += 28;
        a.ord[bt][bs][i >> 3] |= (u64)(HS && (byte(bt, bs, i) & 15) == HB ? kp++ : kf++) << (8 * (i & 7));
    }
    for (int blk = 0; blk < 16; ++blk)
        for (int t = 0; t < 4; ++t)
            for (int s = 0; s < NSLOT; ++s)
                for (int i = 0; i < NTASK; ++i)
                    if (byte(t, s, i) != 0xFF && (byte(t, s, i) & 0x8F) == (0x80 | blk)) a.ord[t][s][i >> 3] |= (u64)n++ << (8 * (i & 7));
    a.ntotal = n;
}

extern "C" void kernel_launch(int rank, int nranks, ncclComm_t comm,
                              hipStream_t stream,
                              void* const* d_in, const int* in_sizes, int n_in,
                              void* d_out, int out_size,
                              const int* in_dtypes, int out_dtype,
                              void* const* d_out_all,
                              void* const* d_ws_all, size_t ws_bytes) {
    if (nranks != NRANK || (unsigned)rank >= (unsigned)NRANK || n_in != 4 || ws_bytes < WS_NEED ||
        in_sizes[0] != MSH * KDIM || in_sizes[1] != KDIM * NSH || out_size != NRANK * MSH * NSH) {
        fprintf(stderr, "unexpected problem shape\n");
        return;
    }
    HIP_CHECK(hipFuncSetAttribute((const void*)ag_gemm_kernel, hipFuncAttributeMaxDynamicSharedMemorySize, LDS_BYTES));
    Args a;
    a.sx = (const float*)d_in[2];
    a.sw = (const float*)d_in[3];
    a.out = (float*)d_out;
    for (int d = 0; d < NRANK; ++d) a.ws[d] = (char*)d_ws_all[rank ^ d];
    for (int i = 0; i < 4; ++i) a.outp[i] = (float*)d_out_all[rank ^ (4 + i)];
    for (int i = 0; i < NTHREADS / 64; ++i) {
        Rec& r = a.rec[i];
        const int d = i & 7;
        r.x = (const signed char*)d_in[0];
        r.w = (const signed char*)d_in[1];
        r.ws = a.ws[0];
        r.peer = a.ws[d];
        r.src = d < 4 ? r.x : r.w + (d & 3) * 32;
        r.dst = r.peer + (size_t)(d * NBLK + (d < 4 ? 0 : 16)) * BLK_BYTES;
        r.cnt = r.ws + CNT_OFF + i * 4;
        r.dst1 = r.dst + XS_PAR;
    }
    build_all(a);
    if (a.npacked < NSTAGE || a.ntotal > NIDX || a.ntotal > KSPLIT + 128 || NSLOT - 3 * NRAW != 4) return;
    a.rank = rank;
    ag_gemm_kernel<<<dim3(256), dim3(NTHREADS), LDS_BYTES, stream>>>(a);
    HIP_CHECK(hipGetLastError());
}
```

```cpp
#include <hip/hip_runtime.h>
#include <rccl/rccl.h>
#include <cstdio>
#include <cstddef>

#define HIP_CHECK(expr) do { hipError_t _e = (expr); if (_e != hipSuccess) { \
    fprintf(stderr, "HIP error %s at %s:%d\n", hipGetErrorString(_e), __FILE__, __LINE__); \
    return; } } while (0)

typedef int i32x2 __attribute__((ext_vector_type(2)));
typedef int i32x4 __attribute__((ext_vector_type(4)));
typedef int i32x16 __attribute__((ext_vector_type(16)));
typedef unsigned u32x2 __attribute__((ext_vector_type(2)));
typedef unsigned u32x4 __attribute__((ext_vector_type(4)));
typedef float f32x4 __attribute__((ext_vector_type(4)));
typedef unsigned long long u64;
typedef unsigned __attribute__((aligned(1))) u32_u;
#define LDS_AS __attribute__((address_space(3)))
#define GLB_AS __attribute__((address_space(1)))
#define CONST_AS __attribute__((address_space(4)))

#define N1 10
#define N3 2
#define NRAW 0
#define PACE 906
#define PACEI 864
#define PACEC 880
#define GOFF 5300
#define TMIN 490
#define DEFER 1400
#define LMAX 115
#define ROT 4
#define TSOFF 500
#define TSTEP 44
#define TA 5.3f
#define FSTART 6.0f
#define FSTEP 2.2f
#define TCOST 1.3f
#define OST0 5.0f
#define TRK 3.95f
#define TRES (TA + (N1 - TRK) * PACEI / 400)
#define FSTRIDE 128
#define PSLEEP 1
#define KSPLIT 144
#define SWZ 3
#define NPOLL 3
#define NSIG 24
#define LDPOS 1
#define PSLACK 160
#define WALL 0
#define PDELAY 60
#define P21 1
#define FASTX 16
#define FASTO 256
#define DELAY1 0
#define DELAYW 12
#define OBSE 1
#define OAUX 0
#define PFIRST 1
#define NSLOW N3
#define K1 96
#define PSLOW (4 * PACEC + K1 * TSTEP / NSLOW)
#define TSTEP1 (NSLOW * PSLOW / K1)
#define KIL 1
#define ODELAY 0
#define FPOS 4
#define GOPOS 1
#define SLEEP0 2
#define SLEEP1 10
#define SLEEP2 8
#define UPAGE(c, f, blk) ((unsigned)((c) ^ ((SWZ & 1 ? (f) : 0) ^ (SWZ & 2 ? (blk) & 3 : 0))) << 12)
#define HAVE_ROLE 1
__host__ __device__ inline int role_of(int b, int& ix) {
    const int r = b >> 3;
    ix = r < 4 ? b : (b & 7) * (r < 6 ? 2 : 26) + r - (r < 6 ? 4 : 6);
    return r < 4 ? 0 : r < 6 ? 1 : 2;
}
constexpr int NRANK = 8;
constexpr int KDIM = 4096, MSH = 512, NSH = 512;
constexpr unsigned BLK_BYTES = 32 * KDIM;
constexpr int NBLK = 20;
constexpr unsigned OSLOT = 4608;
constexpr int NSLOT = 7, NOWN = 6, NTASK = 16, NIDX = 256;
constexpr size_t XS_PAR = (size_t)NRANK * NBLK * BLK_BYTES;
constexpr size_t OS_OFF = 2 * XS_PAR, OS_PAR = (size_t)4 * NIDX * OSLOT;
constexpr size_t FX_OFF = OS_OFF + 2 * OS_PAR;
constexpr size_t FO_OFF = FX_OFF + NRANK * NBLK * 32 * 128;
constexpr size_t RD_OFF = FO_OFF + 4 * NIDX * 128;
constexpr size_t CNT_OFF = RD_OFF + NRANK * 32 * 128;
constexpr size_t TMO_OFF = CNT_OFF + 16384;
constexpr size_t ST_OFF = TMO_OFF + 1024;
constexpr size_t DUMMY_OFF = ST_OFF + 256 * 128;
constexpr size_t WS_NEED = DUMMY_OFF + 256 * 128;
constexpr unsigned DEADLINE_TICKS = 20000000u;
constexpr int TILE_STRIDE = 4352;
constexpr int LDS_MAIN = 131072;
constexpr int LDS_CTR = LDS_MAIN;
constexpr int LDS_BYTES = LDS_CTR + 256;
constexpr int NTHREADS = 768;
static_assert((NTASK + 8) * TILE_STRIDE <= LDS_MAIN, "tiles");
#define DIAG_N 0
#define STAMP(i)
#define TRACE(i, h)
#define TRACE2(i, h)
#define TRACE_OUT()

struct Rec {
    const signed char* src;
    char* dst;
    const char* cnt;
    char* peer;
    const signed char* x;
    const signed char* w;
    char* ws;
    char* dst1;
};
struct Args {
    Rec rec[NTHREADS / 64];
    const float* sx;
    const float* sw;
    float* out;
    char* ws[NRANK];
    float* outp[4];
    u64 tab[5][NSLOT][2];
    u64 ord[4][NSLOT][2];
    int rank;
    int npacked, ntotal;
};

#define STORE_WT(voff, base, val, off) \
    asm volatile("global_store_dwordx4 %0, %2, %1 offset:%3 sc0 sc1\n\ts_nop 1" \
                 :: "v"(voff), "s"(base), "v"(val), "i"(off) : "memory")
#define VMCNT(n) asm volatile("s_waitcnt vmcnt(%0)" :: "i"(n) : "memory")
#define LGKMCNT0() asm volatile("s_waitcnt lgkmcnt(0)" ::: "memory")
#define CBAR() asm volatile("" ::: "memory")

__device__ __forceinline__ void set_flag(char* p, unsigned call, int lane) {
    if (lane == 0) __hip_atomic_store((GLB_AS unsigned*)p, call, __ATOMIC_RELAXED, __HIP_MEMORY_SCOPE_SYSTEM);
}
__device__ __forceinline__ unsigned lds_get(LDS_AS unsigned* p) {
    return __builtin_amdgcn_readfirstlane(__hip_atomic_load(p, __ATOMIC_RELAXED, __HIP_MEMORY_SCOPE_WORKGROUP));
}
__device__ __forceinline__ void lds_wait(LDS_AS unsigned* p, unsigned v) {
    while (lds_get(p) < v) __builtin_amdgcn_s_sleep(1);
    CBAR();
}

#define TMOEVERY 8
__device__ __forceinline__ unsigned tmo_get(GLB_AS unsigned* tmo) {
    return __hip_atomic_load(tmo, __ATOMIC_RELAXED, __HIP_MEMORY_SCOPE_AGENT);
}
__device__ __forceinline__ bool wait_flags(GLB_AS unsigned* p, unsigned f, GLB_AS unsigned* tmo, unsigned call, int lane) {
    if (((unsigned)__builtin_amdgcn_ballot_w64(f >= call) & 15u) == 15u) return true;
    const unsigned t0 = (unsigned)__builtin_amdgcn_s_memrealtime();
    for (unsigned it = 1;; ++it) {
        if (lane < 4) f = __hip_atomic_load(p, __ATOMIC_RELAXED, __HIP_MEMORY_SCOPE_SYSTEM);
        if (((unsigned)__builtin_amdgcn_ballot_w64(f >= call) & 15u) == 15u) return true;
        if (it % TMOEVERY == 0 &&
            (__builtin_amdgcn_readfirstlane(tmo_get(tmo)) == call || (unsigned)__builtin_amdgcn_s_memrealtime() - t0 > DEADLINE_TICKS)) {
            if (lane == 0) __hip_atomic_store(tmo, call, __ATOMIC_RELAXED, __HIP_MEMORY_SCOPE_AGENT);
            return false;
        }
        __builtin_amdgcn_s_sleep(PSLEEP);
    }
}

#define PDEPTH 7
#define PGAP 0
#define POLL(LOAD, CHECK) \
    "v_mov_b64 v[166:167], 0\n\ts_mov_b32 %[it], 0\n\ts_mov_b32 exec_hi, 0\n" \
    "1:\n\ts_add_u32 %[it], %[it], 1\n\ts_and_b32 vcc_lo, %[it], 31\n\ts_cselect_b32 vcc_lo, 0, 16\n\ts_or_b32 exec_lo, %[pend], vcc_lo\n\t" \
    LOAD " %[vo], %[base] sc0 sc1\n\ts_mov_b32 exec_lo, 31\n\ts_memrealtime vcc\n\ts_sleep %[gap]\n\t" \
    "s_cmp_lg_u32 %[fast], 0\n\ts_cbranch_scc1 2f\n\ts_waitcnt vmcnt(0)\n" \
    "2:\n\ts_waitcnt vmcnt(%[dep]) lgkmcnt(0)\n\ts_cmp_eq_u32 %[it], 1\n\ts_cselect_b32 %[t0], vcc_lo, %[t0]\n\ts_sub_u32 vcc_lo, vcc_lo, %[t0]\n\t" \
    "s_cmp_gt_u32 vcc_lo, %[dl]\n\ts_cbranch_scc1 3f\n\tv_cmp_le_u32 vcc, %[call], v166\n\ts_bitcmp1_b32 vcc_lo, 4\n\ts_cbranch_scc1 3f\n\t" \
    CHECK "s_andn2_b32 %[pend], %[pend], vcc_lo\n\ts_cbranch_scc1 1b\n" \
    "3:\n\ts_mov_b64 exec, -1"
__device__ __forceinline__ bool poll_flags(unsigned vo, const char* base, GLB_AS unsigned* tmo, unsigned call, unsigned fast, int lane) {
    unsigned pend = 15, it, t0;
    asm volatile(POLL("global_load_dword v166,", "")
                 : [pend] "+s"(pend), [it] "=&s"(it), [t0] "=&s"(t0)
                 : [vo] "v"(vo), [base] "s"(base), [call] "s"(call), [fast] "s"(fast), [dep] "i"(PDEPTH), [gap] "i"(PGAP), [dl] "i"(DEADLINE_TICKS)
                 : "memory", "vcc", "scc", "v166", "v167");
    if (pend && lane == 0) __hip_atomic_store(tmo, call, __ATOMIC_RELAXED, __HIP_MEMORY_SCOPE_AGENT);
    return !pend;
}
__device__ __forceinline__ unsigned poll_flag64(unsigned vo, const char* base, GLB_AS unsigned* tmo, unsigned call, unsigned fast, int lane) {
    unsigned pend = 1, it, t0, info;
    asm volatile(POLL("global_load_dwordx2 v[166:167],",
                      "v_readfirstlane_b32 %[info], v167\n\ts_xor_b32 vcc_hi, %[info], %[call]\n\ts_and_b32 vcc_hi, vcc_hi, 63\n\ts_cbranch_scc1 1b\n\t")
                 : [pend] "+s"(pend), [it] "=&s"(it), [t0] "=&s"(t0), [info] "=&s"(info)
                 : [vo] "v"(vo), [base] "s"(base), [call] "s"(call), [fast] "s"(fast), [dep] "i"(PDEPTH), [gap] "i"(PGAP), [dl] "i"(DEADLINE_TICKS)
                 : "memory", "vcc", "scc", "v166", "v167");
    if (pend && lane == 0) __hip_atomic_store(tmo, call, __ATOMIC_RELAXED, __HIP_MEMORY_SCOPE_AGENT);
    return pend ? 0 : info;
}

template <int V> struct IC { static constexpr int value = V; };
constexpr unsigned F_WIDE = 1u << 30, F_NAN = 1u << 31, F_SKIP = 1u << 29;
#define BITMASK(nb, m, r) asm("v_bfe_i32 %0, %1, %2, 1" : "=v"(nb) : "v"(m), "i"(r))
#define STEP(x, nb, n) asm("v_mad_i32_i24 %0, %1, %2, %3" : "=v"(x) : "v"(nb), "i"(-(n)), "v"(x))
__device__ __forceinline__ unsigned wave_scan(unsigned x) {
    x += __builtin_amdgcn_update_dpp(0u, x, 0x111, 0xf, 0xf, false);
    x += __builtin_amdgcn_update_dpp(0u, x, 0x112, 0xf, 0xf, false);
    x += __builtin_amdgcn_update_dpp(0u, x, 0x114, 0xf, 0xf, false);
    x += __builtin_amdgcn_update_dpp(0u, x, 0x118, 0xf, 0xf, false);
    x += __builtin_amdgcn_update_dpp(0u, x, 0x142, 0xa, 0xf, false);
    x += __builtin_amdgcn_update_dpp(0u, x, 0x143, 0xc, 0xf, false);
    return x;
}
#define LOAD_LDS_SYS(ldsaddr, voffs, base) \
    asm volatile("s_nop 0\n\tglobal_load_lds_dwordx4 %1, %2 sc0 sc1" :: "{m0}"(ldsaddr), "v"(voffs), "s"(base) : "memory")
#define LOAD_LDS(ldsaddr, voffs, base) \
    asm volatile("s_nop 0\n\tglobal_load_lds_dwordx4 %1, %2" :: "{m0}"(ldsaddr), "v"(voffs), "s"(base) : "memory")

__device__ __forceinline__ u64 flag64_get(const char* p) {
    return __hip_atomic_load((GLB_AS u64*)p, __ATOMIC_RELAXED, __HIP_MEMORY_SCOPE_SYSTEM);
}
__device__ __forceinline__ void flag64_set(char* p, unsigned call, unsigned info, int lane) {
    if (lane == 0) __hip_atomic_store((GLB_AS u64*)p, (u64)info << 32 | call, __ATOMIC_RELAXED, __HIP_MEMORY_SCOPE_SYSTEM);
}
#define IF_DIAG(...)
__global__ void __launch_bounds__(NTHREADS, 3) __attribute__((amdgpu_num_vgpr(83))) ag_gemm_kernel(Args a_) {
    const CONST_AS Args* ap = (const CONST_AS Args*)__builtin_amdgcn_kernarg_segment_ptr();
    unsigned tid = threadIdx.x, w64, call, stmp;
    int b = blockIdx.x;
    const signed char *xin, *win;
    char *myws, *peer;
    u64 t_first, t_top;
#define PRO_(...)
    asm volatile(
        IF_DIAG("s_memrealtime %[t0]\n\t") "v_readfirstlane_b32 %[w], v0\n\ts_load_dwordx16 s[36:51], %[ap], %[w]\n\t"
        "s_cmp_gt_u32 %[b], 31\n\ts_cbranch_scc1 3f\n\ts_sub_u32 %[t], %[w], 64\n\ts_cmpk_lt_u32 %[t], 192\n\ts_cbranch_scc0 2f\n\t"
        "v_lshlrev_b32 v1, 12, v0\n\tv_lshrrev_b32 v10, 1, v0\n\ts_mov_b32 %[t], 0x1f010\n\tv_bitop3_b32 v1, v1, %[t], v10 bitop3:0xc8\n\t"
        "v_lshl_add_u32 v1, %[b], 7, v1\n\ts_waitcnt lgkmcnt(0)\n\tglobal_load_dwordx4 v[2:5], v1, s[36:37]\n\t" PRO_("s_memrealtime %[t2]\n\t")
        "s_lshl_b32 %[t], %[b], 6\n\ts_load_dword %[n], s[40:41], %[t]\n\ts_barrier\n\tglobal_load_dwordx4 v[6:9], v1, s[36:37] offset:32\n\t"
        "v_lshlrev_b32 v10, 4, v0\n\tv_and_b32 v10, 0x3f0, v10\n\tv_lshl_or_b32 v10, %[b], 12, v10\n\ts_movk_i32 m0, 2048-64\n\ts_sleep %[s0]\n\t"
        "global_load_lds_dwordx4 v1, s[36:37] offset:64\n\ts_movk_i32 m0, 3072-96\n\tv_xor_b32 v11, 0x1000, v10\n\tglobal_load_lds_dwordx4 v1, s[36:37] offset:96\n\t"
        "s_waitcnt lgkmcnt(0)\n\t" PRO_("s_memrealtime %[t3]\n\t") "s_bitcmp0_b32 %[n], 0\n\ts_cselect_b64 s[38:39], s[50:51], s[38:39]\n\t"
        "s_waitcnt vmcnt(3)\n\tglobal_store_dwordx4 v10, v[2:5], s[38:39] sc0 sc1\n\ts_memrealtime %[t1]\n\ts_sleep %[s1]\n\ts_waitcnt vmcnt(3)\n\t"
        "global_store_dwordx4 v11, v[6:9], s[38:39] offset:1024 sc0 sc1\n\t" PRO_("s_memrealtime %[t4]\n\t") "s_branch 4f\n\t.p2align 6\n"
        "2:\n\ts_cmpk_gt_u32 %[t], 447\n\ts_cbranch_scc1 3f\n\t"
        "v_lshlrev_b32 v1, 8, v0\n\tv_lshlrev_b32 v10, 4, v0\n\tv_and_b32 v1, 0x3e00, v1\n\tv_and_or_b32 v1, v10, 16, v1\n\t"
        "v_lshl_add_u32 v1, %[b], 16, v1\n\ts_lshl_b32 m0, %[w], 8\n\ts_waitcnt lgkmcnt(0)\n\tglobal_load_lds_dwordx4 v1, s[36:37]\n\t"
        "s_lshl_b32 %[t], %[b], 6\n\ts_barrier\n\t" PRO_("s_memrealtime %[t2]\n\t") "s_load_dword %[n], s[40:41], %[t]\n\t"
        "s_addk_i32 m0, 0x400\n\tv_add_u32 v13, 0x4000, v1\n\tglobal_load_lds_dwordx4 v13, s[36:37]\n\t"
        "v_and_b32 v12, 0x2e0, v10\n\tv_and_or_b32 v12, v0, 16, v12\n\tv_lshlrev_b32 v2, 3, v0\n\tv_and_or_b32 v12, v2, 8, v12\n\tv_lshl_add_u32 v12, %[w], 8, v12\n\t"
        "s_addk_i32 m0, 0x400\n\tv_add_u32 v14, 0x8000, v1\n\tglobal_load_lds_dwordx4 v14, s[36:37]\n\t"
        "s_addk_i32 m0, 0x400\n\tv_add_u32 v1, 0xc000, v1\n\tglobal_load_lds_dwordx4 v1, s[36:37]\n\t"
        "v_and_b32 v10, 0x3f0, v10\n\tv_lshl_or_b32 v10, %[b], 12, v10\n\tv_xor_b32 v11, 0x1000, v10\n\t"
        "s_waitcnt lgkmcnt(0)\n\t" PRO_("s_memrealtime %[t3]\n\t") "s_bitcmp0_b32 %[n], 0\n\ts_cselect_b64 s[38:39], s[50:51], s[38:39]\n\t"
        "s_waitcnt vmcnt(3)\n\tds_read_b64_tr_b8 v[2:3], v12\n\tds_read_b64_tr_b8 v[4:5], v12 offset:256\n\ts_waitcnt lgkmcnt(0)\n\t"
        "global_store_dwordx4 v10, v[2:5], s[38:39] sc0 sc1\n\ts_memrealtime %[t1]\n\ts_sleep %[s2]\n\ts_waitcnt vmcnt(3)\n\t"
        "ds_read_b64_tr_b8 v[6:7], v12 offset:1024\n\tds_read_b64_tr_b8 v[8:9], v12 offset:1280\n\ts_waitcnt lgkmcnt(0)\n\t"
        "global_store_dwordx4 v11, v[6:9], s[38:39] offset:1024 sc0 sc1\n\t" PRO_("s_memrealtime %[t4]\n\t") "s_branch 4f\n\t.p2align 6\n"
        "3:\n\tv_lshlrev_b32 v1, 2, v0\n\tv_and_b32 v1, 0xfc, v1\n\tv_add_u32 v1, %[ctr], v1\n\tv_mov_b32 v2, 0\n\tds_write_b32 v1, v2\n\t"
        "s_lshl_b32 %[t], %[b], 6\n\ts_waitcnt lgkmcnt(0)\n\ts_barrier\n\ts_load_dword %[n], s[40:41], %[t]\n\ts_memrealtime %[t1]\n"
        "4:\n\ts_waitcnt lgkmcnt(0)"
        : [w] "=&s"(w64), [n] "=&s"(call), [t] "=&s"(stmp), [t1] "=&s"(t_first), IF_DIAG([t0] "=&s"(t_top),) [b] "+s"(b), [ap] "+s"(ap),
          PRO_([t2] "=&s"(t_ka), [t3] "=&s"(t_cnt), [t4] "=&s"(t_ack),)
          "=&{s[42:43]}"(peer), "=&{s[44:45]}"(xin), "=&{s[46:47]}"(win), "=&{s[48:49]}"(myws)
        : [ctr] "i"(LDS_CTR), [s0] "i"(SLEEP0), [s1] "i"(SLEEP1), [s2] "i"(SLEEP2), "{v0}"(tid)
        : "memory", "scc", "v1", "v2", "v3", "v4", "v5", "v6", "v7", "v8", "v9", "v10", "v11", "v12", "v13", "v14", "s36", "s37", "s38", "s39", "s40", "s41", "s50", "s51");
    asm volatile("" : "+{v0}"(tid));
    call += 1u;
    const unsigned par = call & 1u;
    const unsigned tstart = (unsigned)t_first;
    static_assert(sizeof(Rec) == 64 && (SWZ & 1) && KDIM == 4096 && NSH == 512, "prologue");
    const CONST_AS Args& a = *ap;
    LDS_AS unsigned char* smem = (LDS_AS unsigned char*)0;
    int lane = tid & 63;
    asm volatile("" : "+v"(lane));
    __builtin_assume(lane >= 0 && lane < 64);
    const int wave = w64 >> 6;
    LDS_AS unsigned* ctr = (LDS_AS unsigned*)(smem + LDS_CTR);
    int j;
    const int role = role_of(b, j);
    STAMP(7);

    GLB_AS unsigned* cnt = (GLB_AS unsigned*)(myws + CNT_OFF) + b * 16 + wave;
    const unsigned voff = lane * 16;
    unsigned xsrc_off, lo[4];
    auto image = [&]() {
        unsigned l = lane;
        asm volatile("" : "+v"(l));
        xsrc_off = (l >> 3) * KDIM + (((l & 7) ^ (l >> 3)) * 16);
#pragma unroll
        for (int f = 0; f < 4; ++f) lo[f] = ((l & 31) * 128 + (((l >> 5) ^ (l & 7)) * 16)) ^ (f * 32);
        asm volatile("" : "+v"(xsrc_off), "+v"(lo[0]), "+v"(lo[1]), "+v"(lo[2]), "+v"(lo[3]));
    };

    const unsigned wtr = (16 * (lane >> 5) + ((lane & 15) >> 1)) * 32 + 16 * ((lane >> 4) & 1) + 8 * (lane & 1);
    if (__builtin_expect(role == 0, 1)) {
        const int c = j;
        const int d = wave < 8 ? wave : wave - 4;
        GLB_AS unsigned* tmo = (GLB_AS unsigned*)(myws + TMO_OFF);
        const char* mine = myws + par * XS_PAR;
        char* fmine = myws + FX_OFF + (size_t)c * FSTRIDE;
        if (wave == 0 || wave >= 8) {
            if (wave >= 7 + NPOLL) __builtin_amdgcn_endpgm();
            for (int i = wave - 7; i > 0; --i) __builtin_amdgcn_s_sleep(14);
            __builtin_amdgcn_s_sleep(PDELAY);
            for (unsigned it = 1;; ++it) {
                u32x4 f = {call, call, call, call};
                u32x2 g = {call, call};
                asm volatile("s_mov_b64 exec, 0xfe\n\tglobal_load_dwordx4 %0, %2, %3 sc0 sc1\n\tglobal_load_dwordx2 %1, %2, %3 offset:16 sc0 sc1\n\ts_mov_b64 exec, -1\n\ts_waitcnt vmcnt(0)"
                             : "+v"(f), "+v"(g) : "v"(lane * 4096), "s"(myws + RD_OFF + c * 128) : "memory");
                unsigned lvl = f[2] >= call ? 3 : f[0] < call ? 0 : f[1] < call ? 1 : 2;
                if ((unsigned)__builtin_amdgcn_s_memrealtime() - tstart > DEADLINE_TICKS || (it % 64 == 0 && (unsigned)__builtin_amdgcn_readfirstlane(tmo_get(tmo)) == call)) {
                    if (lane == 0) __hip_atomic_store(tmo, call, __ATOMIC_RELAXED, __HIP_MEMORY_SCOPE_AGENT);
                    lvl = 4;
                    g = u32x2{call, 1};
                }
                if (lane < 8) {
                    __hip_atomic_fetch_max(ctr + 32 + lane, lvl, __ATOMIC_RELAXED, __HIP_MEMORY_SCOPE_WORKGROUP);
                    if (g[0] == call) ctr[24 + lane] = g[1];
                }
                CBAR();
                if (__builtin_amdgcn_ballot_w64(lvl >= 3 && g[0] == call) == ~0ull) break;
            }
            if (lane == 0) __hip_atomic_store(cnt, call, __ATOMIC_RELAXED, __HIP_MEMORY_SCOPE_AGENT);
            LGKMCNT0();
            STAMP(3);
            __builtin_amdgcn_endpgm();
        }
        auto now16 = [&]() { return (unsigned)__builtin_amdgcn_s_memrealtime() * 16u; };
        const int blk0 = d * NBLK + (d < 4 ? 0 : 16);
        char* prev_flag = peer + FX_OFF + (size_t)(blk0 * 32 + c) * FSTRIDE;
        const char* theirs = peer + par * XS_PAR;
        u32x4 v[4];
        if (GOPOS == 1) set_flag(peer + ST_OFF + (size_t)(d * 32 + c) * 128, call, lane);
        auto second = [&]() {
            const signed char* src = d < 4 ? xin + BLK_BYTES + c * 128 : win + (size_t)c * 128 * NSH + (d & 3) * 32 + 128;
            const unsigned so = d < 4 ? (lane & 31) * KDIM + (lane >> 5) * 16 : (lane >> 1) * NSH + (lane & 1) * 16;
            const int step = d < 4 ? 32 : 32 * NSH;
#pragma unroll
            for (int f = 0; f < 4; ++f) LOAD_LDS(wave * 16384 + 4096 + f * 1024, so, src + f * step);
        };
        if (LDPOS == 1) second();
        VMCNT(DIAG_N + 2 + (GOPOS == 1) + 4 * (LDPOS == 1));
        unsigned tn = tstart * 16u + 2 * PACE;
#pragma unroll
        for (int f = 2; f < 8; ++f) {
            if (f == 4) VMCNT(3 - LDPOS);
            if (d >= 4) {
                LDS_AS unsigned char* bp = smem + wave * 16384 + wtr + f * 1024;
                const i32x2 l2 = __builtin_amdgcn_ds_read_tr8_b64_v2i32((LDS_AS i32x2*)bp);
                const i32x2 h2 = __builtin_amdgcn_ds_read_tr8_b64_v2i32((LDS_AS i32x2*)(bp + 256));
                v[0] = __builtin_bit_cast(u32x4, i32x4{l2[0], l2[1], h2[0], h2[1]});
            } else v[0] = *(LDS_AS u32x4*)(smem + (f < 4 ? 0 : wave * 16384) + f * 1024 + voff);
            int late;
            while ((late = (int)(now16() - tn)) < 0) __builtin_amdgcn_s_sleep(1);
            tn += PACE + max(late - PSLACK, 0);
            IF_DIAG(xw += max(late - PSLACK, 0) >> 4;)
            STORE_WT(voff, theirs + (size_t)(blk0 + (f >> 2)) * BLK_BYTES + UPAGE(c, f & 3, f >> 2), v[0], (f & 3) * 1024);
            if ((WALL >> f) & 1) VMCNT(1);
            if (f == FPOS) { VMCNT(FPOS - 3); set_flag(prev_flag, call, lane); }
            if (f == LDPOS) second();
            TRACE2(f, 0);
            if (GOPOS == f) set_flag(peer + ST_OFF + (size_t)(d * 32 + c) * 128, call, lane);
        }
        prev_flag += 32 * FSTRIDE;
        TRACE(12, 0);
        char* const dummy = myws + DUMMY_OFF + (c * 8 + wave) * 128;
        const unsigned period = d < 4 ? 4 * PACEI : 4 * PACEC;
        while (!(tn = lds_get(ctr + 24 + d))) {
            if (prev_flag != dummy) { VMCNT(0); set_flag(prev_flag, call, lane); prev_flag = dummy; }
            __builtin_amdgcn_s_sleep(1);
        }
        tn += ((c + ROT * d) & 31) * period / 32;
        const int nblk = d < 4 ? N1 + N3 - 2 : 2 + 16 - N1;
#pragma clang loop unroll(disable)
        for (int n = 0; n < nblk; ++n) {
            const int blk = d < 4 ? n + 2 : n < 2 ? 18 + n : 17 - n;
            const int sblk = blk < 16 ? blk : (n + 2) * NBLK + 12 + d;
            if ((int)(tn - now16()) > DEFER) { VMCNT(0); set_flag(prev_flag, call, lane); prev_flag = dummy; }
            unsigned lvl;
            while ((lvl = lds_get(ctr + 32 + d)) < (unsigned)min(n + 1, 3)) __builtin_amdgcn_s_sleep(1);
            CBAR();
            if (lvl > 3) break;
            const char* src = theirs + (size_t)sblk * BLK_BYTES;
            unsigned ta;
            while ((int)((ta = now16()) - tn) < 0) __builtin_amdgcn_s_sleep(1);
#pragma unroll
            for (int f = 0; f < 4; ++f)
                asm volatile("global_load_dwordx4 %0, %1, %2 offset:%3 sc0 sc1" : "=&v"(v[f]) : "v"(voff), "s"(src + UPAGE(c, f, sblk)), "i"(f * 1024) : "memory");
            TRACE(n, 0);
            VMCNT(4);
            set_flag(prev_flag, call, lane);
            const char* dst = mine + (size_t)(d * NBLK + blk) * BLK_BYTES;
#pragma unroll
            for (int f = 0; f < 4; ++f) {
                asm volatile("s_waitcnt vmcnt(4)" : "+v"(v[f]) :: "memory");
                STORE_WT(voff, dst + UPAGE(c, f, blk), v[f], f * 1024);
            }
            TRACE(n, 1);
            tn += period;
            if (d >= 4 && n >= nblk - NSLOW - 1) tn += (n == nblk - NSLOW - 1 ? (c + ROT * d) & 31 : 32) * (PSLOW - 4 * PACEC) / 32;
            if (LMAX) tn += max((int)(now16() - ta) - LMAX * 16, 0);
            prev_flag = fmine + (size_t)(d * NBLK + blk) * 32 * FSTRIDE;
        }
        VMCNT(0);
        set_flag(prev_flag, call, lane);
        if (lane == 0) __hip_atomic_store(cnt, call, __ATOMIC_RELAXED, __HIP_MEMORY_SCOPE_AGENT);
        STAMP(2);
        TRACE_OUT();
        __builtin_amdgcn_endpgm();
    }

        if (role == 1 && wave == 2 && j - 1u < 7u) {
            unsigned f[4] = {0, 0, 0, 0}, el = 0, sum = 0, miss = 32;
            do {
#pragma unroll
                for (int k = 0; k < 4; ++k) {
                    asm volatile("s_mov_b32 exec_hi, 0\n\tglobal_load_dword %0, %2, %3 sc0 sc1\n\ts_mov_b32 exec_hi, -1\n\ts_sleep 3\n\ts_waitcnt vmcnt(3)"
                                 : "+v"(f[k]), "+v"(f[(k + 1) & 3]) : "v"(lane * 128), "s"(myws + ST_OFF + j * 4096) : "memory");
                    const unsigned t = (unsigned)__builtin_amdgcn_s_memrealtime() - tstart;
                    sum += (t - el) * miss;
                    el = t;
                    miss = min(miss, 32u - __builtin_popcount((unsigned)__builtin_amdgcn_ballot_w64(f[(k + 1) & 3] >= call)));
                    if (miss <= 32 - NSIG) miss = 0;
                }
            } while (miss && el < DEADLINE_TICKS);
            asm volatile("s_waitcnt vmcnt(0)" : "+v"(f[0]), "+v"(f[1]), "+v"(f[2]), "+v"(f[3]) :: "memory");
            const unsigned t = tstart * 16u + max((sum >> 1) + GOFF, TMIN * 16u);
            if (lane < 32) __hip_atomic_store((GLB_AS u64*)(myws + RD_OFF + (j * 32 + lane) * 128 + 16), (u64)(t | 1u) << 32 | call, __ATOMIC_RELAXED, __HIP_MEMORY_SCOPE_SYSTEM);
            STAMP(2);
        }
    float scale;
    GLB_AS unsigned* tmo = (GLB_AS unsigned*)(myws + TMO_OFF);
    const float fnan = __builtin_nanf("");
    int rank;

    const int grp = j >> 4, tile = j & 15;
    const int foreign = role == 2 ? !(grp & 1) : 2, slot = grp >> 1;
    const int downer = 4 + (tile >> 2);
    const int su = wave * 208 + j, sq = su >> 5;
    const int se = sq < 10 ? (sq % 5 ? 23 + sq - sq / 5 : 2 + sq / 5) : sq < 18 ? 6 + sq : sq < 20 ? sq - 18 : sq - 16;
    const int sc = se < 16 ? su & 31 : (se & 3) * 8 + ((su & 31) >> 2);
    const int uu = (se < 16 ? se : ((se >> 2) & 3) * NBLK + 16 + (su & 3)) * 32 + sc;
    image();
    const bool selfcopy = role == 2 && su < 1024;
    if (DELAY1) __builtin_amdgcn_s_sleep(DELAY1);
    if (DELAYW) for (int i = 0; i < wave && wave < 8; ++i) __builtin_amdgcn_s_sleep(DELAYW);
    if (selfcopy) {
        const signed char* src = se < 16 ? xin + (size_t)se * BLK_BYTES + sc * 128 : win + (size_t)sc * 128 * NSH + (se & 12) * 32 + (su & 3) * 32;
        const unsigned so = se < 16 ? xsrc_off : (lane >> 1) * NSH + (lane & 1) * 16;
        static_assert(8 * KDIM == 2 * 32 * NSH, "row steps");
#pragma unroll
        for (int i = 0; i < 4; ++i) LOAD_LDS((wave * 16 + i) * 1024, so, src + (i * 64 * NSH >> (se >> 4)));
    }
    const int tsel = foreign ? 1 + (tile & 3) : 0;
    u64 t_lo = a.tab[tsel][slot][0], t_hi = a.tab[tsel][slot][1];
    if (lane == 0) __hip_atomic_store(cnt, call, __ATOMIC_RELAXED, __HIP_MEMORY_SCOPE_AGENT);
    auto src_d = [&](int e) { const int code = (e >> 4) & 7; return foreign && code >= 4 ? downer ^ (code & 3) : code; };
    auto blk_of = [&](int e) { return src_d(e) * NBLK + (e & 15); };

    auto epilogue = [&](int i, int e) {
        const float bad = lds_get(ctr + 17) != 0 ? fnan : 0.0f;
        LDS_AS unsigned char* tp = smem + i * TILE_STRIDE + 128;
        const int srank = rank ^ src_d(e);
        const float* op = (foreign ? a.outp[tile >> 2] : a.out) + (size_t)(srank * MSH + (e & 15) * 32) * NSH + (foreign ? (tile & 3) * 4 + (tile >> 2) : tile) * 32;
        unsigned vo = voff;
        asm volatile("" : "+v"(vo));
        i32x4 q[4];
#pragma unroll
        for (int k = 0; k < 4; ++k) q[k] = *(LDS_AS i32x4*)(tp + k * 1024 + vo);
        vo = (vo >> 7) * (NSH * 4) + (vo & 0x70);
#pragma unroll
        for (int k = 0; k < 4; ++k) {
            f32x4 o;
#pragma unroll
            for (int r = 0; r < 4; ++r) o[r] = fmaxf((float)q[k][r] * scale, 0.0f) + bad;
            STORE_WT(vo, op + (size_t)k * 8 * NSH, o, 0);
        }
    };

    if (role == 2 && wave < 8) {
    bool failed = false;
    const unsigned fxo = (lane & 3) * FSTRIDE;
    auto flag_ptr = [&](int eb) {
        char* p = myws + FX_OFF + (size_t)(eb * 32 + 4 * wave) * FSTRIDE;
        asm("" : "+s"(p));
        return (GLB_AS unsigned*)(p + fxo);
    };
    unsigned aoff = 128 + (4 * (lane >> 5) * 32 + (lane & 31)) * 4;
    asm("" : "+v"(aoff));
    auto ar = __builtin_amdgcn_make_buffer_rsrc((void*)(myws + par * XS_PAR), 0, (unsigned)XS_PAR, 0x00020000);
    i32x4 bf[16];
    if (selfcopy) {
        char* fp = a.ws[lane & 7] + (lane ? RD_OFF + (lane * 32 + sc) * 128 + sq / 5 * 4 : FX_OFF + (size_t)uu * FSTRIDE);
        asm volatile("s_waitcnt vmcnt(0)" : "+v"(fp) :: "memory");
        u32x4 v[4];
        if (se < 16) {
#pragma unroll
            for (int f = 0; f < 4; ++f) v[f] = *(LDS_AS u32x4*)(smem + wave * 16384 + lo[f]);
        } else {
            LDS_AS unsigned char* bp = smem + wave * 16384 + wtr;
#pragma unroll
            for (int f = 0; f < 4; ++f) {
                const i32x2 l2 = __builtin_amdgcn_ds_read_tr8_b64_v2i32((LDS_AS i32x2*)(bp + f * 1024));
                const i32x2 h2 = __builtin_amdgcn_ds_read_tr8_b64_v2i32((LDS_AS i32x2*)(bp + f * 1024 + 256));
                v[f] = __builtin_bit_cast(u32x4, i32x4{l2[0], l2[1], h2[0], h2[1]});
            }
        }
        const char* dst = myws + par * XS_PAR;
#pragma unroll
        for (int f = 0; f < 4; ++f) STORE_WT(voff, dst + UPAGE(uu, f, uu >> 5), v[f], f * 1024);
        VMCNT(0);
        if (lane < 8 && (((sq >= 10 ? 1u : se < 16 ? 15u : 1u | 16u << (su & 3)) >> lane) & 1)) __hip_atomic_store((GLB_AS unsigned*)fp, call, __ATOMIC_RELAXED, __HIP_MEMORY_SCOPE_SYSTEM);
        if (foreign) STAMP(2);
    }
    if (wave * 16384 < NTASK * TILE_STRIDE) {
#pragma unroll
        for (int i = 0; i < 16; ++i) *(LDS_AS u32x4*)(smem + wave * 16384 + i * 1024 + voff) = u32x4{0, 0, 0, 0};
    }
    LGKMCNT0();
    if (lane == 0) __hip_atomic_fetch_add(ctr + 16, 1u, __ATOMIC_RELAXED, __HIP_MEMORY_SCOPE_WORKGROUP);
    u32x4 af[16];
    unsigned fpre;
    auto aload = [&](int k, unsigned abase) { af[k] = __builtin_amdgcn_raw_buffer_load_b128(ar, voff, ((abase | UPAGE(0, 0, abase >> 17)) ^ UPAGE(k & 3, k >> 2, 0)) + (k >> 2) * 1024, 17); };
    auto flag_off = [&](int e) { return lane & 4 ? (unsigned)(TMO_OFF - FX_OFF) : (blk_of(e) * 32 + 4 * wave) * FSTRIDE + fxo; };
    auto peek = [&](int e) { fpre = __hip_atomic_load((GLB_AS unsigned*)(myws + FX_OFF + flag_off(e)), __ATOMIC_RELAXED, __HIP_MEMORY_SCOPE_SYSTEM); };
    if (!foreign) STAMP(2);
    if (ODELAY && !foreign) while ((unsigned)__builtin_amdgcn_s_memrealtime() - tstart < ODELAY) __builtin_amdgcn_s_sleep(8);
#pragma clang loop unroll(disable)
    for (int i = -1; i < NTASK; ++i) {
        asm volatile("" : "+s"(i));
        if (i >= 0) { t_lo = (t_lo >> 8) | (t_hi << 56); t_hi = (t_hi >> 8) | (0xFFull << 56); }
        int en = (int)(t_lo & 0xFF);
        unsigned nbase = (unsigned)blk_of(en) * BLK_BYTES + wave * 16384;
        bool ready = false;
        if (i >= 0) {
        failed = failed || (unsigned)__builtin_amdgcn_readlane(fpre, 4) == call;
        ready = en != 0xFF && ((unsigned)__builtin_amdgcn_ballot_w64(fpre >= call) & 15u) == 15u;
        CBAR();
        i32x16 acc = {};
        if (ready) {
            TRACE(i + 1, 0);
            peek((int)(t_lo >> 8) & 0xFF);
#pragma unroll
            for (int k = 0; k < 16; ++k) {
                __builtin_amdgcn_sched_barrier(0);
                acc = __builtin_amdgcn_mfma_i32_32x32x32_i8(__builtin_bit_cast(i32x4, af[k]), bf[k], acc, 0, 0, 0);
                __builtin_amdgcn_sched_barrier(0);
                aload(k, nbase);
            }
            __builtin_amdgcn_sched_barrier(0);
        } else {
#pragma unroll
            for (int k = 0; k < 16; ++k)
                acc = __builtin_amdgcn_mfma_i32_32x32x32_i8(__builtin_bit_cast(i32x4, af[k]), bf[k], acc, 0, 0, 0);
        }
        if (failed && lane == 0) __hip_atomic_store(ctr + 17, 1u, __ATOMIC_RELAXED, __HIP_MEMORY_SCOPE_WORKGROUP);
        LDS_AS unsigned char* tp = smem + i * TILE_STRIDE + aoff;
#pragma unroll
        for (int r = 0; r < 16; ++r)
            __hip_atomic_fetch_add((LDS_AS int*)(tp + ((r & 3) + 8 * (r >> 2)) * 128), acc[r],
                                   __ATOMIC_RELAXED, __HIP_MEMORY_SCOPE_WORKGROUP);
        LGKMCNT0();
        if (lane == 0) __hip_atomic_fetch_add(ctr + i, 1u, __ATOMIC_RELAXED, __HIP_MEMORY_SCOPE_WORKGROUP);
        TRACE(i, 1);
        }
        if (!ready) {
            if (en == 0xFF) break;
            if (!failed) failed = !poll_flags(flag_off(en), myws + FX_OFF, tmo, call, FASTX < 0 ? 1u : (14u >> ((en >> 4) & 7)) & (((en & 15) + 16 - FASTX) >> 4), lane);
            CBAR();
            TRACE(i + 1, 0);
            peek((int)(t_lo >> 8) & 0xFF);
            __builtin_amdgcn_sched_barrier(0);
#pragma unroll
            for (int k = 0; k < 16; ++k) { aload(k, nbase); __builtin_amdgcn_sched_barrier(0); }
            __builtin_amdgcn_sched_barrier(0);
            if (i < 0) {
                {
                    const int ebw = (foreign ? downer : tile >> 2) * NBLK + 16 + (tile & 3);
                    if (!failed) failed = !wait_flags(flag_ptr(ebw), 0, tmo, call, lane);
                    CBAR();
                    if (foreign) STAMP(6);
                    peek((int)(t_lo >> 8) & 0xFF);
                    CBAR();
                    const unsigned bbase = (unsigned)ebw * BLK_BYTES + wave * 16384;
#pragma unroll
                    for (int k = 0; k < 16; ++k)
                        bf[k] = __builtin_bit_cast(i32x4, __builtin_amdgcn_raw_buffer_load_b128(ar, voff, ((bbase | UPAGE(0, 0, bbase >> 17)) ^ UPAGE(k & 3, k >> 2, 0)) + (k >> 2) * 1024, 17));
                }
                TRACE(0, 0);
            }
            if (i < 0) {
                lds_wait(ctr + 16, 8u);
                __builtin_amdgcn_s_waitcnt(0x0F70);
                STAMP(1);
            }
        }
    }
    STAMP(5);
    TRACE_OUT();
    __builtin_amdgcn_s_waitcnt(0x0F70);
    if (!foreign || slot < 3) __builtin_amdgcn_endpgm();
    }
    asm volatile("" : "+s"(ap));
    peer = ap->ws[4 + ((j >> 2) & 3)];
    rank = ap->rank;
    scale = __builtin_bit_cast(float, __builtin_amdgcn_readfirstlane(__builtin_bit_cast(int, *(const CONST_AS float*)ap->sx * *(const CONST_AS float*)ap->sw)));
    if (role == 2 && wave >= 8 && (!OAUX || foreign || wave < 10)) {
        const u64 o_lo = ap->ord[tile & 3][slot][0], o_hi = ap->ord[tile & 3][slot][1];
        for (int i = wave - 8; i < NTASK; i += OAUX && !foreign ? 2 : 4) {
            const int sh = 8 * (i & 7);
            const int e = (int)(((i & 8 ? t_hi : t_lo) >> sh) & 0xFF);
            if (e == 0xFF) break;
            const unsigned oi = (unsigned)(((i & 8 ? o_hi : o_lo) >> sh) & 0xFF), idx = (tile >> 2) * NIDX + oi;
            lds_wait(ctr + i, 8u);
            if (!foreign || (e & 0x80)) {
                TRACE(i, 0);
                epilogue(i, e);
                if (foreign) {
                    VMCNT(0);
                    flag64_set(peer + FO_OFF + (size_t)idx * 128, call, 64u | (call & 63u), lane);
                }
                TRACE(i, 1);
                continue;
            }
            TRACE(i, 0);
            const bool failed = lds_get(ctr + 17) != 0;
            LDS_AS unsigned char* tp = smem + i * TILE_STRIDE;
            i32x4 q[4];
#pragma unroll
            for (int k = 0; k < 4; ++k) q[k] = *(LDS_AS i32x4*)(tp + 128 + k * 1024 + voff);
            __builtin_amdgcn_s_waitcnt(0xC07F);
            CBAR();
            if (scale < 0.0f) {
#pragma unroll
                for (int k = 0; k < 4; ++k) q[k] = -q[k];
            }
            unsigned m = 0;
            int mx = 0;
#pragma unroll
            for (int r = 15; r >= 0; --r) {
                m = __builtin_amdgcn_alignbit(m, (unsigned)-q[r >> 2][r & 3], 31);
                mx = max(mx, q[r >> 2][r & 3]);
            }
            if (P21) *(LDS_AS u32x4*)(tp + 128 + voff) = u32x4{0, 0, 0, 0};
            const bool wide = __builtin_amdgcn_ballot_w64(mx >= (1 << (P21 ? 21 : 24))) != 0;
            const unsigned c1 = __builtin_popcount(m), inc = wave_scan(c1);
            const unsigned tot = __builtin_amdgcn_readlane(inc, 63);
            const unsigned hbytes = (5 * tot + 31) >> 5 << 2;
            const unsigned nbytes = (128 + (wide ? tot * 4 : P21 ? hbytes + 2 * tot : 3 * tot) + 63) & ~63u;
            *(LDS_AS unsigned short*)(tp + lane * 2) = (unsigned short)m;
            if (wide) {
                unsigned ep = (unsigned)(size_t)tp + 128 + (inc - c1) * 4;
#pragma unroll
                for (int r = 0; r < 16; ++r)
                    asm volatile("v_cmpx_lt_i32 vcc, 0, %1\n\tds_write_b32 %0, %1\n\tv_add_u32 %0, 4, %0\n\ts_mov_b64 exec, -1"
                                 : "+v"(ep) : "v"(q[r >> 2][r & 3]) : "memory", "vcc");
            } else if (P21) {
                const unsigned tb = (unsigned)(size_t)tp + 128, ad = tb + 3072;
                unsigned a2 = tb + hbytes + 2 * (inc - c1), g = 5 * (inc - c1);
#pragma unroll
                for (int r0 = 0; r0 < 16; r0 += 6) {
                    unsigned acc = 0, sh5 = 0;
#pragma unroll
                    for (int r = r0; r < r0 + 6 && r < 16; ++r) {
                        int nb;
                        BITMASK(nb, m, r);
                        asm volatile("ds_write_b16 %0, %1" :: "v"((a2 & nb) | (ad & ~nb)), "v"(q[r >> 2][r & 3]) : "memory");
                        acc |= (((unsigned)q[r >> 2][r & 3] >> 16) & nb) << sh5;
                        STEP(a2, nb, 2);
                        STEP(sh5, nb, 5);
                    }
                    const u64 w = (u64)acc << (g & 31);
                    asm volatile("ds_or_b32 %0, %1\n\tds_or_b32 %0, %2 offset:4" :: "v"(tb + (g >> 5 << 2)), "v"((unsigned)w), "v"((unsigned)(w >> 32)) : "memory");
                    g += sh5;
                }
            } else {
                const unsigned tb = (unsigned)(size_t)tp + 128, c2 = -tb - 4 * tot, ad = tb + 2 * tot + 1536;
                unsigned a1 = tb + 2 * tot + inc - c1;
#pragma unroll
                for (int r = 0; r < 16; ++r) {
                    const unsigned nb = __builtin_amdgcn_sbfe(m, r, 1);
                    const unsigned ae = (a1 & nb) | (ad & ~nb);
                    a1 -= nb;
                    asm volatile("ds_write_b16 %0, %2\n\tds_write_b8_d16_hi %1, %2" :: "v"(ae * 2 + c2), "v"(ae), "v"(q[r >> 2][r & 3]) : "memory");
                }
            }
            const unsigned info = nbytes | (call & 63u) | (unsigned)((tile & 3) << 8 | e) << 16 | (wide ? F_WIDE : 0u) | (failed ? F_NAN : 0u);
            const char* dst = myws + OS_OFF + par * OS_PAR + (size_t)idx * OSLOT + 2048;
            TRACE(i, 1);
            TRACE2(i, 0);
#pragma unroll
            for (int k = 0; k < 5; ++k)
                if (k * 1024 < nbytes) {
                    const u32x4 pv = *(LDS_AS u32x4*)(tp + k * 1024 + voff);
                    if (k * 1024 + voff < nbytes) STORE_WT(voff, dst, pv, k * 1024 - 2048);
                }
            VMCNT(0);
            flag64_set(peer + FO_OFF + (size_t)idx * 128, call, info, lane);
            TRACE2(i, 1);
        }
        STAMP(5);
        TRACE_OUT();
        __builtin_amdgcn_endpgm();
    }
    {
        const int link = j & 3, kstep = role == 1 ? 48 << OAUX : NIDX, kend = role == 1 ? KSPLIT : a.ntotal;
        const int k0 = role == 1 ? (KIL ? wave * 4 + (j >> 2) : (j >> 2) * 12 + wave) : OAUX && !foreign ? 48 + (wave - 10) * 24 + slot * 4 + (tile >> 2) : KSPLIT + wave * 16 + (NSLOT - 1 - slot) * 4 + (tile >> 2);
        LDS_AS unsigned char* tp = smem + (role == 1 ? wave : (wave & 7) + NTASK) * TILE_STRIDE;
        const int dn = 4 + link;
        const int npacked = a.npacked;
        bool failed = false;
        u64 fpre = 0;
        if (role == 1 && wave < 2) {
            const int c = j * 2 + wave;
            GLB_AS unsigned* rdy = (GLB_AS unsigned*)(a.ws[lane & 7] + RD_OFF + ((lane & 7) * 32 + c) * 128);
            GLB_AS unsigned* fl = (GLB_AS unsigned*)(myws + FX_OFF + (size_t)((lane & 16 ? ((lane >> 2) & 3) * NBLK + 16 + (lane & 3) : lane & 15) * 32 + c) * FSTRIDE);
            while ((unsigned)__builtin_amdgcn_s_memrealtime() - tstart < 300) __builtin_amdgcn_s_sleep(8);
            unsigned f = lane < 32 ? 0 : call;
            do {
                if (f < call) f = __hip_atomic_load(fl, __ATOMIC_RELAXED, __HIP_MEMORY_SCOPE_SYSTEM);
                if (__builtin_amdgcn_ballot_w64(f >= call) == ~0ull) {
                    CBAR();
                    if (lane - 1u < 7u) __hip_atomic_store(rdy + 2, call, __ATOMIC_RELAXED, __HIP_MEMORY_SCOPE_SYSTEM);
                    break;
                }
            } while ((unsigned)__builtin_amdgcn_s_memrealtime() - tstart < DEADLINE_TICKS);
            STAMP(2);
        }
        while ((unsigned)__builtin_amdgcn_s_memrealtime() - tstart < 1600) __builtin_amdgcn_s_sleep(8);
        const u64 t64 = flag64_get(myws + RD_OFF + (dn * 32 + (k0 & 31)) * 128 + 16);
        const unsigned turn = __builtin_amdgcn_readfirstlane((unsigned)t64 == call ? (unsigned)(t64 >> 32) + (2 + 16 - N1 - NSLOW) * 4 * PACEC + TSOFF : tstart * 16u);
        const char* theirs = a.ws[dn] + OS_OFF + par * OS_PAR;
        for (int n = 0; k0 + kstep * n < kend; ++n) {
            const int idx = link * NIDX + k0 + kstep * n;
            const unsigned toff = (k0 + kstep * n) * TSTEP + min(k0 + kstep * n, K1) * (TSTEP1 - TSTEP);
            if (!PFIRST)
                while ((int)((unsigned)__builtin_amdgcn_s_memrealtime() * 16u - turn - toff) < 0) __builtin_amdgcn_s_sleep(1);
            else if (n == 0)
                while ((int)((unsigned)__builtin_amdgcn_s_memrealtime() * 16u - turn - toff + 2400) < 0) __builtin_amdgcn_s_sleep(1);
            const unsigned info = (unsigned)fpre >= call ? (unsigned)(fpre >> 32)
                : poll_flag64(lane & 4 ? (unsigned)(TMO_OFF - FO_OFF) : idx * 128, myws + FO_OFF, tmo, call, (unsigned)(k0 + kstep * n + 256 - FASTO) >> 8, lane);
            CBAR();
            if (PFIRST)
                while ((int)((unsigned)__builtin_amdgcn_s_memrealtime() * 16u - turn - toff) < 0) __builtin_amdgcn_s_sleep(1);
            TRACE(n, 0);
            if (role == 1) STAMP(6);
            if (info == 0 || k0 + kstep * n >= npacked) {
                if (info == 0 || (unsigned)__builtin_amdgcn_readfirstlane(tmo_get(tmo)) == call) {
                    a.out[(size_t)rank * MSH * NSH + lane] = fnan;
                    break;
                }
                fpre = 0;
                TRACE(n, 1);
                continue;
            }
            const unsigned e = info >> 16;
            const int code = (e >> 4) & 7;
            const int srank = rank ^ (code < 4 ? dn ^ code : code & 3);
            float* op = a.out + (size_t)(srank * MSH + (e & 15) * 32 + (lane >> 3)) * NSH + (((e >> 8) & 3) * 4 + link) * 32 + (lane & 7) * 4;
            const char* srcp = theirs + (size_t)idx * OSLOT;
            const unsigned nbytes = min(info & 0xFFC0u, 4224u);
#pragma unroll
            for (int k = 0; k < 5; ++k)
                if (k * 1024 + voff < nbytes) LOAD_LDS_SYS((unsigned)(size_t)tp + k * 1024, voff, srcp + k * 1024);
            unsigned tv = tmo_get(tmo);
            VMCNT(0);
            tv = __builtin_amdgcn_readfirstlane(tv);
            asm volatile("" : "+s"(tv) :: "memory");
            fpre = 0;
            if (k0 + kstep * (n + 1) < kend) fpre = flag64_get(myws + FO_OFF + (size_t)(idx + kstep) * 128);
            failed = failed || tv == call;
            const float mul = failed || (info & F_NAN) ? fnan : __builtin_fabsf(scale);
            const unsigned m = *(LDS_AS unsigned short*)(tp + lane * 2);
            const unsigned c1 = __builtin_popcount(m), inc = wave_scan(c1);
            unsigned y[16];
            if (info & F_WIDE) {
                unsigned ix = inc - c1;
#pragma unroll
                for (int r = 0; r < 16; ++r) {
                    const int nb = __builtin_amdgcn_sbfe(m, r, 1);
                    const unsigned w = *(LDS_AS unsigned*)(tp + 128 + ix * 4);
                    ix -= nb;
                    y[r] = __builtin_bit_cast(unsigned, (float)(w & (unsigned)nb) * mul);
                }
            } else if (P21) {
                const unsigned tot = __builtin_amdgcn_readlane(inc, 63);
                const unsigned tb = (unsigned)(size_t)tp + 128;
                unsigned a2 = tb + ((5 * tot + 31) >> 5 << 2) + 2 * (inc - c1), g[3], win[3];
                u32x2 hw[3];
                int nb[16];
#pragma unroll
                for (int k = 0; k < 3; ++k) {
                    g[k] = 5 * (inc - c1 + __builtin_popcount(m & ((1u << 6 * k) - 1u)));
                    const LDS_AS unsigned* hp = (const LDS_AS unsigned*)(size_t)(tb + (g[k] >> 5 << 2));
                    hw[k] = u32x2{hp[0], hp[1]};
                }
#pragma unroll
                for (int r = 0; r < 16; ++r) {
                    y[r] = *(LDS_AS unsigned short*)(size_t)a2;
                    BITMASK(nb[r], m, r);
                    STEP(a2, nb[r], 2);
                }
#pragma unroll
                for (int r = 0; r < 16; r += 8)
                    asm volatile("" : "+v"(y[r]), "+v"(y[r + 1]), "+v"(y[r + 2]), "+v"(y[r + 3]),
                                      "+v"(y[r + 4]), "+v"(y[r + 5]), "+v"(y[r + 6]), "+v"(y[r + 7]) :: "memory");
                asm volatile("" : "+v"(hw[0]), "+v"(hw[1]), "+v"(hw[2]));
#pragma unroll
                for (int k = 0; k < 3; ++k) win[k] = __builtin_amdgcn_alignbit(hw[k][1], hw[k][0], g[k]);
                unsigned sh5 = 0;
#pragma unroll
                for (int r = 0; r < 16; ++r) {
                    if (r % 6 == 0) sh5 = 0;
                    asm("v_lshrrev_b32_sdwa %0, %1, %2 dst_sel:WORD_1 dst_unused:UNUSED_PRESERVE src0_sel:DWORD src1_sel:DWORD" : "+v"(y[r]) : "v"(sh5), "v"(win[r / 6]));
                    y[r] = __builtin_bit_cast(unsigned, (float)(y[r] & (unsigned)nb[r] & 0x1FFFFFu) * mul);
                    STEP(sh5, nb[r], 5);
                }
            } else {
                const unsigned tot = __builtin_amdgcn_readlane(inc, 63);
                const unsigned tb = (unsigned)(size_t)tp + 128, c2 = -tb - 4 * tot;
                unsigned a1 = tb + 2 * tot + inc - c1, wh[16];
#pragma unroll
                for (int r = 0; r < 16; ++r) {
                    y[r] = *(LDS_AS unsigned short*)(size_t)(a1 * 2 + c2);
                    wh[r] = *(LDS_AS unsigned char*)(size_t)a1;
                    a1 -= __builtin_amdgcn_sbfe(m, r, 1);
                }
#pragma unroll
                for (int r = 0; r < 16; r += 4)
                    asm volatile("" : "+v"(y[r]), "+v"(y[r + 1]), "+v"(y[r + 2]), "+v"(y[r + 3]),
                                      "+v"(wh[r]), "+v"(wh[r + 1]), "+v"(wh[r + 2]), "+v"(wh[r + 3]) :: "memory");
#pragma unroll
                for (int r = 0; r < 16; ++r)
                    y[r] = __builtin_bit_cast(unsigned, (float)((wh[r] << 16 | y[r]) & (unsigned)__builtin_amdgcn_sbfe(m, r, 1)) * mul);
            }
            fpre = (u64)(unsigned)__builtin_amdgcn_readfirstlane((unsigned)(fpre >> 32)) << 32 | (unsigned)__builtin_amdgcn_readfirstlane((unsigned)fpre);
            asm volatile("" : "+s"(fpre));
#pragma unroll
            for (int k = 0; k < 4; ++k) {
                const u32x4 o = {y[4 * k], y[4 * k + 1], y[4 * k + 2], y[4 * k + 3]};
                asm volatile("global_store_dwordx4 %0, %1, off sc0 sc1\n\ts_nop 1" :: "v"(op + (size_t)k * 8 * NSH), "v"(o) : "memory");
            }
            TRACE(n, 1);
        }
        STAMP(role == 1 ? 5 : 4);
        TRACE_OUT();
    }
    __builtin_amdgcn_s_waitcnt(0x0F70);
}

static void build_table(int foreign, float tstart, u64 (*tab)[2], float (*fin)[NTASK] = nullptr) {
    struct T { float t; int e, last; } task[128];
    const int nslot = foreign ? NSLOT : NOWN;
    const float tb = PACEI / 400.0f, tx = PACEC / 400.0f;
    int n = 0;
    for (int d = 0; d < 8; ++d)
        for (int blk = 0; blk < 16; ++blk) {
            const int type = blk < N1 ? 1 : blk < N1 + N3 ? 3 : 2;
            float t;
            int direct = 0;
            if (d == 0) { if (foreign && type != 1) continue; t = 3.0f; }
            else if (d < 4) {
                if (type == 2 || (foreign && type == 3)) continue;
                t = TA + tb * blk;
                direct = foreign ? blk >= N1 - NRAW : blk == N1 + N3 - 1;
            } else {
                if (type == 1 || (foreign && (type == 3 || d == 4))) continue;
                t = TA + tx * (19 - blk) + (blk < N1 + NSLOW ? (N1 + NSLOW - blk) * (PSLOW / 1600.0f - tx) : 0.0f);
            }
            task[n].t = t + 0.001f * d; task[n].e = direct << 7 | d << 4 | blk; task[n].last = d && d < 4 && blk == N1 - 1; ++n;
        }
    for (int i = 1; i < n; ++i)
        for (int k = i; k > 0 && task[k].t < task[k - 1].t; --k) { T s = task[k]; task[k] = task[k - 1]; task[k - 1] = s; }
    float fr[NSLOT]; int cntv[NSLOT]; unsigned char lst[NSLOT][NTASK];
    for (int s = 0; s < NSLOT; ++s) { fr[s] = tstart; cntv[s] = 0; for (int i = 0; i < NTASK; ++i) lst[s][i] = 0xFF; }
    for (int i = 0; i < n; ++i) {
        int best = -1;
        for (int s = 0; s < nslot; ++s) {
            if (foreign && task[i].t > TRES && (s < 3) != task[i].last) continue;
            if (cntv[s] < NTASK - 1 && (best < 0 || fr[s] < fr[best])) best = s;
        }
        fr[best] = (fr[best] > task[i].t ? fr[best] : task[i].t) + TCOST;
        if (fin) fin[best][cntv[best]] = fr[best];
        lst[best][cntv[best]++] = (unsigned char)task[i].e;
    }
    for (int s = 0; s < NSLOT; ++s)
        for (int h = 0; h < 2; ++h) {
            u64 v = 0;
            for (int i = 7; i >= 0; --i) v = v << 8 | lst[s][h * 8 + i];
            tab[s][h] = v;
        }
}

static void build_all(Args& a) {
    build_table(0, OST0, a.tab[0]);
    auto byte = [&](int t, int s, int i) { return i >= NTASK ? 0xFF : (int)((a.tab[1 + t][s][i >> 3] >> (8 * (i & 7))) & 0xFF); };
    int n = 0;
    float fin[4][NSLOT][NTASK];
    int next[4][NSLOT];
    for (int t = 0; t < 4; ++t) {
        build_table(1, FSTART + FSTEP * t, a.tab[1 + t], fin[t]);
        for (int s = 0; s < NSLOT; ++s) {
            for (int i = 0; i < NTASK; ++i) n += byte(t, s, i) < 0x80;
            a.ord[t][s][0] = a.ord[t][s][1] = 0;
            next[t][s] = 0;
        }
    }
    a.npacked = n;
    for (int k = 0; k < n; ++k) {
        int bt = -1, bs = 0;
        for (int t = 0; t < 4; ++t)
            for (int s = 0; s < NSLOT; ++s)
                if (byte(t, s, next[t][s]) < 0x80 && (bt < 0 || fin[t][s][next[t][s]] < fin[bt][bs][next[bt][bs]])) { bt = t; bs = s; }
        const int i = next[bt][bs]++;
        a.ord[bt][bs][i >> 3] |= (u64)k << (8 * (i & 7));
    }
    for (int blk = 0; blk < 16; ++blk)
        for (int t = 0; t < 4; ++t)
            for (int s = 0; s < NSLOT; ++s)
                for (int i = 0; i < NTASK; ++i)
                    if (byte(t, s, i) != 0xFF && (byte(t, s, i) & 0x8F) == (0x80 | blk)) a.ord[t][s][i >> 3] |= (u64)n++ << (8 * (i & 7));
    a.ntotal = n;
}

extern "C" void kernel_launch(int rank, int nranks, ncclComm_t comm,
                              hipStream_t stream,
                              void* const* d_in, const int* in_sizes, int n_in,
                              void* d_out, int out_size,
                              const int* in_dtypes, int out_dtype,
                              void* const* d_out_all,
                              void* const* d_ws_all, size_t ws_bytes) {
    if (nranks != NRANK || (unsigned)rank >= (unsigned)NRANK || n_in != 4 || ws_bytes < WS_NEED ||
        in_sizes[0] != MSH * KDIM || in_sizes[1] != KDIM * NSH || out_size != NRANK * MSH * NSH) {
        fprintf(stderr, "unexpected problem shape\n");
        return;
    }
    HIP_CHECK(hipFuncSetAttribute((const void*)ag_gemm_kernel, hipFuncAttributeMaxDynamicSharedMemorySize, LDS_BYTES));
    Args a;
    a.sx = (const float*)d_in[2];
    a.sw = (const float*)d_in[3];
    a.out = (float*)d_out;
    for (int d = 0; d < NRANK; ++d) a.ws[d] = (char*)d_ws_all[rank ^ d];
    for (int i = 0; i < 4; ++i) a.outp[i] = (float*)d_out_all[rank ^ (4 + i)];
    for (int i = 0; i < NTHREADS / 64; ++i) {
        Rec& r = a.rec[i];
        const int d = i & 7;
        r.x = (const signed char*)d_in[0];
        r.w = (const signed char*)d_in[1];
        r.ws = a.ws[0];
        r.peer = a.ws[d];
        r.src = d < 4 ? r.x : r.w + (d & 3) * 32;
        r.dst = r.peer + (size_t)(d * NBLK + (d < 4 ? 0 : 16)) * BLK_BYTES;
        r.cnt = r.ws + CNT_OFF + i * 4;
        r.dst1 = r.dst + XS_PAR;
    }
    build_all(a);
    if (a.ntotal > NIDX || a.ntotal > KSPLIT + 128 || NSLOT != 7 || NRAW > 1) return;
    a.rank = rank;
    ag_gemm_kernel<<<dim3(256), dim3(NTHREADS), LDS_BYTES, stream>>>(a);
    HIP_CHECK(hipGetLastError());
}
```

```cpp
#include <hip/hip_runtime.h>
#include <rccl/rccl.h>
#include <cstdio>
#include <cstddef>

#define HIP_CHECK(expr) do { hipError_t _e = (expr); if (_e != hipSuccess) { \
    fprintf(stderr, "HIP error %s at %s:%d\n", hipGetErrorString(_e), __FILE__, __LINE__); \
    return; } } while (0)

typedef int i32x2 __attribute__((ext_vector_type(2)));
typedef int i32x4 __attribute__((ext_vector_type(4)));
typedef int i32x16 __attribute__((ext_vector_type(16)));
typedef unsigned u32x2 __attribute__((ext_vector_type(2)));
typedef unsigned u32x4 __attribute__((ext_vector_type(4)));
typedef float f32x4 __attribute__((ext_vector_type(4)));
typedef unsigned long long u64;
typedef unsigned __attribute__((aligned(1))) u32_u;
#define LDS_AS __attribute__((address_space(3)))
#define GLB_AS __attribute__((address_space(1)))
#define CONST_AS __attribute__((address_space(4)))

#define N1 10
#define N3 2
#define NRAW 0
#define PACE 906
#define PACEI 864
#define PACEC 880
#define GOFF 6600
#define TMIN 490
#define DEFER 1400
#define LMAX 115
#define ROT 4
#define TSOFF 500
#define TSTEP 44
#define TA 5.3f
#define FSTART 6.0f
#define FSTEP 2.2f
#define TCOST 1.3f
#define OST0 5.0f
#define TRK 3.95f
#define TRES (TA + (N1 - TRK) * PACEI / 400)
#define FSTRIDE 256
#define PSLEEP 1
#define KSPLIT 144
#define SWZ 3
#define NPOLL 3
#define NSIG 24
#define LDPOS 1
#define PSLACK 160
#define WALL 0
#define PDELAY 60
#define P21 1
#define FASTX 16
#define FASTO 256
#define DELAY1 0
#define DELAYW 12
#define OBSE 1
#define OAUX 0
#define PFIRST 1
#define NSLOW N3
#define K1 96
#define PSLOW (4 * PACEC + K1 * TSTEP / NSLOW)
#define TSTEP1 (NSLOW * PSLOW / K1)
#define KIL 1
#define ODELAY 0
#define FPOS 4
#define PAIR 0
#define KROT 0
#define FRAG(k) (UPAGE((k) & 3, (k) >> 2, 0) | ((k) >> 2) * 1024)
#define LAUX 17
#define DIRB 0
#define BDELAY 0
#define BFIRST 0
#define GATE 0
#define NREP 1
#define FXPAD 0
#define OTRK 2.5f
#define OCOST TCOST
#define OSKEW 0
#define HELP 0
#define TRK2 1.95f
#define GOFFC GOFF
#define GOPOS 0
#define SLEEP0 2
#define SLEEP1 10
#define SLEEP2 8
#define UPAGE(c, f, blk) ((unsigned)((c) ^ ((SWZ & 1 ? (f) : 0) ^ (SWZ & 2 ? (blk) & 3 : 0))) << 12)
#define HAVE_ROLE 1
__host__ __device__ inline int role_of(int b, int& ix) {
    const int r = b >> 3;
    ix = r < 4 ? b : (b & 7) * (r < 6 ? 2 : 26) + r - (r < 6 ? 4 : 6);
    if (PAIR && r >= 6) {
        const int own = r < 18, q = own ? (b & 7) * 12 + r - 6 : (b & 7) * 14 + r - 18, g = own ? PAIR & 3 : PAIR >> 2;
        ix = g == 1 ? (q >> 4) * 32 + (q & 15) : g == 3 && own ? (q / 48 * 3 + q % 3) * 32 + q % 48 / 3
           : !own && q >= 64 ? (4 + (q - 64) % 3) * 32 + (q - 64) / 3 : ((q >> 5) * 2 + (q & 1)) * 32 + ((q >> 1) & 15);
        ix += own * 16;
    }
    return r < 4 ? 0 : r < 6 ? 1 : 2;
}
constexpr int NRANK = 8;
constexpr int KDIM = 4096, MSH = 512, NSH = 512;
constexpr unsigned BLK_BYTES = 32 * KDIM;
constexpr int NBLK = 20;
constexpr unsigned OSLOT = 4608;
constexpr int NSLOT = 7, NOWN = 6, NTASK = 16, NIDX = 256;
constexpr size_t XS_PAR = (size_t)NRANK * NBLK * BLK_BYTES;
constexpr size_t OS_OFF = 2 * XS_PAR, OS_PAR = (size_t)4 * NIDX * OSLOT;
constexpr size_t FX_OFF = OS_OFF + 2 * OS_PAR;
constexpr unsigned FX_SIZE = NRANK * NBLK * 32 * FSTRIDE + FXPAD;
constexpr size_t FO_OFF = FX_OFF + NREP * FX_SIZE;
constexpr size_t RD_OFF = FO_OFF + 4 * NIDX * 128;
constexpr size_t CNT_OFF = RD_OFF + NRANK * 32 * 128;
constexpr size_t TMO_OFF = CNT_OFF + 16384;
constexpr size_t ST_OFF = TMO_OFF + 1024;
constexpr size_t DUMMY_OFF = ST_OFF + 256 * 128;
constexpr size_t WS_NEED = DUMMY_OFF + NREP * FX_SIZE;
constexpr unsigned DEADLINE_TICKS = 20000000u;
constexpr int TILE_STRIDE = 4352;
constexpr int LDS_MAIN = 131072;
constexpr int LDS_CTR = LDS_MAIN;
constexpr int LDS_BYTES = LDS_CTR + 256;
constexpr int NTHREADS = 768;
static_assert((NTASK + 8) * TILE_STRIDE <= LDS_MAIN, "tiles");
#define DIAG_N 0
#define STAMP(i)
#define TRACE(i, h)
#define TRACE2(i, h)
#define TRACE_OUT()

struct Rec {
    const signed char* src;
    char* dst;
    const char* cnt;
    char* peer;
    const signed char* x;
    const signed char* w;
    char* ws;
    char* dst1;
};
struct Args {
    Rec rec[NTHREADS / 64];
    const float* sx;
    const float* sw;
    float* out;
    char* ws[NRANK];
    float* outp[4];
    u64 tab[5][NSLOT][2];
    u64 ord[4][NSLOT][2];
    int rank;
    int npacked, ntotal;
};

#define STORE_WT(voff, base, val, off) \
    asm volatile("global_store_dwordx4 %0, %2, %1 offset:%3 sc0 sc1\n\ts_nop 1" \
                 :: "v"(voff), "s"(base), "v"(val), "i"(off) : "memory")
#define VMCNT(n) asm volatile("s_waitcnt vmcnt(%0)" :: "i"(n) : "memory")
#define LGKMCNT0() asm volatile("s_waitcnt lgkmcnt(0)" ::: "memory")
#define CBAR() asm volatile("" ::: "memory")

__device__ __forceinline__ void set_flag(char* p, unsigned call, int lane) {
    if (lane == 0) __hip_atomic_store((GLB_AS unsigned*)p, call, __ATOMIC_RELAXED, __HIP_MEMORY_SCOPE_SYSTEM);
}
__device__ __forceinline__ unsigned lds_get(LDS_AS unsigned* p) {
    return __builtin_amdgcn_readfirstlane(__hip_atomic_load(p, __ATOMIC_RELAXED, __HIP_MEMORY_SCOPE_WORKGROUP));
}
__device__ __forceinline__ void lds_wait(LDS_AS unsigned* p, unsigned v) {
    while (lds_get(p) < v) __builtin_amdgcn_s_sleep(1);
    CBAR();
}

#define TMOEVERY 8
__device__ __forceinline__ unsigned tmo_get(GLB_AS unsigned* tmo) {
    return __hip_atomic_load(tmo, __ATOMIC_RELAXED, __HIP_MEMORY_SCOPE_AGENT);
}
__device__ __forceinline__ bool wait_flags(GLB_AS unsigned* p, unsigned f, GLB_AS unsigned* tmo, unsigned call, int lane) {
    if (((unsigned)__builtin_amdgcn_ballot_w64(f >= call) & 15u) == 15u) return true;
    const unsigned t0 = (unsigned)__builtin_amdgcn_s_memrealtime();
    for (unsigned it = 1;; ++it) {
        if (lane < 4) f = __hip_atomic_load(p, __ATOMIC_RELAXED, __HIP_MEMORY_SCOPE_SYSTEM);
        if (((unsigned)__builtin_amdgcn_ballot_w64(f >= call) & 15u) == 15u) return true;
        if (it % TMOEVERY == 0 &&
            (__builtin_amdgcn_readfirstlane(tmo_get(tmo)) == call || (unsigned)__builtin_amdgcn_s_memrealtime() - t0 > DEADLINE_TICKS)) {
            if (lane == 0) __hip_atomic_store(tmo, call, __ATOMIC_RELAXED, __HIP_MEMORY_SCOPE_AGENT);
            return false;
        }
        __builtin_amdgcn_s_sleep(PSLEEP);
    }
}

#define PDEPTH 7
#define PGAP 0
#define POLL(LOAD, CHECK) \
    "v_mov_b64 v[166:167], 0\n\ts_mov_b32 %[it], 0\n\ts_mov_b32 exec_hi, 0\n" \
    "1:\n\ts_add_u32 %[it], %[it], 1\n\ts_and_b32 vcc_lo, %[it], 31\n\ts_cselect_b32 vcc_lo, 0, 16\n\ts_or_b32 exec_lo, %[pend], vcc_lo\n\t" \
    LOAD " %[vo], %[base] sc0 sc1\n\ts_mov_b32 exec_lo, 31\n\ts_memrealtime vcc\n\ts_sleep %[gap]\n\t" \
    "s_cmp_lg_u32 %[fast], 0\n\ts_cbranch_scc1 2f\n\ts_waitcnt vmcnt(0)\n" \
    "2:\n\ts_waitcnt vmcnt(%[dep]) lgkmcnt(0)\n\ts_cmp_eq_u32 %[it], 1\n\ts_cselect_b32 %[t0], vcc_lo, %[t0]\n\ts_sub_u32 vcc_lo, vcc_lo, %[t0]\n\t" \
    "s_cmp_gt_u32 vcc_lo, %[dl]\n\ts_cbranch_scc1 3f\n\tv_cmp_le_u32 vcc, %[call], v166\n\ts_bitcmp1_b32 vcc_lo, 4\n\ts_cbranch_scc1 3f\n\t" \
    CHECK "s_andn2_b32 %[pend], %[pend], vcc_lo\n\ts_cbranch_scc1 1b\n" \
    "3:\n\ts_mov_b64 exec, -1"
__device__ __forceinline__ bool poll_flags(unsigned vo, const char* base, GLB_AS unsigned* tmo, unsigned call, unsigned fast, int lane) {
    unsigned pend = 15, it, t0;
    asm volatile(POLL("global_load_dword v166,", "")
                 : [pend] "+s"(pend), [it] "=&s"(it), [t0] "=&s"(t0)
                 : [vo] "v"(vo), [base] "s"(base), [call] "s"(call), [fast] "s"(fast), [dep] "i"(PDEPTH), [gap] "i"(PGAP), [dl] "i"(DEADLINE_TICKS)
                 : "memory", "vcc", "scc", "v166", "v167");
    if (pend && lane == 0) __hip_atomic_store(tmo, call, __ATOMIC_RELAXED, __HIP_MEMORY_SCOPE_AGENT);
    return !pend;
}
__device__ __forceinline__ unsigned poll_flag64(unsigned vo, const char* base, GLB_AS unsigned* tmo, unsigned call, unsigned fast, int lane) {
    unsigned pend = 1, it, t0, info;
    asm volatile(POLL("global_load_dwordx2 v[166:167],",
                      "v_readfirstlane_b32 %[info], v167\n\ts_xor_b32 vcc_hi, %[info], %[call]\n\ts_and_b32 vcc_hi, vcc_hi, 63\n\ts_cbranch_scc1 1b\n\t")
                 : [pend] "+s"(pend), [it] "=&s"(it), [t0] "=&s"(t0), [info] "=&s"(info)
                 : [vo] "v"(vo), [base] "s"(base), [call] "s"(call), [fast] "s"(fast), [dep] "i"(PDEPTH), [gap] "i"(PGAP), [dl] "i"(DEADLINE_TICKS)
                 : "memory", "vcc", "scc", "v166", "v167");
    if (pend && lane == 0) __hip_atomic_store(tmo, call, __ATOMIC_RELAXED, __HIP_MEMORY_SCOPE_AGENT);
    return pend ? 0 : info;
}

template <int V> struct IC { static constexpr int value = V; };
constexpr unsigned F_WIDE = 1u << 30, F_NAN = 1u << 31, F_SKIP = 1u << 29;
#define BITMASK(nb, m, r) asm("v_bfe_i32 %0, %1, %2, 1" : "=v"(nb) : "v"(m), "i"(r))
#define STEP(x, nb, n) asm("v_mad_i32_i24 %0, %1, %2, %3" : "=v"(x) : "v"(nb), "i"(-(n)), "v"(x))
__device__ __forceinline__ unsigned wave_scan(unsigned x) {
    x += __builtin_amdgcn_update_dpp(0u, x, 0x111, 0xf, 0xf, false);
    x += __builtin_amdgcn_update_dpp(0u, x, 0x112, 0xf, 0xf, false);
    x += __builtin_amdgcn_update_dpp(0u, x, 0x114, 0xf, 0xf, false);
    x += __builtin_amdgcn_update_dpp(0u, x, 0x118, 0xf, 0xf, false);
    x += __builtin_amdgcn_update_dpp(0u, x, 0x142, 0xa, 0xf, false);
    x += __builtin_amdgcn_update_dpp(0u, x, 0x143, 0xc, 0xf, false);
    return x;
}
#define SIG_A "s_lshl_b32 %[t], %[b], 1\n\ts_add_u32 %[t], %[t], %[w]\n\ts_lshl_b32 %[t], %[t], 6\n\ts_add_u32 %[t], %[t], %[st]\n\tv_mov_b32 v13, %[t]\n\t"
#define SIG_B "v_add_u32_e64 v14, %[n], 1\n\ts_mov_b64 exec, 1\n\tglobal_store_dword v13, v14, s[42:43] sc0 sc1\n\ts_mov_b64 exec, -1\n\t"
#define SIG_W "s_waitcnt vmcnt(4)\n\t"
#define LOAD_LDS_SYS(ldsaddr, voffs, base) \
    asm volatile("s_nop 0\n\tglobal_load_lds_dwordx4 %1, %2 sc0 sc1" :: "{m0}"(ldsaddr), "v"(voffs), "s"(base) : "memory")
#define LOAD_LDS(ldsaddr, voffs, base) \
    asm volatile("s_nop 0\n\tglobal_load_lds_dwordx4 %1, %2" :: "{m0}"(ldsaddr), "v"(voffs), "s"(base) : "memory")

__device__ __forceinline__ u64 flag64_get(const char* p) {
    return __hip_atomic_load((GLB_AS u64*)p, __ATOMIC_RELAXED, __HIP_MEMORY_SCOPE_SYSTEM);
}
__device__ __forceinline__ void flag64_set(char* p, unsigned call, unsigned info, int lane) {
    if (lane == 0) __hip_atomic_store((GLB_AS u64*)p, (u64)info << 32 | call, __ATOMIC_RELAXED, __HIP_MEMORY_SCOPE_SYSTEM);
}
#define IF_DIAG(...)
__global__ void __launch_bounds__(NTHREADS, 3) __attribute__((amdgpu_num_vgpr(83))) ag_gemm_kernel(Args a_) {
    const CONST_AS Args* ap = (const CONST_AS Args*)__builtin_amdgcn_kernarg_segment_ptr();
    unsigned tid = threadIdx.x, w64, call, stmp;
    int b = blockIdx.x;
    const signed char *xin, *win;
    char *myws, *peer;
    u64 t_first, t_top;
#define PRO_(...)
    asm volatile(
        IF_DIAG("s_memrealtime %[t0]\n\t") "v_readfirstlane_b32 %[w], v0\n\ts_load_dwordx16 s[36:51], %[ap], %[w]\n\t"
        "s_cmp_gt_u32 %[b], 31\n\ts_cbranch_scc1 3f\n\ts_sub_u32 %[t], %[w], 64\n\ts_cmpk_lt_u32 %[t], 192\n\ts_cbranch_scc0 2f\n\t"
        "v_lshlrev_b32 v1, 12, v0\n\tv_lshrrev_b32 v10, 1, v0\n\ts_mov_b32 %[t], 0x1f010\n\tv_bitop3_b32 v1, v1, %[t], v10 bitop3:0xc8\n\t"
        "v_lshl_add_u32 v1, %[b], 7, v1\n\ts_waitcnt lgkmcnt(0)\n\tglobal_load_dwordx4 v[2:5], v1, s[36:37]\n\t" PRO_("s_memrealtime %[t2]\n\t")
        "s_lshl_b32 %[t], %[b], 6\n\ts_load_dword %[n], s[40:41], %[t]\n\ts_barrier\n\tglobal_load_dwordx4 v[6:9], v1, s[36:37] offset:32\n\t"
        "v_lshlrev_b32 v10, 4, v0\n\tv_and_b32 v10, 0x3f0, v10\n\tv_lshl_or_b32 v10, %[b], 12, v10\n\ts_movk_i32 m0, 2048-64\n\ts_sleep %[s0]\n\t"
        "global_load_lds_dwordx4 v1, s[36:37] offset:64\n\ts_movk_i32 m0, 3072-96\n\tv_xor_b32 v11, 0x1000, v10\n\tglobal_load_lds_dwordx4 v1, s[36:37] offset:96\n\t"
        SIG_A "s_waitcnt lgkmcnt(0)\n\t" PRO_("s_memrealtime %[t3]\n\t") "s_bitcmp0_b32 %[n], 0\n\ts_cselect_b64 s[38:39], s[50:51], s[38:39]\n\t" SIG_B
        SIG_W "global_store_dwordx4 v10, v[2:5], s[38:39] sc0 sc1\n\ts_memrealtime %[t1]\n\ts_sleep %[s1]\n\t" SIG_W
        "global_store_dwordx4 v11, v[6:9], s[38:39] offset:1024 sc0 sc1\n\t" PRO_("s_memrealtime %[t4]\n\t") "s_branch 4f\n\t.p2align 6\n"
        "2:\n\ts_cmpk_gt_u32 %[t], 447\n\ts_cbranch_scc1 3f\n\t"
        "v_lshlrev_b32 v1, 8, v0\n\tv_lshlrev_b32 v10, 4, v0\n\tv_and_b32 v1, 0x3e00, v1\n\tv_and_or_b32 v1, v10, 16, v1\n\t"
        "v_lshl_add_u32 v1, %[b], 16, v1\n\ts_lshl_b32 m0, %[w], 8\n\ts_waitcnt lgkmcnt(0)\n\tglobal_load_lds_dwordx4 v1, s[36:37]\n\t"
        "s_lshl_b32 %[t], %[b], 6\n\ts_barrier\n\t" PRO_("s_memrealtime %[t2]\n\t") "s_load_dword %[n], s[40:41], %[t]\n\t"
        "s_addk_i32 m0, 0x400\n\tv_add_u32 v13, 0x4000, v1\n\tglobal_load_lds_dwordx4 v13, s[36:37]\n\t"
        "v_and_b32 v12, 0x2e0, v10\n\tv_and_or_b32 v12, v0, 16, v12\n\tv_lshlrev_b32 v2, 3, v0\n\tv_and_or_b32 v12, v2, 8, v12\n\tv_lshl_add_u32 v12, %[w], 8, v12\n\t"
        "s_addk_i32 m0, 0x400\n\tv_add_u32 v14, 0x8000, v1\n\tglobal_load_lds_dwordx4 v14, s[36:37]\n\t"
        "s_addk_i32 m0, 0x400\n\tv_add_u32 v1, 0xc000, v1\n\tglobal_load_lds_dwordx4 v1, s[36:37]\n\t"
        "v_and_b32 v10, 0x3f0, v10\n\tv_lshl_or_b32 v10, %[b], 12, v10\n\tv_xor_b32 v11, 0x1000, v10\n\t"
        SIG_A "s_waitcnt lgkmcnt(0)\n\t" PRO_("s_memrealtime %[t3]\n\t") "s_bitcmp0_b32 %[n], 0\n\ts_cselect_b64 s[38:39], s[50:51], s[38:39]\n\t" SIG_B
        SIG_W "ds_read_b64_tr_b8 v[2:3], v12\n\tds_read_b64_tr_b8 v[4:5], v12 offset:256\n\ts_waitcnt lgkmcnt(0)\n\t"
        "global_store_dwordx4 v10, v[2:5], s[38:39] sc0 sc1\n\ts_memrealtime %[t1]\n\ts_sleep %[s2]\n\t" SIG_W
        "ds_read_b64_tr_b8 v[6:7], v12 offset:1024\n\tds_read_b64_tr_b8 v[8:9], v12 offset:1280\n\ts_waitcnt lgkmcnt(0)\n\t"
        "global_store_dwordx4 v11, v[6:9], s[38:39] offset:1024 sc0 sc1\n\t" PRO_("s_memrealtime %[t4]\n\t") "s_branch 4f\n\t.p2align 6\n"
        "3:\n\tv_lshlrev_b32 v1, 2, v0\n\tv_and_b32 v1, 0xfc, v1\n\tv_add_u32 v1, %[ctr], v1\n\tv_mov_b32 v2, 0\n\tds_write_b32 v1, v2\n\t"
        "s_lshl_b32 %[t], %[b], 6\n\ts_waitcnt lgkmcnt(0)\n\ts_barrier\n\ts_load_dword %[n], s[40:41], %[t]\n\ts_memrealtime %[t1]\n"
        "4:\n\ts_waitcnt lgkmcnt(0)"
        : [w] "=&s"(w64), [n] "=&s"(call), [t] "=&s"(stmp), [t1] "=&s"(t_first), IF_DIAG([t0] "=&s"(t_top),) [b] "+s"(b), [ap] "+s"(ap),
          PRO_([t2] "=&s"(t_ka), [t3] "=&s"(t_cnt), [t4] "=&s"(t_ack),)
          "=&{s[42:43]}"(peer), "=&{s[44:45]}"(xin), "=&{s[46:47]}"(win), "=&{s[48:49]}"(myws)
        : [ctr] "i"(LDS_CTR), [st] "i"((unsigned)ST_OFF), [s0] "i"(SLEEP0), [s1] "i"(SLEEP1), [s2] "i"(SLEEP2), "{v0}"(tid)
        : "memory", "scc", "v1", "v2", "v3", "v4", "v5", "v6", "v7", "v8", "v9", "v10", "v11", "v12", "v13", "v14", "s36", "s37", "s38", "s39", "s40", "s41", "s50", "s51");
    asm volatile("" : "+{v0}"(tid));
    call += 1u;
    const unsigned par = call & 1u;
    const unsigned tstart = (unsigned)t_first;
    static_assert(sizeof(Rec) == 64 && (SWZ & 1) && KDIM == 4096 && NSH == 512, "prologue");
    const CONST_AS Args& a = *ap;
    LDS_AS unsigned char* smem = (LDS_AS unsigned char*)0;
    int lane = tid & 63;
    asm volatile("" : "+v"(lane));
    __builtin_assume(lane >= 0 && lane < 64);
    const int wave = w64 >> 6;
    LDS_AS unsigned* ctr = (LDS_AS unsigned*)(smem + LDS_CTR);
    int j;
    const int role = role_of(b, j);
    STAMP(7);

    GLB_AS unsigned* cnt = (GLB_AS unsigned*)(myws + CNT_OFF) + b * 16 + wave;
    const unsigned voff = lane * 16;
    unsigned xsrc_off, lo[4];
    auto image = [&]() {
        unsigned l = lane;
        asm volatile("" : "+v"(l));
        xsrc_off = (l >> 3) * KDIM + (((l & 7) ^ (l >> 3)) * 16);
#pragma unroll
        for (int f = 0; f < 4; ++f) lo[f] = ((l & 31) * 128 + (((l >> 5) ^ (l & 7)) * 16)) ^ (f * 32);
        asm volatile("" : "+v"(xsrc_off), "+v"(lo[0]), "+v"(lo[1]), "+v"(lo[2]), "+v"(lo[3]));
    };

    const unsigned wtr = (16 * (lane >> 5) + ((lane & 15) >> 1)) * 32 + 16 * ((lane >> 4) & 1) + 8 * (lane & 1);
    if (__builtin_expect(role == 0, 1)) {
        const int c = j;
        const int d = wave < 8 ? wave : wave - 4;
        GLB_AS unsigned* tmo = (GLB_AS unsigned*)(myws + TMO_OFF);
        const char* mine = myws + par * XS_PAR;
        char* fmine = myws + FX_OFF + (size_t)c * FSTRIDE;
        if (wave == 0 || wave >= 8) {
            if (wave >= 7 + NPOLL) __builtin_amdgcn_endpgm();
            for (int i = wave - 7; i > 0; --i) __builtin_amdgcn_s_sleep(14);
            __builtin_amdgcn_s_sleep(PDELAY);
            for (unsigned it = 1;; ++it) {
                u32x4 f = {call, call, call, call};
                u32x2 g = {call, call};
                asm volatile("s_mov_b64 exec, 0xfe\n\tglobal_load_dwordx4 %0, %2, %3 sc0 sc1\n\tglobal_load_dwordx2 %1, %2, %3 offset:16 sc0 sc1\n\ts_mov_b64 exec, -1\n\ts_waitcnt vmcnt(0)"
                             : "+v"(f), "+v"(g) : "v"(lane * 4096), "s"(myws + RD_OFF + c * 128) : "memory");
                unsigned lvl = f[2] >= call ? 3 : f[0] < call ? 0 : f[1] < call ? 1 : 2;
                if ((unsigned)__builtin_amdgcn_s_memrealtime() - tstart > DEADLINE_TICKS || (it % 64 == 0 && (unsigned)__builtin_amdgcn_readfirstlane(tmo_get(tmo)) == call)) {
                    if (lane == 0) __hip_atomic_store(tmo, call, __ATOMIC_RELAXED, __HIP_MEMORY_SCOPE_AGENT);
                    lvl = 4;
                    g = u32x2{call, 1};
                }
                if (lane < 8) {
                    __hip_atomic_fetch_max(ctr + 32 + lane, lvl, __ATOMIC_RELAXED, __HIP_MEMORY_SCOPE_WORKGROUP);
                    if (g[0] == call) ctr[24 + lane] = g[1];
                }
                CBAR();
                if (__builtin_amdgcn_ballot_w64(lvl >= 3 && g[0] == call) == ~0ull) break;
            }
            if (lane == 0) __hip_atomic_store(cnt, call, __ATOMIC_RELAXED, __HIP_MEMORY_SCOPE_AGENT);
            LGKMCNT0();
            STAMP(3);
            __builtin_amdgcn_endpgm();
        }
        auto now16 = [&]() { return (unsigned)__builtin_amdgcn_s_memrealtime() * 16u; };
        const int blk0 = d * NBLK + (d < 4 ? 0 : 16);
        char* prev_flag = peer + FX_OFF + (size_t)(blk0 * 32 + c) * FSTRIDE;
        const char* theirs = peer + par * XS_PAR;
        u32x4 v[4];
        if (GOPOS == 1) set_flag(peer + ST_OFF + (size_t)(d * 32 + c) * 128, call, lane);
        auto second = [&]() {
            const signed char* src = d < 4 ? xin + BLK_BYTES + c * 128 : win + (size_t)c * 128 * NSH + (d & 3) * 32 + 128;
            const unsigned so = d < 4 ? (lane & 31) * KDIM + (lane >> 5) * 16 : (lane >> 1) * NSH + (lane & 1) * 16;
            const int step = d < 4 ? 32 : 32 * NSH;
#pragma unroll
            for (int f = 0; f < 4; ++f) LOAD_LDS(wave * 16384 + 4096 + f * 1024, so, src + f * step);
        };
        if (LDPOS == 1) second();
        VMCNT(DIAG_N + 2 + (GOPOS <= 1) + 4 * (LDPOS == 1));
        unsigned tn = tstart * 16u + 2 * PACE;
#pragma unroll
        for (int f = 2; f < 8; ++f) {
            if (f == 4) VMCNT(3 - LDPOS);
            if (d >= 4) {
                LDS_AS unsigned char* bp = smem + wave * 16384 + wtr + f * 1024;
                const i32x2 l2 = __builtin_amdgcn_ds_read_tr8_b64_v2i32((LDS_AS i32x2*)bp);
                const i32x2 h2 = __builtin_amdgcn_ds_read_tr8_b64_v2i32((LDS_AS i32x2*)(bp + 256));
                v[0] = __builtin_bit_cast(u32x4, i32x4{l2[0], l2[1], h2[0], h2[1]});
            } else v[0] = *(LDS_AS u32x4*)(smem + (f < 4 ? 0 : wave * 16384) + f * 1024 + voff);
            int late;
            while ((late = (int)(now16() - tn)) < 0) __builtin_amdgcn_s_sleep(1);
            tn += PACE + max(late - PSLACK, 0);
            IF_DIAG(xw += max(late - PSLACK, 0) >> 4;)
            STORE_WT(voff, theirs + (size_t)(blk0 + (f >> 2)) * BLK_BYTES + UPAGE(c, f & 3, f >> 2), v[0], (f & 3) * 1024);
            if ((WALL >> f) & 1) VMCNT(1);
            if (f == FPOS) { VMCNT(FPOS - 3); set_flag(prev_flag, call, lane); }
            if (f == LDPOS) second();
            TRACE2(f, 0);
            if (GOPOS == f) set_flag(peer + ST_OFF + (size_t)(d * 32 + c) * 128, call, lane);
        }
        prev_flag += 32 * FSTRIDE;
        TRACE(12, 0);
        auto set_flags = [&](char* p) { if (lane < NREP) asm volatile("global_store_dword %0, %1, %2 sc0 sc1" :: "v"(lane * FX_SIZE), "v"(call), "s"(p) : "memory"); };
        char* const dummy = myws + DUMMY_OFF + (c * 8 + wave) * 128;
        const unsigned period = d < 4 ? 4 * PACEI : 4 * PACEC;
        while (!(tn = lds_get(ctr + 24 + d))) {
            if (prev_flag != dummy) { VMCNT(0); set_flags(prev_flag); prev_flag = dummy; }
            __builtin_amdgcn_s_sleep(1);
        }
        tn += ((c + ROT * d) & 31) * period / 32;
        const int nblk = d < 4 ? N1 + N3 - 2 : 2 + 16 - N1;
#pragma clang loop unroll(disable)
        for (int n = 0; n < nblk; ++n) {
            const int blk = d < 4 ? n + 2 : n < 2 ? 18 + n : 17 - n;
            const int sblk = blk < 16 ? blk : (n + 2) * NBLK + 12 + d;
            if ((int)(tn - now16()) > DEFER) { VMCNT(0); set_flags(prev_flag); prev_flag = dummy; }
            unsigned lvl;
            while ((lvl = lds_get(ctr + 32 + d)) < (unsigned)min(n + 1, 3)) __builtin_amdgcn_s_sleep(1);
            CBAR();
            if (lvl > 3) break;
            const char* src = theirs + (size_t)sblk * BLK_BYTES;
            unsigned ta;
            while ((int)((ta = now16()) - tn) < 0) __builtin_amdgcn_s_sleep(1);
#pragma unroll
            for (int f = 0; f < 4; ++f)
                asm volatile("global_load_dwordx4 %0, %1, %2 offset:%3 sc0 sc1" : "=&v"(v[f]) : "v"(voff), "s"(src + UPAGE(c, f, sblk)), "i"(f * 1024) : "memory");
            TRACE(n, 0);
            VMCNT(4);
            set_flags(prev_flag);
            const char* dst = mine + (size_t)(d * NBLK + blk) * BLK_BYTES;
#pragma unroll
            for (int f = 0; f < 4; ++f) {
                asm volatile("s_waitcnt vmcnt(4)" : "+v"(v[f]) :: "memory");
                STORE_WT(voff, dst + UPAGE(c, f, blk), v[f], f * 1024);
            }
            TRACE(n, 1);
            tn += period;
            if (d >= 4 && n >= nblk - NSLOW - 1) tn += (n == nblk - NSLOW - 1 ? (c + ROT * d) & 31 : 32) * (PSLOW - 4 * PACEC) / 32;
            if (LMAX) tn += max((int)(now16() - ta) - LMAX * 16, 0);
            prev_flag = fmine + (size_t)(d * NBLK + blk) * 32 * FSTRIDE;
        }
        VMCNT(0);
        set_flags(prev_flag);
        if (lane == 0) __hip_atomic_store(cnt, call, __ATOMIC_RELAXED, __HIP_MEMORY_SCOPE_AGENT);
        STAMP(2);
        TRACE_OUT();
        __builtin_amdgcn_endpgm();
    }

        if (role == 1 && wave == 2 && j - 1u < 7u) {
            unsigned f[4] = {0, 0, 0, 0}, el = 0, sum = 0, miss = 32;
            do {
#pragma unroll
                for (int k = 0; k < 4; ++k) {
                    asm volatile("s_mov_b32 exec_hi, 0\n\tglobal_load_dword %0, %2, %3 sc0 sc1\n\ts_mov_b32 exec_hi, -1\n\ts_sleep 3\n\ts_waitcnt vmcnt(3)"
                                 : "+v"(f[k]), "+v"(f[(k + 1) & 3]) : "v"(lane * 128), "s"(myws + ST_OFF + j * 4096) : "memory");
                    const unsigned t = (unsigned)__builtin_amdgcn_s_memrealtime() - tstart;
                    sum += (t - el) * miss;
                    el = t;
                    miss = min(miss, 32u - __builtin_popcount((unsigned)__builtin_amdgcn_ballot_w64(f[(k + 1) & 3] >= call)));
                    if (miss <= 32 - NSIG) miss = 0;
                }
            } while (miss && el < DEADLINE_TICKS);
            asm volatile("s_waitcnt vmcnt(0)" : "+v"(f[0]), "+v"(f[1]), "+v"(f[2]), "+v"(f[3]) :: "memory");
            const unsigned t = tstart * 16u + max((sum >> 1) + (j < 4 ? GOFF : GOFFC), TMIN * 16u);
            if (lane < 32) __hip_atomic_store((GLB_AS u64*)(myws + RD_OFF + (j * 32 + lane) * 128 + 16), (u64)(t | 1u) << 32 | call, __ATOMIC_RELAXED, __HIP_MEMORY_SCOPE_SYSTEM);
            STAMP(2);
        }
    float scale;
    GLB_AS unsigned* tmo = (GLB_AS unsigned*)(myws + TMO_OFF);
    const float fnan = __builtin_nanf("");
    int rank;

    const bool helper = HELP && j >= 160 && !(j & 18);
    const int grp = j >> 4, tile = helper ? (j >> 3 & 8) + (j >> 1 & 6) + (j & 1) : j & 15;
    const int foreign = role == 2 ? !(grp & 1) && !helper : 2, slot = helper ? 6 : grp >> 1;
    const int downer = 4 + (tile >> 2);
    const int su = wave * 208 + j, sq = su >> 5;
    const int se = sq < 10 ? (sq % 5 ? 23 + sq - sq / 5 : 2 + sq / 5) : DIRB ? (sq < 12 ? sq - 10 : sq - 8) : sq < 18 ? 6 + sq : sq < 20 ? sq - 18 : sq - 16;
    const int sc = se < 16 ? su & 31 : (se & 3) * 8 + ((su & 31) >> 2);
    const int uu = (se < 16 ? se : ((se >> 2) & 3) * NBLK + 16 + (su & 3)) * 32 + sc;
    image();
    const bool selfcopy = role == 2 && su < (DIRB ? 768 : 1024);
    if (DELAY1) __builtin_amdgcn_s_sleep(DELAY1);
    if (DELAYW) for (int i = 0; i < wave && wave < 8; ++i) __builtin_amdgcn_s_sleep(DELAYW);
    if (selfcopy) {
        const signed char* src = se < 16 ? xin + (size_t)se * BLK_BYTES + sc * 128 : win + (size_t)sc * 128 * NSH + (se & 12) * 32 + (su & 3) * 32;
        const unsigned so = se < 16 ? xsrc_off : (lane >> 1) * NSH + (lane & 1) * 16;
        static_assert(8 * KDIM == 2 * 32 * NSH, "row steps");
#pragma unroll
        for (int i = 0; i < 4; ++i) LOAD_LDS((wave * 16 + i) * 1024, so, src + (i * 64 * NSH >> (se >> 4)));
    }
    const int tsel = foreign ? 1 + (tile & 3) : 0;
    u64 t_lo = a.tab[tsel][slot][0], t_hi = a.tab[tsel][slot][1];
    if (lane == 0) __hip_atomic_store(cnt, call, __ATOMIC_RELAXED, __HIP_MEMORY_SCOPE_AGENT);
    auto src_d = [&](int e) { const int code = (e >> 4) & 7; return foreign && code >= 4 ? downer ^ (code & 3) : code; };
    auto blk_of = [&](int e) { return src_d(e) * NBLK + (e & 15); };

    auto epilogue = [&](int i, int e) {
        const float bad = lds_get(ctr + 17) != 0 ? fnan : 0.0f;
        LDS_AS unsigned char* tp = smem + i * TILE_STRIDE + 128;
        const int srank = rank ^ src_d(e);
        const float* op = (foreign ? a.outp[tile >> 2] : a.out) + (size_t)(srank * MSH + (e & 15) * 32) * NSH + (foreign ? (tile & 3) * 4 + (tile >> 2) : tile) * 32;
        unsigned vo = voff;
        asm volatile("" : "+v"(vo));
        i32x4 q[4];
#pragma unroll
        for (int k = 0; k < 4; ++k) q[k] = *(LDS_AS i32x4*)(tp + k * 1024 + vo);
        vo = (vo >> 7) * (NSH * 4) + (vo & 0x70);
#pragma unroll
        for (int k = 0; k < 4; ++k) {
            f32x4 o;
#pragma unroll
            for (int r = 0; r < 4; ++r) o[r] = fmaxf((float)q[k][r] * scale, 0.0f) + bad;
            STORE_WT(vo, op + (size_t)k * 8 * NSH, o, 0);
        }
    };

    if (role == 2 && wave < 8) {
    bool failed = false;
    const unsigned fxo = (lane & 3) * FSTRIDE;
    auto flag_ptr = [&](int eb) {
        char* p = myws + FX_OFF + (size_t)(eb * 32 + 4 * wave) * FSTRIDE;
        asm("" : "+s"(p));
        return (GLB_AS unsigned*)(p + fxo);
    };
    unsigned aoff = 128 + (4 * (lane >> 5) * 32 + (lane & 31)) * 4;
    asm("" : "+v"(aoff));
    auto ar = __builtin_amdgcn_make_buffer_rsrc((void*)(myws + par * XS_PAR), 0, (unsigned)XS_PAR, 0x00020000);
    i32x4 bf[16];
    if (selfcopy) {
        char* fp = a.ws[lane & 7] + (lane ? RD_OFF + (lane * 32 + sc) * 128 + sq / 5 * 4 : FX_OFF + (size_t)uu * FSTRIDE);
        asm volatile("s_waitcnt vmcnt(0)" : "+v"(fp) :: "memory");
        u32x4 v[4];
        if (se < 16) {
#pragma unroll
            for (int f = 0; f < 4; ++f) v[f] = *(LDS_AS u32x4*)(smem + wave * 16384 + lo[f]);
        } else {
            LDS_AS unsigned char* bp = smem + wave * 16384 + wtr;
#pragma unroll
            for (int f = 0; f < 4; ++f) {
                const i32x2 l2 = __builtin_amdgcn_ds_read_tr8_b64_v2i32((LDS_AS i32x2*)(bp + f * 1024));
                const i32x2 h2 = __builtin_amdgcn_ds_read_tr8_b64_v2i32((LDS_AS i32x2*)(bp + f * 1024 + 256));
                v[f] = __builtin_bit_cast(u32x4, i32x4{l2[0], l2[1], h2[0], h2[1]});
            }
        }
        const char* dst = myws + par * XS_PAR;
#pragma unroll
        for (int f = 0; f < 4; ++f) STORE_WT(voff, dst + UPAGE(uu, f, uu >> 5), v[f], f * 1024);
        VMCNT(0);
        if (lane < 8 && (((sq >= 10 ? 1u : se < 16 ? 15u : 1u | 16u << (su & 3)) >> lane) & 1)) __hip_atomic_store((GLB_AS unsigned*)fp, call, __ATOMIC_RELAXED, __HIP_MEMORY_SCOPE_SYSTEM);
        if (foreign) STAMP(2);
    }
    if (DIRB && !foreign) {
        if (BDELAY) while ((unsigned)__builtin_amdgcn_s_memrealtime() - tstart < BDELAY) __builtin_amdgcn_s_sleep(4);
        const signed char* src = win + (size_t)wave * 512 * NSH + tile * 32;
        const unsigned so = (lane >> 1) * NSH + (lane & 1) * 16;
#pragma unroll
        for (int i = 0; i < 16; ++i) LOAD_LDS((wave * 16 + i) * 1024, so, src + i * 32 * NSH);
        VMCNT(0);
#pragma unroll
        for (int k = 0; k < 16; ++k) {
            LDS_AS unsigned char* bp = smem + wave * 16384 + wtr + (k & 3) * 4096 + (k >> 2) * 1024;
            const i32x2 l2 = __builtin_amdgcn_ds_read_tr8_b64_v2i32((LDS_AS i32x2*)bp);
            const i32x2 h2 = __builtin_amdgcn_ds_read_tr8_b64_v2i32((LDS_AS i32x2*)(bp + 256));
            bf[k] = i32x4{l2[0], l2[1], h2[0], h2[1]};
        }
        LGKMCNT0();
    }
    if (wave * 16384 < NTASK * TILE_STRIDE) {
#pragma unroll
        for (int i = 0; i < 16; ++i) *(LDS_AS u32x4*)(smem + wave * 16384 + i * 1024 + voff) = u32x4{0, 0, 0, 0};
    }
    LGKMCNT0();
    if (lane == 0) __hip_atomic_fetch_add(ctr + 16, 1u, __ATOMIC_RELAXED, __HIP_MEMORY_SCOPE_WORKGROUP);
    u32x4 af[16];
    const unsigned krot = KROT == 1 ? FRAG(tile) : KROT == 2 ? FRAG(tile ^ (slot & 1) * 8) : KROT == 3 ? FRAG(tile & 12) : 0;
    unsigned fpre;
    auto aload = [&](int k, unsigned abase) { af[k] = __builtin_amdgcn_raw_buffer_load_b128(ar, voff, (abase | UPAGE(0, 0, abase >> 17)) ^ krot ^ FRAG(k), LAUX); };
    auto flag_off = [&](int e) { return lane & 4 ? (unsigned)(TMO_OFF - FX_OFF) : (blk_of(e) * 32 + 4 * wave) * FSTRIDE + fxo + ((e & 0x70) && (e & 0x4F) ? ((tile + (foreign ? NREP / 2 + (tile >> 2) : 0)) & (NREP - 1)) * FX_SIZE : 0); };
    auto peek = [&](int e) { fpre = __hip_atomic_load((GLB_AS unsigned*)(myws + FX_OFF + flag_off(e)), __ATOMIC_RELAXED, __HIP_MEMORY_SCOPE_SYSTEM); };
    auto gate = [&](int d, int n) {
        if (!GATE || d == 0 || n < 0) return;
        const int x = n * 32 + ((4 * wave + ROT * d) & 31) + 3;
        const unsigned t = tstart + GATE + (d < 4 ? x * PACEI : x * PACEC + max(x - (18 - N1 - NSLOW) * 32, 0) * (PSLOW / 4 - PACEC)) / 128;
        while ((int)((unsigned)__builtin_amdgcn_s_memrealtime() - t) < 0) __builtin_amdgcn_s_sleep(2);
    };
    if (!foreign) STAMP(2);
    if (ODELAY && !foreign) while ((unsigned)__builtin_amdgcn_s_memrealtime() - tstart < ODELAY) __builtin_amdgcn_s_sleep(8);
    const bool bfirst = BFIRST && (!foreign || (tile & 3) < BFIRST - 1);
    if (bfirst) {
        const int ebw = (foreign ? downer : tile >> 2) * NBLK + 16 + (tile & 3);
        failed = !wait_flags(flag_ptr(ebw), 0, tmo, call, lane);
        CBAR();
        const unsigned bbase = (unsigned)ebw * BLK_BYTES + wave * 16384;
#pragma unroll
        for (int k = 0; k < 16; ++k)
            bf[k] = __builtin_bit_cast(i32x4, __builtin_amdgcn_raw_buffer_load_b128(ar, voff, (bbase | UPAGE(0, 0, bbase >> 17)) ^ krot ^ FRAG(k), LAUX));
        CBAR();
    }
#pragma clang loop unroll(disable)
    for (int i = -1; i < NTASK; ++i) {
        asm volatile("" : "+s"(i));
        if (i >= 0) { t_lo = (t_lo >> 8) | (t_hi << 56); t_hi = (t_hi >> 8) | (0xFFull << 56); }
        int en = (int)(t_lo & 0xFF);
        unsigned nbase = (unsigned)blk_of(en) * BLK_BYTES + wave * 16384;
        bool ready = false;
        if (i >= 0) {
        failed = failed || (unsigned)__builtin_amdgcn_readlane(fpre, 4) == call;
        ready = en != 0xFF && ((unsigned)__builtin_amdgcn_ballot_w64(fpre >= call) & 15u) == 15u;
        CBAR();
        i32x16 acc = {};
        if (ready) {
            TRACE(i + 1, 0);
            peek((int)(t_lo >> 8) & 0xFF);
#pragma unroll
            for (int k = 0; k < 16; ++k) {
                __builtin_amdgcn_sched_barrier(0);
                acc = __builtin_amdgcn_mfma_i32_32x32x32_i8(__builtin_bit_cast(i32x4, af[k]), bf[k], acc, 0, 0, 0);
                __builtin_amdgcn_sched_barrier(0);
                aload(k, nbase);
            }
            __builtin_amdgcn_sched_barrier(0);
        } else {
#pragma unroll
            for (int k = 0; k < 16; ++k)
                acc = __builtin_amdgcn_mfma_i32_32x32x32_i8(__builtin_bit_cast(i32x4, af[k]), bf[k], acc, 0, 0, 0);
        }
        if (failed && lane == 0) __hip_atomic_store(ctr + 17, 1u, __ATOMIC_RELAXED, __HIP_MEMORY_SCOPE_WORKGROUP);
        LDS_AS unsigned char* tp = smem + i * TILE_STRIDE + aoff;
#pragma unroll
        for (int r = 0; r < 16; ++r)
            __hip_atomic_fetch_add((LDS_AS int*)(tp + ((r & 3) + 8 * (r >> 2)) * 128), acc[r],
                                   __ATOMIC_RELAXED, __HIP_MEMORY_SCOPE_WORKGROUP);
        LGKMCNT0();
        if (lane == 0) __hip_atomic_fetch_add(ctr + i, 1u, __ATOMIC_RELAXED, __HIP_MEMORY_SCOPE_WORKGROUP);
        TRACE(i, 1);
        }
        if (!ready) {
            if (en == 0xFF) break;
            gate(src_d(en), src_d(en) < 4 ? (en & 15) - 2 : 17 - (en & 15));
            if (!failed) failed = !poll_flags(flag_off(en), myws + FX_OFF, tmo, call, FASTX < 0 ? 1u : (14u >> ((en >> 4) & 7)) & (((en & 15) + 16 - FASTX) >> 4), lane);
            CBAR();
            TRACE(i + 1, 0);
            peek((int)(t_lo >> 8) & 0xFF);
            __builtin_amdgcn_sched_barrier(0);
#pragma unroll
            for (int k = 0; k < 16; ++k) { aload(k, nbase); __builtin_amdgcn_sched_barrier(0); }
            __builtin_amdgcn_sched_barrier(0);
            if (i < 0) {
                if ((!DIRB || foreign) && !bfirst) {
                    const int ebw = (foreign ? downer : tile >> 2) * NBLK + 16 + (tile & 3);
                    if (foreign) gate(downer, (tile & 3) - 2);
                    if (!failed) failed = !wait_flags(flag_ptr(ebw), 0, tmo, call, lane);
                    CBAR();
                    if (foreign) STAMP(6);
                    peek((int)(t_lo >> 8) & 0xFF);
                    CBAR();
                    const unsigned bbase = (unsigned)ebw * BLK_BYTES + wave * 16384;
#pragma unroll
                    for (int k = 0; k < 16; ++k)
                        bf[k] = __builtin_bit_cast(i32x4, __builtin_amdgcn_raw_buffer_load_b128(ar, voff, (bbase | UPAGE(0, 0, bbase >> 17)) ^ krot ^ FRAG(k), LAUX));
                }
                TRACE(0, 0);
            }
            if (i < 0) {
                lds_wait(ctr + 16, 8u);
                __builtin_amdgcn_s_waitcnt(0x0F70);
                STAMP(1);
            }
        }
    }
    STAMP(5);
    TRACE_OUT();
    __builtin_amdgcn_s_waitcnt(0x0F70);
    if (!foreign || slot < 3) __builtin_amdgcn_endpgm();
    }
    asm volatile("" : "+s"(ap));
    peer = ap->ws[4 + ((j >> 2) & 3)];
    rank = ap->rank;
    scale = __builtin_bit_cast(float, __builtin_amdgcn_readfirstlane(__builtin_bit_cast(int, *(const CONST_AS float*)ap->sx * *(const CONST_AS float*)ap->sw)));
    if (role == 2 && wave >= 8 && (!OAUX || foreign || wave < 10)) {
        const u64 o_lo = ap->ord[tile & 3][slot][0], o_hi = ap->ord[tile & 3][slot][1];
        for (int i = wave - 8; i < NTASK; i += OAUX && !foreign ? 2 : 4) {
            const int sh = 8 * (i & 7);
            const int e = (int)(((i & 8 ? t_hi : t_lo) >> sh) & 0xFF);
            if (e == 0xFF) break;
            const unsigned oi = (unsigned)(((i & 8 ? o_hi : o_lo) >> sh) & 0xFF), idx = (tile >> 2) * NIDX + oi;
            lds_wait(ctr + i, 8u);
            if (!foreign || (e & 0x80)) {
                TRACE(i, 0);
                epilogue(i, e);
                if (foreign) {
                    VMCNT(0);
                    flag64_set(peer + FO_OFF + (size_t)idx * 128, call, 64u | (call & 63u), lane);
                }
                TRACE(i, 1);
                continue;
            }
            TRACE(i, 0);
            const bool failed = lds_get(ctr + 17) != 0;
            LDS_AS unsigned char* tp = smem + i * TILE_STRIDE;
            i32x4 q[4];
#pragma unroll
            for (int k = 0; k < 4; ++k) q[k] = *(LDS_AS i32x4*)(tp + 128 + k * 1024 + voff);
            __builtin_amdgcn_s_waitcnt(0xC07F);
            CBAR();
            if (scale < 0.0f) {
#pragma unroll
                for (int k = 0; k < 4; ++k) q[k] = -q[k];
            }
            unsigned m = 0;
            int mx = 0;
#pragma unroll
            for (int r = 15; r >= 0; --r) {
                m = __builtin_amdgcn_alignbit(m, (unsigned)-q[r >> 2][r & 3], 31);
                mx = max(mx, q[r >> 2][r & 3]);
            }
            if (P21) *(LDS_AS u32x4*)(tp + 128 + voff) = u32x4{0, 0, 0, 0};
            const bool wide = __builtin_amdgcn_ballot_w64(mx >= (1 << (P21 ? 21 : 24))) != 0;
            const unsigned c1 = __builtin_popcount(m), inc = wave_scan(c1);
            const unsigned tot = __builtin_amdgcn_readlane(inc, 63);
            const unsigned hbytes = (5 * tot + 31) >> 5 << 2;
            const unsigned nbytes = (128 + (wide ? tot * 4 : P21 ? hbytes + 2 * tot : 3 * tot) + 63) & ~63u;
            *(LDS_AS unsigned short*)(tp + lane * 2) = (unsigned short)m;
            if (wide) {
                unsigned ep = (unsigned)(size_t)tp + 128 + (inc - c1) * 4;
#pragma unroll
                for (int r = 0; r < 16; ++r)
                    asm volatile("v_cmpx_lt_i32 vcc, 0, %1\n\tds_write_b32 %0, %1\n\tv_add_u32 %0, 4, %0\n\ts_mov_b64 exec, -1"
                                 : "+v"(ep) : "v"(q[r >> 2][r & 3]) : "memory", "vcc");
            } else if (P21) {
                const unsigned tb = (unsigned)(size_t)tp + 128, ad = tb + 3072;
                unsigned a2 = tb + hbytes + 2 * (inc - c1), g = 5 * (inc - c1);
#pragma unroll
                for (int r0 = 0; r0 < 16; r0 += 6) {
                    unsigned acc = 0, sh5 = 0;
#pragma unroll
                    for (int r = r0; r < r0 + 6 && r < 16; ++r) {
                        int nb;
                        BITMASK(nb, m, r);
                        asm volatile("ds_write_b16 %0, %1" :: "v"((a2 & nb) | (ad & ~nb)), "v"(q[r >> 2][r & 3]) : "memory");
                        acc |= (((unsigned)q[r >> 2][r & 3] >> 16) & nb) << sh5;
                        STEP(a2, nb, 2);
                        STEP(sh5, nb, 5);
                    }
                    const u64 w = (u64)acc << (g & 31);
                    asm volatile("ds_or_b32 %0, %1\n\tds_or_b32 %0, %2 offset:4" :: "v"(tb + (g >> 5 << 2)), "v"((unsigned)w), "v"((unsigned)(w >> 32)) : "memory");
                    g += sh5;
                }
            } else {
                const unsigned tb = (unsigned)(size_t)tp + 128, c2 = -tb - 4 * tot, ad = tb + 2 * tot + 1536;
                unsigned a1 = tb + 2 * tot + inc - c1;
#pragma unroll
                for (int r = 0; r < 16; ++r) {
                    const unsigned nb = __builtin_amdgcn_sbfe(m, r, 1);
                    const unsigned ae = (a1 & nb) | (ad & ~nb);
                    a1 -= nb;
                    asm volatile("ds_write_b16 %0, %2\n\tds_write_b8_d16_hi %1, %2" :: "v"(ae * 2 + c2), "v"(ae), "v"(q[r >> 2][r & 3]) : "memory");
                }
            }
            const unsigned info = nbytes | (call & 63u) | (unsigned)((tile & 3) << 8 | e) << 16 | (wide ? F_WIDE : 0u) | (failed ? F_NAN : 0u);
            const char* dst = myws + OS_OFF + par * OS_PAR + (size_t)idx * OSLOT + 2048;
            TRACE(i, 1);
            TRACE2(i, 0);
#pragma unroll
            for (int k = 0; k < 5; ++k)
                if (k * 1024 < nbytes) {
                    const u32x4 pv = *(LDS_AS u32x4*)(tp + k * 1024 + voff);
                    if (k * 1024 + voff < nbytes) STORE_WT(voff, dst, pv, k * 1024 - 2048);
                }
            VMCNT(0);
            flag64_set(peer + FO_OFF + (size_t)idx * 128, call, info, lane);
            TRACE2(i, 1);
        }
        STAMP(5);
        TRACE_OUT();
        __builtin_amdgcn_endpgm();
    }
    {
        const int link = HELP && role == 2 ? tile >> 2 : j & 3, kstep = role == 1 ? 48 << OAUX : NIDX, kend = role == 1 ? KSPLIT : a.ntotal;
        const int k0 = role == 1 ? (KIL ? wave * 4 + (j >> 2) : (j >> 2) * 12 + wave) : OAUX && !foreign ? 48 + (wave - 10) * 24 + slot * 4 + (tile >> 2) : HELP ? KSPLIT + wave * 12 + (slot < 5 ? slot * 4 - 12 : slot * 2 - 4) + (tile & 3) : KSPLIT + wave * 16 + (NSLOT - 1 - slot) * 4 + (tile >> 2);
        LDS_AS unsigned char* tp = smem + (role == 1 ? wave : (wave & 7) + NTASK) * TILE_STRIDE;
        const int dn = 4 + link;
        const int npacked = a.npacked;
        bool failed = false;
        u64 fpre = 0;
        if (role == 1 && wave < 2) {
            const int c = j * 2 + wave;
            GLB_AS unsigned* rdy = (GLB_AS unsigned*)(a.ws[lane & 7] + RD_OFF + ((lane & 7) * 32 + c) * 128);
            GLB_AS unsigned* fl = (GLB_AS unsigned*)(myws + FX_OFF + (size_t)((lane & 16 ? ((lane >> 2) & 3) * NBLK + 16 + (lane & 3) : lane & 15) * 32 + c) * FSTRIDE);
            while ((unsigned)__builtin_amdgcn_s_memrealtime() - tstart < 300) __builtin_amdgcn_s_sleep(8);
            unsigned f = lane < 32 && !(DIRB && (lane >> 3) == 2) ? 0 : call;
            do {
                if (f < call) f = __hip_atomic_load(fl, __ATOMIC_RELAXED, __HIP_MEMORY_SCOPE_SYSTEM);
                if (__builtin_amdgcn_ballot_w64(f >= call) == ~0ull) {
                    CBAR();
                    if (lane - 1u < 7u) __hip_atomic_store(rdy + 2, call, __ATOMIC_RELAXED, __HIP_MEMORY_SCOPE_SYSTEM);
                    break;
                }
            } while ((unsigned)__builtin_amdgcn_s_memrealtime() - tstart < DEADLINE_TICKS);
            STAMP(2);
        }
        while ((unsigned)__builtin_amdgcn_s_memrealtime() - tstart < 1600) __builtin_amdgcn_s_sleep(8);
        const u64 t64 = flag64_get(myws + RD_OFF + (dn * 32 + (k0 & 31)) * 128 + 16);
        const unsigned turn = __builtin_amdgcn_readfirstlane((unsigned)t64 == call ? (unsigned)(t64 >> 32) + (2 + 16 - N1 - NSLOW) * 4 * PACEC + TSOFF : tstart * 16u);
        const char* theirs = a.ws[dn] + OS_OFF + par * OS_PAR;
        for (int n = 0; k0 + kstep * n < kend; ++n) {
            const int idx = link * NIDX + k0 + kstep * n;
            const unsigned toff = (k0 + kstep * n) * TSTEP + min(k0 + kstep * n, K1) * (TSTEP1 - TSTEP);
            if (!PFIRST)
                while ((int)((unsigned)__builtin_amdgcn_s_memrealtime() * 16u - turn - toff) < 0) __builtin_amdgcn_s_sleep(1);
            else if (n == 0)
                while ((int)((unsigned)__builtin_amdgcn_s_memrealtime() * 16u - turn - toff + 2400) < 0) __builtin_amdgcn_s_sleep(1);
            const unsigned info = (unsigned)fpre >= call ? (unsigned)(fpre >> 32)
                : poll_flag64(lane & 4 ? (unsigned)(TMO_OFF - FO_OFF) : idx * 128, myws + FO_OFF, tmo, call, (unsigned)(k0 + kstep * n + 256 - FASTO) >> 8, lane);
            CBAR();
            if (PFIRST)
                while ((int)((unsigned)__builtin_amdgcn_s_memrealtime() * 16u - turn - toff) < 0) __builtin_amdgcn_s_sleep(1);
            TRACE(n, 0);
            if (info == 0 || k0 + kstep * n >= npacked) {
                if (info == 0 || (unsigned)__builtin_amdgcn_readfirstlane(tmo_get(tmo)) == call) {
                    a.out[(size_t)rank * MSH * NSH + lane] = fnan;
                    break;
                }
                fpre = 0;
                TRACE(n, 1);
                continue;
            }
            const unsigned e = info >> 16;
            const int code = (e >> 4) & 7;
            const int srank = rank ^ (code < 4 ? dn ^ code : code & 3);
            float* op = a.out + (size_t)(srank * MSH + (e & 15) * 32 + (lane >> 3)) * NSH + (((e >> 8) & 3) * 4 + link) * 32 + (lane & 7) * 4;
            const char* srcp = theirs + (size_t)idx * OSLOT;
            const unsigned nbytes = min(info & 0xFFC0u, 4224u);
#pragma unroll
            for (int k = 0; k < 5; ++k)
                if (k * 1024 + voff < nbytes) LOAD_LDS_SYS((unsigned)(size_t)tp + k * 1024, voff, srcp + k * 1024);
            unsigned tv = tmo_get(tmo);
            VMCNT(0);
            tv = __builtin_amdgcn_readfirstlane(tv);
            asm volatile("" : "+s"(tv) :: "memory");
            fpre = 0;
            if (k0 + kstep * (n + 1) < kend) fpre = flag64_get(myws + FO_OFF + (size_t)(idx + kstep) * 128);
            failed = failed || tv == call;
            const float mul = failed || (info & F_NAN) ? fnan : __builtin_fabsf(scale);
            const unsigned m = *(LDS_AS unsigned short*)(tp + lane * 2);
            const unsigned c1 = __builtin_popcount(m), inc = wave_scan(c1);
            unsigned y[16];
            if (info & F_WIDE) {
                unsigned ix = inc - c1;
#pragma unroll
                for (int r = 0; r < 16; ++r) {
                    const int nb = __builtin_amdgcn_sbfe(m, r, 1);
                    const unsigned w = *(LDS_AS unsigned*)(tp + 128 + ix * 4);
                    ix -= nb;
                    y[r] = __builtin_bit_cast(unsigned, (float)(w & (unsigned)nb) * mul);
                }
            } else if (P21) {
                const unsigned tot = __builtin_amdgcn_readlane(inc, 63);
                const unsigned tb = (unsigned)(size_t)tp + 128;
                unsigned a2 = tb + ((5 * tot + 31) >> 5 << 2) + 2 * (inc - c1), g[3], win[3];
                u32x2 hw[3];
                int nb[16];
#pragma unroll
                for (int k = 0; k < 3; ++k) {
                    g[k] = 5 * (inc - c1 + __builtin_popcount(m & ((1u << 6 * k) - 1u)));
                    const LDS_AS unsigned* hp = (const LDS_AS unsigned*)(size_t)(tb + (g[k] >> 5 << 2));
                    hw[k] = u32x2{hp[0], hp[1]};
                }
#pragma unroll
                for (int r = 0; r < 16; ++r) {
                    y[r] = *(LDS_AS unsigned short*)(size_t)a2;
                    BITMASK(nb[r], m, r);
                    STEP(a2, nb[r], 2);
                }
#pragma unroll
                for (int r = 0; r < 16; r += 8)
                    asm volatile("" : "+v"(y[r]), "+v"(y[r + 1]), "+v"(y[r + 2]), "+v"(y[r + 3]),
                                      "+v"(y[r + 4]), "+v"(y[r + 5]), "+v"(y[r + 6]), "+v"(y[r + 7]) :: "memory");
                asm volatile("" : "+v"(hw[0]), "+v"(hw[1]), "+v"(hw[2]));
#pragma unroll
                for (int k = 0; k < 3; ++k) win[k] = __builtin_amdgcn_alignbit(hw[k][1], hw[k][0], g[k]);
                unsigned sh5 = 0;
#pragma unroll
                for (int r = 0; r < 16; ++r) {
                    if (r % 6 == 0) sh5 = 0;
                    asm("v_lshrrev_b32_sdwa %0, %1, %2 dst_sel:WORD_1 dst_unused:UNUSED_PRESERVE src0_sel:DWORD src1_sel:DWORD" : "+v"(y[r]) : "v"(sh5), "v"(win[r / 6]));
                    y[r] = __builtin_bit_cast(unsigned, (float)(y[r] & (unsigned)nb[r] & 0x1FFFFFu) * mul);
                    STEP(sh5, nb[r], 5);
                }
            } else {
                const unsigned tot = __builtin_amdgcn_readlane(inc, 63);
                const unsigned tb = (unsigned)(size_t)tp + 128, c2 = -tb - 4 * tot;
                unsigned a1 = tb + 2 * tot + inc - c1, wh[16];
#pragma unroll
                for (int r = 0; r < 16; ++r) {
                    y[r] = *(LDS_AS unsigned short*)(size_t)(a1 * 2 + c2);
                    wh[r] = *(LDS_AS unsigned char*)(size_t)a1;
                    a1 -= __builtin_amdgcn_sbfe(m, r, 1);
                }
#pragma unroll
                for (int r = 0; r < 16; r += 4)
                    asm volatile("" : "+v"(y[r]), "+v"(y[r + 1]), "+v"(y[r + 2]), "+v"(y[r + 3]),
                                      "+v"(wh[r]), "+v"(wh[r + 1]), "+v"(wh[r + 2]), "+v"(wh[r + 3]) :: "memory");
#pragma unroll
                for (int r = 0; r < 16; ++r)
                    y[r] = __builtin_bit_cast(unsigned, (float)((wh[r] << 16 | y[r]) & (unsigned)__builtin_amdgcn_sbfe(m, r, 1)) * mul);
            }
            fpre = (u64)(unsigned)__builtin_amdgcn_readfirstlane((unsigned)(fpre >> 32)) << 32 | (unsigned)__builtin_amdgcn_readfirstlane((unsigned)fpre);
            asm volatile("" : "+s"(fpre));
#pragma unroll
            for (int k = 0; k < 4; ++k) {
                const u32x4 o = {y[4 * k], y[4 * k + 1], y[4 * k + 2], y[4 * k + 3]};
                asm volatile("global_store_dwordx4 %0, %1, off sc0 sc1\n\ts_nop 1" :: "v"(op + (size_t)k * 8 * NSH), "v"(o) : "memory");
            }
            TRACE(n, 1);
        }
        STAMP(role == 1 ? 5 : 4);
        TRACE_OUT();
    }
    __builtin_amdgcn_s_waitcnt(0x0F70);
}

static void build_table(int foreign, float tstart, u64 (*tab)[2], float (*fin)[NTASK] = nullptr) {
    struct T { float t; int e, last; } task[128];
    const int nslot = HELP ? (foreign == 1 || foreign == 2 ? 5 : 7) : foreign ? NSLOT : NOWN;
    const float tb = PACEI / 400.0f, tx = PACEC / 400.0f;
    int n = 0;
    for (int d = 0; d < 8; ++d)
        for (int blk = 0; blk < 16; ++blk) {
            const int type = blk < N1 ? 1 : blk < N1 + N3 ? 3 : 2;
            float t;
            int direct = 0;
            if (d == 0) { if (foreign && type != 1) continue; t = 3.0f; }
            else if (d < 4) {
                if (type == 2 || (foreign && type == 3)) continue;
                t = TA + tb * blk;
                direct = foreign ? blk >= N1 - NRAW : blk == N1 + N3 - 1;
            } else {
                if (type == 1 || (foreign && (type == 3 || d == 4))) continue;
                t = TA + tx * (19 - blk) + (blk < N1 + NSLOW ? (N1 + NSLOW - blk) * (PSLOW / 1600.0f - tx) : 0.0f);
            }
            task[n].t = t + 0.001f * d; task[n].e = direct << 7 | d << 4 | blk; task[n].last = d && d < 4 && blk == N1 - 1; ++n;
        }
    for (int i = 1; i < n; ++i)
        for (int k = i; k > 0 && task[k].t < task[k - 1].t; --k) { T s = task[k]; task[k] = task[k - 1]; task[k - 1] = s; }
    float fr[NSLOT]; int cntv[NSLOT]; unsigned char lst[NSLOT][NTASK];
    for (int s = 0; s < NSLOT; ++s) { fr[s] = tstart + (!foreign && s < OSKEW ? 1.01f * OCOST : 0.0f); cntv[s] = 0; for (int i = 0; i < NTASK; ++i) lst[s][i] = 0xFF; }
    for (int i = 0; i < n; ++i) {
        int best = -1;
        for (int s = 0; s < nslot; ++s) {
            if (foreign && task[i].t > TRES + (nslot == 5 ? (TRK - TRK2) * tb : 0.0f) && (s < 3) != task[i].last) continue;
            if (!foreign && task[i].t > TA + (N1 + N3 - OTRK) * tb && (task[i].e & 0x40 ? s < 3 : s + 1 != ((task[i].e >> 4) & 3))) continue;
            if (cntv[s] < NTASK - 1 && (best < 0 || fr[s] < fr[best])) best = s;
        }
        fr[best] = (fr[best] > task[i].t ? fr[best] : task[i].t) + (foreign ? TCOST : OCOST);
        if (fin) fin[best][cntv[best]] = fr[best];
        lst[best][cntv[best]++] = (unsigned char)task[i].e;
    }
    for (int s = 0; s < NSLOT; ++s)
        for (int h = 0; h < 2; ++h) {
            u64 v = 0;
            for (int i = 7; i >= 0; --i) v = v << 8 | lst[s][h * 8 + i];
            tab[s][h] = v;
        }
}

static void build_all(Args& a) {
    build_table(0, OST0, a.tab[0]);
    auto byte = [&](int t, int s, int i) { return i >= NTASK ? 0xFF : (int)((a.tab[1 + t][s][i >> 3] >> (8 * (i & 7))) & 0xFF); };
    int n = 0;
    float fin[4][NSLOT][NTASK];
    int next[4][NSLOT];
    for (int t = 0; t < 4; ++t) {
        build_table(1 + t, FSTART + FSTEP * t, a.tab[1 + t], fin[t]);
        for (int s = 0; s < NSLOT; ++s) {
            for (int i = 0; i < NTASK; ++i) n += byte(t, s, i) < 0x80;
            a.ord[t][s][0] = a.ord[t][s][1] = 0;
            next[t][s] = 0;
        }
    }
    a.npacked = n;
    for (int k = 0; k < n; ++k) {
        int bt = -1, bs = 0;
        for (int t = 0; t < 4; ++t)
            for (int s = 0; s < NSLOT; ++s)
                if (byte(t, s, next[t][s]) < 0x80 && (bt < 0 || fin[t][s][next[t][s]] < fin[bt][bs][next[bt][bs]])) { bt = t; bs = s; }
        const int i = next[bt][bs]++;
        a.ord[bt][bs][i >> 3] |= (u64)k << (8 * (i & 7));
    }
    for (int blk = 0; blk < 16; ++blk)
        for (int t = 0; t < 4; ++t)
            for (int s = 0; s < NSLOT; ++s)
                for (int i = 0; i < NTASK; ++i)
                    if (byte(t, s, i) != 0xFF && (byte(t, s, i) & 0x8F) == (0x80 | blk)) a.ord[t][s][i >> 3] |= (u64)n++ << (8 * (i & 7));
    a.ntotal = n;
}

extern "C" void kernel_launch(int rank, int nranks, ncclComm_t comm,
                              hipStream_t stream,
                              void* const* d_in, const int* in_sizes, int n_in,
                              void* d_out, int out_size,
                              const int* in_dtypes, int out_dtype,
                              void* const* d_out_all,
                              void* const* d_ws_all, size_t ws_bytes) {
    if (nranks != NRANK || (unsigned)rank >= (unsigned)NRANK || n_in != 4 || ws_bytes < WS_NEED ||
        in_sizes[0] != MSH * KDIM || in_sizes[1] != KDIM * NSH || out_size != NRANK * MSH * NSH) {
        fprintf(stderr, "unexpected problem shape\n");
        return;
    }
    HIP_CHECK(hipFuncSetAttribute((const void*)ag_gemm_kernel, hipFuncAttributeMaxDynamicSharedMemorySize, LDS_BYTES));
    Args a;
    a.sx = (const float*)d_in[2];
    a.sw = (const float*)d_in[3];
    a.out = (float*)d_out;
    for (int d = 0; d < NRANK; ++d) a.ws[d] = (char*)d_ws_all[rank ^ d];
    for (int i = 0; i < 4; ++i) a.outp[i] = (float*)d_out_all[rank ^ (4 + i)];
    for (int i = 0; i < NTHREADS / 64; ++i) {
        Rec& r = a.rec[i];
        const int d = i & 7;
        r.x = (const signed char*)d_in[0];
        r.w = (const signed char*)d_in[1];
        r.ws = a.ws[0];
        r.peer = a.ws[d];
        r.src = d < 4 ? r.x : r.w + (d & 3) * 32;
        r.dst = r.peer + (size_t)(d * NBLK + (d < 4 ? 0 : 16)) * BLK_BYTES;
        r.cnt = r.ws + CNT_OFF + i * 4;
        r.dst1 = r.dst + XS_PAR;
    }
    build_all(a);
    if (a.ntotal > NIDX || a.ntotal > KSPLIT + (HELP ? 96 : 128) || NSLOT != 7 || NRAW > 1) return;
    a.rank = rank;
    ag_gemm_kernel<<<dim3(256), dim3(NTHREADS), LDS_BYTES, stream>>>(a);
    HIP_CHECK(hipGetLastError());
}
```

```cpp
#include <hip/hip_runtime.h>
#include <rccl/rccl.h>
#include <cstdio>
#include <cstddef>

#define HIP_CHECK(expr) do { hipError_t _e = (expr); if (_e != hipSuccess) { \
    fprintf(stderr, "HIP error %s at %s:%d\n", hipGetErrorString(_e), __FILE__, __LINE__); \
    return; } } while (0)

typedef int i32x2 __attribute__((ext_vector_type(2)));
typedef int i32x4 __attribute__((ext_vector_type(4)));
typedef int i32x16 __attribute__((ext_vector_type(16)));
typedef unsigned u32x2 __attribute__((ext_vector_type(2)));
typedef unsigned u32x4 __attribute__((ext_vector_type(4)));
typedef float f32x4 __attribute__((ext_vector_type(4)));
typedef unsigned long long u64;
typedef unsigned __attribute__((aligned(1))) u32_u;
#define LDS_AS __attribute__((address_space(3)))
#define GLB_AS __attribute__((address_space(1)))
#define CONST_AS __attribute__((address_space(4)))

#define N1 10
#define N3 2
#define PACE 906
#define PACEI 864
#define PACEC 880
#define GOFF 6600
#define TMIN 490
#define DEFER 1400
#define LMAX 115
#define ROT 4
#define TSOFF 500
#define TSTEP 44
#define TA 5.3f
#define FSTART 6.0f
#define FSTEP 2.2f
#define TCOST 1.3f
#define OST0 5.0f
#define TRK 3.95f
#define TRES (TA + (N1 - TRK) * PACEI / 400)
#define FSTRIDE 256
#define PSLEEP 1
#define KSPLIT 144
#define SWZ 3
#define NPOLL 3
#define NSIG 24
#define PSLACK 160
#define PDELAY 60
#define P21 1
#define FASTX 16
#define FASTO 256
#define DELAYW 12
#define NSLOW N3
#define K1 96
#define PSLOW (4 * PACEC + K1 * TSTEP / NSLOW)
#define TSTEP1 (NSLOW * PSLOW / K1)
#define FPOS 4
#define FRAG(k) (UPAGE((k) & 3, (k) >> 2, 0) | ((k) >> 2) * 1024)
#define OTRK 2.5f
#define SLEEP0 2
#define SLEEP1 10
#define SLEEP2 8
#define UPAGE(c, f, blk) ((unsigned)((c) ^ ((SWZ & 1 ? (f) : 0) ^ (SWZ & 2 ? (blk) & 3 : 0))) << 12)
#define HAVE_ROLE 1
__host__ __device__ inline int role_of(int b, int& ix) {
    const int r = b >> 3;
    ix = r < 4 ? b : (b & 7) * (r < 6 ? 2 : 26) + r - (r < 6 ? 4 : 6);
    return r < 4 ? 0 : r < 6 ? 1 : 2;
}
constexpr int NRANK = 8;
constexpr int KDIM = 4096, MSH = 512, NSH = 512;
constexpr unsigned BLK_BYTES = 32 * KDIM;
constexpr int NBLK = 20;
constexpr unsigned OSLOT = 4608;
constexpr int NSLOT = 7, NOWN = 6, NTASK = 16, NIDX = 256;
constexpr size_t XS_PAR = (size_t)NRANK * NBLK * BLK_BYTES;
constexpr size_t OS_OFF = 2 * XS_PAR, OS_PAR = (size_t)4 * NIDX * OSLOT;
constexpr size_t FX_OFF = OS_OFF + 2 * OS_PAR;
constexpr unsigned FX_SIZE = NRANK * NBLK * 32 * FSTRIDE;
constexpr size_t FO_OFF = FX_OFF + FX_SIZE;
constexpr size_t RD_OFF = FO_OFF + 4 * NIDX * 128;
constexpr size_t CNT_OFF = RD_OFF + NRANK * 32 * 128;
constexpr size_t TMO_OFF = CNT_OFF + 16384;
constexpr size_t ST_OFF = TMO_OFF + 1024;
constexpr size_t DUMMY_OFF = ST_OFF + 256 * 128;
constexpr size_t WS_NEED = DUMMY_OFF + 256 * 128;
constexpr unsigned DEADLINE_TICKS = 20000000u;
constexpr int TILE_STRIDE = 4352;
constexpr int LDS_MAIN = 131072;
constexpr int LDS_CTR = LDS_MAIN;
constexpr int LDS_BYTES = LDS_CTR + 256;
constexpr int NTHREADS = 768;
static_assert((NTASK + 8) * TILE_STRIDE <= LDS_MAIN, "tiles");
#define DIAG_N 0
#define STAMP(i)
#define TRACE(i, h)
#define TRACE2(i, h)
#define TRACE_OUT()

struct Rec {
    const signed char* src;
    char* dst;
    const char* cnt;
    char* peer;
    const signed char* x;
    const signed char* w;
    char* ws;
    char* dst1;
};
struct Args {
    Rec rec[NTHREADS / 64];
    const float* sx;
    const float* sw;
    float* out;
    char* ws[NRANK];
    u64 tab[5][NSLOT][2];
    u64 ord[4][NSLOT][2];
    int rank;
    int npacked, ntotal;
};

#define STORE_WT(voff, base, val, off) \
    asm volatile("global_store_dwordx4 %0, %2, %1 offset:%3 sc0 sc1\n\ts_nop 1" \
                 :: "v"(voff), "s"(base), "v"(val), "i"(off) : "memory")
#define VMCNT(n) asm volatile("s_waitcnt vmcnt(%0)" :: "i"(n) : "memory")
#define LGKMCNT0() asm volatile("s_waitcnt lgkmcnt(0)" ::: "memory")
#define CBAR() asm volatile("" ::: "memory")

__device__ __forceinline__ void set_flag(char* p, unsigned call, int lane) {
    if (lane == 0) __hip_atomic_store((GLB_AS unsigned*)p, call, __ATOMIC_RELAXED, __HIP_MEMORY_SCOPE_SYSTEM);
}
__device__ __forceinline__ unsigned lds_get(LDS_AS unsigned* p) {
    return __builtin_amdgcn_readfirstlane(__hip_atomic_load(p, __ATOMIC_RELAXED, __HIP_MEMORY_SCOPE_WORKGROUP));
}
__device__ __forceinline__ void lds_wait(LDS_AS unsigned* p, unsigned v) {
    while (lds_get(p) < v) __builtin_amdgcn_s_sleep(1);
    CBAR();
}

#define TMOEVERY 8
__device__ __forceinline__ unsigned tmo_get(GLB_AS unsigned* tmo) {
    return __hip_atomic_load(tmo, __ATOMIC_RELAXED, __HIP_MEMORY_SCOPE_AGENT);
}
__device__ __forceinline__ bool wait_flags(GLB_AS unsigned* p, unsigned f, GLB_AS unsigned* tmo, unsigned call, int lane) {
    if (((unsigned)__builtin_amdgcn_ballot_w64(f >= call) & 15u) == 15u) return true;
    const unsigned t0 = (unsigned)__builtin_amdgcn_s_memrealtime();
    for (unsigned it = 1;; ++it) {
        if (lane < 4) f = __hip_atomic_load(p, __ATOMIC_RELAXED, __HIP_MEMORY_SCOPE_SYSTEM);
        if (((unsigned)__builtin_amdgcn_ballot_w64(f >= call) & 15u) == 15u) return true;
        if (it % TMOEVERY == 0 &&
            (__builtin_amdgcn_readfirstlane(tmo_get(tmo)) == call || (unsigned)__builtin_amdgcn_s_memrealtime() - t0 > DEADLINE_TICKS)) {
            if (lane == 0) __hip_atomic_store(tmo, call, __ATOMIC_RELAXED, __HIP_MEMORY_SCOPE_AGENT);
            return false;
        }
        __builtin_amdgcn_s_sleep(PSLEEP);
    }
}

#define PDEPTH 7
#define PGAP 0
#define POLL(LOAD, CHECK) \
    "v_mov_b64 v[166:167], 0\n\ts_mov_b32 %[it], 0\n\ts_mov_b32 exec_hi, 0\n" \
    "1:\n\ts_add_u32 %[it], %[it], 1\n\ts_and_b32 vcc_lo, %[it], 31\n\ts_cselect_b32 vcc_lo, 0, 16\n\ts_or_b32 exec_lo, %[pend], vcc_lo\n\t" \
    LOAD " %[vo], %[base] sc0 sc1\n\ts_mov_b32 exec_lo, 31\n\ts_memrealtime vcc\n\ts_sleep %[gap]\n\t" \
    "s_cmp_lg_u32 %[fast], 0\n\ts_cbranch_scc1 2f\n\ts_waitcnt vmcnt(0)\n" \
    "2:\n\ts_waitcnt vmcnt(%[dep]) lgkmcnt(0)\n\ts_cmp_eq_u32 %[it], 1\n\ts_cselect_b32 %[t0], vcc_lo, %[t0]\n\ts_sub_u32 vcc_lo, vcc_lo, %[t0]\n\t" \
    "s_cmp_gt_u32 vcc_lo, %[dl]\n\ts_cbranch_scc1 3f\n\tv_cmp_le_u32 vcc, %[call], v166\n\ts_bitcmp1_b32 vcc_lo, 4\n\ts_cbranch_scc1 3f\n\t" \
    CHECK "s_andn2_b32 %[pend], %[pend], vcc_lo\n\ts_cbranch_scc1 1b\n" \
    "3:\n\ts_mov_b64 exec, -1"
__device__ __forceinline__ bool poll_flags(unsigned vo, const char* base, GLB_AS unsigned* tmo, unsigned call, unsigned fast, int lane) {
    unsigned pend = 15, it, t0;
    asm volatile(POLL("global_load_dword v166,", "")
                 : [pend] "+s"(pend), [it] "=&s"(it), [t0] "=&s"(t0)
                 : [vo] "v"(vo), [base] "s"(base), [call] "s"(call), [fast] "s"(fast), [dep] "i"(PDEPTH), [gap] "i"(PGAP), [dl] "i"(DEADLINE_TICKS)
                 : "memory", "vcc", "scc", "v166", "v167");
    if (pend && lane == 0) __hip_atomic_store(tmo, call, __ATOMIC_RELAXED, __HIP_MEMORY_SCOPE_AGENT);
    return !pend;
}
__device__ __forceinline__ unsigned poll_flag64(unsigned vo, const char* base, GLB_AS unsigned* tmo, unsigned call, unsigned fast, int lane) {
    unsigned pend = 1, it, t0, info;
    asm volatile(POLL("global_load_dwordx2 v[166:167],",
                      "v_readfirstlane_b32 %[info], v167\n\ts_xor_b32 vcc_hi, %[info], %[call]\n\ts_and_b32 vcc_hi, vcc_hi, 63\n\ts_cbranch_scc1 1b\n\t")
                 : [pend] "+s"(pend), [it] "=&s"(it), [t0] "=&s"(t0), [info] "=&s"(info)
                 : [vo] "v"(vo), [base] "s"(base), [call] "s"(call), [fast] "s"(fast), [dep] "i"(PDEPTH), [gap] "i"(PGAP), [dl] "i"(DEADLINE_TICKS)
                 : "memory", "vcc", "scc", "v166", "v167");
    if (pend && lane == 0) __hip_atomic_store(tmo, call, __ATOMIC_RELAXED, __HIP_MEMORY_SCOPE_AGENT);
    return pend ? 0 : info;
}

template <int V> struct IC { static constexpr int value = V; };
constexpr unsigned F_WIDE = 1u << 30, F_NAN = 1u << 31, F_SKIP = 1u << 29;
#define BITMASK(nb, m, r) asm("v_bfe_i32 %0, %1, %2, 1" : "=v"(nb) : "v"(m), "i"(r))
#define STEP(x, nb, n) asm("v_mad_i32_i24 %0, %1, %2, %3" : "=v"(x) : "v"(nb), "i"(-(n)), "v"(x))
__device__ __forceinline__ unsigned wave_scan(unsigned x) {
    x += __builtin_amdgcn_update_dpp(0u, x, 0x111, 0xf, 0xf, false);
    x += __builtin_amdgcn_update_dpp(0u, x, 0x112, 0xf, 0xf, false);
    x += __builtin_amdgcn_update_dpp(0u, x, 0x114, 0xf, 0xf, false);
    x += __builtin_amdgcn_update_dpp(0u, x, 0x118, 0xf, 0xf, false);
    x += __builtin_amdgcn_update_dpp(0u, x, 0x142, 0xa, 0xf, false);
    x += __builtin_amdgcn_update_dpp(0u, x, 0x143, 0xc, 0xf, false);
    return x;
}
#define SIG_A "s_lshl_b32 %[t], %[b], 1\n\ts_add_u32 %[t], %[t], %[w]\n\ts_lshl_b32 %[t], %[t], 6\n\ts_add_u32 %[t], %[t], %[st]\n\tv_mov_b32 v13, %[t]\n\t"
#define SIG_B "v_add_u32_e64 v14, %[n], 1\n\ts_mov_b64 exec, 1\n\tglobal_store_dword v13, v14, s[42:43] sc0 sc1\n\ts_mov_b64 exec, -1\n\t"
#define SIG_W "s_waitcnt vmcnt(4)\n\t"
#define LOAD_LDS_SYS(ldsaddr, voffs, base) \
    asm volatile("s_nop 0\n\tglobal_load_lds_dwordx4 %1, %2 sc0 sc1" :: "{m0}"(ldsaddr), "v"(voffs), "s"(base) : "memory")
#define LOAD_LDS(ldsaddr, voffs, base) \
    asm volatile("s_nop 0\n\tglobal_load_lds_dwordx4 %1, %2" :: "{m0}"(ldsaddr), "v"(voffs), "s"(base) : "memory")

__device__ __forceinline__ u64 flag64_get(const char* p) {
    return __hip_atomic_load((GLB_AS u64*)p, __ATOMIC_RELAXED, __HIP_MEMORY_SCOPE_SYSTEM);
}
__device__ __forceinline__ void flag64_set(char* p, unsigned call, unsigned info, int lane) {
    if (lane == 0) __hip_atomic_store((GLB_AS u64*)p, (u64)info << 32 | call, __ATOMIC_RELAXED, __HIP_MEMORY_SCOPE_SYSTEM);
}
#define IF_DIAG(...)
__global__ void __launch_bounds__(NTHREADS, 3) __attribute__((amdgpu_num_vgpr(83))) ag_gemm_kernel(Args a_) {
    const CONST_AS Args* ap = (const CONST_AS Args*)__builtin_amdgcn_kernarg_segment_ptr();
    unsigned tid = threadIdx.x, w64, call, stmp;
    int b = blockIdx.x;
    const signed char *xin, *win;
    char *myws, *peer;
    u64 t_first, t_top;
#define PRO_(...)
    asm volatile(
        IF_DIAG("s_memrealtime %[t0]\n\t") "v_readfirstlane_b32 %[w], v0\n\ts_load_dwordx16 s[36:51], %[ap], %[w]\n\t"
        "s_cmp_gt_u32 %[b], 31\n\ts_cbranch_scc1 3f\n\ts_sub_u32 %[t], %[w], 64\n\ts_cmpk_lt_u32 %[t], 192\n\ts_cbranch_scc0 2f\n\t"
        "v_lshlrev_b32 v1, 12, v0\n\tv_lshrrev_b32 v10, 1, v0\n\ts_mov_b32 %[t], 0x1f010\n\tv_bitop3_b32 v1, v1, %[t], v10 bitop3:0xc8\n\t"
        "v_lshl_add_u32 v1, %[b], 7, v1\n\ts_waitcnt lgkmcnt(0)\n\tglobal_load_dwordx4 v[2:5], v1, s[36:37]\n\t" PRO_("s_memrealtime %[t2]\n\t")
        "s_lshl_b32 %[t], %[b], 6\n\ts_load_dword %[n], s[40:41], %[t]\n\ts_barrier\n\tglobal_load_dwordx4 v[6:9], v1, s[36:37] offset:32\n\t"
        "v_lshlrev_b32 v10, 4, v0\n\tv_and_b32 v10, 0x3f0, v10\n\tv_lshl_or_b32 v10, %[b], 12, v10\n\ts_movk_i32 m0, 2048-64\n\ts_sleep %[s0]\n\t"
        "global_load_lds_dwordx4 v1, s[36:37] offset:64\n\ts_movk_i32 m0, 3072-96\n\tv_xor_b32 v11, 0x1000, v10\n\tglobal_load_lds_dwordx4 v1, s[36:37] offset:96\n\t"
        SIG_A "s_waitcnt lgkmcnt(0)\n\t" PRO_("s_memrealtime %[t3]\n\t") "s_bitcmp0_b32 %[n], 0\n\ts_cselect_b64 s[38:39], s[50:51], s[38:39]\n\t" SIG_B
        SIG_W "global_store_dwordx4 v10, v[2:5], s[38:39] sc0 sc1\n\ts_memrealtime %[t1]\n\ts_sleep %[s1]\n\t" SIG_W
        "global_store_dwordx4 v11, v[6:9], s[38:39] offset:1024 sc0 sc1\n\t" PRO_("s_memrealtime %[t4]\n\t") "s_branch 4f\n\t.p2align 6\n"
        "2:\n\ts_cmpk_gt_u32 %[t], 447\n\ts_cbranch_scc1 3f\n\t"
        "v_lshlrev_b32 v1, 8, v0\n\tv_lshlrev_b32 v10, 4, v0\n\tv_and_b32 v1, 0x3e00, v1\n\tv_and_or_b32 v1, v10, 16, v1\n\t"
        "v_lshl_add_u32 v1, %[b], 16, v1\n\ts_lshl_b32 m0, %[w], 8\n\ts_waitcnt lgkmcnt(0)\n\tglobal_load_lds_dwordx4 v1, s[36:37]\n\t"
        "s_lshl_b32 %[t], %[b], 6\n\ts_barrier\n\t" PRO_("s_memrealtime %[t2]\n\t") "s_load_dword %[n], s[40:41], %[t]\n\t"
        "s_addk_i32 m0, 0x400\n\tv_add_u32 v13, 0x4000, v1\n\tglobal_load_lds_dwordx4 v13, s[36:37]\n\t"
        "v_and_b32 v12, 0x2e0, v10\n\tv_and_or_b32 v12, v0, 16, v12\n\tv_lshlrev_b32 v2, 3, v0\n\tv_and_or_b32 v12, v2, 8, v12\n\tv_lshl_add_u32 v12, %[w], 8, v12\n\t"
        "s_addk_i32 m0, 0x400\n\tv_add_u32 v14, 0x8000, v1\n\tglobal_load_lds_dwordx4 v14, s[36:37]\n\t"
        "s_addk_i32 m0, 0x400\n\tv_add_u32 v1, 0xc000, v1\n\tglobal_load_lds_dwordx4 v1, s[36:37]\n\t"
        "v_and_b32 v10, 0x3f0, v10\n\tv_lshl_or_b32 v10, %[b], 12, v10\n\tv_xor_b32 v11, 0x1000, v10\n\t"
        SIG_A "s_waitcnt lgkmcnt(0)\n\t" PRO_("s_memrealtime %[t3]\n\t") "s_bitcmp0_b32 %[n], 0\n\ts_cselect_b64 s[38:39], s[50:51], s[38:39]\n\t" SIG_B
        SIG_W "ds_read_b64_tr_b8 v[2:3], v12\n\tds_read_b64_tr_b8 v[4:5], v12 offset:256\n\ts_waitcnt lgkmcnt(0)\n\t"
        "global_store_dwordx4 v10, v[2:5], s[38:39] sc0 sc1\n\ts_memrealtime %[t1]\n\ts_sleep %[s2]\n\t" SIG_W
        "ds_read_b64_tr_b8 v[6:7], v12 offset:1024\n\tds_read_b64_tr_b8 v[8:9], v12 offset:1280\n\ts_waitcnt lgkmcnt(0)\n\t"
        "global_store_dwordx4 v11, v[6:9], s[38:39] offset:1024 sc0 sc1\n\t" PRO_("s_memrealtime %[t4]\n\t") "s_branch 4f\n\t.p2align 6\n"
        "3:\n\tv_lshlrev_b32 v1, 2, v0\n\tv_and_b32 v1, 0xfc, v1\n\tv_add_u32 v1, %[ctr], v1\n\tv_mov_b32 v2, 0\n\tds_write_b32 v1, v2\n\t"
        "s_lshl_b32 %[t], %[b], 6\n\ts_waitcnt lgkmcnt(0)\n\ts_barrier\n\ts_load_dword %[n], s[40:41], %[t]\n\ts_memrealtime %[t1]\n"
        "4:\n\ts_waitcnt lgkmcnt(0)"
        : [w] "=&s"(w64), [n] "=&s"(call), [t] "=&s"(stmp), [t1] "=&s"(t_first), IF_DIAG([t0] "=&s"(t_top),) [b] "+s"(b), [ap] "+s"(ap),
          PRO_([t2] "=&s"(t_ka), [t3] "=&s"(t_cnt), [t4] "=&s"(t_ack),)
          "=&{s[42:43]}"(peer), "=&{s[44:45]}"(xin), "=&{s[46:47]}"(win), "=&{s[48:49]}"(myws)
        : [ctr] "i"(LDS_CTR), [st] "i"((unsigned)ST_OFF), [s0] "i"(SLEEP0), [s1] "i"(SLEEP1), [s2] "i"(SLEEP2), "{v0}"(tid)
        : "memory", "scc", "v1", "v2", "v3", "v4", "v5", "v6", "v7", "v8", "v9", "v10", "v11", "v12", "v13", "v14", "s36", "s37", "s38", "s39", "s40", "s41", "s50", "s51");
    asm volatile("" : "+{v0}"(tid));
    call += 1u;
    const unsigned par = call & 1u;
    const unsigned tstart = (unsigned)t_first;
    static_assert(sizeof(Rec) == 64 && (SWZ & 1) && KDIM == 4096 && NSH == 512, "prologue");
    const CONST_AS Args& a = *ap;
    LDS_AS unsigned char* smem = (LDS_AS unsigned char*)0;
    int lane = tid & 63;
    asm volatile("" : "+v"(lane));
    __builtin_assume(lane >= 0 && lane < 64);
    const int wave = w64 >> 6;
    LDS_AS unsigned* ctr = (LDS_AS unsigned*)(smem + LDS_CTR);
    int j;
    const int role = role_of(b, j);
    STAMP(7);

    GLB_AS unsigned* cnt = (GLB_AS unsigned*)(myws + CNT_OFF) + b * 16 + wave;
    const unsigned voff = lane * 16;
    unsigned xsrc_off, lo[4];
    auto image = [&]() {
        unsigned l = lane;
        asm volatile("" : "+v"(l));
        xsrc_off = (l >> 3) * KDIM + (((l & 7) ^ (l >> 3)) * 16);
#pragma unroll
        for (int f = 0; f < 4; ++f) lo[f] = ((l & 31) * 128 + (((l >> 5) ^ (l & 7)) * 16)) ^ (f * 32);
        asm volatile("" : "+v"(xsrc_off), "+v"(lo[0]), "+v"(lo[1]), "+v"(lo[2]), "+v"(lo[3]));
    };

    const unsigned wtr = (16 * (lane >> 5) + ((lane & 15) >> 1)) * 32 + 16 * ((lane >> 4) & 1) + 8 * (lane & 1);
    if (__builtin_expect(role == 0, 1)) {
        const int c = j;
        const int d = wave < 8 ? wave : wave - 4;
        GLB_AS unsigned* tmo = (GLB_AS unsigned*)(myws + TMO_OFF);
        const char* mine = myws + par * XS_PAR;
        char* fmine = myws + FX_OFF + (size_t)c * FSTRIDE;
        if (wave == 0 || wave >= 8) {
            if (wave >= 7 + NPOLL) __builtin_amdgcn_endpgm();
            for (int i = wave - 7; i > 0; --i) __builtin_amdgcn_s_sleep(14);
            __builtin_amdgcn_s_sleep(PDELAY);
            for (unsigned it = 1;; ++it) {
                u32x4 f = {call, call, call, call};
                u32x2 g = {call, call};
                asm volatile("s_mov_b64 exec, 0xfe\n\tglobal_load_dwordx4 %0, %2, %3 sc0 sc1\n\tglobal_load_dwordx2 %1, %2, %3 offset:16 sc0 sc1\n\ts_mov_b64 exec, -1\n\ts_waitcnt vmcnt(0)"
                             : "+v"(f), "+v"(g) : "v"(lane * 4096), "s"(myws + RD_OFF + c * 128) : "memory");
                unsigned lvl = f[2] >= call ? 3 : f[0] < call ? 0 : f[1] < call ? 1 : 2;
                if ((unsigned)__builtin_amdgcn_s_memrealtime() - tstart > DEADLINE_TICKS || (it % 64 == 0 && (unsigned)__builtin_amdgcn_readfirstlane(tmo_get(tmo)) == call)) {
                    if (lane == 0) __hip_atomic_store(tmo, call, __ATOMIC_RELAXED, __HIP_MEMORY_SCOPE_AGENT);
                    lvl = 4;
                    g = u32x2{call, 1};
                }
                if (lane < 8) {
                    __hip_atomic_fetch_max(ctr + 32 + lane, lvl, __ATOMIC_RELAXED, __HIP_MEMORY_SCOPE_WORKGROUP);
                    if (g[0] == call) ctr[24 + lane] = g[1];
                }
                CBAR();
                if (__builtin_amdgcn_ballot_w64(lvl >= 3 && g[0] == call) == ~0ull) break;
            }
            if (lane == 0) __hip_atomic_store(cnt, call, __ATOMIC_RELAXED, __HIP_MEMORY_SCOPE_AGENT);
            LGKMCNT0();
            STAMP(3);
            __builtin_amdgcn_endpgm();
        }
        auto now16 = [&]() { return (unsigned)__builtin_amdgcn_s_memrealtime() * 16u; };
        const int blk0 = d * NBLK + (d < 4 ? 0 : 16);
        char* prev_flag = peer + FX_OFF + (size_t)(blk0 * 32 + c) * FSTRIDE;
        const char* theirs = peer + par * XS_PAR;
        u32x4 v[4];
        auto second = [&]() {
            const signed char* src = d < 4 ? xin + BLK_BYTES + c * 128 : win + (size_t)c * 128 * NSH + (d & 3) * 32 + 128;
            const unsigned so = d < 4 ? (lane & 31) * KDIM + (lane >> 5) * 16 : (lane >> 1) * NSH + (lane & 1) * 16;
            const int step = d < 4 ? 32 : 32 * NSH;
#pragma unroll
            for (int f = 0; f < 4; ++f) LOAD_LDS(wave * 16384 + 4096 + f * 1024, so, src + f * step);
        };
        second();
        VMCNT(DIAG_N + 7);
        unsigned tn = tstart * 16u + 2 * PACE;
#pragma unroll
        for (int f = 2; f < 8; ++f) {
            if (f == 4) VMCNT(2);
            if (d >= 4) {
                LDS_AS unsigned char* bp = smem + wave * 16384 + wtr + f * 1024;
                const i32x2 l2 = __builtin_amdgcn_ds_read_tr8_b64_v2i32((LDS_AS i32x2*)bp);
                const i32x2 h2 = __builtin_amdgcn_ds_read_tr8_b64_v2i32((LDS_AS i32x2*)(bp + 256));
                v[0] = __builtin_bit_cast(u32x4, i32x4{l2[0], l2[1], h2[0], h2[1]});
            } else v[0] = *(LDS_AS u32x4*)(smem + (f < 4 ? 0 : wave * 16384) + f * 1024 + voff);
            int late;
            while ((late = (int)(now16() - tn)) < 0) __builtin_amdgcn_s_sleep(1);
            tn += PACE + max(late - PSLACK, 0);
            IF_DIAG(xw += max(late - PSLACK, 0) >> 4;)
            STORE_WT(voff, theirs + (size_t)(blk0 + (f >> 2)) * BLK_BYTES + UPAGE(c, f & 3, f >> 2), v[0], (f & 3) * 1024);
            if (f == FPOS) { VMCNT(FPOS - 3); set_flag(prev_flag, call, lane); }
            TRACE2(f, 0);
        }
        prev_flag += 32 * FSTRIDE;
        TRACE(12, 0);
        auto set_flags = [&](char* p) { asm volatile("s_mov_b64 exec, 1\n\tglobal_store_dword %0, %1, %2 sc0 sc1\n\ts_mov_b64 exec, -1" :: "v"(lane * FX_SIZE), "v"(call), "s"(p) : "memory"); };
        char* const dummy = myws + DUMMY_OFF + (c * 8 + wave) * 128;
        const unsigned period = d < 4 ? 4 * PACEI : 4 * PACEC;
        while (!(tn = lds_get(ctr + 24 + d))) {
            if (prev_flag != dummy) { VMCNT(0); set_flags(prev_flag); prev_flag = dummy; }
            __builtin_amdgcn_s_sleep(1);
        }
        tn += ((c + ROT * d) & 31) * period / 32;
        const int nblk = d < 4 ? N1 + N3 - 2 : 2 + 16 - N1;
#pragma clang loop unroll(disable)
        for (int n = 0; n < nblk; ++n) {
            const int blk = d < 4 ? n + 2 : n < 2 ? 18 + n : 17 - n;
            const int sblk = blk < 16 ? blk : (n + 2) * NBLK + 12 + d;
            if ((int)(tn - now16()) > DEFER) { VMCNT(0); set_flags(prev_flag); prev_flag = dummy; }
            unsigned lvl;
            while ((lvl = lds_get(ctr + 32 + d)) < (unsigned)min(n + 1, 3)) __builtin_amdgcn_s_sleep(1);
            CBAR();
            if (lvl > 3) break;
            const char* src = theirs + (size_t)sblk * BLK_BYTES;
            unsigned ta;
            while ((int)((ta = now16()) - tn) < 0) __builtin_amdgcn_s_sleep(1);
#pragma unroll
            for (int f = 0; f < 4; ++f)
                asm volatile("global_load_dwordx4 %0, %1, %2 offset:%3 sc0 sc1" : "=&v"(v[f]) : "v"(voff), "s"(src + UPAGE(c, f, sblk)), "i"(f * 1024) : "memory");
            TRACE(n, 0);
            VMCNT(4);
            set_flags(prev_flag);
            const char* dst = mine + (size_t)(d * NBLK + blk) * BLK_BYTES;
#pragma unroll
            for (int f = 0; f < 4; ++f) {
                asm volatile("s_waitcnt vmcnt(4)" : "+v"(v[f]) :: "memory");
                STORE_WT(voff, dst + UPAGE(c, f, blk), v[f], f * 1024);
            }
            TRACE(n, 1);
            tn += period;
            if (d >= 4 && n >= nblk - NSLOW - 1) tn += (n == nblk - NSLOW - 1 ? (c + ROT * d) & 31 : 32) * (PSLOW - 4 * PACEC) / 32;
            if (LMAX) tn += max((int)(now16() - ta) - LMAX * 16, 0);
            prev_flag = fmine + (size_t)(d * NBLK + blk) * 32 * FSTRIDE;
        }
        VMCNT(0);
        set_flags(prev_flag);
        if (lane == 0) __hip_atomic_store(cnt, call, __ATOMIC_RELAXED, __HIP_MEMORY_SCOPE_AGENT);
        STAMP(2);
        TRACE_OUT();
        __builtin_amdgcn_endpgm();
    }

        if (role == 1 && wave == 2 && j - 1u < 7u) {
            unsigned f[4] = {0, 0, 0, 0}, el = 0, sum = 0, miss = 32;
            do {
#pragma unroll
                for (int k = 0; k < 4; ++k) {
                    asm volatile("s_mov_b32 exec_hi, 0\n\tglobal_load_dword %0, %2, %3 sc0 sc1\n\ts_mov_b32 exec_hi, -1\n\ts_sleep 3\n\ts_waitcnt vmcnt(3)"
                                 : "+v"(f[k]), "+v"(f[(k + 1) & 3]) : "v"(lane * 128), "s"(myws + ST_OFF + j * 4096) : "memory");
                    const unsigned t = (unsigned)__builtin_amdgcn_s_memrealtime() - tstart;
                    sum += (t - el) * miss;
                    el = t;
                    miss = min(miss, 32u - __builtin_popcount((unsigned)__builtin_amdgcn_ballot_w64(f[(k + 1) & 3] >= call)));
                    if (miss <= 32 - NSIG) miss = 0;
                }
            } while (miss && el < DEADLINE_TICKS);
            asm volatile("s_waitcnt vmcnt(0)" : "+v"(f[0]), "+v"(f[1]), "+v"(f[2]), "+v"(f[3]) :: "memory");
            const unsigned t = tstart * 16u + max((sum >> 1) + GOFF, TMIN * 16u);
            if (lane < 32) __hip_atomic_store((GLB_AS u64*)(myws + RD_OFF + (j * 32 + lane) * 128 + 16), (u64)(t | 1u) << 32 | call, __ATOMIC_RELAXED, __HIP_MEMORY_SCOPE_SYSTEM);
            STAMP(2);
        }
    float scale;
    GLB_AS unsigned* tmo = (GLB_AS unsigned*)(myws + TMO_OFF);
    const float fnan = __builtin_nanf("");
    int rank;

    const int grp = j >> 4, tile = j & 15;
    const int foreign = role == 2 ? !(grp & 1) : 2, slot = grp >> 1;
    const int downer = 4 + (tile >> 2);
    const int su = wave * 208 + j, sq = su >> 5;
    const int se = sq < 10 ? (sq % 5 ? 23 + sq - sq / 5 : 2 + sq / 5) : sq < 18 ? 6 + sq : sq < 20 ? sq - 18 : sq - 16;
    const int sc = se < 16 ? su & 31 : (se & 3) * 8 + ((su & 31) >> 2);
    const int uu = (se < 16 ? se : ((se >> 2) & 3) * NBLK + 16 + (su & 3)) * 32 + sc;
    image();
    const bool selfcopy = role == 2 && su < 1024;
    if (DELAYW) for (int i = 0; i < wave && wave < 8; ++i) __builtin_amdgcn_s_sleep(DELAYW);
    if (selfcopy) {
        const signed char* src = se < 16 ? xin + (size_t)se * BLK_BYTES + sc * 128 : win + (size_t)sc * 128 * NSH + (se & 12) * 32 + (su & 3) * 32;
        const unsigned so = se < 16 ? xsrc_off : (lane >> 1) * NSH + (lane & 1) * 16;
        static_assert(8 * KDIM == 2 * 32 * NSH, "row steps");
#pragma unroll
        for (int i = 0; i < 4; ++i) LOAD_LDS((wave * 16 + i) * 1024, so, src + (i * 64 * NSH >> (se >> 4)));
    }
    const int tsel = foreign ? 1 + (tile & 3) : 0;
    u64 t_lo = a.tab[tsel][slot][0], t_hi = a.tab[tsel][slot][1];
    if (lane == 0) __hip_atomic_store(cnt, call, __ATOMIC_RELAXED, __HIP_MEMORY_SCOPE_AGENT);
    auto src_d = [&](int e) { const int code = (e >> 4) & 7; return foreign && code >= 4 ? downer ^ (code & 3) : code; };
    auto blk_of = [&](int e) { return src_d(e) * NBLK + (e & 15); };

    auto epilogue = [&](int i, int e) {
        const float bad = lds_get(ctr + 17) != 0 ? fnan : 0.0f;
        LDS_AS unsigned char* tp = smem + i * TILE_STRIDE + 128;
        const int srank = rank ^ src_d(e);
        const float* op = a.out + (size_t)(srank * MSH + (e & 15) * 32) * NSH + tile * 32;
        unsigned vo = voff;
        asm volatile("" : "+v"(vo));
        i32x4 q[4];
#pragma unroll
        for (int k = 0; k < 4; ++k) q[k] = *(LDS_AS i32x4*)(tp + k * 1024 + vo);
        vo = (vo >> 7) * (NSH * 4) + (vo & 0x70);
#pragma unroll
        for (int k = 0; k < 4; ++k) {
            f32x4 o;
#pragma unroll
            for (int r = 0; r < 4; ++r) o[r] = fmaxf((float)q[k][r] * scale, 0.0f) + bad;
            STORE_WT(vo, op + (size_t)k * 8 * NSH, o, 0);
        }
    };

    if (role == 2 && wave < 8) {
    bool failed = false;
    const unsigned fxo = (lane & 3) * FSTRIDE;
    auto flag_ptr = [&](int eb) {
        char* p = myws + FX_OFF + (size_t)(eb * 32 + 4 * wave) * FSTRIDE;
        asm("" : "+s"(p));
        return (GLB_AS unsigned*)(p + fxo);
    };
    unsigned aoff = 128 + (4 * (lane >> 5) * 32 + (lane & 31)) * 4;
    asm("" : "+v"(aoff));
    auto ar = __builtin_amdgcn_make_buffer_rsrc((void*)(myws + par * XS_PAR), 0, (unsigned)XS_PAR, 0x00020000);
    i32x4 bf[16];
    if (selfcopy) {
        char* fp = a.ws[lane & 7] + (lane ? RD_OFF + (lane * 32 + sc) * 128 + sq / 5 * 4 : FX_OFF + (size_t)uu * FSTRIDE);
        asm volatile("s_waitcnt vmcnt(0)" : "+v"(fp) :: "memory");
        u32x4 v[4];
        if (se < 16) {
#pragma unroll
            for (int f = 0; f < 4; ++f) v[f] = *(LDS_AS u32x4*)(smem + wave * 16384 + lo[f]);
        } else {
            LDS_AS unsigned char* bp = smem + wave * 16384 + wtr;
#pragma unroll
            for (int f = 0; f < 4; ++f) {
                const i32x2 l2 = __builtin_amdgcn_ds_read_tr8_b64_v2i32((LDS_AS i32x2*)(bp + f * 1024));
                const i32x2 h2 = __builtin_amdgcn_ds_read_tr8_b64_v2i32((LDS_AS i32x2*)(bp + f * 1024 + 256));
                v[f] = __builtin_bit_cast(u32x4, i32x4{l2[0], l2[1], h2[0], h2[1]});
            }
        }
        const char* dst = myws + par * XS_PAR;
#pragma unroll
        for (int f = 0; f < 4; ++f) STORE_WT(voff, dst + UPAGE(uu, f, uu >> 5), v[f], f * 1024);
        VMCNT(0);
        if (lane < 8 && (((sq >= 10 ? 1u : se < 16 ? 15u : 1u | 16u << (su & 3)) >> lane) & 1)) __hip_atomic_store((GLB_AS unsigned*)fp, call, __ATOMIC_RELAXED, __HIP_MEMORY_SCOPE_SYSTEM);
        if (foreign) STAMP(2);
    }
    if (wave * 16384 < NTASK * TILE_STRIDE) {
#pragma unroll
        for (int i = 0; i < 16; ++i) *(LDS_AS u32x4*)(smem + wave * 16384 + i * 1024 + voff) = u32x4{0, 0, 0, 0};
    }
    LGKMCNT0();
    if (lane == 0) __hip_atomic_fetch_add(ctr + 16, 1u, __ATOMIC_RELAXED, __HIP_MEMORY_SCOPE_WORKGROUP);
    u32x4 af[16];
    unsigned fpre;
    auto aload = [&](int k, unsigned abase) { af[k] = __builtin_amdgcn_raw_buffer_load_b128(ar, voff, (abase | UPAGE(0, 0, abase >> 17)) ^ FRAG(k), 17); };
    auto flag_off = [&](int e) { return lane & 4 ? (unsigned)(TMO_OFF - FX_OFF) : (blk_of(e) * 32 + 4 * wave) * FSTRIDE + fxo; };
    auto peek = [&](int e) { fpre = __hip_atomic_load((GLB_AS unsigned*)(myws + FX_OFF + flag_off(e)), __ATOMIC_RELAXED, __HIP_MEMORY_SCOPE_SYSTEM); };
    if (!foreign) STAMP(2);
#pragma clang loop unroll(disable)
    for (int i = -1; i < NTASK; ++i) {
        asm volatile("" : "+s"(i));
        if (i >= 0) { t_lo = (t_lo >> 8) | (t_hi << 56); t_hi = (t_hi >> 8) | (0xFFull << 56); }
        int en = (int)(t_lo & 0xFF);
        unsigned nbase = (unsigned)blk_of(en) * BLK_BYTES + wave * 16384;
        bool ready = false;
        if (i >= 0) {
        failed = failed || (unsigned)__builtin_amdgcn_readlane(fpre, 4) == call;
        ready = en != 0xFF && ((unsigned)__builtin_amdgcn_ballot_w64(fpre >= call) & 15u) == 15u;
        CBAR();
        i32x16 acc = {};
        if (ready) {
            TRACE(i + 1, 0);
            peek((int)(t_lo >> 8) & 0xFF);
#pragma unroll
            for (int k = 0; k < 16; ++k) {
                __builtin_amdgcn_sched_barrier(0);
                acc = __builtin_amdgcn_mfma_i32_32x32x32_i8(__builtin_bit_cast(i32x4, af[k]), bf[k], acc, 0, 0, 0);
                __builtin_amdgcn_sched_barrier(0);
                aload(k, nbase);
            }
            __builtin_amdgcn_sched_barrier(0);
        } else {
#pragma unroll
            for (int k = 0; k < 16; ++k)
                acc = __builtin_amdgcn_mfma_i32_32x32x32_i8(__builtin_bit_cast(i32x4, af[k]), bf[k], acc, 0, 0, 0);
        }
        if (failed && lane == 0) __hip_atomic_store(ctr + 17, 1u, __ATOMIC_RELAXED, __HIP_MEMORY_SCOPE_WORKGROUP);
        LDS_AS unsigned char* tp = smem + i * TILE_STRIDE + aoff;
#pragma unroll
        for (int r = 0; r < 16; ++r)
            __hip_atomic_fetch_add((LDS_AS int*)(tp + ((r & 3) + 8 * (r >> 2)) * 128), acc[r],
                                   __ATOMIC_RELAXED, __HIP_MEMORY_SCOPE_WORKGROUP);
        LGKMCNT0();
        if (lane == 0) __hip_atomic_fetch_add(ctr + i, 1u, __ATOMIC_RELAXED, __HIP_MEMORY_SCOPE_WORKGROUP);
        TRACE(i, 1);
        }
        if (!ready) {
            if (en == 0xFF) break;
            if (!failed) failed = !poll_flags(flag_off(en), myws + FX_OFF, tmo, call, FASTX < 0 ? 1u : (14u >> ((en >> 4) & 7)) & (((en & 15) + 16 - FASTX) >> 4), lane);
            CBAR();
            TRACE(i + 1, 0);
            peek((int)(t_lo >> 8) & 0xFF);
            __builtin_amdgcn_sched_barrier(0);
#pragma unroll
            for (int k = 0; k < 16; ++k) { aload(k, nbase); __builtin_amdgcn_sched_barrier(0); }
            __builtin_amdgcn_sched_barrier(0);
            if (i < 0) {
                {
                    const int ebw = (foreign ? downer : tile >> 2) * NBLK + 16 + (tile & 3);
                    if (!failed) failed = !wait_flags(flag_ptr(ebw), 0, tmo, call, lane);
                    CBAR();
                    if (foreign) STAMP(6);
                    peek((int)(t_lo >> 8) & 0xFF);
                    CBAR();
                    const unsigned bbase = (unsigned)ebw * BLK_BYTES + wave * 16384;
#pragma unroll
                    for (int k = 0; k < 16; ++k)
                        bf[k] = __builtin_bit_cast(i32x4, __builtin_amdgcn_raw_buffer_load_b128(ar, voff, (bbase | UPAGE(0, 0, bbase >> 17)) ^ FRAG(k), 17));
                }
                TRACE(0, 0);
            }
            if (i < 0) {
                lds_wait(ctr + 16, 8u);
                __builtin_amdgcn_s_waitcnt(0x0F70);
                STAMP(1);
            }
        }
    }
    STAMP(5);
    TRACE_OUT();
    __builtin_amdgcn_s_waitcnt(0x0F70);
    if (!foreign || slot < 3) __builtin_amdgcn_endpgm();
    }
    asm volatile("" : "+s"(ap));
    peer = ap->ws[4 + ((j >> 2) & 3)];
    rank = ap->rank;
    scale = __builtin_bit_cast(float, __builtin_amdgcn_readfirstlane(__builtin_bit_cast(int, *(const CONST_AS float*)ap->sx * *(const CONST_AS float*)ap->sw)));
    if (role == 2 && wave >= 8) {
        const u64 o_lo = ap->ord[tile & 3][slot][0], o_hi = ap->ord[tile & 3][slot][1];
        for (int i = wave - 8; i < NTASK; i += 4) {
            const int sh = 8 * (i & 7);
            const int e = (int)(((i & 8 ? t_hi : t_lo) >> sh) & 0xFF);
            if (e == 0xFF) break;
            const unsigned oi = (unsigned)(((i & 8 ? o_hi : o_lo) >> sh) & 0xFF), idx = (tile >> 2) * NIDX + oi;
            lds_wait(ctr + i, 8u);
            if (!foreign) {
                TRACE(i, 0);
                epilogue(i, e);
                TRACE(i, 1);
                continue;
            }
            TRACE(i, 0);
            const bool failed = lds_get(ctr + 17) != 0;
            LDS_AS unsigned char* tp = smem + i * TILE_STRIDE;
            i32x4 q[4];
#pragma unroll
            for (int k = 0; k < 4; ++k) q[k] = *(LDS_AS i32x4*)(tp + 128 + k * 1024 + voff);
            __builtin_amdgcn_s_waitcnt(0xC07F);
            CBAR();
            if (scale < 0.0f) {
#pragma unroll
                for (int k = 0; k < 4; ++k) q[k] = -q[k];
            }
            unsigned m = 0;
            int mx = 0;
#pragma unroll
            for (int r = 15; r >= 0; --r) {
                m = __builtin_amdgcn_alignbit(m, (unsigned)-q[r >> 2][r & 3], 31);
                mx = max(mx, q[r >> 2][r & 3]);
            }
            if (P21) *(LDS_AS u32x4*)(tp + 128 + voff) = u32x4{0, 0, 0, 0};
            const bool wide = __builtin_amdgcn_ballot_w64(mx >= (1 << (P21 ? 21 : 24))) != 0;
            const unsigned c1 = __builtin_popcount(m), inc = wave_scan(c1);
            const unsigned tot = __builtin_amdgcn_readlane(inc, 63);
            const unsigned hbytes = (5 * tot + 31) >> 5 << 2;
            const unsigned nbytes = (128 + (wide ? tot * 4 : P21 ? hbytes + 2 * tot : 3 * tot) + 63) & ~63u;
            *(LDS_AS unsigned short*)(tp + lane * 2) = (unsigned short)m;
            if (wide) {
                unsigned ep = (unsigned)(size_t)tp + 128 + (inc - c1) * 4;
#pragma unroll
                for (int r = 0; r < 16; ++r)
                    asm volatile("v_cmpx_lt_i32 vcc, 0, %1\n\tds_write_b32 %0, %1\n\tv_add_u32 %0, 4, %0\n\ts_mov_b64 exec, -1"
                                 : "+v"(ep) : "v"(q[r >> 2][r & 3]) : "memory", "vcc");
            } else if (P21) {
                const unsigned tb = (unsigned)(size_t)tp + 128, ad = tb + 3072;
                unsigned a2 = tb + hbytes + 2 * (inc - c1), g = 5 * (inc - c1);
#pragma unroll
                for (int r0 = 0; r0 < 16; r0 += 6) {
                    unsigned acc = 0, sh5 = 0;
#pragma unroll
                    for (int r = r0; r < r0 + 6 && r < 16; ++r) {
                        int nb;
                        BITMASK(nb, m, r);
                        asm volatile("ds_write_b16 %0, %1" :: "v"((a2 & nb) | (ad & ~nb)), "v"(q[r >> 2][r & 3]) : "memory");
                        acc |= (((unsigned)q[r >> 2][r & 3] >> 16) & nb) << sh5;
                        STEP(a2, nb, 2);
                        STEP(sh5, nb, 5);
                    }
                    const u64 w = (u64)acc << (g & 31);
                    asm volatile("ds_or_b32 %0, %1\n\tds_or_b32 %0, %2 offset:4" :: "v"(tb + (g >> 5 << 2)), "v"((unsigned)w), "v"((unsigned)(w >> 32)) : "memory");
                    g += sh5;
                }
            } else {
                const unsigned tb = (unsigned)(size_t)tp + 128, c2 = -tb - 4 * tot, ad = tb + 2 * tot + 1536;
                unsigned a1 = tb + 2 * tot + inc - c1;
#pragma unroll
                for (int r = 0; r < 16; ++r) {
                    const unsigned nb = __builtin_amdgcn_sbfe(m, r, 1);
                    const unsigned ae = (a1 & nb) | (ad & ~nb);
                    a1 -= nb;
                    asm volatile("ds_write_b16 %0, %2\n\tds_write_b8_d16_hi %1, %2" :: "v"(ae * 2 + c2), "v"(ae), "v"(q[r >> 2][r & 3]) : "memory");
                }
            }
            const unsigned info = nbytes | (call & 63u) | (unsigned)((tile & 3) << 8 | e) << 16 | (wide ? F_WIDE : 0u) | (failed ? F_NAN : 0u);
            const char* dst = myws + OS_OFF + par * OS_PAR + (size_t)idx * OSLOT + 2048;
            TRACE(i, 1);
            TRACE2(i, 0);
#pragma unroll
            for (int k = 0; k < 5; ++k)
                if (k * 1024 < nbytes) {
                    const u32x4 pv = *(LDS_AS u32x4*)(tp + k * 1024 + voff);
                    if (k * 1024 + voff < nbytes) STORE_WT(voff, dst, pv, k * 1024 - 2048);
                }
            VMCNT(0);
            flag64_set(peer + FO_OFF + (size_t)idx * 128, call, info, lane);
            TRACE2(i, 1);
        }
        STAMP(5);
        TRACE_OUT();
        __builtin_amdgcn_endpgm();
    }
    {
        const int link = j & 3, kstep = role == 1 ? 48 : NIDX, kend = role == 1 ? KSPLIT : a.ntotal;
        const int k0 = role == 1 ? wave * 4 + (j >> 2) : KSPLIT + wave * 16 + (NSLOT - 1 - slot) * 4 + (tile >> 2);
        LDS_AS unsigned char* tp = smem + (role == 1 ? wave : (wave & 7) + NTASK) * TILE_STRIDE;
        const int dn = 4 + link;
        bool failed = false;
        u64 fpre = 0;
        if (role == 1 && wave < 2) {
            const int c = j * 2 + wave;
            GLB_AS unsigned* rdy = (GLB_AS unsigned*)(a.ws[lane & 7] + RD_OFF + ((lane & 7) * 32 + c) * 128);
            GLB_AS unsigned* fl = (GLB_AS unsigned*)(myws + FX_OFF + (size_t)((lane & 16 ? ((lane >> 2) & 3) * NBLK + 16 + (lane & 3) : lane & 15) * 32 + c) * FSTRIDE);
            while ((unsigned)__builtin_amdgcn_s_memrealtime() - tstart < 300) __builtin_amdgcn_s_sleep(8);
            unsigned f = lane < 32 ? 0 : call;
            do {
                if (f < call) f = __hip_atomic_load(fl, __ATOMIC_RELAXED, __HIP_MEMORY_SCOPE_SYSTEM);
                if (__builtin_amdgcn_ballot_w64(f >= call) == ~0ull) {
                    CBAR();
                    if (lane - 1u < 7u) __hip_atomic_store(rdy + 2, call, __ATOMIC_RELAXED, __HIP_MEMORY_SCOPE_SYSTEM);
                    break;
                }
            } while ((unsigned)__builtin_amdgcn_s_memrealtime() - tstart < DEADLINE_TICKS);
            STAMP(2);
        }
        while ((unsigned)__builtin_amdgcn_s_memrealtime() - tstart < 1600) __builtin_amdgcn_s_sleep(8);
        const u64 t64 = flag64_get(myws + RD_OFF + (dn * 32 + (k0 & 31)) * 128 + 16);
        const unsigned turn = __builtin_amdgcn_readfirstlane((unsigned)t64 == call ? (unsigned)(t64 >> 32) + (2 + 16 - N1 - NSLOW) * 4 * PACEC + TSOFF : tstart * 16u);
        const char* theirs = a.ws[dn] + OS_OFF + par * OS_PAR;
        for (int n = 0; k0 + kstep * n < kend; ++n) {
            const int idx = link * NIDX + k0 + kstep * n;
            const unsigned toff = (k0 + kstep * n) * TSTEP + min(k0 + kstep * n, K1) * (TSTEP1 - TSTEP);
            if (n == 0)
                while ((int)((unsigned)__builtin_amdgcn_s_memrealtime() * 16u - turn - toff + 2400) < 0) __builtin_amdgcn_s_sleep(1);
            const unsigned info = (unsigned)fpre >= call ? (unsigned)(fpre >> 32)
                : poll_flag64(lane & 4 ? (unsigned)(TMO_OFF - FO_OFF) : idx * 128, myws + FO_OFF, tmo, call, (unsigned)(k0 + kstep * n + 256 - FASTO) >> 8, lane);
            CBAR();
            while ((int)((unsigned)__builtin_amdgcn_s_memrealtime() * 16u - turn - toff) < 0) __builtin_amdgcn_s_sleep(1);
            TRACE(n, 0);
            if (info == 0) {
                a.out[(size_t)rank * MSH * NSH + lane] = fnan;
                break;
            }
            const unsigned e = info >> 16;
            const int code = (e >> 4) & 7;
            const int srank = rank ^ (code < 4 ? dn ^ code : code & 3);
            float* op = a.out + (size_t)(srank * MSH + (e & 15) * 32 + (lane >> 3)) * NSH + (((e >> 8) & 3) * 4 + link) * 32 + (lane & 7) * 4;
            const char* srcp = theirs + (size_t)idx * OSLOT;
            const unsigned nbytes = min(info & 0xFFC0u, 4224u);
#pragma unroll
            for (int k = 0; k < 5; ++k)
                if (k * 1024 + voff < nbytes) LOAD_LDS_SYS((unsigned)(size_t)tp + k * 1024, voff, srcp + k * 1024);
            unsigned tv = tmo_get(tmo);
            VMCNT(0);
            tv = __builtin_amdgcn_readfirstlane(tv);
            asm volatile("" : "+s"(tv) :: "memory");
            fpre = 0;
            if (k0 + kstep * (n + 1) < kend) fpre = flag64_get(myws + FO_OFF + (size_t)(idx + kstep) * 128);
            failed = failed || tv == call;
            const float mul = failed || (info & F_NAN) ? fnan : __builtin_fabsf(scale);
            const unsigned m = *(LDS_AS unsigned short*)(tp + lane * 2);
            const unsigned c1 = __builtin_popcount(m), inc = wave_scan(c1);
            unsigned y[16];
            if (info & F_WIDE) {
                unsigned ix = inc - c1;
#pragma unroll
                for (int r = 0; r < 16; ++r) {
                    const int nb = __builtin_amdgcn_sbfe(m, r, 1);
                    const unsigned w = *(LDS_AS unsigned*)(tp + 128 + ix * 4);
                    ix -= nb;
                    y[r] = __builtin_bit_cast(unsigned, (float)(w & (unsigned)nb) * mul);
                }
            } else if (P21) {
                const unsigned tot = __builtin_amdgcn_readlane(inc, 63);
                const unsigned tb = (unsigned)(size_t)tp + 128;
                unsigned a2 = tb + ((5 * tot + 31) >> 5 << 2) + 2 * (inc - c1), g[3], win[3];
                u32x2 hw[3];
                int nb[16];
#pragma unroll
                for (int k = 0; k < 3; ++k) {
                    g[k] = 5 * (inc - c1 + __builtin_popcount(m & ((1u << 6 * k) - 1u)));
                    const LDS_AS unsigned* hp = (const LDS_AS unsigned*)(size_t)(tb + (g[k] >> 5 << 2));
                    hw[k] = u32x2{hp[0], hp[1]};
                }
#pragma unroll
                for (int r = 0; r < 16; ++r) {
                    y[r] = *(LDS_AS unsigned short*)(size_t)a2;
                    BITMASK(nb[r], m, r);
                    STEP(a2, nb[r], 2);
                }
#pragma unroll
                for (int r = 0; r < 16; r += 8)
                    asm volatile("" : "+v"(y[r]), "+v"(y[r + 1]), "+v"(y[r + 2]), "+v"(y[r + 3]),
                                      "+v"(y[r + 4]), "+v"(y[r + 5]), "+v"(y[r + 6]), "+v"(y[r + 7]) :: "memory");
                asm volatile("" : "+v"(hw[0]), "+v"(hw[1]), "+v"(hw[2]));
#pragma unroll
                for (int k = 0; k < 3; ++k) win[k] = __builtin_amdgcn_alignbit(hw[k][1], hw[k][0], g[k]);
                unsigned sh5 = 0;
#pragma unroll
                for (int r = 0; r < 16; ++r) {
                    if (r % 6 == 0) sh5 = 0;
                    asm("v_lshrrev_b32_sdwa %0, %1, %2 dst_sel:WORD_1 dst_unused:UNUSED_PRESERVE src0_sel:DWORD src1_sel:DWORD" : "+v"(y[r]) : "v"(sh5), "v"(win[r / 6]));
                    y[r] = __builtin_bit_cast(unsigned, (float)(y[r] & (unsigned)nb[r] & 0x1FFFFFu) * mul);
                    STEP(sh5, nb[r], 5);
                }
            } else {
                const unsigned tot = __builtin_amdgcn_readlane(inc, 63);
                const unsigned tb = (unsigned)(size_t)tp + 128, c2 = -tb - 4 * tot;
                unsigned a1 = tb + 2 * tot + inc - c1, wh[16];
#pragma unroll
                for (int r = 0; r < 16; ++r) {
                    y[r] = *(LDS_AS unsigned short*)(size_t)(a1 * 2 + c2);
                    wh[r] = *(LDS_AS unsigned char*)(size_t)a1;
                    a1 -= __builtin_amdgcn_sbfe(m, r, 1);
                }
#pragma unroll
                for (int r = 0; r < 16; r += 4)
                    asm volatile("" : "+v"(y[r]), "+v"(y[r + 1]), "+v"(y[r + 2]), "+v"(y[r + 3]),
                                      "+v"(wh[r]), "+v"(wh[r + 1]), "+v"(wh[r + 2]), "+v"(wh[r + 3]) :: "memory");
#pragma unroll
                for (int r = 0; r < 16; ++r)
                    y[r] = __builtin_bit_cast(unsigned, (float)((wh[r] << 16 | y[r]) & (unsigned)__builtin_amdgcn_sbfe(m, r, 1)) * mul);
            }
            fpre = (u64)(unsigned)__builtin_amdgcn_readfirstlane((unsigned)(fpre >> 32)) << 32 | (unsigned)__builtin_amdgcn_readfirstlane((unsigned)fpre);
            asm volatile("" : "+s"(fpre));
#pragma unroll
            for (int k = 0; k < 4; ++k) {
                const u32x4 o = {y[4 * k], y[4 * k + 1], y[4 * k + 2], y[4 * k + 3]};
                asm volatile("global_store_dwordx4 %0, %1, off sc0 sc1\n\ts_nop 1" :: "v"(op + (size_t)k * 8 * NSH), "v"(o) : "memory");
            }
            TRACE(n, 1);
        }
        STAMP(role == 1 ? 5 : 4);
        TRACE_OUT();
    }
    __builtin_amdgcn_s_waitcnt(0x0F70);
}

static void build_table(int foreign, float tstart, u64 (*tab)[2], float (*fin)[NTASK] = nullptr) {
    struct T { float t; int e, last; } task[128];
    const int nslot = foreign ? NSLOT : NOWN;
    const float tb = PACEI / 400.0f, tx = PACEC / 400.0f;
    int n = 0;
    for (int d = 0; d < 8; ++d)
        for (int blk = 0; blk < 16; ++blk) {
            const int type = blk < N1 ? 1 : blk < N1 + N3 ? 3 : 2;
            float t;
            int direct = 0;
            if (d == 0) { if (foreign && type != 1) continue; t = 3.0f; }
            else if (d < 4) {
                if (type == 2 || (foreign && type == 3)) continue;
                t = TA + tb * blk;
                direct = !foreign && blk == N1 + N3 - 1;
            } else {
                if (type == 1 || (foreign && (type == 3 || d == 4))) continue;
                t = TA + tx * (19 - blk) + (blk < N1 + NSLOW ? (N1 + NSLOW - blk) * (PSLOW / 1600.0f - tx) : 0.0f);
            }
            task[n].t = t + 0.001f * d; task[n].e = direct << 7 | d << 4 | blk; task[n].last = d && d < 4 && blk == N1 - 1; ++n;
        }
    for (int i = 1; i < n; ++i)
        for (int k = i; k > 0 && task[k].t < task[k - 1].t; --k) { T s = task[k]; task[k] = task[k - 1]; task[k - 1] = s; }
    float fr[NSLOT]; int cntv[NSLOT]; unsigned char lst[NSLOT][NTASK];
    for (int s = 0; s < NSLOT; ++s) { fr[s] = tstart; cntv[s] = 0; for (int i = 0; i < NTASK; ++i) lst[s][i] = 0xFF; }
    for (int i = 0; i < n; ++i) {
        int best = -1;
        for (int s = 0; s < nslot; ++s) {
            if (foreign && task[i].t > TRES && (s < 3) != task[i].last) continue;
            if (!foreign && task[i].t > TA + (N1 + N3 - OTRK) * tb && (task[i].e & 0x40 ? s < 3 : s + 1 != ((task[i].e >> 4) & 3))) continue;
            if (cntv[s] < NTASK - 1 && (best < 0 || fr[s] < fr[best])) best = s;
        }
        fr[best] = (fr[best] > task[i].t ? fr[best] : task[i].t) + TCOST;
        if (fin) fin[best][cntv[best]] = fr[best];
        lst[best][cntv[best]++] = (unsigned char)task[i].e;
    }
    for (int s = 0; s < NSLOT; ++s)
        for (int h = 0; h < 2; ++h) {
            u64 v = 0;
            for (int i = 7; i >= 0; --i) v = v << 8 | lst[s][h * 8 + i];
            tab[s][h] = v;
        }
}

static void build_all(Args& a) {
    build_table(0, OST0, a.tab[0]);
    auto byte = [&](int t, int s, int i) { return i >= NTASK ? 0xFF : (int)((a.tab[1 + t][s][i >> 3] >> (8 * (i & 7))) & 0xFF); };
    int n = 0;
    float fin[4][NSLOT][NTASK];
    int next[4][NSLOT];
    for (int t = 0; t < 4; ++t) {
        build_table(1 + t, FSTART + FSTEP * t, a.tab[1 + t], fin[t]);
        for (int s = 0; s < NSLOT; ++s) {
            for (int i = 0; i < NTASK; ++i) n += byte(t, s, i) < 0x80;
            a.ord[t][s][0] = a.ord[t][s][1] = 0;
            next[t][s] = 0;
        }
    }
    a.npacked = n;
    for (int k = 0; k < n; ++k) {
        int bt = -1, bs = 0;
        for (int t = 0; t < 4; ++t)
            for (int s = 0; s < NSLOT; ++s)
                if (byte(t, s, next[t][s]) < 0x80 && (bt < 0 || fin[t][s][next[t][s]] < fin[bt][bs][next[bt][bs]])) { bt = t; bs = s; }
        const int i = next[bt][bs]++;
        a.ord[bt][bs][i >> 3] |= (u64)k << (8 * (i & 7));
    }
    for (int blk = 0; blk < 16; ++blk)
        for (int t = 0; t < 4; ++t)
            for (int s = 0; s < NSLOT; ++s)
                for (int i = 0; i < NTASK; ++i)
                    if (byte(t, s, i) != 0xFF && (byte(t, s, i) & 0x8F) == (0x80 | blk)) a.ord[t][s][i >> 3] |= (u64)n++ << (8 * (i & 7));
    a.ntotal = n;
}

extern "C" void kernel_launch(int rank, int nranks, ncclComm_t comm,
                              hipStream_t stream,
                              void* const* d_in, const int* in_sizes, int n_in,
                              void* d_out, int out_size,
                              const int* in_dtypes, int out_dtype,
                              void* const* d_out_all,
                              void* const* d_ws_all, size_t ws_bytes) {
    if (nranks != NRANK || (unsigned)rank >= (unsigned)NRANK || n_in != 4 || ws_bytes < WS_NEED ||
        in_sizes[0] != MSH * KDIM || in_sizes[1] != KDIM * NSH || out_size != NRANK * MSH * NSH) {
        fprintf(stderr, "unexpected problem shape\n");
        return;
    }
    HIP_CHECK(hipFuncSetAttribute((const void*)ag_gemm_kernel, hipFuncAttributeMaxDynamicSharedMemorySize, LDS_BYTES));
    Args a;
    a.sx = (const float*)d_in[2];
    a.sw = (const float*)d_in[3];
    a.out = (float*)d_out;
    for (int d = 0; d < NRANK; ++d) a.ws[d] = (char*)d_ws_all[rank ^ d];
    for (int i = 0; i < NTHREADS / 64; ++i) {
        Rec& r = a.rec[i];
        const int d = i & 7;
        r.x = (const signed char*)d_in[0];
        r.w = (const signed char*)d_in[1];
        r.ws = a.ws[0];
        r.peer = a.ws[d];
        r.src = d < 4 ? r.x : r.w + (d & 3) * 32;
        r.dst = r.peer + (size_t)(d * NBLK + (d < 4 ? 0 : 16)) * BLK_BYTES;
        r.cnt = r.ws + CNT_OFF + i * 4;
        r.dst1 = r.dst + XS_PAR;
    }
    build_all(a);
    if (a.ntotal > NIDX || a.ntotal > KSPLIT + 128 || NSLOT != 7) return;
    a.rank = rank;
    ag_gemm_kernel<<<dim3(256), dim3(NTHREADS), LDS_BYTES, stream>>>(a);
    HIP_CHECK(hipGetLastError());
}
```
